```python
import numpy as np

kernel_name = "gpu_gemm_rs_m4096_k4096_n16384_bf16_relu"


ABSMAX_RTOL = 0.008

COMPUTE_DTYPE = "bfloat16"
EPILOGUE = "relu"
IN_QUANT = None
OUT_QUANT = None
IN_DTYPES = {"x": "bfloat16", "w_mat": "bfloat16"}
WORLD_SIZE = 8

def _to_bf16(x):
    u = np.ascontiguousarray(x, dtype=np.float32).view(np.uint32)
    return ((u + 0x7FFF + ((u >> 16) & 1)) & 0xFFFF0000).view(np.float32)

def setup_inputs():

    x = _to_bf16(np.random.randn(4096, 4096).astype(np.float32))
    w_mat = _to_bf16((np.random.randn(4096, 16384) / np.sqrt(4096)).astype(np.float32))
    return {"x": x, "w_mat": w_mat}

def _epilogue(y):
    return np.maximum(y, 0.0).astype(np.float32)

def reference(x, w_mat):
    y = (x @ w_mat).astype(np.float32)
    return _epilogue(y)
```

```cpp
#include <hip/hip_runtime.h>
#include <rccl/rccl.h>
#include <cstdio>
#include <cstdint>

#define HIP_CHECK(expr) do { hipError_t _e = (expr); if (_e != hipSuccess) { \
    fprintf(stderr, "HIP error %s at %s:%d\n", hipGetErrorString(_e), __FILE__, __LINE__); \
    return; } } while (0)

namespace {

typedef unsigned u32;
typedef unsigned long long u64;
typedef unsigned short u16;
typedef u32 u32x4 __attribute__((ext_vector_type(4)));
typedef u32 u32x2 __attribute__((ext_vector_type(2)));
typedef u32 u32x32 __attribute__((ext_vector_type(32)));
typedef u16 u16x2 __attribute__((ext_vector_type(2)));
typedef float f32x4 __attribute__((ext_vector_type(4)));
typedef float f32x2 __attribute__((ext_vector_type(2)));
typedef __bf16 bf16x2 __attribute__((ext_vector_type(2)));
typedef __bf16 bf16x8 __attribute__((ext_vector_type(8)));
typedef __amdgpu_buffer_rsrc_t rsrc_t;
#define LAS __attribute__((address_space(3)))
#define GAS __attribute__((address_space(1)))
#define DEV __device__ __forceinline__

#define FORCE_RAW_IN 0

constexpr int NR = 8;
constexpr int NTOT = 16384, KSH = 512;
constexpr int NWG = 256, NCOMM = 32, NPULL = 28, NGEMM = NWG - NCOMM, NTHREADS = 768;
constexpr int NCHIN = 384;
constexpr int NMSG = 224;
constexpr int NQIN = NCHIN / 4, NQY = NMSG / 4, NQ = NQIN + NQY;
constexpr int NWORK = 10;

constexpr u32 WS_CNT = 0, WS_TMO = 128;
constexpr u32 WS_FIN = 4096;
constexpr u32 WS_FY = 32768;
constexpr u32 WS_XR = 49152;
constexpr u32 WS_WR = 57344;
constexpr u32 WS_DBG = 65536;
constexpr u32 CHB = 16384;
constexpr u32 WS_PK = 1u << 20;
constexpr u32 PK_PAR = 1280u * CHB;
constexpr u32 WS_XA = WS_PK + 2 * PK_PAR;
constexpr u32 WS_WB = WS_XA + (32u << 20);
constexpr u32 YSLOT = 10752;
constexpr u32 WS_YP = WS_WB + (16u << 20);
constexpr u32 WS_END = WS_YP + 8u * NMSG * YSLOT;
constexpr u32 PKD_BYTES = 12288 + 128;

constexpr u32 STG = 28672, NSTG = 5;
constexpr u32 LBIG = 16384, NBIG = 6, LSML = 5632, NSML = 8;
constexpr u32 L_SML0 = LBIG * NBIG;
constexpr u32 L_CTL = 143360;
static_assert(STG * NSTG <= L_CTL && L_SML0 + LSML * NSML <= L_CTL, "ring");
constexpr u32 C_CALL = 0, C_ABORT = 1, C_TICKET = 2, C_QN = 3, C_FULL = 8, C_FREE = 16, C_BUSY = 24,
              C_Q = 64, C_FQ = C_Q + 160, C_HDR = C_FQ + 160, C_WORDS = C_HDR + 4 * 32;
constexpr u32 LDS_BYTES = L_CTL + C_WORDS * 4;
static_assert(LDS_BYTES <= 160 * 1024 && NQ <= 160, "lds");

constexpr u64 DEADLINE = 40000000ull;
constexpr u64 DEADLINE_LDS = 100000000ull;

constexpr u32 T_INP = 0, T_INR = 1, T_YC = 2, T_YR = 3;
constexpr u32 YM_RAW = 1u << 16, YM_BAD = 1u << 17;

struct Order { u16 v[7][NCHIN]; };
constexpr Order make_order() {
    Order o{};
    for (int d = 1; d <= 7; ++d) {
        int n = 0;
        for (int s = 0; s < 8; ++s) {
            for (int c = 0; c < 16; ++c) o.v[d - 1][n++] = (u16)(0x8000 | s << 8 | c);
            for (int i = 0; i < 32; ++i) o.v[d - 1][n++] = (u16)(s << 8 | ((((i >> 2) + d) & 7) << 2 | (i & 3)));
        }
    }
    return o;
}
__constant__ const Order ORDER = make_order();

struct Args { const u16* x; const u16* w; u16* out; char* ws[NR]; int rank; int pad; };

DEV u64 rt() { return __builtin_amdgcn_s_memrealtime(); }
DEV u32 rfl(u32 v) { return __builtin_amdgcn_readfirstlane(v); }
DEV u32 perm(u32 hi, u32 lo, u32 sel) { return __builtin_amdgcn_perm(hi, lo, sel); }
DEV u32 bfi(u32 m, u32 a, u32 b) { return (m & a) | (~m & b); }
#define CBAR() asm volatile("" ::: "memory")
#define WAIT_VM0() asm volatile("s_waitcnt vmcnt(0)" ::: "memory")
#define WAIT_LGKM0() do { CBAR(); __builtin_amdgcn_s_waitcnt(0xc07f); CBAR(); } while (0)

#define STAMPW(c, i) do {} while (0)

struct Ctx {
    const u16* x; const u16* w; u16* out; char* me; char* peer_lane;
    rsrc_t rs;
    LAS char* lds; volatile LAS u32* ctl;
    int rank, wg, wave, lane;
};

DEV u32 ld_sys(const void* p) { return __hip_atomic_load((const GAS u32*)p, __ATOMIC_RELAXED, __HIP_MEMORY_SCOPE_SYSTEM); }
DEV u64 ld_sys64(const void* p) { return __hip_atomic_load((const GAS u64*)p, __ATOMIC_RELAXED, __HIP_MEMORY_SCOPE_SYSTEM); }
DEV void st_sys(void* p, u32 v) { __hip_atomic_store((GAS u32*)p, v, __ATOMIC_RELAXED, __HIP_MEMORY_SCOPE_SYSTEM); }
DEV void st_sys64(void* p, u64 v) { __hip_atomic_store((GAS u64*)p, v, __ATOMIC_RELAXED, __HIP_MEMORY_SCOPE_SYSTEM); }
DEV void st16(const Ctx& c, u32x4 v, u32 voff, u32 soff) { __builtin_amdgcn_raw_buffer_store_b128(v, c.rs, voff + soff, 0, 17); }
DEV bool tmo_set(const Ctx& c, u32 call) { return rfl(__hip_atomic_load((const GAS u32*)(c.me + WS_TMO), __ATOMIC_RELAXED, __HIP_MEMORY_SCOPE_AGENT)) == call; }
DEV void tmo_raise(const Ctx& c, u32 call) { if (c.lane == 0) __hip_atomic_store((GAS u32*)(c.me + WS_TMO), call, __ATOMIC_RELAXED, __HIP_MEMORY_SCOPE_AGENT); }

DEV bool lds_wait(const Ctx& c, u32 word, u32 want, bool look_abort, int nap) {
    if (rfl(c.ctl[word]) >= want) return true;
    const u64 t0 = rt();
    for (u32 n = 1;; ++n) {
        if (rfl(c.ctl[word]) >= want) return true;
        if (nap) __builtin_amdgcn_s_sleep(1);
        if ((n & 63) == 0) {
            if (look_abort && rfl(c.ctl[C_ABORT])) return false;
            if (rt() - t0 > DEADLINE_LDS) { c.ctl[C_ABORT] = 1; return false; }
        }
    }
}
DEV u32 get_call(const Ctx& c) { lds_wait(c, C_CALL, 1, false, 1); return rfl(c.ctl[C_CALL]); }

DEV void glds4(rsrc_t rs, u32 voff, u32 soff, u32 ldsaddr) {
    u32 keep;
    asm volatile("s_mov_b32 %0, m0\n\ts_mov_b32 m0, %4\n\ts_nop 2\n\t"
                 "buffer_load_dwordx4 %1, %2, %3 offen sc0 sc1 lds\n\t"
                 "buffer_load_dwordx4 %1, %2, %3 offen offset:1024 sc0 sc1 lds\n\t"
                 "buffer_load_dwordx4 %1, %2, %3 offen offset:2048 sc0 sc1 lds\n\t"
                 "buffer_load_dwordx4 %1, %2, %3 offen offset:3072 sc0 sc1 lds\n\t"
                 "s_mov_b32 m0, %0" : "=&s"(keep) : "v"(voff), "s"(rs), "s"(soff), "s"(ldsaddr) : "memory");
}
DEV void glds1(rsrc_t rs, u32 voff, u32 soff, u32 ldsaddr) {
    u32 keep;
    asm volatile("s_mov_b32 %0, m0\n\ts_mov_b32 m0, %4\n\ts_nop 2\n\t"
                 "buffer_load_dwordx4 %1, %2, %3 offen sc0 sc1 lds\n\t"
                 "s_mov_b32 m0, %0" : "=&s"(keep) : "v"(voff), "s"(rs), "s"(soff), "s"(ldsaddr) : "memory");
}
#define VMC(n) case n: asm volatile("s_waitcnt vmcnt(" #n ")" ::: "memory"); break;
#define VMC8(a, b, c, d, e, f, g, h) VMC(a) VMC(b) VMC(c) VMC(d) VMC(e) VMC(f) VMC(g) VMC(h)
DEV void wait_vm(u32 n) {
    switch (n) {
        VMC8(1, 2, 3, 4, 5, 6, 7, 8) VMC8(9, 10, 11, 12, 13, 14, 15, 16) VMC8(17, 18, 19, 20, 21, 22, 23, 24)
        VMC8(25, 26, 27, 28, 29, 30, 31, 32) VMC8(33, 34, 35, 36, 37, 38, 39, 40) VMC8(41, 42, 43, 44, 45, 46, 47, 48)
        default: WAIT_VM0(); break;
    }
}

DEV u32 wave_max(u32 v) {
    v = max(v, (u32)__builtin_amdgcn_update_dpp(0, (int)v, 0x111, 0xf, 0xf, true));
    v = max(v, (u32)__builtin_amdgcn_update_dpp(0, (int)v, 0x112, 0xf, 0xf, true));
    v = max(v, (u32)__builtin_amdgcn_update_dpp(0, (int)v, 0x114, 0xf, 0xf, true));
    v = max(v, (u32)__builtin_amdgcn_update_dpp(0, (int)v, 0x118, 0xf, 0xf, true));
    v = max(v, (u32)__builtin_amdgcn_update_dpp(0, (int)v, 0x142, 0xa, 0xf, false));
    v = max(v, (u32)__builtin_amdgcn_update_dpp(0, (int)v, 0x143, 0xc, 0xf, false));
    return __builtin_amdgcn_readlane(v, 63);
}

DEV void store_raw(const Ctx& c, const u32x4 (&R)[16], u32 off0, u32 off1) {
#pragma unroll
    for (int f = 0; f < 16; ++f) st16(c, R[f], c.lane * 16, (f < 8 ? off0 : off1) + (f & 7) * 1024);
}
DEV u32x4 join8(u32 ha, u32 la, u32 hb, u32 lb) {
    return (u32x4){perm(ha, la, 0x05010400u), perm(ha, la, 0x07030602u), perm(hb, lb, 0x05010400u), perm(hb, lb, 0x07030602u)};
}
DEV u32 pack_store(const Ctx& c, const u32x4 (&R)[16], u32 dst) {
    u32x32 H, L;
    u16x2 m0 = {0, 0}, m1 = {0, 0};
#pragma unroll
    for (int f = 0; f < 16; ++f) {
        L[2 * f] = perm(R[f][1], R[f][0], 0x06040200u); L[2 * f + 1] = perm(R[f][3], R[f][2], 0x06040200u);
        H[2 * f] = perm(R[f][1], R[f][0], 0x07050301u); H[2 * f + 1] = perm(R[f][3], R[f][2], 0x07050301u);
    }
#pragma unroll
    for (int i = 0; i < 8; ++i) st16(c, (u32x4){L[4 * i], L[4 * i + 1], L[4 * i + 2], L[4 * i + 3]}, c.lane * 16, dst + i * 1024);
#pragma unroll
    for (int i = 0; i < 32; ++i) {
        const u32 t = H[i] & 0x7f7f7f7fu;
        m0 = __builtin_elementwise_max(m0, __builtin_bit_cast(u16x2, t));
        m1 = __builtin_elementwise_max(m1, __builtin_bit_cast(u16x2, t << 8));
    }
    m0 = __builtin_elementwise_max(m0, m1);
    const u32 top = wave_max(max((u32)m0[0], (u32)m0[1])) >> 8;
    const u32 C = (0x80u + top) * 0x01010101u;
    u32 gm = 0;
#pragma unroll
    for (int i = 0; i < 4; ++i) {
        u32x4 code;
        u32 chk[2] = {0, 0};
#pragma unroll
        for (int j = 0; j < 4; ++j) {
            const u32 d0 = C - H[8 * i + 2 * j], d1 = C - H[8 * i + 2 * j + 1];
            chk[j >> 1] |= d0 | d1;
            code[j] = bfi(0x87878787u, d0, __builtin_amdgcn_alignbit(d1, d1, 4));
        }
        st16(c, code, c.lane * 16, dst + 8192 + i * 1024);
        gm |= (__any(chk[0] & 0x78787878u) ? 0xfu : 0u) << (8 * i) | (__any(chk[1] & 0x78787878u) ? 0xf0u : 0u) << (8 * i);
    }
    u32 nexc = 0;
    volatile LAS u32* hdr = c.ctl + C_HDR + (c.wave & 3) * 32;
#pragma unroll 1
    while (gm && nexc <= 31) {
        const int i = __builtin_ctz(gm);
        gm &= gm - 1;
        const u32 h = H[i], d = (C - h) & 0x78787878u;
        if (!__any(d)) continue;
        const u32 l = L[i];
#pragma unroll 1
        for (int b = 0; b < 4; ++b) {
            const bool e = (d >> (8 * b)) & 0xff;
            const u64 m = __ballot(e);
            const u32 idx = nexc + __builtin_amdgcn_mbcnt_hi((u32)(m >> 32), __builtin_amdgcn_mbcnt_lo((u32)m, 0));
            const u32 pos = (u32)(i >> 1) << 9 | (u32)c.lane << 3 | (u32)(i & 1) << 2 | (u32)b;
            if (e && idx < 31) hdr[1 + idx] = pos << 16 | ((h >> (8 * b)) & 0xff) << 8 | ((l >> (8 * b)) & 0xff);
            nexc += __popcll(m);
        }
    }
    if (FORCE_RAW_IN || nexc > 31) {
#pragma unroll
        for (int f = 0; f < 16; ++f) st16(c, join8(H[2 * f], L[2 * f], H[2 * f + 1], L[2 * f + 1]), c.lane * 16, dst + f * 1024);
        return 1;
    }
    if (c.lane == 0) hdr[0] = top | nexc << 8;
    if (c.lane < 8) {
        const u32x4 hv = {hdr[4 * c.lane], hdr[4 * c.lane + 1], hdr[4 * c.lane + 2], hdr[4 * c.lane + 3]};
        st16(c, hv, c.lane * 16, dst + 12288);
    }
    return 0;
}

DEV u32 xa_off(int ks, int h, int rowfrag) { return WS_XA + (u32)((ks * 2 + h) * 256 + rowfrag) * 1024u; }
DEV u32 wb_off(int ks, int h, int colfrag) { return WS_WB + (u32)((ks * 2 + h) * 128 + colfrag) * 1024u; }

DEV void pack_job(const Ctx& c, int job) {
    const int s = job / 160, q = job % 160, ks = c.rank * 8 + s;
    const int l15 = c.lane & 15, lq = c.lane >> 4;
    const bool isx = q < 32;
    const int dst = (c.rank + 1 + (q - 32)) & 7, cc = (q - 32) >> 3;
    u32x4 R[16];
    if (isx) {
        const GAS char* src = (const GAS char*)c.x + (size_t)(q * 128 + l15) * (KSH * 2) + (s * 64 + lq * 8) * 2;
#pragma unroll
        for (int f = 0; f < 16; ++f) R[f] = *(const GAS u32x4*)(src + (f & 7) * 16 * (KSH * 2) + (f >> 3) * 64);
    } else {
        const GAS char* src = (const GAS char*)c.w + (size_t)(s * 64 + lq * 8) * (NTOT * 2) + (dst * 2048 + cc * 128 + l15 * 4) * 2;
        u32x2 r[4][8];
#pragma unroll
        for (int hg = 0; hg < 4; ++hg)
#pragma unroll
            for (int j = 0; j < 8; ++j) r[hg][j] = *(const GAS u32x2*)(src + (size_t)((hg >> 1) * 32 + j) * (NTOT * 2) + (hg & 1) * 128);
#pragma unroll
        for (int hg = 0; hg < 4; ++hg)
#pragma unroll
            for (int ii = 0; ii < 4; ++ii)
#pragma unroll
                for (int t = 0; t < 4; ++t)
                    R[hg * 4 + ii][t] = perm(r[hg][2 * t + 1][ii >> 1], r[hg][2 * t][ii >> 1], (ii & 1) ? 0x07060302u : 0x05040100u);
    }
    const bool mine = isx || dst == c.rank, theirs = isx || dst != c.rank;
    if (mine) store_raw(c, R, isx ? xa_off(ks, 0, q * 8) : wb_off(ks, 0, cc * 8), isx ? xa_off(ks, 1, q * 8) : wb_off(ks, 1, cc * 8));
    const u32 call = get_call(c), ch = isx ? s * 32 + q : 256 + s * 16 + cc;
    u32 raw = 0;
    if (theirs) raw = pack_store(c, R, WS_PK + (call & 1) * PK_PAR + (isx ? ch : ch + dst * 128) * CHB);
    WAIT_VM0();
    if (c.lane < NR && (isx || c.lane == dst)) {
        if (c.lane == c.rank) st_sys(c.me + (isx ? WS_XR + q * 256 : WS_WR + cc * 256) + ks * 4, call);
        else st_sys(c.peer_lane + WS_FIN + (((call & 1) * NR + c.rank) * NCHIN + ch) * 4, 2 * call + raw);
    }
}

struct Tile { int cc, big, o0, j; };
DEV int item_rf(const Ctx& c, int it) { return (((it / 3) + c.rank) & 7) * 32 + 20 + (it % 3) * 4; }
DEV int wave_rf(const Ctx& c, const Tile& t, int wr) { return t.big ? ((t.o0 + c.rank) & 7) * 32 + wr * 5 : item_rf(c, t.j * 4 + wr); }

template <int NF>
DEV void loader(const Ctx& c, const Tile& t, int L, u32 call) {
    constexpr int NI = (NF + 2) * 4;
    const int rf0 = wave_rf(c, t, 0), rf1 = wave_rf(c, t, 1), rf2 = wave_rf(c, t, 2), rf3 = wave_rf(c, t, 3);
    const int rcA = rf0 >> 3, rcB = NF == 5 ? rcA + 1 : ((rf1 >> 3) != rcA ? rf1 >> 3 : rf2 >> 3), rcC = NF == 5 ? rcA + 2 : rf3 >> 3;
    const int which = c.lane & 3, myks = L * 16 + (c.lane >> 2);
    const char* fp = c.me + (which == 3 ? WS_WR + t.cc * 256 : WS_XR + (which == 0 ? rcA : which == 1 ? rcB : rcC) * 256) + myks * 4;
    const u32 ldsbase = (u32)(size_t)c.lds, voff = c.lane * 16;
    u64 todo = 0x1111111111111111ull, avail = 0;
    int pend = -1;
    bool ok = true;
    while (todo && ok) {
        if (!avail) {
            if (pend >= 0) { WAIT_VM0(); c.ctl[C_FULL + pend % NSTG] = pend + 1; pend = -1; }
            const u64 t0 = rt();
            for (u32 n = 1;; ++n) {
                u64 m = __ballot(ld_sys(fp) >= call);
                m &= m >> 1; m &= m >> 2;
                avail = m & todo;
                if (avail) break;
                if ((n & 15) == 0) {
                    if (rfl(c.ctl[C_ABORT]) || tmo_set(c, call)) { ok = false; break; }
                    if (rt() - t0 > DEADLINE) { tmo_raise(c, call); ok = false; break; }
                }
                __builtin_amdgcn_s_sleep(4);
            }
            if (!ok) break;
            if (avail == todo && tmo_set(c, call)) { ok = false; break; }
        }
        const int bit = __builtin_ctzll(avail), ks = L * 16 + (bit >> 2);
        avail &= avail - 1; todo &= ~(1ull << bit);
        u32 tk = 0;
        if (c.lane == 0) tk = __hip_atomic_fetch_add((LAS u32*)&c.ctl[C_TICKET], 2u, __ATOMIC_RELAXED, __HIP_MEMORY_SCOPE_WORKGROUP);
        tk = rfl(tk);
#pragma unroll
        for (int h = 0; h < 2; ++h) {
            const u32 tt = tk + h, slot = tt % NSTG, need = 8 * (tt / NSTG);
            if (rfl(c.ctl[C_FREE + slot]) < need) {
                if (pend >= 0) { WAIT_VM0(); c.ctl[C_FULL + pend % NSTG] = pend + 1; pend = -1; }
                if (!lds_wait(c, C_FREE + slot, need, true, 1)) { ok = false; break; }
            }
            const u32 l = ldsbase + slot * STG;
            if (NF == 5) {
#pragma unroll
                for (int g = 0; g < 5; ++g) glds4(c.rs, voff, xa_off(ks, h, rf0 + g * 4), l + g * 4096);
            } else {
                glds4(c.rs, voff, xa_off(ks, h, rf0), l); glds4(c.rs, voff, xa_off(ks, h, rf1), l + 4096);
                glds4(c.rs, voff, xa_off(ks, h, rf2), l + 8192); glds4(c.rs, voff, xa_off(ks, h, rf3), l + 12288);
            }
            glds4(c.rs, voff, wb_off(ks, h, t.cc * 8), l + 20480); glds4(c.rs, voff, wb_off(ks, h, t.cc * 8 + 4), l + 24576);
            if (pend >= 0) {
                if (NI == 28) asm volatile("s_waitcnt vmcnt(28)" ::: "memory"); else asm volatile("s_waitcnt vmcnt(24)" ::: "memory");
                c.ctl[C_FULL + pend % NSTG] = pend + 1;
            }
            pend = tt;
        }
    }
    WAIT_VM0();
    if (!ok) c.ctl[C_ABORT] = 1;
    else if (pend >= 0) c.ctl[C_FULL + pend % NSTG] = pend + 1;
}

DEV u32 pk_relu(float a, float b) {
    const f32x2 v = __builtin_elementwise_maximum((f32x2){a, b}, (f32x2){0.f, 0.f});
    return __builtin_bit_cast(u32, __builtin_convertvector(v, bf16x2));
}

template <int NF>
DEV void consumer(const Ctx& c, const Tile& t, int wr, int wc) {
    f32x4 acc[NF][4];
#pragma unroll
    for (int f = 0; f < NF; ++f)
#pragma unroll
        for (int i = 0; i < 4; ++i) acc[f][i] = (f32x4){0.f, 0.f, 0.f, 0.f};
    const LAS char* pa = c.lds + c.lane * 16 + wr * NF * 1024;
    const LAS char* pb = c.lds + c.lane * 16 + 20480 + wc * 4096;
    u32 slot = 0, seen = 0;
    bool ok = true;
#pragma unroll 1
    for (u32 tk = 0; tk < 128; ++tk) {
        if (ok && rfl(seen) < tk + 1) ok = lds_wait(c, C_FULL + slot, tk + 1, true, 0);
        CBAR();
        if (tk == 0) STAMPW(c, 1);
        if (tk == 64) STAMPW(c, 4);
        if (tk == 96) STAMPW(c, 5);
        if (tk == 112) STAMPW(c, 6);
        if (tk == 120) STAMPW(c, 7);
        u32x4 a[NF], b[4];
#pragma unroll
        for (int i = 0; i < 4; ++i) b[i] = *(const LAS u32x4*)(pb + slot * STG + i * 1024);
#pragma unroll
        for (int f = 0; f < NF; ++f) a[f] = *(const LAS u32x4*)(pa + slot * STG + f * 1024);
        WAIT_LGKM0();
        if (c.lane == 0) __hip_atomic_fetch_add((LAS u32*)&c.ctl[C_FREE + slot], 1u, __ATOMIC_RELAXED, __HIP_MEMORY_SCOPE_WORKGROUP);
        slot = slot == NSTG - 1 ? 0 : slot + 1;
        seen = c.ctl[C_FULL + slot];
        __builtin_amdgcn_sched_barrier(0);
#pragma unroll
        for (int f = 0; f < NF; ++f)
#pragma unroll
            for (int i = 0; i < 4; ++i)
                acc[f][i] = __builtin_amdgcn_mfma_f32_16x16x32_bf16(__builtin_bit_cast(bf16x8, a[f]), __builtin_bit_cast(bf16x8, b[i]), acc[f][i], 0, 0, 0);
    }
    STAMPW(c, 2);
    ok = ok && !rfl(c.ctl[C_ABORT]);
    const u32 call = get_call(c);
    const int rf = wave_rf(c, t, wr), owner = rf >> 5;
    u32x2 P[NF][4];
#pragma unroll
    for (int f = 0; f < NF; ++f)
#pragma unroll
        for (int r = 0; r < 4; ++r) P[f][r] = (u32x2){pk_relu(acc[f][0][r], acc[f][1][r]), pk_relu(acc[f][2][r], acc[f][3][r])};
    if (owner == c.rank) {
        GAS char* o = (GAS char*)c.out + (size_t)((rf & 31) * 16 + (c.lane >> 4) * 4) * (NTOT * 2) + (c.rank * 2048 + t.cc * 128 + wc * 64 + (c.lane & 15) * 4) * 2;
#pragma unroll
        for (int f = 0; f < NF; ++f)
#pragma unroll
            for (int r = 0; r < 4; ++r) *(GAS u32x2*)(o + (size_t)(f * 16 + r) * (NTOT * 2)) = ok ? P[f][r] : (u32x2){0x7fc07fc0u, 0x7fc07fc0u};
    } else {
        const int rg = NF == 5 ? wr : 4 + (((rf & 31) - 20) >> 2), msg = (t.cc * 7 + rg) * 2 + wc;
        const u32 dst = WS_YP + (owner * NMSG + msg) * YSLOT;
#pragma unroll
        for (int f = 0; f < NF; ++f)
#pragma unroll
            for (int rp = 0; rp < 2; ++rp)
                st16(c, (u32x4){P[f][2 * rp][0], P[f][2 * rp][1], P[f][2 * rp + 1][0], P[f][2 * rp + 1][1]}, c.lane * 16, dst + (f * 2 + rp) * 1024);
        WAIT_VM0();
        const u32 meta = ok ? (YM_RAW | NF * 128) : YM_BAD;
        if (c.lane == owner) st_sys64(c.peer_lane + WS_FY + (c.rank * NMSG + msg) * 8, (u64)call << 32 | meta);
    }
    STAMPW(c, 3);
}

DEV void gemm_wg(const Ctx& c) {
    const int g = c.wg - NCOMM, xcd = g & 7, idx = g >> 3, half = xcd >> 2, k = idx % 7;
    Tile t;
    t.cc = (xcd & 3) * 4 + idx / 7; t.big = k < 4; t.o0 = half * 4 + k; t.j = half * 3 + (k - 4);
    if (c.wave >= 8) {
        const int L = c.wave - 8;
        for (int i = 0; i < L; ++i) __builtin_amdgcn_s_sleep(24);
#pragma unroll 1
        for (int job = L * NGEMM + g; job < 1280; job += 4 * NGEMM) { pack_job(c, job); if (job < 4 * NGEMM) STAMPW(c, 3); }
        STAMPW(c, 1);
        const u32 call = get_call(c);
        if (t.big) loader<5>(c, t, L, call); else loader<4>(c, t, L, call);
        STAMPW(c, 2);
    } else {
        if (t.big) consumer<5>(c, t, c.wave >> 1, c.wave & 1); else consumer<4>(c, t, c.wave >> 1, c.wave & 1);
    }
}

constexpr u32 WIN = 26;

DEV void q_publish(const Ctx& c, u32 n) { WAIT_LGKM0(); c.ctl[C_QN] = n; }

DEV void watcher(const Ctx& c, int d, int sub) {
    const u32 call = get_call(c);
    const int p = (c.rank + d) & 7;
    bool bad = false;
    for (int base = 0; base < NQIN; base += 64) {
        const int n = base + c.lane, cnt = min(64, NQIN - base);
        const u32 code = ORDER.v[d - 1][min(n, NQIN - 1) * 4 + sub], s = (code >> 8) & 7, ix = code & 31;
        const u32 id = (code & 0x8000) ? 256 + s * 16 + ix : s * 32 + ((((ix >> 2) + c.rank) & 7) << 2 | (ix & 3));
        const char* fp = c.me + WS_FIN + (((call & 1) * NR + p) * NCHIN + id) * 4;
        int done = 0;
        u64 t0 = rt();
        for (u32 it = 1; done < cnt; ++it) {
            const u32 v = bad ? 0 : ld_sys(fp);
            const u64 m = __ballot(bad || v >= 2 * call || c.lane < done || c.lane >= cnt);
            const int upto = min(cnt, m == ~0ull ? 64 : (int)__builtin_ctzll(~m));
            if (upto > done) {
                const u32 raw = v & 1;
                if (c.lane >= done && c.lane < upto)
                    c.ctl[C_Q + n] = 0x80000000u | id | (bad ? 1u << 14 : (raw ? T_INR << 12 | 1024u << 16 : T_INP << 12 | (PKD_BYTES / 16) << 16));
                q_publish(c, base + upto);
                done = upto; t0 = rt();
            } else {
                if ((it & 15) == 0) {
                    if (tmo_set(c, call)) bad = true;
                    else if (rt() - t0 > DEADLINE) { tmo_raise(c, call); bad = true; }
                }
                __builtin_amdgcn_s_sleep(2);
            }
        }
    }
    STAMPW(c, 1);
    {
        const u32 id = sub + 4 * min(c.lane, NQY - 1);
        const char* fp = c.me + WS_FY + (p * NMSG + id) * 8;
        u64 pending = (1ull << NQY) - 1, t0 = rt();
        u32 qn = NQIN;
        for (u32 it = 1; pending; ++it) {
            const u64 v = bad ? 0 : ld_sys64(fp);
            const bool mine = ((pending >> c.lane) & 1) && (bad || (u32)(v >> 32) >= call);
            const u64 m = __ballot(mine);
            if (m) {
                const bool poison = bad || tmo_set(c, call);
                const u32 meta = (u32)v, pos = qn + __builtin_amdgcn_mbcnt_hi((u32)(m >> 32), __builtin_amdgcn_mbcnt_lo((u32)m, 0));
                if (mine) c.ctl[C_Q + pos] = 0x80000000u | id | ((poison || (meta & YM_BAD)) ? (T_YR << 12 | 1u << 14)
                                                                 : (((meta & YM_RAW) ? T_YR : T_YC) << 12 | min(meta & 0x7ff, YSLOT / 16) << 16));
                qn += __popcll(m); pending &= ~m; t0 = rt();
                q_publish(c, qn);
            } else {
                if ((it & 15) == 0) {
                    if (tmo_set(c, call)) bad = true;
                    else if (rt() - t0 > DEADLINE) { tmo_raise(c, call); bad = true; }
                }
                __builtin_amdgcn_s_sleep(2);
            }
        }
    }
    STAMPW(c, 2);
}

DEV void fetcher(const Ctx& c, char* peer) {
    const u32 call = get_call(c);
    const rsrc_t rp = __builtin_amdgcn_make_buffer_rsrc(peer, 0, WS_END, 0x00020000);
    const u32 ldsbase = (u32)(size_t)c.lds, voff = c.lane * 16;
    u32 nxt = 0, pub = 0, out = 0, qn = 0, kb = 0, ksm = 0, alt = 0;
    u64 fn = 0, fs = 0;
    u64 t0 = rt();
    for (u32 it = 1; pub < NQ; ++it) {
        bool issued = false;
        if (nxt < NQ) {
            if (nxt >= qn) qn = rfl(c.ctl[C_QN]);
            if (nxt < qn && nxt - pub < 12) {
                const u32 e = rfl(c.ctl[C_Q + nxt]), bytes = ((e >> 16) & 0x7ff) * 16, ni = (bytes + 1023) >> 10;
                const bool small = bytes <= LSML && (alt % 7) < 4;
                const u32 slot = small ? NBIG + ksm : kb;
                if ((out + ni <= WIN || out == 0) && rfl(c.ctl[C_BUSY + slot]) == 0) {
                    c.ctl[C_BUSY + slot] = 1;
                    const u32 id = e & 0xfff, type = (e >> 12) & 3;
                    const u32 src = type >= T_YC ? WS_YP + (c.rank * NMSG + id) * YSLOT
                                                 : WS_PK + (call & 1) * PK_PAR + (id < 256 ? id : id + c.rank * 128) * CHB;
                    const u32 l = ldsbase + (small ? L_SML0 + ksm * LSML : kb * LBIG);
                    u32 o = 0;
                    for (; o + 4096 <= bytes; o += 4096) glds4(rp, voff, src + o, l + o);
                    for (; o + 1024 <= bytes; o += 1024) glds1(rp, voff, src + o, l + o);
                    if (o < bytes) { if (voff < bytes - o) glds1(rp, voff, src + o, l + o); }
                    fn |= (u64)ni << (5 * (nxt - pub)); fs |= (u64)slot << (4 * (nxt - pub));
                    out += ni; ++nxt; ++alt; issued = true;
                    if (small) ksm = ksm == NSML - 1 ? 0 : ksm + 1; else kb = kb == NBIG - 1 ? 0 : kb + 1;
                }
            }
        }
        if (issued) continue;
        if (pub < nxt) {
            out -= (u32)fn & 31;
            wait_vm(out);
            c.ctl[C_FQ + pub] = ((u32)fs & 15) + 1;
            fn >>= 5; fs >>= 4; ++pub; t0 = rt();
            if (pub == 1) STAMPW(c, 1);
            if (pub == NQIN / 2) STAMPW(c, 2);
            if (pub == NQIN) STAMPW(c, 3);
            if (pub == NQIN + 1) STAMPW(c, 4);
        } else {
            __builtin_amdgcn_s_sleep(1);
            if ((it & 255) == 0 && rt() - t0 > DEADLINE_LDS) { tmo_raise(c, call); break; }
        }
    }
    WAIT_VM0();
    STAMPW(c, 5);
}

DEV void unpack_in(const Ctx& c, u32 e, u32 slot, int p, u32 call) {
    const LAS char* l = c.lds + slot * LBIG + c.lane * 16;
    const u32 id = e & 0xfff;
    const int s = id < 256 ? id >> 5 : (id - 256) >> 4, ks = p * 8 + s;
    const u32 d0 = id < 256 ? xa_off(ks, 0, (id & 31) * 8) : wb_off(ks, 0, (id & 15) * 8);
    const u32 d1 = id < 256 ? xa_off(ks, 1, (id & 31) * 8) : wb_off(ks, 1, (id & 15) * 8);
    u32x4 R[16];
    if (((e >> 12) & 3) == T_INR) {
#pragma unroll
        for (int f = 0; f < 16; ++f) R[f] = *(const LAS u32x4*)(l + f * 1024);
        WAIT_LGKM0();
        c.ctl[C_BUSY + slot] = 0;
        store_raw(c, R, d0, d1);
    } else {
        u32x4 L[8], code[4];
#pragma unroll
        for (int i = 0; i < 4; ++i) code[i] = *(const LAS u32x4*)(l + 8192 + i * 1024);
#pragma unroll
        for (int i = 0; i < 8; ++i) L[i] = *(const LAS u32x4*)(l + i * 1024);
        const u32 hv = *(const LAS u32*)(c.lds + slot * LBIG + 12288 + (c.lane & 31) * 4);
        WAIT_LGKM0();
        c.ctl[C_BUSY + slot] = 0;
        const u32 h0 = rfl(hv), C = (0x80u + (h0 & 0x7f)) * 0x01010101u, nexc = min((h0 >> 8) & 0xff, 31u);
#pragma unroll
        for (int f = 0; f < 16; ++f) {
            const u32 n = code[f >> 2][f & 3];
            const u32 ha = C - (n & 0x87878787u), hb = C - (__builtin_amdgcn_alignbit(n, n, 28) & 0x87878787u);
            const u32 la = L[f >> 1][(f & 1) * 2], lb = L[f >> 1][(f & 1) * 2 + 1];
            R[f] = join8(ha, la, hb, lb);
        }
        store_raw(c, R, d0, d1);
#pragma unroll 1
        for (u32 i = 0; i < nexc; ++i) {
            const u32 x = __builtin_amdgcn_readlane(hv, i + 1), pos = x >> 16, f = (pos >> 9) & 15, ln = (pos >> 3) & 63;
            if ((u32)c.lane == ln) __builtin_amdgcn_raw_buffer_store_b16((u16)x, c.rs, ln * 16 + (f & 7) * 1024 + (pos & 7) * 2, f < 8 ? d0 : d1, 17);
        }
    }
    WAIT_VM0();
    if (c.lane == 0) st_sys(c.me + (id < 256 ? WS_XR + (id & 31) * 256 : WS_WR + (id & 15) * 256) + ks * 4, call);
}

DEV void decode_y(const Ctx& c, u32 e, u32 slot, int p) {
    const u32 id = e & 0xfff, wc = id & 1, rg = (id >> 1) % 7, cc = (id >> 1) / 7;
    const int nf = rg < 4 ? 5 : 4, row0 = rg < 4 ? rg * 80 : 320 + (rg - 4) * 64;
    GAS char* o = (GAS char*)c.out + (size_t)(row0 + (c.lane >> 4) * 4) * (NTOT * 2) + (p * 2048 + cc * 128 + wc * 64 + (c.lane & 15) * 4) * 2;
    const LAS char* l = c.lds + (slot < NBIG ? slot * LBIG : L_SML0 + (slot - NBIG) * LSML) + c.lane * 16;
    const bool bad = (e >> 14) & 1;
    u32x4 v[10];
#pragma unroll
    for (int i = 0; i < 10; ++i) v[i] = (bad || i >= 2 * nf) ? (u32x4){0x7fc07fc0u, 0x7fc07fc0u, 0x7fc07fc0u, 0x7fc07fc0u} : *(const LAS u32x4*)(l + i * 1024);
    WAIT_LGKM0();
    c.ctl[C_BUSY + slot] = 0;
#pragma unroll
    for (int i = 0; i < 10; ++i)
        if (i < 2 * nf) {
            *(GAS u32x2*)(o + (size_t)((i >> 1) * 16 + (i & 1) * 2) * (NTOT * 2)) = (u32x2){v[i][0], v[i][1]};
            *(GAS u32x2*)(o + (size_t)((i >> 1) * 16 + (i & 1) * 2 + 1) * (NTOT * 2)) = (u32x2){v[i][2], v[i][3]};
        }
}

DEV void worker(const Ctx& c, int p, int w) {
    const u32 call = get_call(c);
#pragma unroll 1
    for (int i = w; i < NQ; i += NWORK) {
        if (!lds_wait(c, C_FQ + i, 1, false, 1)) { tmo_raise(c, call); break; }
        CBAR();
        const u32 slot = rfl(c.ctl[C_FQ + i]) - 1, e = rfl(c.ctl[C_Q + i]);
        if (i < NQIN) {
            if ((e >> 14) & 1) c.ctl[C_BUSY + slot] = 0; else unpack_in(c, e, slot, p, call);
            if (i + NWORK >= NQIN) STAMPW(c, 1);
        } else decode_y(c, e, slot, p);
    }
    STAMPW(c, 2);
}

__global__ void __launch_bounds__(NTHREADS, 3) fused_kernel(Args a) {
    extern __shared__ __attribute__((aligned(128))) char smem[];
    Ctx c;
    c.x = a.x; c.w = a.w; c.out = a.out; c.rank = a.rank & 7; c.me = a.ws[c.rank];
    c.wg = blockIdx.x; c.wave = rfl(threadIdx.x >> 6); c.lane = threadIdx.x & 63;
    c.peer_lane = a.ws[c.lane & 7];
    c.rs = __builtin_amdgcn_make_buffer_rsrc(c.me, 0, WS_END, 0x00020000);
    c.lds = (LAS char*)smem; c.ctl = (volatile LAS u32*)(c.lds + L_CTL);
    STAMPW(c, 0);
    u32 ticket = 0;
    if (threadIdx.x == 0) ticket = __hip_atomic_fetch_add((GAS u32*)(c.me + WS_CNT), 1u, __ATOMIC_RELAXED, __HIP_MEMORY_SCOPE_AGENT);
    if (threadIdx.x < C_WORDS) c.ctl[threadIdx.x] = 0;
    WAIT_LGKM0();
    __builtin_amdgcn_s_barrier();
    CBAR();
    if (threadIdx.x == 0) c.ctl[C_CALL] = ticket / NWG + 1;
    if (c.wg >= NCOMM) gemm_wg(c);
    else if (c.wg < NPULL) {
        const int d = 1 + c.wg % 7, sub = c.wg / 7, p = (c.rank + d) & 7;
        if (c.wave == 0) fetcher(c, a.ws[p]);
        else if (c.wave == 1) watcher(c, d, sub);
        else worker(c, p, c.wave - 2);
    }
}
static_assert(C_WORDS <= NTHREADS, "control words are zeroed by one store per thread");

}

extern "C" void kernel_launch(int rank, int nranks, ncclComm_t comm, hipStream_t stream,
                              void* const* d_in, const int* in_sizes, int n_in,
                              void* d_out, int out_size, const int* in_dtypes, int out_dtype,
                              void* const* d_out_all, void* const* d_ws_all, size_t ws_bytes) {
    if (nranks != NR || (unsigned)rank >= (unsigned)NR || n_in != 2 || ws_bytes < WS_END || in_sizes[0] != 4096 * KSH ||
        in_sizes[1] != KSH * NTOT || out_size != 512 * NTOT) {
        fprintf(stderr, "unexpected problem shape\n");
        return;
    }
    HIP_CHECK(hipFuncSetAttribute((const void*)fused_kernel, hipFuncAttributeMaxDynamicSharedMemorySize, LDS_BYTES));
    Args a;
    a.x = (const u16*)d_in[0]; a.w = (const u16*)d_in[1]; a.out = (u16*)d_out;
    for (int p = 0; p < NR; ++p) a.ws[p] = (char*)d_ws_all[p];
    a.rank = rank; a.pad = 0;
    hipLaunchKernelGGL(fused_kernel, dim3(NWG), dim3(NTHREADS), LDS_BYTES, stream, a);
    HIP_CHECK(hipGetLastError());
}
```

```cpp
#include <hip/hip_runtime.h>
#include <rccl/rccl.h>
#include <cstdio>
#include <cstdint>

#define HIP_CHECK(expr) do { hipError_t _e = (expr); if (_e != hipSuccess) { \
    fprintf(stderr, "HIP error %s at %s:%d\n", hipGetErrorString(_e), __FILE__, __LINE__); \
    return; } } while (0)

extern "C" __device__ unsigned writelane(unsigned value, unsigned lane, unsigned old) __asm("llvm.amdgcn.writelane.i32");

namespace {

typedef unsigned u32;
typedef unsigned long long u64;
typedef unsigned short u16;
typedef u32 u32x4 __attribute__((ext_vector_type(4)));
typedef u32 u32x2 __attribute__((ext_vector_type(2)));
typedef u32 u32x8 __attribute__((ext_vector_type(8)));
typedef u32 u32x32 __attribute__((ext_vector_type(32)));
typedef u16 u16x2 __attribute__((ext_vector_type(2)));
typedef float f32x4 __attribute__((ext_vector_type(4)));
typedef float f32x2 __attribute__((ext_vector_type(2)));
typedef __bf16 bf16x2 __attribute__((ext_vector_type(2)));
typedef __bf16 bf16x8 __attribute__((ext_vector_type(8)));
typedef __amdgpu_buffer_rsrc_t rsrc_t;
#define LAS __attribute__((address_space(3)))
#define GAS __attribute__((address_space(1)))
#define DEV __device__ __forceinline__

#define PSLEEP 3
#define PREF 1
#define FORCE_RAW_Y 0
#define FORCE_RAW_IN 0

constexpr int NR = 8;
constexpr int NTOT = 16384, KSH = 512;
constexpr int NWG = 256, NCOMM = 32, NPULL = 28, NGEMM = NWG - NCOMM, NTHREADS = 768;
constexpr int NCHIN = 384;
constexpr int NMSG = 224;
constexpr int NQY = NMSG / 4, YQ0 = 448, NQ = YQ0 + NQY;
constexpr int NWORK = 9;

constexpr u32 WS_CNT = 0, WS_TMO = 128, WS_CLM = 1024;
constexpr u32 WS_FIN = 4096;
constexpr u32 WS_FY = 32768;
constexpr u32 WS_XR = 49152;
constexpr u32 WS_WR = 57344;
constexpr u32 WS_DBG = 65536;
constexpr u32 CHB = 16384;
constexpr u32 WS_PK = 1u << 20;
constexpr u32 PK_PAR = 1280u * CHB;
constexpr u32 WS_XA = WS_PK + 2 * PK_PAR;
constexpr u32 WS_WB = WS_XA + (32u << 20);
constexpr u32 YSLOT = 10752;
constexpr u32 WS_YP = WS_WB + (16u << 20);
constexpr u32 WS_END = WS_YP + 8u * NMSG * YSLOT;
constexpr u32 PKD_BYTES = 12288 + 128;

constexpr u32 STG = 28672, NSTG = 5;
constexpr u32 LBIG = 16384, NBIG = 6, LSML = 5120, NSML = 8;
constexpr u32 L_SML0 = LBIG * NBIG;
constexpr u32 L_CTL = 143360;
static_assert(STG * NSTG <= L_CTL && L_SML0 + LSML * NSML <= L_CTL, "ring");
constexpr u32 C_CALL = 0, C_ABORT = 1, C_TICKET = 2, C_ISS = 3, C_FULL = 8, C_FREE = 16, C_BUSY = 32,
              C_DUMP = 64, C_Q = 64, C_FQ = C_Q + 576, C_FE = C_FQ + 576, C_WORDS = C_FE + 576,
              C_LUT = C_WORDS;
constexpr u32 LDS_BYTES = L_CTL + (C_LUT + 128) * 4;
constexpr u32 L_DUMP = L_CTL + C_DUMP * 4, L_LUT = L_CTL + C_LUT * 4, L_YST = 3 * STG, YST = 5632;
static_assert(L_LUT % 512 == 0 && L_YST + 8 * YST <= 5 * STG && 4 * (32 + 64 * 21) <= YST, "y staging");
static_assert(LDS_BYTES <= 160 * 1024 && NQ + 1 + 64 <= 576 && NCHIN + NWORK < YQ0, "lds");
constexpr u32 FQ_END = 0xff;

constexpr u64 DEADLINE = 40000000ull;
constexpr u64 DEADLINE_LDS = 100000000ull;

constexpr u32 T_INP = 0, T_INR = 1, T_YC = 2, T_YR = 3;
constexpr u32 YM_RAW = 1u << 16, YM_BAD = 1u << 17;

struct Order { u16 v[7][NCHIN]; };
#define R1V 3
#define R1Q 4
#define R2V 3
constexpr Order make_order() {
    Order o{};
    for (int d = 1; d <= 7; ++d) {
        int n = 0;
        for (int pass = 0; pass < 3; ++pass)
            for (int s = 0; s < 8; ++s) {
                if (pass == 0) for (int c = 0; c < 16; ++c) o.v[d - 1][n++] = (u16)(0x8000 | s << 8 | c);
                for (int i = 0; i < 32; ++i) {
                    const int own = ((i >> 2) + d) & 7, q = i & 3;
                    const int cls = (own == 0 && q < R1Q && s >= 8 - R1V) ? 2 : (q == 3 && s >= 8 - R2V) ? 1 : 0;
                    if (cls == pass) o.v[d - 1][n++] = (u16)(s << 8 | own << 2 | q);
                }
            }
    }
    return o;
}
__constant__ const Order ORDER = make_order();

struct Args { const u16* x; const u16* w; u16* out; char* ws[NR]; int rank; int pad; };

DEV u64 rt() { return __builtin_amdgcn_s_memrealtime(); }
DEV u32 rfl(u32 v) { return __builtin_amdgcn_readfirstlane(v); }
DEV u32 perm(u32 hi, u32 lo, u32 sel) { return __builtin_amdgcn_perm(hi, lo, sel); }
DEV u32 bfi(u32 m, u32 a, u32 b) { return (m & a) | (~m & b); }
#define CBAR() asm volatile("" ::: "memory")
#define WAIT_VM0() asm volatile("s_waitcnt vmcnt(0)" ::: "memory")
#define WAIT_LGKM0() do { CBAR(); __builtin_amdgcn_s_waitcnt(0xc07f); CBAR(); } while (0)

#define STAMPW(c, i) do {} while (0)

struct Ctx {
    const u16* x; const u16* w; u16* out; char* me; char* peer_lane;
    rsrc_t rs;
    LAS char* lds; volatile LAS u32* ctl;
    int rank, wg, wave, lane;
};

DEV u32 ld_sys(const void* p) { return __hip_atomic_load((const GAS u32*)p, __ATOMIC_RELAXED, __HIP_MEMORY_SCOPE_SYSTEM); }
DEV u64 ld_sys64(const void* p) { return __hip_atomic_load((const GAS u64*)p, __ATOMIC_RELAXED, __HIP_MEMORY_SCOPE_SYSTEM); }
DEV void st_sys(void* p, u32 v) { __hip_atomic_store((GAS u32*)p, v, __ATOMIC_RELAXED, __HIP_MEMORY_SCOPE_SYSTEM); }
DEV void st_sys64(void* p, u64 v) { __hip_atomic_store((GAS u64*)p, v, __ATOMIC_RELAXED, __HIP_MEMORY_SCOPE_SYSTEM); }
DEV void st16(const Ctx& c, u32x4 v, u32 voff, u32 soff) { __builtin_amdgcn_raw_buffer_store_b128(v, c.rs, voff + soff, 0, 17); }
DEV void st_out(GAS char* p, u32x2 v) { __hip_atomic_store((GAS u64*)p, __builtin_bit_cast(u64, v), __ATOMIC_RELAXED, __HIP_MEMORY_SCOPE_SYSTEM); }
DEV bool tmo_set(const Ctx& c, u32 call) { return rfl(__hip_atomic_load((const GAS u32*)(c.me + WS_TMO), __ATOMIC_RELAXED, __HIP_MEMORY_SCOPE_AGENT)) == call; }
DEV void tmo_raise(const Ctx& c, u32 call) { if (c.lane == 0) __hip_atomic_store((GAS u32*)(c.me + WS_TMO), call, __ATOMIC_RELAXED, __HIP_MEMORY_SCOPE_AGENT); }

DEV bool expired(u64& t0, u64 limit) { const u64 t = rt(); if (!t0) t0 = t; return t - t0 > limit; }

DEV bool lds_wait(const Ctx& c, u32 word, u32 want, bool look_abort, int nap) {
    u64 t0 = 0;
    for (u32 n = 1;; ++n) {
        if (rfl(c.ctl[word]) >= want) return true;
        if (nap) __builtin_amdgcn_s_sleep(1);
        if ((n & 63) == 0) {
            if (look_abort && rfl(c.ctl[C_ABORT])) return false;
            if (expired(t0, DEADLINE_LDS)) { c.ctl[C_ABORT] = 1; return false; }
        }
    }
}
DEV u32 get_call(const Ctx& c) { lds_wait(c, C_CALL, 1, false, 1); return rfl(c.ctl[C_CALL]); }

DEV void glds4(rsrc_t rs, u32 voff, u32 soff, u32 ldsaddr) {
    u32 keep;
    asm volatile("s_mov_b32 %0, m0\n\ts_mov_b32 m0, %4\n\ts_nop 2\n\t"
                 "buffer_load_dwordx4 %1, %2, %3 offen sc0 sc1 lds\n\t"
                 "buffer_load_dwordx4 %1, %2, %3 offen offset:1024 sc0 sc1 lds\n\t"
                 "buffer_load_dwordx4 %1, %2, %3 offen offset:2048 sc0 sc1 lds\n\t"
                 "buffer_load_dwordx4 %1, %2, %3 offen offset:3072 sc0 sc1 lds\n\t"
                 "s_mov_b32 m0, %0" : "=&s"(keep) : "v"(voff), "s"(rs), "s"(soff), "s"(ldsaddr) : "memory");
}
DEV void touch(rsrc_t rs, u32 voff, u32 soff, u32 ldsaddr) {
    u32 keep;
    asm volatile("s_mov_b32 %0, m0\n\ts_mov_b32 m0, %4\n\ts_nop 2\n\tbuffer_load_dword %1, %2, %3 offen sc0 sc1 lds\n\t"
                 "s_mov_b32 m0, %0" : "=&s"(keep) : "v"(voff), "s"(rs), "s"(soff), "s"(ldsaddr) : "memory");
}
#define GL(o) "buffer_load_dwordx4 %[v], %[rs], %[so] offen offset:" #o " sc0 sc1 lds\n\t"
#define GL4 GL(3072) GL(2048) GL(1024) GL(0)
#define GLN "s_sub_u32 m0, m0, %[k]\n\ts_sub_u32 %[so], %[so], %[k]\n\t"
DEV void glds_n(u32 lo, u32 hi, u32 voff, u32 src, u32 bytes, u32 ni, u32 ldsaddr) {
    const u32 g = (ni - 1) >> 2 & 3, joff = 16 + 8 * (((16 - ni) & 15) + 3 - g);
    const u32x4 rs = {lo, hi, min(src + bytes, WS_END), 0x00020000u};
    u32 so = src + g * 4096, keep;
    asm volatile("s_mov_b32 %[keep], m0\n\ts_getpc_b64 vcc\n\ts_add_u32 vcc_lo, vcc_lo, %[j]\n\ts_addc_u32 vcc_hi, vcc_hi, 0\n\t"
                 "s_mov_b32 m0, %[l]\n\ts_setpc_b64 vcc\n\t" GL4 GLN GL4 GLN GL4 GLN GL4 "s_mov_b32 m0, %[keep]"
                 : [keep] "=&s"(keep), [so] "+s"(so)
                 : [v] "v"(voff), [rs] "s"(rs), [j] "s"(joff), [l] "s"(ldsaddr + g * 4096), [k] "s"(4096u) : "memory", "vcc", "scc");
}
#define VW(n) "s_waitcnt vmcnt(" #n ")\n\ts_branch .Lvm%=\n\t"
#define VW8(a, b, c, d, e, f, g, h) VW(a) VW(b) VW(c) VW(d) VW(e) VW(f) VW(g) VW(h)
DEV void wait_vm(u32 n) {
    asm volatile("s_getpc_b64 vcc\n\ts_add_u32 vcc_lo, vcc_lo, %0\n\ts_addc_u32 vcc_hi, vcc_hi, 0\n\ts_setpc_b64 vcc\n\t"
                 VW8(0, 1, 2, 3, 4, 5, 6, 7) VW8(8, 9, 10, 11, 12, 13, 14, 15) VW8(16, 17, 18, 19, 20, 21, 22, 23)
                 VW8(24, 25, 26, 27, 28, 29, 30, 31) VW8(32, 33, 34, 35, 36, 37, 38, 39) VW8(40, 41, 42, 43, 44, 45, 46, 47)
                 ".Lvm%=:" :: "s"(12 + 8 * min(n, 47u)) : "memory", "vcc", "scc");
}

DEV u32 wave_max(u32 v) {
    v = max(v, (u32)__builtin_amdgcn_update_dpp(0, (int)v, 0x111, 0xf, 0xf, true));
    v = max(v, (u32)__builtin_amdgcn_update_dpp(0, (int)v, 0x112, 0xf, 0xf, true));
    v = max(v, (u32)__builtin_amdgcn_update_dpp(0, (int)v, 0x114, 0xf, 0xf, true));
    v = max(v, (u32)__builtin_amdgcn_update_dpp(0, (int)v, 0x118, 0xf, 0xf, true));
    v = max(v, (u32)__builtin_amdgcn_update_dpp(0, (int)v, 0x142, 0xa, 0xf, false));
    v = max(v, (u32)__builtin_amdgcn_update_dpp(0, (int)v, 0x143, 0xc, 0xf, false));
    return __builtin_amdgcn_readlane(v, 63);
}

DEV u32 wave_scan(u32 v) {
    v += (u32)__builtin_amdgcn_update_dpp(0, (int)v, 0x111, 0xf, 0xf, true);
    v += (u32)__builtin_amdgcn_update_dpp(0, (int)v, 0x112, 0xf, 0xf, true);
    v += (u32)__builtin_amdgcn_update_dpp(0, (int)v, 0x114, 0xf, 0xf, true);
    v += (u32)__builtin_amdgcn_update_dpp(0, (int)v, 0x118, 0xf, 0xf, true);
    v += (u32)__builtin_amdgcn_update_dpp(0, (int)v, 0x142, 0xa, 0xf, false);
    v += (u32)__builtin_amdgcn_update_dpp(0, (int)v, 0x143, 0xc, 0xf, false);
    return v;
}
DEV u32 lane_rank(u64 m) { return __builtin_amdgcn_mbcnt_hi((u32)(m >> 32), __builtin_amdgcn_mbcnt_lo((u32)m, 0)); }

DEV void store_raw(const Ctx& c, const u32x4 (&R)[16], u32 off0, u32 off1) {
#pragma unroll
    for (int f = 0; f < 16; ++f) st16(c, R[f], c.lane * 16, (f < 8 ? off0 : off1) + (f & 7) * 1024);
}
DEV u32x4 join8(u32 ha, u32 la, u32 hb, u32 lb) {
    return (u32x4){perm(ha, la, 0x05010400u), perm(ha, la, 0x07030602u), perm(hb, lb, 0x05010400u), perm(hb, lb, 0x07030602u)};
}
DEV u32 pack_store(const Ctx& c, const u32x4 (&R)[16], u32 dst) {
    u32 H[32], L[32];
    u16x2 m0 = {0, 0}, m1 = {0, 0};
#pragma unroll
    for (int f = 0; f < 16; ++f) {
        L[2 * f] = perm(R[f][1], R[f][0], 0x06040200u); L[2 * f + 1] = perm(R[f][3], R[f][2], 0x06040200u);
        H[2 * f] = perm(R[f][1], R[f][0], 0x07050301u); H[2 * f + 1] = perm(R[f][3], R[f][2], 0x07050301u);
    }
#pragma unroll
    for (int i = 0; i < 8; ++i) st16(c, (u32x4){L[4 * i], L[4 * i + 1], L[4 * i + 2], L[4 * i + 3]}, c.lane * 16, dst + i * 1024);
#pragma unroll
    for (int i = 0; i < 32; ++i) {
        const u32 t = H[i] & 0x7f7f7f7fu;
        m0 = __builtin_elementwise_max(m0, __builtin_bit_cast(u16x2, t));
        m1 = __builtin_elementwise_max(m1, __builtin_bit_cast(u16x2, t << 8));
    }
    m0 = __builtin_elementwise_max(m0, m1);
    const u32 top = wave_max(max((u32)m0[0], (u32)m0[1])) >> 8;
    const u32 C = (0x80u + top) * 0x01010101u;
    u32 nexc = 0, hv = 0;
#pragma unroll
    for (int i = 0; i < 4; ++i) {
        u32x4 code;
        u32 chk = 0;
#pragma unroll
        for (int j = 0; j < 4; ++j) {
            const u32 d0 = C - H[8 * i + 2 * j], d1 = C - H[8 * i + 2 * j + 1];
            chk |= d0 | d1;
            code[j] = bfi(0x87878787u, d0, __builtin_amdgcn_alignbit(d1, d1, 4));
        }
        st16(c, code, c.lane * 16, dst + 8192 + i * 1024);
        for (u64 m = __ballot(chk & 0x78787878u); m && nexc <= 31; m &= m - 1) {
            const u32 ln = __builtin_ctzll(m);
            u32 th = top * 0x01010101u, tl = 0;
#pragma unroll
            for (int k = 0; k < 8; ++k) {
                th = writelane(__builtin_amdgcn_readlane(H[8 * i + k], ln), k, th);
                tl = writelane(__builtin_amdgcn_readlane(L[8 * i + k], ln), k, tl);
            }
            for (u32 mk = (u32)__ballot((C - th) & 0x78787878u); mk; mk &= mk - 1) {
                const u32 k = __builtin_ctz(mk), h = __builtin_amdgcn_readlane(th, k), l = __builtin_amdgcn_readlane(tl, k);
                for (u32 d = (C - h) & 0x78787878u; d;) {
                    const u32 b8 = __builtin_ctz(d) & 24, pos = (4 * i + (k >> 1)) << 9 | ln << 3 | (k & 1) << 2 | b8 >> 3;
                    d &= ~(0xffu << b8);
                    ++nexc;
                    hv = writelane(pos << 16 | ((h >> b8) & 0xff) << 8 | ((l >> b8) & 0xff), nexc & 63, hv);
                }
            }
        }
    }
    if (FORCE_RAW_IN || nexc > 31) {
#pragma unroll
        for (int f = 0; f < 16; ++f) st16(c, join8(H[2 * f], L[2 * f], H[2 * f + 1], L[2 * f + 1]), c.lane * 16, dst + f * 1024);
        return 1;
    }
    hv = writelane(top | nexc << 8, 0, hv);
    if (c.lane < 32) __builtin_amdgcn_raw_buffer_store_b32(hv, c.rs, c.lane * 4 + dst + 12288, 0, 17);
    return 0;
}

DEV u32 xa_off(int ks, int h, int rowfrag) { return WS_XA + (u32)((ks * 2 + h) * 256 + rowfrag) * 1024u; }
DEV u32 wb_off(int ks, int h, int colfrag) { return WS_WB + (u32)((ks * 2 + h) * 128 + colfrag) * 1024u; }

DEV void pack_job(const Ctx& c, int job) {
    const int s = job / 160, q = job % 160, ks = c.rank * 8 + s;
    const int l15 = c.lane & 15, lq = c.lane >> 4;
    const bool isx = q < 32;
    const int dst = (c.rank + 1 + (q - 32)) & 7, cc = (q - 32) >> 3;
    u32x4 R[16];
    if (isx) {
        const GAS char* src = (const GAS char*)c.x + (size_t)(q * 128 + l15) * (KSH * 2) + (s * 64 + lq * 8) * 2;
#pragma unroll
        for (int f = 0; f < 16; ++f) R[f] = *(const GAS u32x4*)(src + (f & 7) * 16 * (KSH * 2) + (f >> 3) * 64);
    } else {
        const GAS char* src = (const GAS char*)c.w + (size_t)(s * 64 + lq * 8) * (NTOT * 2) + (dst * 2048 + cc * 128 + l15 * 4) * 2;
        u32x2 r[4][8];
#pragma unroll
        for (int hg = 0; hg < 4; ++hg)
#pragma unroll
            for (int j = 0; j < 8; ++j) r[hg][j] = *(const GAS u32x2*)(src + (size_t)((hg >> 1) * 32 + j) * (NTOT * 2) + (hg & 1) * 128);
#pragma unroll
        for (int hg = 0; hg < 4; ++hg)
#pragma unroll
            for (int ii = 0; ii < 4; ++ii)
#pragma unroll
                for (int t = 0; t < 4; ++t)
                    R[hg * 4 + ii][t] = perm(r[hg][2 * t + 1][ii >> 1], r[hg][2 * t][ii >> 1], (ii & 1) ? 0x07060302u : 0x05040100u);
    }
    const bool mine = isx || dst == c.rank, theirs = isx || dst != c.rank;
    if (mine) store_raw(c, R, isx ? xa_off(ks, 0, q * 8) : wb_off(ks, 0, cc * 8), isx ? xa_off(ks, 1, q * 8) : wb_off(ks, 1, cc * 8));
    const u32 call = get_call(c), ch = isx ? s * 32 + q : 256 + s * 16 + cc;
    u32 raw = 0;
    if (theirs) raw = pack_store(c, R, WS_PK + (call & 1) * PK_PAR + (isx ? ch : ch + dst * 128) * CHB);
    if (job < 160) STAMPW(c, 5);
    WAIT_VM0();
    if (c.lane < NR && (isx || c.lane == dst)) {
        if (c.lane == c.rank) st_sys(c.me + (isx ? WS_XR + q * 256 : WS_WR + cc * 256) + ks * 4, call);
        else st_sys(c.peer_lane + WS_FIN + (((call & 1) * NR + c.rank) * NCHIN + ch) * 4, 2 * call + raw);
    }
}

struct Tile { int cc, big, o0, j; };
DEV int item_rf(const Ctx& c, int it) { return (((it / 3) + c.rank) & 7) * 32 + 20 + (it % 3) * 4; }
DEV int wave_rf(const Ctx& c, const Tile& t, int wr) { return t.big ? ((t.o0 + c.rank) & 7) * 32 + wr * 5 : item_rf(c, t.j * 4 + wr); }

template <int NF>
DEV void loader(const Ctx& c, const Tile& t, int L, u32 call) {
    constexpr int NI = (NF + 2) * 4;
    const int rf0 = wave_rf(c, t, 0), rf1 = wave_rf(c, t, 1), rf2 = wave_rf(c, t, 2), rf3 = wave_rf(c, t, 3);
    const int rcA = rf0 >> 3, rcB = NF == 5 ? rcA + 1 : ((rf1 >> 3) != rcA ? rf1 >> 3 : rf2 >> 3), rcC = NF == 5 ? rcA + 2 : rf3 >> 3;
    const int which = c.lane & 3, myks = L * 16 + (c.lane >> 2);
    const char* fp = c.me + (which == 3 ? WS_WR + t.cc * 256 : WS_XR + (which == 0 ? rcA : which == 1 ? rcB : rcC) * 256) + myks * 4;
    const u32 ldsbase = (u32)(size_t)c.lds, voff = c.lane * 16;
    const u32 pa0 = (NF == 5 ? rf0 : c.lane < 32 ? rf0 : rf1 - 4) * 1024 + c.lane * 128, pa1 = (NF == 5 ? rf0 + 8 : c.lane < 32 ? rf2 : rf3 - 4) * 1024 + c.lane * 128;
    const u32 pa2 = (rf0 + 16) * 1024 + (c.lane & 31) * 128, pb = t.cc * 8192 + c.lane * 128;
    u64 todo = 0x1111111111111111ull, avail = 0;
    int pend = -1;
    bool ok = true;
    while (todo && ok) {
        if (!avail) {
            if (pend >= 0) { WAIT_VM0(); c.ctl[C_FULL + pend % NSTG] = pend + 1; pend = -1; }
            u64 t0 = 0;
            for (u32 n = 1;; ++n) {
                u64 m = __ballot(ld_sys(fp) >= call);
                m &= m >> 1; m &= m >> 2;
                avail = m & todo;
                if (avail) break;
                if ((n & 15) == 0) {
                    if (rfl(c.ctl[C_ABORT]) || tmo_set(c, call)) { ok = false; break; }
                    if (expired(t0, DEADLINE)) { tmo_raise(c, call); ok = false; break; }
                }
                __builtin_amdgcn_s_sleep(4);
            }
            if (!ok) break;
            if (avail == todo && tmo_set(c, call)) { ok = false; break; }
        }
        const int bit = __builtin_ctzll(avail), ks = L * 16 + (bit >> 2);
        avail &= avail - 1; todo &= ~(1ull << bit);
        u32 tk = 0;
        if (c.lane == 0) tk = __hip_atomic_fetch_add((LAS u32*)&c.ctl[C_TICKET], 2u, __ATOMIC_RELAXED, __HIP_MEMORY_SCOPE_WORKGROUP);
        tk = rfl(tk);
        bool pre = false;
#pragma unroll
        for (int h = 0; h < 2; ++h) {
            const u32 tt = tk + h, slot = tt % NSTG, need = 8 * (tt / NSTG);
            if (rfl(c.ctl[C_FREE + slot]) < need) {
                for (int hh = h; hh < 2 && !pre; ++hh) {
                    touch(c.rs, pa0, xa_off(ks, hh, 0), ldsbase + L_DUMP); touch(c.rs, pa1, xa_off(ks, hh, 0), ldsbase + L_DUMP);
                    if (NF == 5) touch(c.rs, pa2, xa_off(ks, hh, 0), ldsbase + L_DUMP);
                    touch(c.rs, pb, wb_off(ks, hh, 0), ldsbase + L_DUMP);
                }
                pre = true;
                if (pend >= 0) { WAIT_VM0(); c.ctl[C_FULL + pend % NSTG] = pend + 1; pend = -1; }
                if (!lds_wait(c, C_FREE + slot, need, true, 1)) { ok = false; break; }
            }
            const u32 l = ldsbase + slot * STG;
            if (NF == 5) {
#pragma unroll
                for (int g = 0; g < 5; ++g) glds4(c.rs, voff, xa_off(ks, h, rf0 + g * 4), l + g * 4096);
            } else {
                glds4(c.rs, voff, xa_off(ks, h, rf0), l); glds4(c.rs, voff, xa_off(ks, h, rf1), l + 4096);
                glds4(c.rs, voff, xa_off(ks, h, rf2), l + 8192); glds4(c.rs, voff, xa_off(ks, h, rf3), l + 12288);
            }
            glds4(c.rs, voff, wb_off(ks, h, t.cc * 8), l + 20480); glds4(c.rs, voff, wb_off(ks, h, t.cc * 8 + 4), l + 24576);
            if (pend >= 0) {
                if (NI == 28) asm volatile("s_waitcnt vmcnt(28)" ::: "memory"); else asm volatile("s_waitcnt vmcnt(24)" ::: "memory");
                c.ctl[C_FULL + pend % NSTG] = pend + 1;
            }
            pend = tt;
        }
    }
    WAIT_VM0();
    if (!ok) c.ctl[C_ABORT] = 1;
    else if (pend >= 0) c.ctl[C_FULL + pend % NSTG] = pend + 1;
}

DEV u32 pk_relu(float a, float b) {
    const f32x2 v = __builtin_elementwise_maximum((f32x2){a, b}, (f32x2){0.f, 0.f});
    return __builtin_bit_cast(u32, __builtin_convertvector(v, bf16x2));
}

DEV u32 nz8(u32 cb) { return (cb + 0x07070707u) & 0x08080808u; }
template <int NF>
DEV u32 encode_y(const Ctx& c, const u32 (&P)[NF * 8], u32 dst) {
    constexpr int G = NF * 4;
    constexpr u32 CODEB = NF == 5 ? 2048 : 1536;
    u16x2 mx = {0, 0}, mn[NF];
#pragma unroll
    for (int f = 0; f < NF; ++f) {
        mn[f] = (u16x2){0xffff, 0xffff};
#pragma unroll
        for (int j = 0; j < 8; ++j) {
            const u16x2 v = __builtin_bit_cast(u16x2, P[f * 8 + j]);
            mx = __builtin_elementwise_max(mx, v);
            mn[f] = __builtin_elementwise_min(mn[f], (u16x2)(v - (u16x2){1, 1}));
        }
    }
    const u32 base = max(wave_max(max((u32)mx[0], (u32)mx[1])) >> 8, 7u) - 7, lim = ((base + 1) << 8) - 1;
    const u32 ldsbase = (u32)(size_t)c.lds, stg = L_YST + c.wave * YST;
    u32 nexc = 0, hv = 0;
#pragma unroll
    for (int f = 0; f < NF; ++f)
        for (u64 m = __ballot(min((u32)mn[f][0], (u32)mn[f][1]) < lim); m && nexc <= 31; m &= m - 1) {
            const u32 ln = __builtin_ctzll(m);
            u32 t = 0;
#pragma unroll
            for (int k = 0; k < 8; ++k) t = writelane(__builtin_amdgcn_readlane(P[f * 8 + k], ln), k, t);
            for (u32 mk = (u32)__ballot((t & 0xffff) - 1 < lim) | (u32)__ballot((t >> 16) - 1 < lim) << 8; mk; mk &= mk - 1) {
                const u32 k = __builtin_ctz(mk), v = (__builtin_amdgcn_readlane(t, k & 7) >> (k & 8) * 2) & 0xffff;
                ++nexc;
                hv = writelane((f * 16 + (k & 7) * 2 + (k >> 3)) << 22 | ln << 16 | v, nexc & 63, hv);
            }
        }
    if (FORCE_RAW_Y || nexc > 31) {
#pragma unroll
        for (int i = 0; i < NF * 2; ++i) st16(c, (u32x4){P[4 * i], P[4 * i + 1], P[4 * i + 2], P[4 * i + 3]}, c.lane * 16, dst + i * 1024);
        return YM_RAW | NF * 128;
    }
    hv = writelane(base | nexc << 8, 0, hv);
    if ((u32)c.lane <= nexc) *(LAS u32*)(c.lds + stg + c.lane * 4) = hv;
    u32 cb[G], lb[G], cnt = 0;
    const u16x2 B = {(u16)(base << 8), (u16)(base << 8)};
#pragma unroll
    for (int g = 0; g < G; ++g) {
        const u32 t0 = __builtin_bit_cast(u32, __builtin_elementwise_sub_sat(__builtin_bit_cast(u16x2, P[2 * g]), B));
        const u32 t1 = __builtin_bit_cast(u32, __builtin_elementwise_sub_sat(__builtin_bit_cast(u16x2, P[2 * g + 1]), B));
        cb[g] = perm(t1, t0, 0x07050301u); lb[g] = perm(t1, t0, 0x06040200u);
        cnt += __builtin_popcount(nz8(cb[g]));
    }
    u32 D[8];
#pragma unroll
    for (int h = 0; h < 2; ++h) {
        const u32* q = cb + 8 * h;
        D[3 * h] = q[0] | q[1] << 3 | (q[6] & 0x03030303u) << 6;
        D[3 * h + 1] = q[2] | q[3] << 3 | (q[7] & 0x03030303u) << 6;
        D[3 * h + 2] = q[4] | q[5] << 3 | (q[6] & 0x04040404u) << 4 | (q[7] & 0x04040404u) << 5;
    }
    st16(c, (u32x4){D[0], D[1], D[2], D[3]}, c.lane * 16, dst);
    if (NF == 5) st16(c, (u32x4){D[4], D[5], cb[G - 4] | cb[G - 3] << 3, cb[G - 2] | cb[G - 1] << 3}, c.lane * 16, dst + 1024);
    else __builtin_amdgcn_raw_buffer_store_b64((u32x2){D[4], D[5]}, c.rs, c.lane * 8 + dst + 1024, 0, 17);
    const u32 dw = (cnt >> 2) + 1, incl = wave_scan(dw), total = 1 + nexc + __builtin_amdgcn_readlane(incl, 63);
    u32 pos = ldsbase + stg + 4 * (1 + nexc + incl - dw), adr[G];
    u32x2 sel[G];
#pragma unroll
    for (int g = 0; g < G; ++g) {
        const u32 nz = nz8(cb[g]);
        adr[g] = pos & ~3u;
        sel[g] = *(const LAS u32x2*)(size_t)__builtin_amdgcn_udot4(nz, 0x20100804u, ((pos & 3) << 3) | (ldsbase + L_LUT), false);
        pos += __builtin_popcount(nz);
    }
    __builtin_amdgcn_sched_barrier(0);
    u32 cur = 0;
#pragma unroll
    for (int g = 0; g < G; ++g) {
        const u32 o = perm(lb[g], cur, sel[g][0]);
        *(LAS u32*)(size_t)adr[g] = o;
        cur = perm(lb[g], o, sel[g][1]);
    }
    *(LAS u32*)(size_t)(pos & ~3u) = cur;
    CBAR();
    u32x4 v[6];
#pragma unroll
    for (int i = 0; i < 6; ++i) if (i < 3 || total > i * 256u) v[i] = *(const LAS u32x4*)(c.lds + stg + c.lane * 16 + min(i * 1024u, YST - 1024));
#pragma unroll
    for (int i = 0; i < 6; ++i) if (i < 3 || total > i * 256u) st16(c, v[i], c.lane * 16, dst + CODEB + min(i * 1024u, YST - 1024));
    return (CODEB + total * 4 + 15) >> 4;
}

template <int NF>
DEV void consumer(const Ctx& c, const Tile& t, int wr, int wc) {
    f32x4 acc[NF][4];
#pragma unroll
    for (int f = 0; f < NF; ++f)
#pragma unroll
        for (int i = 0; i < 4; ++i) acc[f][i] = (f32x4){0.f, 0.f, 0.f, 0.f};
    const LAS char* pa = c.lds + c.lane * 16 + wr * NF * 1024;
    const LAS char* pb = c.lds + c.lane * 16 + 20480 + wc * 4096;
    u32 slot = 0, seen = 0;
    bool ok = true;
#pragma unroll 1
    for (u32 tk = 0; tk < 128; ++tk) {
        if (ok && rfl(seen) < tk + 1) ok = lds_wait(c, C_FULL + slot, tk + 1, true, 0);
        CBAR();
        if (tk == 0) STAMPW(c, 1);
        if (tk == 64) STAMPW(c, 4);
        if (tk == 96) STAMPW(c, 5);
        if (tk == 112) STAMPW(c, 6);
        if (tk == 120) STAMPW(c, 7);
        u32x4 a[NF], b[4];
#pragma unroll
        for (int i = 0; i < 4; ++i) b[i] = *(const LAS u32x4*)(pb + slot * STG + i * 1024);
#pragma unroll
        for (int f = 0; f < NF; ++f) a[f] = *(const LAS u32x4*)(pa + slot * STG + f * 1024);
        WAIT_LGKM0();
        if (c.lane == 0) __hip_atomic_fetch_add((LAS u32*)&c.ctl[C_FREE + slot], 1u, __ATOMIC_RELAXED, __HIP_MEMORY_SCOPE_WORKGROUP);
        slot = slot == NSTG - 1 ? 0 : slot + 1;
        seen = c.ctl[C_FULL + slot];
        __builtin_amdgcn_sched_barrier(0);
#pragma unroll
        for (int f = 0; f < NF; ++f)
#pragma unroll
            for (int i = 0; i < 4; ++i)
                acc[f][i] = __builtin_amdgcn_mfma_f32_16x16x32_bf16(__builtin_bit_cast(bf16x8, a[f]), __builtin_bit_cast(bf16x8, b[i]), acc[f][i], 0, 0, 0);
    }
    STAMPW(c, 2);
    ok = ok && !rfl(c.ctl[C_ABORT]);
    const u32 call = get_call(c);
    const int rf = wave_rf(c, t, wr), owner = rf >> 5;
    u32 P[NF * 8];
#pragma unroll
    for (int g = 0; g < NF * 4; ++g) {
        P[2 * g] = pk_relu(acc[g >> 2][0][g & 3], acc[g >> 2][1][g & 3]);
        P[2 * g + 1] = pk_relu(acc[g >> 2][2][g & 3], acc[g >> 2][3][g & 3]);
    }
    if (owner == c.rank) {
        GAS char* o = (GAS char*)c.out + (size_t)((rf & 31) * 16 + (c.lane >> 4) * 4) * (NTOT * 2) + (c.rank * 2048 + t.cc * 128 + wc * 64 + (c.lane & 15) * 4) * 2;
#pragma unroll
        for (int g = 0; g < NF * 4; ++g)
            st_out(o + (size_t)((g >> 2) * 16 + (g & 3)) * (NTOT * 2), ok ? (u32x2){P[2 * g], P[2 * g + 1]} : (u32x2){0x7fc07fc0u, 0x7fc07fc0u});
    } else {
        const int rg = NF == 5 ? wr : 4 + (((rf & 31) - 20) >> 2), msg = (t.cc * 7 + rg) * 2 + wc;
        ok = ok && lds_wait(c, C_FREE + 3, 200, true, 0) && lds_wait(c, C_FREE + 4, 200, true, 0);
        CBAR();
        STAMPW(c, 8);
        u32 meta = YM_BAD;
        if (ok) meta = encode_y<NF>(c, P, WS_YP + (owner * NMSG + msg) * YSLOT);
        STAMPW(c, 9);
        WAIT_VM0();
        if (c.lane == owner) st_sys64(c.peer_lane + WS_FY + (c.rank * NMSG + msg) * 8, (u64)call << 32 | meta);
    }
    STAMPW(c, 3);
}

DEV void gemm_wg(const Ctx& c) {
    const int g = c.wg - NCOMM, xcd = g & 7, idx = g >> 3, half = xcd >> 2, k = idx % 7;
    Tile t;
    t.cc = (xcd & 3) * 4 + idx / 7; t.big = k < 4; t.o0 = half * 4 + k; t.j = half * 3 + (k - 4);
    if (c.wave >= 8) {
        const int L = c.wave - 8;
        if (L * NGEMM + g >= 160) for (int i = 0; i < PSLEEP + L; ++i) __builtin_amdgcn_s_sleep(24);
#pragma unroll 1
        for (int job = L * NGEMM + g; job < 1280; job += 4 * NGEMM) { pack_job(c, job); if (job < 4 * NGEMM) STAMPW(c, 3); }
        STAMPW(c, 1);
        const u32 call = get_call(c);
        if (t.big) loader<5>(c, t, L, call); else loader<4>(c, t, L, call);
        STAMPW(c, 2);
    } else {
        if (t.big) consumer<5>(c, t, c.wave >> 1, c.wave & 1); else consumer<4>(c, t, c.wave >> 1, c.wave & 1);
    }
}

#define WINV 26
#define WINYV 15
constexpr u32 WINY = WINYV;
constexpr u32 WIN = WINV;

DEV u32 slot_off(u32 slot) { return min(slot * LBIG, L_SML0 - NBIG * LSML + slot * LSML); }
DEV void q_put(const Ctx& c, u32 n, u32 info, u32 src, u32 size16) {
    c.ctl[C_Q + n] = info;
    c.ctl[C_FE + n] = 0x80000000u | size16 << 19 | src >> 8;
}

DEV void watcher(const Ctx& c, int d, int sub) {
    const u32 call = get_call(c);
    const int p = (c.rank + d) & 7;
    bool bad = false;
    {
        GAS u64* ctr = (GAS u64*)(c.me + WS_CLM + d * 128);
        if (c.lane == 0) __hip_atomic_fetch_max(ctr, (u64)call << 32, __ATOMIC_RELAXED, __HIP_MEMORY_SCOPE_AGENT);
        WAIT_VM0();
        u32 enq = 0, blk = ~0u, id = 0, v = 0;
        u64 rdy = 0;
        for (;;) {
            if (enq > 2) lds_wait(c, C_ISS, enq - 2, false, 1);
            u32 n = 0;
            if (c.lane == 0) n = (u32)__hip_atomic_fetch_add(ctr, 2ull, __ATOMIC_RELAXED, __HIP_MEMORY_SCOPE_AGENT);
            n = rfl(n);
            if (n >= NCHIN) break;
            if (n >> 6 != blk) {
                blk = n >> 6; rdy = 0;
                const u32 code = ORDER.v[d - 1][blk * 64 + c.lane], s = (code >> 8) & 7, ix = code & 31;
                id = (code & 0x8000) ? 256 + s * 16 + ix : s * 32 + ((((ix >> 2) + c.rank) & 7) << 2 | (ix & 3));
            }
            const char* fp = c.me + WS_FIN + (((call & 1) * NR + p) * NCHIN + id) * 4;
#pragma unroll 1
            for (u32 ln = n & 63; ln < (n & 63) + 2; ++ln, ++enq) {
                u64 t0 = 0;
                for (u32 it = 1; !((rdy >> ln) & 1); ++it) {
                    if (it > 1) __builtin_amdgcn_s_sleep(2);
                    if ((it & 15) == 0) {
                        if (tmo_set(c, call)) bad = true;
                        else if (expired(t0, DEADLINE)) { tmo_raise(c, call); bad = true; }
                    }
                    v = bad ? 0 : ld_sys(fp);
                    rdy = __ballot(bad || v >= 2 * call);
                }
                if ((u32)c.lane == ln)
                    q_put(c, enq, id | (bad ? 1u << 14 : ((v & 1) ? T_INR : T_INP) << 12), WS_PK + (call & 1) * PK_PAR + (id < 256 ? id : id + c.rank * 128) * CHB,
                          bad ? 0 : (v & 1) ? 1024 : PKD_BYTES / 16);
            }
        }
        c.ctl[C_FE + enq] = 0xc0000000u;
    }
    STAMPW(c, 1);
    {
        const u32 id = sub + 4 * min(c.lane, NQY - 1);
        const char* fp = c.me + WS_FY + (p * NMSG + id) * 8;
        u64 pending = (1ull << NQY) - 1, t0 = 0;
        u32 qn = YQ0;
        for (u32 it = 1; pending; ++it) {
            const u64 v = bad ? 0 : ld_sys64(fp);
            const bool mine = ((pending >> c.lane) & 1) && (bad || (u32)(v >> 32) >= call);
            const u64 m = __ballot(mine);
            if (m) {
                const bool poison = bad || tmo_set(c, call);
                const u32 meta = (u32)v;
                const bool nan = poison || (meta & YM_BAD);
                if (mine) q_put(c, qn + lane_rank(m), id | (nan ? 1u << 14 : 0) | ((meta & YM_RAW) ? T_YR : T_YC) << 12, WS_YP + (c.rank * NMSG + id) * YSLOT,
                                nan ? 0 : min(meta & 0x7ff, YSLOT / 16));
                qn += __popcll(m); pending &= ~m; t0 = 0;
            } else {
                if ((it & 15) == 0) {
                    if (tmo_set(c, call)) bad = true;
                    else if (expired(t0, DEADLINE)) { tmo_raise(c, call); bad = true; }
                }
                __builtin_amdgcn_s_sleep(2);
            }
        }
    }
    STAMPW(c, 2);
}

DEV void fetcher(const Ctx& c, char* peer, int who) {
    __builtin_amdgcn_s_setprio(3);
    const u32 call = get_call(c);
    const u32 plo = (u32)(size_t)peer, phi = (u32)((size_t)peer >> 32) & 0xffff;
    const u32 ldsbase = (u32)(size_t)c.lds, voff = c.lane * 16;
    u32 nxt = who * (YQ0 + 1), pub = nxt, out = 0, qb = nxt, sh = 0, freem = 0, vq = 0, nend = ~0u;
    u64 ff = 0, t0 = 0;
    for (u32 it = 1; pub < NQ;) {
        const u32 e = __builtin_amdgcn_readlane(vq, nxt - qb), bytes = (e >> 15) & 0x7ff0, ni = (bytes + 1023) >> 10;
        const u32 m = (ni <= LSML / 1024 && freem >> NBIG) ? freem & ~((1u << NBIG) - 1) : freem & ((1u << NBIG) - 1);
        const bool yph = nxt >= YQ0;
        if (e >> 30 == 2 && m && (out + ni <= (yph ? WINY : WIN) || !out) && sh < 7 * 9) {
            u32 slot = __builtin_ctz(m);
            asm("" : "+s"(slot));
            freem &= ~(1u << slot);
            c.ctl[C_BUSY + slot] = 1;
            if (ni) glds_n(plo, phi, voff, (e & 0x7ffff) << 8, bytes, min(ni, 16u), ldsbase + slot_off(slot));
            ff |= (u64)(ni | slot << 5) << sh;
            sh += 9; out += ni; nxt += yph ? 2 : 1;
            if (!yph) c.ctl[C_ISS] = nxt;
            continue;
        }
        if (e >> 30 == 3) {
            if (c.lane < NWORK) c.ctl[C_FQ + nxt + c.lane] = FQ_END;
            nend = nxt; nxt = YQ0;
            if (pub == nend) pub = YQ0;
        }
        vq = c.ctl[C_FE + nxt + c.lane];
        const u32 vb = c.ctl[C_BUSY + (c.lane & 15)];
        qb = nxt;
        if (sh) {
            out -= (u32)ff & 31;
            wait_vm(out);
            c.ctl[C_FQ + pub] = (((u32)ff >> 5) & 15) + 1;
            ff >>= 9; sh -= 9; pub += pub >= YQ0 ? 2 : 1; t0 = 0;
            if (pub == 1) STAMPW(c, 1);
            if (pub <= YQ0) STAMPW(c, 3);
            if (pub == YQ0 + 2) STAMPW(c, 4);
            if (pub == nend) pub = YQ0;
        } else {
            __builtin_amdgcn_s_sleep(1);
            if ((++it & 1023) == 0 && expired(t0, DEADLINE_LDS)) { tmo_raise(c, call); break; }
        }
        freem = (u32)__ballot(vb == 0) & (who ? 0x3c20u : 0x3dfu);
    }
    WAIT_VM0();
    STAMPW(c, 5);
}

DEV void unpack_in(const Ctx& c, u32 e, u32 slot, int p, u32 call) {
    const LAS char* l = c.lds + slot * LBIG + c.lane * 16;
    const u32 id = e & 0xfff;
    const int s = id < 256 ? id >> 5 : (id - 256) >> 4, ks = p * 8 + s;
    const u32 d0 = id < 256 ? xa_off(ks, 0, (id & 31) * 8) : wb_off(ks, 0, (id & 15) * 8);
    const u32 d1 = id < 256 ? xa_off(ks, 1, (id & 31) * 8) : wb_off(ks, 1, (id & 15) * 8);
    u32x4 R[16];
    if (((e >> 12) & 3) == T_INR) {
#pragma unroll
        for (int f = 0; f < 16; ++f) R[f] = *(const LAS u32x4*)(l + f * 1024);
        WAIT_LGKM0();
        c.ctl[C_BUSY + slot] = 0;
        store_raw(c, R, d0, d1);
    } else {
        u32x4 L[8], code[4];
#pragma unroll
        for (int i = 0; i < 4; ++i) code[i] = *(const LAS u32x4*)(l + 8192 + i * 1024);
#pragma unroll
        for (int i = 0; i < 8; ++i) L[i] = *(const LAS u32x4*)(l + i * 1024);
        const u32 hv = *(const LAS u32*)(c.lds + slot * LBIG + 12288 + (c.lane & 31) * 4);
        WAIT_LGKM0();
        c.ctl[C_BUSY + slot] = 0;
        const u32 h0 = rfl(hv), C = (0x80u + (h0 & 0x7f)) * 0x01010101u, nexc = min((h0 >> 8) & 0xff, 31u);
#pragma unroll
        for (int f = 0; f < 16; ++f) {
            const u32 n = code[f >> 2][f & 3];
            const u32 ha = C - (n & 0x87878787u), hb = C - (__builtin_amdgcn_alignbit(n, n, 28) & 0x87878787u);
            const u32 la = L[f >> 1][(f & 1) * 2], lb = L[f >> 1][(f & 1) * 2 + 1];
            R[f] = join8(ha, la, hb, lb);
        }
        store_raw(c, R, d0, d1);
#pragma unroll 1
        for (u32 i = 0; i < nexc; ++i) {
            const u32 x = __builtin_amdgcn_readlane(hv, i + 1), pos = x >> 16, f = (pos >> 9) & 15, ln = (pos >> 3) & 63;
            if ((u32)c.lane == ln) __builtin_amdgcn_raw_buffer_store_b16((u16)x, c.rs, ln * 16 + (f & 7) * 1024 + (pos & 7) * 2, f < 8 ? d0 : d1, 17);
        }
    }
    WAIT_VM0();
    if (c.lane == 0) st_sys(c.me + (id < 256 ? WS_XR + (id & 31) * 256 : WS_WR + (id & 15) * 256) + ks * 4, call);
}

template <int NF>
DEV void decode_y(const Ctx& c, u32 e, u32 slot, GAS char* o) {
    constexpr int G = NF * 4;
    constexpr u32 CODEB = NF == 5 ? 2048 : 1536;
    const u32 lo = slot_off(slot), ldsbase = (u32)(size_t)c.lds;
    const LAS char* l = c.lds + lo;
    u32x2 P[G];
    u32 hv = 0, nexc = 0;
    if ((e >> 14) & 1) {
#pragma unroll
        for (int g = 0; g < G; ++g) P[g] = (u32x2){0x7fc07fc0u, 0x7fc07fc0u};
    } else if (((e >> 12) & 3) == T_YR) {
#pragma unroll
        for (int i = 0; i < G / 2; ++i) {
            const u32x4 v = *(const LAS u32x4*)(l + c.lane * 16 + i * 1024);
            P[2 * i] = (u32x2){v[0], v[1]}; P[2 * i + 1] = (u32x2){v[2], v[3]};
        }
    } else {
        const u32x4 da = *(const LAS u32x4*)(l + c.lane * 16);
        u32x4 db;
        if (NF == 5) db = *(const LAS u32x4*)(l + 1024 + c.lane * 16);
        else { const u32x2 t = *(const LAS u32x2*)(l + 1024 + c.lane * 8); db = (u32x4){t[0], t[1], 0, 0}; }
        hv = *(const LAS u32*)(l + CODEB + (c.lane & 31) * 4);
        const u32 D[8] = {da[0], da[1], da[2], da[3], db[0], db[1], db[2], db[3]};
        u32 cb[G], cnt = 0;
#pragma unroll
        for (int h = 0; h < 2; ++h) {
            u32* q = cb + 8 * h;
#pragma unroll
            for (int k = 0; k < 3; ++k) { q[2 * k] = D[3 * h + k] & 0x07070707u; q[2 * k + 1] = (D[3 * h + k] >> 3) & 0x07070707u; }
            q[6] = bfi(0x03030303u, D[3 * h] >> 6, D[3 * h + 2] >> 4) & 0x07070707u;
            q[7] = bfi(0x03030303u, D[3 * h + 1] >> 6, D[3 * h + 2] >> 5) & 0x07070707u;
        }
        if (NF == 5) { cb[G - 4] = D[6] & 0x07070707u; cb[G - 3] = (D[6] >> 3) & 0x07070707u; cb[G - 2] = D[7] & 0x07070707u; cb[G - 1] = (D[7] >> 3) & 0x07070707u; }
#pragma unroll
        for (int g = 0; g < G; ++g) cnt += __builtin_popcount(nz8(cb[g]));
        const u32 h0 = rfl(hv), B = (h0 & 0xff) * 0x01010101u, dw = (cnt >> 2) + 1;
        nexc = min((h0 >> 8) & 0xff, 31u);
        u32 pos = ldsbase + lo + CODEB + 4 * (1 + nexc + wave_scan(dw) - dw), ps[G];
        u32x2 sel[G];
#pragma unroll
        for (int g = 0; g < G; ++g) {
            const u32 nz = nz8(cb[g]);
            const LAS u32* q = (const LAS u32*)(size_t)(pos & ~3u);
            sel[g] = *(const LAS u32x2*)(size_t)__builtin_amdgcn_udot4(nz, 0x08040201u, ldsbase + L_LUT, false);
            P[g] = (u32x2){q[0], q[1]};
            ps[g] = pos;
            pos += __builtin_popcount(nz);
        }
        __builtin_amdgcn_sched_barrier(0);
#pragma unroll
        for (int g = 0; g < G; ++g) {
            const u32 by = __builtin_amdgcn_alignbyte(P[g][1], P[g][0], ps[g]), lb = perm(by, by, sel[g][0]), hb = (cb[g] + B) & sel[g][1];
            P[g] = (u32x2){perm(hb, lb, 0x05010400u), perm(hb, lb, 0x07030602u)};
        }
    }
    WAIT_LGKM0();
    c.ctl[C_BUSY + slot] = 0;
#pragma unroll
    for (int g = 0; g < G; ++g) {
        st_out(o + (size_t)((g >> 2) * 16 + (g & 3)) * (NTOT * 2), P[g]);
    }
#pragma unroll 1
    for (u32 i = 0; i < nexc; ++i) {
        const u32 x = __builtin_amdgcn_readlane(hv, i + 1), j = min(x >> 22, (u32)G * 4 - 1);
        if ((u32)c.lane == ((x >> 16) & 63))
            __hip_atomic_store((GAS u16*)(o + (size_t)((j >> 4) * 16 + ((j >> 2) & 3)) * (NTOT * 2) + (j & 3) * 2), (u16)x, __ATOMIC_RELAXED, __HIP_MEMORY_SCOPE_SYSTEM);
    }
}
DEV void decode_y(const Ctx& c, u32 e, u32 slot, int p) {
    const u32 id = e & 0xfff, wc = id & 1, rg = (id >> 1) % 7, cc = (id >> 1) / 7;
    const int row0 = rg < 4 ? rg * 80 : 320 + (rg - 4) * 64;
    GAS char* o = (GAS char*)c.out + (size_t)(row0 + (c.lane >> 4) * 4) * (NTOT * 2) + (p * 2048 + cc * 128 + wc * 64 + (c.lane & 15) * 4) * 2;
    if (rg < 4) decode_y<5>(c, e, slot, o); else decode_y<4>(c, e, slot, o);
}

DEV void worker(const Ctx& c, int p, int w) {
    const u32 call = get_call(c);
#pragma unroll 1
    for (int i = w; i < NQ; i += NWORK) {
        if (!lds_wait(c, C_FQ + i, 1, false, 1)) { tmo_raise(c, call); break; }
        CBAR();
        const u32 slot = rfl(c.ctl[C_FQ + i]) - 1, e = rfl(c.ctl[C_Q + i]);
        if (i < YQ0) {
            if (slot == FQ_END - 1) { STAMPW(c, 1); i = YQ0 + w - NWORK; }
            else if ((e >> 14) & 1) c.ctl[C_BUSY + slot] = 0;
            else unpack_in(c, e, slot, p, call);
        } else {
            decode_y(c, e, slot, p);
        }
    }
    STAMPW(c, 2);
}

__global__ void __launch_bounds__(NTHREADS, 3) fused_kernel(Args a) {
    extern __shared__ __attribute__((aligned(128))) char smem[];
    Ctx c;
    c.x = a.x; c.w = a.w; c.out = a.out; c.rank = a.rank & 7; c.me = a.ws[c.rank];
    c.wg = blockIdx.x; c.wave = rfl(threadIdx.x >> 6); c.lane = threadIdx.x & 63;
    c.peer_lane = a.ws[c.lane & 7];
    c.rs = __builtin_amdgcn_make_buffer_rsrc(c.me, 0, WS_END, 0x00020000);
    c.lds = (LAS char*)smem; c.ctl = (volatile LAS u32*)(c.lds + L_CTL);
    STAMPW(c, 0);
    u32 ticket = 0;
    if (threadIdx.x == 0) ticket = __hip_atomic_fetch_add((GAS u32*)(c.me + WS_CNT), 1u, __ATOMIC_RELAXED, __HIP_MEMORY_SCOPE_AGENT);
    for (u32 i = threadIdx.x; i < C_WORDS; i += NTHREADS) c.ctl[i] = 0;
    if (c.wave == 11) {
        u32 s0, s1;
        if (c.wg >= NCOMM) {
            const u32 mask = c.lane >> 2;
            u32 n = c.lane & 3;
            u64 sel = 0x0c0c0c0c03020100ull;
            for (int b = 0; b < 4; ++b)
                if ((mask >> b) & 1) { sel = (sel & ~(0xffull << (8 * n))) | (u64)(4 + b) << (8 * n); ++n; }
            s0 = (u32)sel; s1 = n >= 4 ? (u32)(sel >> 32) : 0x03020100u;
        } else {
            u32 n = 0;
            s0 = 0x0c0c0c0cu; s1 = 0;
            for (int b = 0; b < 4; ++b)
                if ((c.lane >> b) & 1) { s0 = (s0 & ~(0xffu << (8 * b))) | n << (8 * b); s1 |= 0xffu << (8 * b); ++n; }
        }
        c.ctl[C_LUT + 2 * c.lane] = s0; c.ctl[C_LUT + 2 * c.lane + 1] = s1;
    }
    WAIT_LGKM0();
    __builtin_amdgcn_s_barrier();
    CBAR();
    if (threadIdx.x == 0) c.ctl[C_CALL] = ticket / NWG + 1;
    if (c.wg >= NCOMM) gemm_wg(c);
    else if (c.wg < NPULL) {
        const int d = 1 + c.wg % 7, sub = c.wg / 7, p = (c.rank + d) & 7;
        if (c.wave == 1) watcher(c, d, sub);
        else if (c.wave < 3) fetcher(c, a.ws[p], c.wave >> 1);
        else worker(c, p, c.wave - 3);
    }
}

}

extern "C" void kernel_launch(int rank, int nranks, ncclComm_t comm, hipStream_t stream,
                              void* const* d_in, const int* in_sizes, int n_in,
                              void* d_out, int out_size, const int* in_dtypes, int out_dtype,
                              void* const* d_out_all, void* const* d_ws_all, size_t ws_bytes) {
    if (nranks != NR || (unsigned)rank >= (unsigned)NR || n_in != 2 || ws_bytes < WS_END || in_sizes[0] != 4096 * KSH ||
        in_sizes[1] != KSH * NTOT || out_size != 512 * NTOT) {
        fprintf(stderr, "unexpected problem shape\n");
        return;
    }
    HIP_CHECK(hipFuncSetAttribute((const void*)fused_kernel, hipFuncAttributeMaxDynamicSharedMemorySize, LDS_BYTES));
    Args a;
    a.x = (const u16*)d_in[0]; a.w = (const u16*)d_in[1]; a.out = (u16*)d_out;
    for (int p = 0; p < NR; ++p) a.ws[p] = (char*)d_ws_all[p];
    a.rank = rank; a.pad = 0;
    hipLaunchKernelGGL(fused_kernel, dim3(NWG), dim3(NTHREADS), LDS_BYTES, stream, a);
    HIP_CHECK(hipGetLastError());
}
```

```cpp
#include <hip/hip_runtime.h>
#include <rccl/rccl.h>
#include <cstdio>
#include <cstdint>

#define HIP_CHECK(expr) do { hipError_t _e = (expr); if (_e != hipSuccess) { \
    fprintf(stderr, "HIP error %s at %s:%d\n", hipGetErrorString(_e), __FILE__, __LINE__); \
    return; } } while (0)

extern "C" __device__ unsigned writelane(unsigned value, unsigned lane, unsigned old) __asm("llvm.amdgcn.writelane.i32");

namespace {

typedef unsigned u32;
typedef unsigned long long u64;
typedef unsigned short u16;
typedef u32 u32x4 __attribute__((ext_vector_type(4)));
typedef u32 u32x2 __attribute__((ext_vector_type(2)));
typedef u32 u32x8 __attribute__((ext_vector_type(8)));
typedef u32 u32x32 __attribute__((ext_vector_type(32)));
typedef u16 u16x2 __attribute__((ext_vector_type(2)));
typedef float f32x4 __attribute__((ext_vector_type(4)));
typedef float f32x2 __attribute__((ext_vector_type(2)));
typedef __bf16 bf16x2 __attribute__((ext_vector_type(2)));
typedef __bf16 bf16x8 __attribute__((ext_vector_type(8)));
typedef __amdgpu_buffer_rsrc_t rsrc_t;
#define LAS __attribute__((address_space(3)))
#define GAS __attribute__((address_space(1)))
#define DEV __device__ __forceinline__

#define PSLEEP 3
#define OAUX 17
#define XROT 1
#define FORCE_RAW_Y 0
#define FORCE_RAW_IN 0

constexpr int NR = 8;
constexpr int NTOT = 16384, KSH = 512;
constexpr int NWG = 256, NCOMM = 32, NPULL = 28, NGEMM = NWG - NCOMM, NTHREADS = 768;
constexpr int NCHIN = 384;
constexpr int NMSG = 256;
constexpr u32 NQY = NMSG / 4;
constexpr int NWORK = 11;

constexpr u32 WS_CNT = 0, WS_TMO = 128, WS_CLM = 1024;
constexpr u32 WS_FIN = 4096;
constexpr u32 WS_FY = 32768;
constexpr u32 WS_XR = 49152;
constexpr u32 WS_WR = 57344;
constexpr u32 WS_DBG = 65536;
constexpr u32 CHB = 16384;
constexpr u32 WS_PK = 1u << 20;
constexpr u32 PK_PAR = 1280u * CHB;
constexpr u32 WS_XA = WS_PK + 2 * PK_PAR;
constexpr u32 WS_WB = WS_XA + (32u << 20);
constexpr u32 YSLOT = 8192;
constexpr u32 WS_YP = WS_WB + (16u << 20);
constexpr u32 WS_END = WS_YP + 8u * NMSG * YSLOT;
constexpr u32 PKD_BYTES = 12288 + 128;

constexpr u32 STG = 28672, NSTG = 5;
constexpr u32 LSLOT = 6144;
constexpr u32 L_CTL = 143360;
static_assert(STG * NSTG <= L_CTL && NWORK * 2 * LSLOT <= L_CTL, "ring");
constexpr u32 C_CALL = 0, C_ABORT = 1, C_TICKET = 2, C_TICKET2 = 3, C_FULL = 8, C_FREE = 16,
              C_TK = 2, C_IS = 3, C_LAND = 4, C_YGO = 5, C_YT = 6, C_Q = 64, C_WORDS = 128,
              C_LUT = C_WORDS;
constexpr u32 LDS_BYTES = L_CTL + (C_LUT + 128) * 4;
#define ST2(c, i, v) do {} while (0)
constexpr u32 L_LUT = L_CTL + C_LUT * 4, L_YST = 3 * STG, YST = 4608;
static_assert(L_LUT % 512 == 0 && L_YST + 8 * YST <= 5 * STG && 4 * (32 + 64 * 17) <= YST && 1536 + 4 * (32 + 64 * 17) <= LSLOT, "y staging");
static_assert(LDS_BYTES <= 160 * 1024, "lds");

constexpr u64 DEADLINE = 40000000ull;
constexpr u64 DEADLINE_LDS = 100000000ull;

constexpr u32 YM_RAW = 1u << 16, YM_BAD = 1u << 17;

struct Order { u16 v[7][NCHIN]; };
#define D1V 6
#define D2V 2
#define D3V 0
constexpr Order make_order() {
    Order o{};
    for (int d = 1; d <= 7; ++d) {
        int n = 0;
        for (int pass = 0; pass < 4; ++pass)
            for (int s = 0; s < 8; ++s) {
                if (pass == (s >= 8 - D1V ? 3 : 0) && s) for (int q = 0; q < 4; ++q) o.v[d - 1][n++] = (u16)(s << 8 | q);
                for (int c = 0; c < 16; ++c) if (pass == ((c & 2) && s >= 8 - D3V ? 1 : 0)) o.v[d - 1][n++] = (u16)(0x8000 | s << 8 | c);
                if (pass == (s >= 8 - D1V ? 3 : 0) && !s) for (int q = 0; q < 4; ++q) o.v[d - 1][n++] = (u16)(s << 8 | q);
                for (int i = 0; i < 28; ++i) {
                    const int q = (i / 14) * 2 + (i & 1), own = 1 + ((i % 14) / 2 + d) % 7;
                    if (pass == (q >= 2 && own != 4 && s >= 8 - D2V ? 2 : 0)) o.v[d - 1][n++] = (u16)(s << 8 | own << 2 | q);
                }
            }
    }
    return o;
}
__constant__ const Order ORDER = make_order();

struct Args { const u16* x; const u16* w; u16* out; char* ws[NR]; int rank; int pad; };

DEV u64 rt() { return __builtin_amdgcn_s_memrealtime(); }
DEV u32 rfl(u32 v) { return __builtin_amdgcn_readfirstlane(v); }
DEV u32 perm(u32 hi, u32 lo, u32 sel) { return __builtin_amdgcn_perm(hi, lo, sel); }
DEV u32 bfi(u32 m, u32 a, u32 b) { return (m & a) | (~m & b); }
#define CBAR() asm volatile("" ::: "memory")
#define WAIT_VM0() asm volatile("s_waitcnt vmcnt(0)" ::: "memory")
#define WAIT_LGKM0() do { CBAR(); __builtin_amdgcn_s_waitcnt(0xc07f); CBAR(); } while (0)

#define STAMPW(c, i) do {} while (0)

struct Ctx {
    const u16* x; const u16* w; u16* out; char* me; char* peer_lane;
    rsrc_t rs, ro;
    LAS char* lds; volatile LAS u32* ctl;
    int rank, wg, wave, lane;
};

DEV u32 ld_sys(const void* p) { return __hip_atomic_load((const GAS u32*)p, __ATOMIC_RELAXED, __HIP_MEMORY_SCOPE_SYSTEM); }
DEV u64 ld_sys64(const void* p) { return __hip_atomic_load((const GAS u64*)p, __ATOMIC_RELAXED, __HIP_MEMORY_SCOPE_SYSTEM); }
DEV void st_sys(void* p, u32 v) { __hip_atomic_store((GAS u32*)p, v, __ATOMIC_RELAXED, __HIP_MEMORY_SCOPE_SYSTEM); }
DEV void st_sys64(void* p, u64 v) { __hip_atomic_store((GAS u64*)p, v, __ATOMIC_RELAXED, __HIP_MEMORY_SCOPE_SYSTEM); }
DEV void st16(const Ctx& c, u32x4 v, u32 voff, u32 soff) { __builtin_amdgcn_raw_buffer_store_b128(v, c.rs, voff + soff, 0, 17); }
DEV void st_out(const Ctx& c, u32 off, u32x2 v) { __builtin_amdgcn_raw_buffer_store_b64(v, c.ro, off, 0, OAUX); }
DEV u32 out_off(const Ctx& c, u32 row0, u32 col0) { return (row0 + (c.lane >> 4) * 4) * (NTOT * 2) + (col0 + (c.lane & 15) * 4) * 2; }
DEV bool tmo_set(const Ctx& c, u32 call) { return rfl(__hip_atomic_load((const GAS u32*)(c.me + WS_TMO), __ATOMIC_RELAXED, __HIP_MEMORY_SCOPE_AGENT)) == call; }
DEV void tmo_raise(const Ctx& c, u32 call) { if (c.lane == 0) __hip_atomic_store((GAS u32*)(c.me + WS_TMO), call, __ATOMIC_RELAXED, __HIP_MEMORY_SCOPE_AGENT); }

DEV bool expired(u64& t0, u64 limit) { const u64 t = rt(); if (!t0) t0 = t; return t - t0 > limit; }

DEV u32 lds_wait(const Ctx& c, u32 word, u32 want, bool look_abort, int nap) {
    u64 t0 = 0;
    for (u32 n = 1;; ++n) {
        const u32 v = rfl(c.ctl[word]);
        if (v >= want) return v;
        if (nap) __builtin_amdgcn_s_sleep(1);
        if ((n & 63) == 0) {
            if (look_abort && rfl(c.ctl[C_ABORT])) return 0;
            if (expired(t0, DEADLINE_LDS)) { c.ctl[C_ABORT] = 1; return 0; }
        }
    }
}
DEV u32 get_call(const Ctx& c) { return lds_wait(c, C_CALL, 1, false, 1); }

DEV void glds4(rsrc_t rs, u32 voff, u32 soff, u32 ldsaddr) {
    u32 keep;
    asm volatile("s_mov_b32 %0, m0\n\ts_mov_b32 m0, %4\n\ts_nop 2\n\t"
                 "buffer_load_dwordx4 %1, %2, %3 offen sc0 sc1 lds\n\t"
                 "buffer_load_dwordx4 %1, %2, %3 offen offset:1024 sc0 sc1 lds\n\t"
                 "buffer_load_dwordx4 %1, %2, %3 offen offset:2048 sc0 sc1 lds\n\t"
                 "buffer_load_dwordx4 %1, %2, %3 offen offset:3072 sc0 sc1 lds\n\t"
                 "s_mov_b32 m0, %0" : "=&s"(keep) : "v"(voff), "s"(rs), "s"(soff), "s"(ldsaddr) : "memory");
}
DEV void glds2(rsrc_t rs, u32 voff, u32 soff, u32 ldsaddr) {
    u32 keep;
    asm volatile("s_mov_b32 %0, m0\n\ts_mov_b32 m0, %4\n\ts_nop 2\n\t"
                 "buffer_load_dwordx4 %1, %2, %3 offen sc0 sc1 lds\n\t"
                 "buffer_load_dwordx4 %1, %2, %3 offen offset:1024 sc0 sc1 lds\n\t"
                 "s_mov_b32 m0, %0" : "=&s"(keep) : "v"(voff), "s"(rs), "s"(soff), "s"(ldsaddr) : "memory");
}
DEV u32x4 ld16(rsrc_t rs, u32 voff, u32 soff) { return __builtin_amdgcn_raw_buffer_load_b128(rs, voff, soff, 17); }
DEV u32 lds_inc(const Ctx& c, u32 word) {
    u32 v = 0;
    if (c.lane == 0) v = __hip_atomic_fetch_add((LAS u32*)&c.ctl[word], 1u, __ATOMIC_RELAXED, __HIP_MEMORY_SCOPE_WORKGROUP);
    return rfl(v);
}

DEV u32 wave_max(u32 v) {
    v = max(v, (u32)__builtin_amdgcn_update_dpp(0, (int)v, 0x111, 0xf, 0xf, true));
    v = max(v, (u32)__builtin_amdgcn_update_dpp(0, (int)v, 0x112, 0xf, 0xf, true));
    v = max(v, (u32)__builtin_amdgcn_update_dpp(0, (int)v, 0x114, 0xf, 0xf, true));
    v = max(v, (u32)__builtin_amdgcn_update_dpp(0, (int)v, 0x118, 0xf, 0xf, true));
    v = max(v, (u32)__builtin_amdgcn_update_dpp(0, (int)v, 0x142, 0xa, 0xf, false));
    v = max(v, (u32)__builtin_amdgcn_update_dpp(0, (int)v, 0x143, 0xc, 0xf, false));
    return __builtin_amdgcn_readlane(v, 63);
}

DEV u32 wave_scan(u32 v) {
    v += (u32)__builtin_amdgcn_update_dpp(0, (int)v, 0x111, 0xf, 0xf, true);
    v += (u32)__builtin_amdgcn_update_dpp(0, (int)v, 0x112, 0xf, 0xf, true);
    v += (u32)__builtin_amdgcn_update_dpp(0, (int)v, 0x114, 0xf, 0xf, true);
    v += (u32)__builtin_amdgcn_update_dpp(0, (int)v, 0x118, 0xf, 0xf, true);
    v += (u32)__builtin_amdgcn_update_dpp(0, (int)v, 0x142, 0xa, 0xf, false);
    v += (u32)__builtin_amdgcn_update_dpp(0, (int)v, 0x143, 0xc, 0xf, false);
    return v;
}
DEV u32 lane_rank(u64 m) { return __builtin_amdgcn_mbcnt_hi((u32)(m >> 32), __builtin_amdgcn_mbcnt_lo((u32)m, 0)); }

DEV void store_raw(const Ctx& c, const u32x4 (&R)[16], u32 off0, u32 off1) {
#pragma unroll
    for (int f = 0; f < 16; ++f) st16(c, R[f], c.lane * 16, (f < 8 ? off0 : off1) + (f & 7) * 1024);
}
DEV u32x4 join8(u32 ha, u32 la, u32 hb, u32 lb) {
    return (u32x4){perm(ha, la, 0x05010400u), perm(ha, la, 0x07030602u), perm(hb, lb, 0x05010400u), perm(hb, lb, 0x07030602u)};
}
DEV u32 pack_store(const Ctx& c, const u32x4 (&R)[16], u32 dst) {
    u32 H[32], L[32];
    u16x2 m0 = {0, 0}, m1 = {0, 0};
#pragma unroll
    for (int f = 0; f < 16; ++f) {
        L[2 * f] = perm(R[f][1], R[f][0], 0x06040200u); L[2 * f + 1] = perm(R[f][3], R[f][2], 0x06040200u);
        H[2 * f] = perm(R[f][1], R[f][0], 0x07050301u); H[2 * f + 1] = perm(R[f][3], R[f][2], 0x07050301u);
    }
#pragma unroll
    for (int i = 0; i < 8; ++i) st16(c, (u32x4){L[4 * i], L[4 * i + 1], L[4 * i + 2], L[4 * i + 3]}, c.lane * 16, dst + i * 1024);
#pragma unroll
    for (int i = 0; i < 32; ++i) {
        const u32 t = H[i] & 0x7f7f7f7fu;
        m0 = __builtin_elementwise_max(m0, __builtin_bit_cast(u16x2, t));
        m1 = __builtin_elementwise_max(m1, __builtin_bit_cast(u16x2, t << 8));
    }
    m0 = __builtin_elementwise_max(m0, m1);
    const u32 top = wave_max(max((u32)m0[0], (u32)m0[1])) >> 8;
    const u32 C = (0x80u + top) * 0x01010101u;
    u32 nexc = 0, hv = 0;
#pragma unroll
    for (int i = 0; i < 4; ++i) {
        u32x4 code;
        u32 chk = 0;
#pragma unroll
        for (int j = 0; j < 4; ++j) {
            const u32 d0 = C - H[8 * i + 2 * j], d1 = C - H[8 * i + 2 * j + 1];
            chk |= d0 | d1;
            code[j] = bfi(0x87878787u, d0, __builtin_amdgcn_alignbit(d1, d1, 4));
        }
        st16(c, code, c.lane * 16, dst + 8192 + i * 1024);
        for (u64 m = __ballot(chk & 0x78787878u); m && nexc <= 31; m &= m - 1) {
            const u32 ln = __builtin_ctzll(m);
            u32 th = top * 0x01010101u, tl = 0;
#pragma unroll
            for (int k = 0; k < 8; ++k) {
                th = writelane(__builtin_amdgcn_readlane(H[8 * i + k], ln), k, th);
                tl = writelane(__builtin_amdgcn_readlane(L[8 * i + k], ln), k, tl);
            }
            for (u32 mk = (u32)__ballot((C - th) & 0x78787878u); mk; mk &= mk - 1) {
                const u32 k = __builtin_ctz(mk), h = __builtin_amdgcn_readlane(th, k), l = __builtin_amdgcn_readlane(tl, k);
                for (u32 d = (C - h) & 0x78787878u; d;) {
                    const u32 b8 = __builtin_ctz(d) & 24, pos = (4 * i + (k >> 1)) << 9 | ln << 3 | (k & 1) << 2 | b8 >> 3;
                    d &= ~(0xffu << b8);
                    ++nexc;
                    hv = writelane(pos << 16 | ((h >> b8) & 0xff) << 8 | ((l >> b8) & 0xff), nexc & 63, hv);
                }
            }
        }
    }
    if (FORCE_RAW_IN || nexc > 31) {
#pragma unroll
        for (int f = 0; f < 16; ++f) st16(c, join8(H[2 * f], L[2 * f], H[2 * f + 1], L[2 * f + 1]), c.lane * 16, dst + f * 1024);
        return 1;
    }
    hv = writelane(top | nexc << 8, 0, hv);
    if (c.lane < 32) __builtin_amdgcn_raw_buffer_store_b32(hv, c.rs, c.lane * 4 + dst + 12288, 0, 17);
    return 0;
}

DEV u32 xa_off(int ks, int h, int rowfrag) { return WS_XA + (u32)((ks * 2 + h) * 256 + rowfrag) * 1024u; }
DEV u32 wb_off(int ks, int h, int colfrag) { return WS_WB + (u32)((ks * 2 + h) * 128 + colfrag) * 1024u; }

DEV void pack_job(const Ctx& c, int job) {
    const int s = job / 160, q = job % 160, ks = c.rank * 8 + s;
    const int l15 = c.lane & 15, lq = c.lane >> 4;
    const bool isx = q < 32;
    const int dst = (c.rank + 1 + (q - 32)) & 7, cc = (q - 32) >> 3;
    u32x4 R[16];
    if (isx) {
        const GAS char* src = (const GAS char*)c.x + (size_t)(q * 128 + l15) * (KSH * 2) + (s * 64 + lq * 8) * 2;
#pragma unroll
        for (int f = 0; f < 16; ++f) R[f] = *(const GAS u32x4*)(src + (f & 7) * 16 * (KSH * 2) + (f >> 3) * 64);
    } else {
        const GAS char* src = (const GAS char*)c.w + (size_t)(s * 64 + lq * 8) * (NTOT * 2) + (dst * 2048 + cc * 128 + l15 * 4) * 2;
        u32x2 r[4][8];
#pragma unroll
        for (int hg = 0; hg < 4; ++hg)
#pragma unroll
            for (int j = 0; j < 8; ++j) r[hg][j] = *(const GAS u32x2*)(src + (size_t)((hg >> 1) * 32 + j) * (NTOT * 2) + (hg & 1) * 128);
#pragma unroll
        for (int hg = 0; hg < 4; ++hg)
#pragma unroll
            for (int ii = 0; ii < 4; ++ii)
#pragma unroll
                for (int t = 0; t < 4; ++t)
                    R[hg * 4 + ii][t] = perm(r[hg][2 * t + 1][ii >> 1], r[hg][2 * t][ii >> 1], (ii & 1) ? 0x07060302u : 0x05040100u);
    }
    const bool mine = isx || dst == c.rank, theirs = isx || dst != c.rank;
    if (mine) store_raw(c, R, isx ? xa_off(ks, 0, q * 8) : wb_off(ks, 0, cc * 8), isx ? xa_off(ks, 1, q * 8) : wb_off(ks, 1, cc * 8));
    const u32 call = get_call(c), ch = isx ? s * 32 + q : 256 + s * 16 + cc;
    u32 raw = 0;
    if (theirs) raw = pack_store(c, R, WS_PK + (call & 1) * PK_PAR + (isx ? ch : ch + dst * 128) * CHB);
    if (job < 160) STAMPW(c, 5);
    WAIT_VM0();
    if (c.lane < NR && (isx || c.lane == dst)) {
        if (c.lane == c.rank) st_sys(c.me + (isx ? WS_XR + q * 256 : WS_WR + cc * 256) + ks * 4, call);
        else st_sys(c.peer_lane + WS_FIN + (((call & 1) * NR + c.rank) * NCHIN + ch) * 4, 2 * call + raw);
    }
}

struct Tile { int cc, owner, h, rf, rfo; };
constexpr u32 TK2 = 130;

template <bool BIG>
DEV void loader(const Ctx& c, const Tile& t, int L, u32 xr, u32 call) {
    const int which = c.lane & 3, myks = L * 16 + ((c.lane >> 2) ^ xr), rc = BIG && which == 2 ? t.rfo >> 3 : (t.rf >> 3) + (which & 1);
    const char* fp = c.me + (which == 3 ? WS_WR + t.cc * 256 : WS_XR + rc * 256) + myks * 4;
    const u32 ldsbase = (u32)(size_t)c.lds, voff = c.lane * 16;
    u64 todo = 0x1111111111111111ull, avail = 0, own = 0, late = 0;
    u32 pend = 0, pslot = 0;
    bool ok = true, second = false;
    while (ok) {
        if (!todo) {
            if (!BIG || second || !late) break;
            second = true; todo = late;
        }
        if (!avail) {
            if (pend) { WAIT_VM0(); c.ctl[C_FULL + pslot] = pend; pend = 0; }
            u64 t0 = 0;
            for (u32 n = 1;; ++n) {
                const u64 m = __ballot(ld_sys(fp) >= call);
                own = m >> 2;
                avail = (second ? own : m & m >> 1 & m >> 3) & todo;
                if (avail) break;
                if ((n & 15) == 0) {
                    if (rfl(c.ctl[C_ABORT]) || tmo_set(c, call)) { ok = false; break; }
                    if (expired(t0, DEADLINE)) { tmo_raise(c, call); ok = false; break; }
                }
                __builtin_amdgcn_s_sleep(4);
            }
            if (!ok) break;
            if (avail == todo && tmo_set(c, call)) { ok = false; break; }
        }
        const int bit = __builtin_ctzll(avail), ks = L * 16 + ((bit >> 2) ^ xr);
        avail &= avail - 1; todo &= ~(1ull << bit);
        const u32 full = BIG && !second && ((own >> bit) & 1);
        if (BIG && !second && !full) late |= 1ull << bit;
        u32 tk = 0;
        if (c.lane == 0) tk = __hip_atomic_fetch_add((LAS u32*)&c.ctl[second ? C_TICKET2 : C_TICKET], second ? 1u : 2u, __ATOMIC_RELAXED, __HIP_MEMORY_SCOPE_WORKGROUP);
        tk = rfl(tk) + (second ? TK2 : 0);
#pragma unroll
        for (int h = 0; h < 2; ++h) {
            const u32 tt = tk + h, slot = tt % NSTG, need = 8 * (tt / NSTG);
            if (rfl(c.ctl[C_FREE + slot]) < need) {
                if (pend) { WAIT_VM0(); c.ctl[C_FULL + pslot] = pend; pend = 0; }
                if (!lds_wait(c, C_FREE + slot, need, true, 1)) { ok = false; break; }
            }
            const u32 l = ldsbase + slot * STG;
            if (BIG && second) {
                glds4(c.rs, voff, xa_off(ks, 0, t.rfo), l); glds4(c.rs, voff, xa_off(ks, 1, t.rfo), l + 4096);
#pragma unroll
                for (int g = 0; g < 4; ++g) glds4(c.rs, voff, wb_off(ks, g >> 1, t.cc * 8 + (g & 1) * 4), l + 8192 + g * 4096);
            } else {
#pragma unroll
                for (int g = 0; g < 4; ++g) glds4(c.rs, voff, xa_off(ks, h, t.rf + g * 4), l + g * 4096);
                glds4(c.rs, voff, wb_off(ks, h, t.cc * 8), l + 20480); glds4(c.rs, voff, wb_off(ks, h, t.cc * 8 + 4), l + 24576);
            }
            if (full) {
                glds4(c.rs, voff, xa_off(ks, h, t.rfo), l + 16384);
                asm volatile("s_waitcnt vmcnt(28)" ::: "memory");
            } else asm volatile("s_waitcnt vmcnt(24)" ::: "memory");
            if (pend) c.ctl[C_FULL + pslot] = pend;
            pend = 2 * tt + 2 + full; pslot = slot;
            if (BIG && second) break;
        }
    }
    WAIT_VM0();
    if (!ok) c.ctl[C_ABORT] = 1;
    else if (pend) c.ctl[C_FULL + pslot] = pend;
}

DEV u32 pk_relu(float a, float b) {
    const f32x2 v = __builtin_elementwise_maximum((f32x2){a, b}, (f32x2){0.f, 0.f});
    return __builtin_bit_cast(u32, __builtin_convertvector(v, bf16x2));
}

DEV u32 nz8(u32 cb) { return (cb + 0x07070707u) & 0x08080808u; }
template <int NF>
DEV u32 encode_y(const Ctx& c, const u32 (&P)[NF * 8], u32 dst) {
    constexpr int G = NF * 4;
    constexpr u32 CODEB = NF == 5 ? 2048 : 1536;
    u16x2 mx = {0, 0}, mn[NF];
#pragma unroll
    for (int f = 0; f < NF; ++f) {
        mn[f] = (u16x2){0xffff, 0xffff};
#pragma unroll
        for (int j = 0; j < 8; ++j) {
            const u16x2 v = __builtin_bit_cast(u16x2, P[f * 8 + j]);
            mx = __builtin_elementwise_max(mx, v);
            mn[f] = __builtin_elementwise_min(mn[f], (u16x2)(v - (u16x2){1, 1}));
        }
    }
    const u32 base = max(wave_max(max((u32)mx[0], (u32)mx[1])) >> 8, 7u) - 7, lim = ((base + 1) << 8) - 1;
    const u32 ldsbase = (u32)(size_t)c.lds, stg = L_YST + c.wave * YST;
    u32 nexc = 0, hv = 0;
#pragma unroll
    for (int f = 0; f < NF; ++f)
        for (u64 m = __ballot(min((u32)mn[f][0], (u32)mn[f][1]) < lim); m && nexc <= 31; m &= m - 1) {
            const u32 ln = __builtin_ctzll(m);
            u32 t = 0;
#pragma unroll
            for (int k = 0; k < 8; ++k) t = writelane(__builtin_amdgcn_readlane(P[f * 8 + k], ln), k, t);
            for (u32 mk = (u32)__ballot((t & 0xffff) - 1 < lim) | (u32)__ballot((t >> 16) - 1 < lim) << 8; mk; mk &= mk - 1) {
                const u32 k = __builtin_ctz(mk), v = (__builtin_amdgcn_readlane(t, k & 7) >> (k & 8) * 2) & 0xffff;
                ++nexc;
                hv = writelane((f * 16 + (k & 7) * 2 + (k >> 3)) << 22 | ln << 16 | v, nexc & 63, hv);
            }
        }
    if (FORCE_RAW_Y || nexc > 31) {
#pragma unroll
        for (int i = 0; i < NF * 2; ++i) st16(c, (u32x4){P[4 * i], P[4 * i + 1], P[4 * i + 2], P[4 * i + 3]}, c.lane * 16, dst + i * 1024);
        return YM_RAW | NF * 128;
    }
    hv = writelane(base | nexc << 8, 0, hv);
    if ((u32)c.lane <= nexc) *(LAS u32*)(c.lds + stg + c.lane * 4) = hv;
    u32 cb[G], lb[G], cnt = 0;
    const u16x2 B = {(u16)(base << 8), (u16)(base << 8)};
#pragma unroll
    for (int g = 0; g < G; ++g) {
        const u32 t0 = __builtin_bit_cast(u32, __builtin_elementwise_sub_sat(__builtin_bit_cast(u16x2, P[2 * g]), B));
        const u32 t1 = __builtin_bit_cast(u32, __builtin_elementwise_sub_sat(__builtin_bit_cast(u16x2, P[2 * g + 1]), B));
        cb[g] = perm(t1, t0, 0x07050301u); lb[g] = perm(t1, t0, 0x06040200u);
        cnt += __builtin_popcount(nz8(cb[g]));
    }
    u32 D[8];
#pragma unroll
    for (int h = 0; h < 2; ++h) {
        const u32* q = cb + 8 * h;
        D[3 * h] = q[0] | q[1] << 3 | (q[6] & 0x03030303u) << 6;
        D[3 * h + 1] = q[2] | q[3] << 3 | (q[7] & 0x03030303u) << 6;
        D[3 * h + 2] = q[4] | q[5] << 3 | (q[6] & 0x04040404u) << 4 | (q[7] & 0x04040404u) << 5;
    }
    st16(c, (u32x4){D[0], D[1], D[2], D[3]}, c.lane * 16, dst);
    if (NF == 5) st16(c, (u32x4){D[4], D[5], cb[G - 4] | cb[G - 3] << 3, cb[G - 2] | cb[G - 1] << 3}, c.lane * 16, dst + 1024);
    else __builtin_amdgcn_raw_buffer_store_b64((u32x2){D[4], D[5]}, c.rs, c.lane * 8 + dst + 1024, 0, 17);
    const u32 dw = (cnt >> 2) + 1, incl = wave_scan(dw), total = 1 + nexc + __builtin_amdgcn_readlane(incl, 63);
    u32 pos = ldsbase + stg + 4 * (1 + nexc + incl - dw), adr[G];
    u32x2 sel[G];
#pragma unroll
    for (int g = 0; g < G; ++g) {
        const u32 nz = nz8(cb[g]);
        adr[g] = pos & ~3u;
        sel[g] = *(const LAS u32x2*)(size_t)__builtin_amdgcn_udot4(nz, 0x20100804u, ((pos & 3) << 3) | (ldsbase + L_LUT), false);
        pos += __builtin_popcount(nz);
    }
    __builtin_amdgcn_sched_barrier(0);
    u32 cur = 0;
#pragma unroll
    for (int g = 0; g < G; ++g) {
        const u32 o = perm(lb[g], cur, sel[g][0]);
        *(LAS u32*)(size_t)adr[g] = o;
        cur = perm(lb[g], o, sel[g][1]);
    }
    *(LAS u32*)(size_t)(pos & ~3u) = cur;
    CBAR();
    u32x4 v[5];
#pragma unroll
    for (int i = 0; i < 5; ++i) if (i < 3 || total > i * 256u) v[i] = *(const LAS u32x4*)(c.lds + stg + c.lane * 16 + min(i * 1024u, YST - 1024));
#pragma unroll
    for (int i = 0; i < 5; ++i) if (i < 3 || total > i * 256u) st16(c, v[i], c.lane * 16, dst + CODEB + min(i * 1024u, YST - 1024));
    return (CODEB + total * 4 + 15) >> 4;
}

DEV f32x4 mfma(u32x4 a, u32x4 b, f32x4 acc) { return __builtin_amdgcn_mfma_f32_16x16x32_bf16(__builtin_bit_cast(bf16x8, a), __builtin_bit_cast(bf16x8, b), acc, 0, 0, 0); }
DEV void slot_free(const Ctx& c, u32 slot) { if (c.lane == 0) __hip_atomic_fetch_add((LAS u32*)&c.ctl[C_FREE + slot], 1u, __ATOMIC_RELAXED, __HIP_MEMORY_SCOPE_WORKGROUP); }

template <bool BIG>
DEV void consumer(const Ctx& c, const Tile& t, int wr, int wc) {
    f32x4 acc[4][4], oacc[4];
#pragma unroll
    for (int i = 0; i < 4; ++i) {
        oacc[i] = (f32x4){0.f, 0.f, 0.f, 0.f};
#pragma unroll
        for (int f = 0; f < 4; ++f) acc[f][i] = (f32x4){0.f, 0.f, 0.f, 0.f};
    }
    const LAS char* pa = c.lds + c.lane * 16 + wr * 4096;
    const LAS char* po = c.lds + c.lane * 16 + wr * 1024;
    const LAS char* pb = c.lds + c.lane * 16 + wc * 4096;
    u32 slot = 0, seen = 0, nfull = 0;
    bool ok = true;
#pragma unroll 1
    for (u32 tk = 0; tk < 128; ++tk) {
        u32 sv = rfl(seen);
        if (ok && sv < 2 * tk + 2) { sv = lds_wait(c, C_FULL + slot, 2 * tk + 2, true, 0); ok = sv != 0; }
        CBAR();
        if (tk == 0) STAMPW(c, 1);
        if (tk == 64) STAMPW(c, 4);
        if (tk == 96) STAMPW(c, 5);
        if (tk == 112) STAMPW(c, 6);
        if (tk == 120) STAMPW(c, 7);
        u32x4 a[4], b[4], ao;
#pragma unroll
        for (int i = 0; i < 4; ++i) b[i] = *(const LAS u32x4*)(pb + slot * STG + 20480 + i * 1024);
#pragma unroll
        for (int f = 0; f < 4; ++f) a[f] = *(const LAS u32x4*)(pa + slot * STG + f * 1024);
        if (BIG) ao = *(const LAS u32x4*)(po + slot * STG + 16384);
        WAIT_LGKM0();
        slot_free(c, slot);
        slot = slot == NSTG - 1 ? 0 : slot + 1;
        seen = c.ctl[C_FULL + slot];
        __builtin_amdgcn_sched_barrier(0);
#pragma unroll
        for (int f = 0; f < 4; ++f)
#pragma unroll
            for (int i = 0; i < 4; ++i) acc[f][i] = mfma(a[f], b[i], acc[f][i]);
        if (BIG && (sv & 1)) {
            ++nfull;
#pragma unroll
            for (int i = 0; i < 4; ++i) oacc[i] = mfma(ao, b[i], oacc[i]);
        }
    }
    STAMPW(c, 2);
    ok = ok && !rfl(c.ctl[C_ABORT]);
    {
        const u32 call = get_call(c), msg = (t.cc * 2 + t.h) * 8 + c.wave;
        u32 P[32];
#pragma unroll
        for (int g = 0; g < 16; ++g) {
            P[2 * g] = pk_relu(acc[g >> 2][0][g & 3], acc[g >> 2][1][g & 3]);
            P[2 * g + 1] = pk_relu(acc[g >> 2][2][g & 3], acc[g >> 2][3][g & 3]);
        }
        ok = ok && lds_wait(c, C_FREE + 3, 200, true, 0) && lds_wait(c, C_FREE + 4, 200, true, 0);
        CBAR();
        STAMPW(c, 8);
        u32 meta = YM_BAD;
        if (ok) meta = encode_y<4>(c, P, WS_YP + (t.owner * NMSG + msg) * YSLOT);
        STAMPW(c, 9);
        WAIT_VM0();
        if (c.lane == t.owner) st_sys64(c.peer_lane + WS_FY + (c.rank * NMSG + msg) * 8, (u64)call << 32 | meta);
    }
    STAMPW(c, 3);
    if (BIG) {
        slot_free(c, 3); slot_free(c, 4);
        slot = 0;
#pragma unroll 1
        for (u32 tk = TK2; ok && tk < TK2 + 64 - (nfull >> 1); ++tk) {
            ok = lds_wait(c, C_FULL + slot, 2 * tk + 2, true, 0) != 0;
            CBAR();
            u32x4 a[2], b[8];
#pragma unroll
            for (int i = 0; i < 8; ++i) b[i] = *(const LAS u32x4*)(pb + slot * STG + 8192 + (i >> 2) * 8192 + (i & 3) * 1024);
#pragma unroll
            for (int f = 0; f < 2; ++f) a[f] = *(const LAS u32x4*)(po + slot * STG + f * 4096);
            WAIT_LGKM0();
            slot_free(c, slot);
            slot = slot == NSTG - 1 ? 0 : slot + 1;
#pragma unroll
            for (int i = 0; i < 8; ++i) oacc[i & 3] = mfma(a[i >> 2], b[i], oacc[i & 3]);
        }
        ok = ok && !rfl(c.ctl[C_ABORT]);
        const u32 o = out_off(c, ((t.rfo + wr) & 31) * 16, c.rank * 2048 + t.cc * 128 + wc * 64);
#pragma unroll
        for (int r = 0; r < 4; ++r)
            st_out(c, o + r * (NTOT * 2), ok ? (u32x2){pk_relu(oacc[0][r], oacc[1][r]), pk_relu(oacc[2][r], oacc[3][r])} : (u32x2){0x7fc07fc0u, 0x7fc07fc0u});
        STAMPW(c, 10);
    }
}

DEV void gemm_wg(const Ctx& c) {
    const int g = c.wg - NCOMM, xcd = g & 7, idx = g >> 3, grp = xcd >> 2, k = idx % 7;
    const bool big = k < 4;
    Tile t;
    t.cc = (xcd & 3) * 4 + idx / 7;
    t.h = grp * 4 + k >= 7 || k >= 4;
    t.owner = (c.rank + (grp ? (k < 3 ? 5 + k : 1 + k) : (k < 4 ? 1 + k : k - 3))) & 7;
    t.rf = t.owner * 32 + t.h * 16; t.rfo = c.rank * 32 + (grp * 4 + (k & 3)) * 4;
    if (c.wave >= 8) {
        const int L = c.wave - 8;
        if (L * NGEMM + g >= 160) for (int i = 0; i < PSLEEP + L; ++i) __builtin_amdgcn_s_sleep(24);
#pragma unroll 1
        for (int job = L * NGEMM + g; job < 1280; job += 4 * NGEMM) { pack_job(c, job); if (job < 4 * NGEMM) STAMPW(c, 3); }
        STAMPW(c, 1);
        const u32 call = get_call(c), xr = XROT ? (t.cc & 1) | (t.cc & 2) << 2 | (k & 3) << 1 : 0;
        if (big) loader<true>(c, t, L, xr, call); else loader<false>(c, t, L, xr, call);
        STAMPW(c, 2);
    } else {
        if (big) consumer<true>(c, t, c.wave >> 1, c.wave & 1); else consumer<false>(c, t, c.wave >> 1, c.wave & 1);
    }
}

#define WINV 2
#define WSUM 8
#define WCLV 3
constexpr u32 WIN = WINV, WCL = WINV + WCLV;

DEV void watcher(const Ctx& c, int p, int sub) {
    static_assert(NQY == 64, "one message per lane");
    const u32 call = get_call(c), id = sub + 4 * c.lane;
    const char* fp = c.me + WS_FY + (p * NMSG + id) * 8;
    bool bad = !lds_wait(c, C_YGO, 1, false, 1);
    STAMPW(c, 1);
    u64 pending = ~0ull, t0 = 0;
    u32 qn = 0;
    for (u32 it = 1; pending; ++it) {
        const u64 v = bad ? 0 : ld_sys64(fp);
        const bool mine = ((pending >> c.lane) & 1) && (bad || (u32)(v >> 32) >= call);
        const u64 m = __ballot(mine);
        if (m) {
            const bool poison = bad || tmo_set(c, call);
            if (mine) c.ctl[C_Q + qn + lane_rank(m)] = 0x80000000u | id << 20 | (poison ? YM_BAD : (u32)v & (YM_BAD | YM_RAW | 0x3ff));
            qn += __popcll(m); pending &= ~m; t0 = 0;
        } else {
            if ((it & 15) == 0) {
                if (tmo_set(c, call)) bad = true;
                else if (expired(t0, DEADLINE)) { tmo_raise(c, call); bad = true; }
            }
            __builtin_amdgcn_s_sleep(2);
        }
    }
    STAMPW(c, 2);
}

DEV void pull_inputs(const Ctx& c, rsrc_t pr, int d, int p, u32 call) {
    GAS u64* ctr = (GAS u64*)(c.me + WS_CLM + d * 128);
    if (c.lane == 0) __hip_atomic_fetch_max(ctr, (u64)call << 32, __ATOMIC_RELAXED, __HIP_MEMORY_SCOPE_AGENT);
    WAIT_VM0();
    const u32 voff = c.lane * 16, win = WSUM / 4 + (c.wg / 7 < WSUM % 4);
    bool bad = false;
#pragma unroll 1
    for (;;) {
        const u32 t = lds_inc(c, C_TK);
        if (t >= WCL) lds_wait(c, C_LAND, t - WCL + 1, false, 1);
        u32 n = 0;
        if (c.lane == 0) n = (u32)__hip_atomic_fetch_add(ctr, 1ull, __ATOMIC_RELAXED, __HIP_MEMORY_SCOPE_AGENT);
        n = rfl(n);
        if (n + 64 >= NCHIN) c.ctl[C_YGO] = 1;
        if (n >= NCHIN) break;
        const u32 code = rfl(ORDER.v[d - 1][n]), s = (code >> 8) & 7, ix = code & 31;
        const bool isw = code & 0x8000;
        const u32 rc = isw ? ix : ((((ix >> 2) + c.rank) & 7) << 2 | (ix & 3)), id = isw ? 256 + s * 16 + rc : s * 32 + rc;
        const char* fp = c.me + WS_FIN + (((call & 1) * NR + p) * NCHIN + id) * 4;
        u32 v = 0;
        u64 t0 = 0;
        for (u32 it = 1; !bad; ++it) {
            v = rfl(ld_sys(fp));
            if (v >= 2 * call) break;
            if ((it & 15) == 0) {
                if (tmo_set(c, call)) bad = true;
                else if (expired(t0, DEADLINE)) { tmo_raise(c, call); bad = true; }
            }
            __builtin_amdgcn_s_sleep(2);
        }
        if (bad) { lds_inc(c, C_LAND); continue; }
        const u32 u = lds_inc(c, C_IS);
        if (u >= win) lds_wait(c, C_LAND, u - win + 1, false, 0);
        ST2(c, 1024 + u, 13);
        const int ks = p * 8 + s;
        const u32 src = WS_PK + (call & 1) * PK_PAR + (isw ? id + c.rank * 128 : id) * CHB;
        const u32 d0 = isw ? wb_off(ks, 0, rc * 8) : xa_off(ks, 0, rc * 8), d1 = isw ? wb_off(ks, 1, rc * 8) : xa_off(ks, 1, rc * 8);
        u32x4 R[16];
        if (v & 1) {
#pragma unroll
            for (int f = 0; f < 16; ++f) R[f] = ld16(pr, voff + (f & 3) * 1024, src + (f >> 2) * 4096);
            WAIT_VM0();
            lds_inc(c, C_LAND);
            store_raw(c, R, d0, d1);
        } else {
            u32x4 L[8], code[4];
            const u32 hv = __builtin_amdgcn_raw_buffer_load_b32(pr, (c.lane & 31) * 4, src + 12288, 17);
#pragma unroll
            for (int i = 0; i < 4; ++i) code[i] = ld16(pr, voff + i * 1024, src + 8192);
#pragma unroll
            for (int i = 0; i < 8; ++i) L[i] = ld16(pr, voff + (i & 3) * 1024, src + (i >> 2) * 4096);
            WAIT_VM0();
            lds_inc(c, C_LAND);
            ST2(c, 1536 + u, 0); ST2(c, u, 0);
            const u32 h0 = rfl(hv), C = (0x80u + (h0 & 0x7f)) * 0x01010101u, nexc = min((h0 >> 8) & 0xff, 31u);
#pragma unroll
            for (int f = 0; f < 16; ++f) {
                const u32 n = code[f >> 2][f & 3];
                const u32 ha = C - (n & 0x87878787u), hb = C - (__builtin_amdgcn_alignbit(n, n, 28) & 0x87878787u);
                const u32 la = L[f >> 1][(f & 1) * 2], lb = L[f >> 1][(f & 1) * 2 + 1];
                R[f] = join8(ha, la, hb, lb);
            }
            store_raw(c, R, d0, d1);
#pragma unroll 1
            for (u32 i = 0; i < nexc; ++i) {
                const u32 x = __builtin_amdgcn_readlane(hv, i + 1), pos = x >> 16, f = (pos >> 9) & 15, ln = (pos >> 3) & 63;
                if ((u32)c.lane == ln) __builtin_amdgcn_raw_buffer_store_b16((u16)x, c.rs, ln * 16 + (f & 7) * 1024 + (pos & 7) * 2, f < 8 ? d0 : d1, 17);
            }
        }
        STAMPW(c, 3);
        WAIT_VM0();
        if (c.lane == 0) st_sys(c.me + (isw ? WS_WR : WS_XR) + rc * 256 + ks * 4, call);
        ST2(c, 512 + u, 0);
    }
    lds_inc(c, C_LAND);
}

template <int NF>
DEV void decode_y(const Ctx& c, u32 e, u32 lo, rsrc_t pr, u32 src, u32 o, u32 tk) {
    constexpr int G = NF * 4;
    constexpr u32 CODEB = NF == 5 ? 2048 : 1536;
    const u32 ldsbase = (u32)(size_t)c.lds;
    const LAS char* l = c.lds + lo;
    u32x2 P[G];
    u32 hv = 0, nexc = 0;
    if (e & YM_BAD) {
#pragma unroll
        for (int g = 0; g < G; ++g) P[g] = (u32x2){0x7fc07fc0u, 0x7fc07fc0u};
    } else if (e & YM_RAW) {
#pragma unroll
        for (int i = 0; i < G / 2; ++i) {
            const u32x4 v = ld16(pr, c.lane * 16 + (i & 3) * 1024, src + (i >> 2) * 4096);
            P[2 * i] = (u32x2){v[0], v[1]}; P[2 * i + 1] = (u32x2){v[2], v[3]};
        }
        __builtin_amdgcn_s_waitcnt(0x0f70);
    } else {
        const u32x4 da = *(const LAS u32x4*)(l + c.lane * 16);
        u32x4 db;
        if (NF == 5) db = *(const LAS u32x4*)(l + 1024 + c.lane * 16);
        else { const u32x2 t = *(const LAS u32x2*)(l + 1024 + c.lane * 8); db = (u32x4){t[0], t[1], 0, 0}; }
        hv = *(const LAS u32*)(l + CODEB + (c.lane & 31) * 4);
        const u32 D[8] = {da[0], da[1], da[2], da[3], db[0], db[1], db[2], db[3]};
        u32 cb[G], cnt = 0;
#pragma unroll
        for (int h = 0; h < 2; ++h) {
            u32* q = cb + 8 * h;
#pragma unroll
            for (int k = 0; k < 3; ++k) { q[2 * k] = D[3 * h + k] & 0x07070707u; q[2 * k + 1] = (D[3 * h + k] >> 3) & 0x07070707u; }
            q[6] = bfi(0x03030303u, D[3 * h] >> 6, D[3 * h + 2] >> 4) & 0x07070707u;
            q[7] = bfi(0x03030303u, D[3 * h + 1] >> 6, D[3 * h + 2] >> 5) & 0x07070707u;
        }
        if (NF == 5) { cb[G - 4] = D[6] & 0x07070707u; cb[G - 3] = (D[6] >> 3) & 0x07070707u; cb[G - 2] = D[7] & 0x07070707u; cb[G - 1] = (D[7] >> 3) & 0x07070707u; }
#pragma unroll
        for (int g = 0; g < G; ++g) cnt += __builtin_popcount(nz8(cb[g]));
        const u32 h0 = rfl(hv), B = (h0 & 0xff) * 0x01010101u, dw = (cnt >> 2) + 1;
        nexc = min((h0 >> 8) & 0xff, 31u);
        u32 pos = ldsbase + lo + CODEB + 4 * (1 + nexc + wave_scan(dw) - dw), ps[G];
        u32x2 sel[G];
#pragma unroll
        for (int g = 0; g < G; ++g) {
            const u32 nz = nz8(cb[g]);
            const LAS u32* q = (const LAS u32*)(size_t)(pos & ~3u);
            sel[g] = *(const LAS u32x2*)(size_t)__builtin_amdgcn_udot4(nz, 0x08040201u, ldsbase + L_LUT, false);
            P[g] = (u32x2){q[0], q[1]};
            ps[g] = pos;
            pos += __builtin_popcount(nz);
        }
        __builtin_amdgcn_sched_barrier(0);
#pragma unroll
        for (int g = 0; g < G; ++g) {
            const u32 by = __builtin_amdgcn_alignbyte(P[g][1], P[g][0], ps[g]), lb = perm(by, by, sel[g][0]), hb = (cb[g] + B) & sel[g][1];
            P[g] = (u32x2){perm(hb, lb, 0x05010400u), perm(hb, lb, 0x07030602u)};
        }
    }
#pragma unroll
    for (int g = 0; g < G; ++g) st_out(c, o + ((g >> 2) * 16 + (g & 3)) * (NTOT * 2), P[g]);
    ST2(c, 512 + 384 + tk, nexc);
#pragma unroll 1
    for (u32 i = 0; i < nexc; ++i) {
        const u32 x = __builtin_amdgcn_readlane(hv, i + 1), j = min(x >> 22, (u32)G * 4 - 1);
        if ((u32)c.lane == ((x >> 16) & 63)) __builtin_amdgcn_raw_buffer_store_b16((u16)x, c.ro, o + ((j >> 4) * 16 + ((j >> 2) & 3)) * (NTOT * 2) + (j & 3) * 2, 0, OAUX);
    }
}
DEV void fetch_y(const Ctx& c, char* peer, u32 e, u32 lo) {
    if (e & (YM_BAD | YM_RAW)) return;
    const u32 src = WS_YP + (c.rank * NMSG + ((e >> 20) & 0xff)) * YSLOT, bytes = min((e & 0x3ff) * 16, LSLOT);
    const rsrc_t rs = __builtin_amdgcn_make_buffer_rsrc(peer, 0, src + bytes, 0x00020000);
    glds4(rs, c.lane * 16, src, (u32)(size_t)c.lds + lo);
    if (bytes > 4096) glds2(rs, c.lane * 16, src + 4096, (u32)(size_t)c.lds + lo + 4096);
}

DEV void worker(const Ctx& c, char* peer, int d, int p, int w) {
    const u32 call = get_call(c);
    const rsrc_t pr = __builtin_amdgcn_make_buffer_rsrc(peer, 0, WS_END, 0x00020000);
    pull_inputs(c, pr, d, p, call);
    STAMPW(c, 1);
    u32 lo = w * 2 * LSLOT, lo2 = lo + LSLOT, e = 0;
#pragma unroll 1
    for (u32 tk = lds_inc(c, C_YT); tk < NQY;) {
        if (e) asm volatile("s_waitcnt vmcnt(16)" ::: "memory");
        else {
            e = lds_wait(c, C_Q + tk, 1, false, 1);
            if (!e) { tmo_raise(c, call); break; }
            ST2(c, 1024 + 448 + tk, 4);
            fetch_y(c, peer, e, lo);
            WAIT_VM0();
        }
        ST2(c, 1536 + 448 + tk, 0); ST2(c, 448 + tk, 0);
        if (tk == 0) STAMPW(c, 4);
        STAMPW(c, 5);
        const u32 nt = lds_inc(c, C_YT), ne = nt < NQY ? rfl(c.ctl[C_Q + nt]) : 0;
        if (ne) { ST2(c, 1024 + 448 + nt, 5); fetch_y(c, peer, ne, lo2); }
        const u32 id = (e >> 20) & 0xff, wc = id & 1, rg = (id >> 1) & 7, cc = id >> 4;
        decode_y<4>(c, e, lo, pr, WS_YP + (c.rank * NMSG + id) * YSLOT, out_off(c, rg * 64, p * 2048 + cc * 128 + wc * 64), tk);
        ST2(c, 512 + 448 + tk, 0);
        tk = nt; e = ne;
        const u32 sw = lo; lo = lo2; lo2 = sw;
    }
    STAMPW(c, 2);
}

__global__ void __launch_bounds__(NTHREADS, 3) fused_kernel(Args a) {
    extern __shared__ __attribute__((aligned(128))) char smem[];
    Ctx c;
    c.x = a.x; c.w = a.w; c.out = a.out; c.rank = a.rank & 7; c.me = a.ws[c.rank];
    c.wg = blockIdx.x; c.wave = rfl(threadIdx.x >> 6); c.lane = threadIdx.x & 63;
    c.peer_lane = a.ws[c.lane & 7];
    c.rs = __builtin_amdgcn_make_buffer_rsrc(c.me, 0, WS_END, 0x00020000);
    c.ro = __builtin_amdgcn_make_buffer_rsrc(c.out, 0, 512 * NTOT * 2, 0x00020000);
    c.lds = (LAS char*)smem; c.ctl = (volatile LAS u32*)(c.lds + L_CTL);
    STAMPW(c, 0);
    u32 ticket = 0;
    if (threadIdx.x == 0) ticket = __hip_atomic_fetch_add((GAS u32*)(c.me + WS_CNT), 1u, __ATOMIC_RELAXED, __HIP_MEMORY_SCOPE_AGENT);
    for (u32 i = threadIdx.x; i < C_WORDS; i += NTHREADS) c.ctl[i] = 0;
    if (c.wave == 11) {
        u32 s0, s1;
        if (c.wg >= NCOMM) {
            const u32 mask = c.lane >> 2;
            u32 n = c.lane & 3;
            u64 sel = 0x0c0c0c0c03020100ull;
            for (int b = 0; b < 4; ++b)
                if ((mask >> b) & 1) { sel = (sel & ~(0xffull << (8 * n))) | (u64)(4 + b) << (8 * n); ++n; }
            s0 = (u32)sel; s1 = n >= 4 ? (u32)(sel >> 32) : 0x03020100u;
        } else {
            u32 n = 0;
            s0 = 0x0c0c0c0cu; s1 = 0;
            for (int b = 0; b < 4; ++b)
                if ((c.lane >> b) & 1) { s0 = (s0 & ~(0xffu << (8 * b))) | n << (8 * b); s1 |= 0xffu << (8 * b); ++n; }
        }
        c.ctl[C_LUT + 2 * c.lane] = s0; c.ctl[C_LUT + 2 * c.lane + 1] = s1;
    }
    WAIT_LGKM0();
    __builtin_amdgcn_s_barrier();
    CBAR();
    if (threadIdx.x == 0) c.ctl[C_CALL] = ticket / NWG + 1;
    if (c.wg >= NCOMM) gemm_wg(c);
    else if (c.wg < NPULL) {
        const int d = 1 + c.wg % 7, sub = c.wg / 7, p = (c.rank + d) & 7;
        if (c.wave == 0) watcher(c, p, sub);
        else worker(c, a.ws[p], d, p, c.wave - 1);
    }
}

}

extern "C" void kernel_launch(int rank, int nranks, ncclComm_t comm, hipStream_t stream,
                              void* const* d_in, const int* in_sizes, int n_in,
                              void* d_out, int out_size, const int* in_dtypes, int out_dtype,
                              void* const* d_out_all, void* const* d_ws_all, size_t ws_bytes) {
    if (nranks != NR || (unsigned)rank >= (unsigned)NR || n_in != 2 || ws_bytes < WS_END || in_sizes[0] != 4096 * KSH ||
        in_sizes[1] != KSH * NTOT || out_size != 512 * NTOT) {
        fprintf(stderr, "unexpected problem shape\n");
        return;
    }
    HIP_CHECK(hipFuncSetAttribute((const void*)fused_kernel, hipFuncAttributeMaxDynamicSharedMemorySize, LDS_BYTES));
    Args a;
    a.x = (const u16*)d_in[0]; a.w = (const u16*)d_in[1]; a.out = (u16*)d_out;
    for (int p = 0; p < NR; ++p) a.ws[p] = (char*)d_ws_all[p];
    a.rank = rank; a.pad = 0;
    hipLaunchKernelGGL(fused_kernel, dim3(NWG), dim3(NTHREADS), LDS_BYTES, stream, a);
    HIP_CHECK(hipGetLastError());
}
```

```cpp
#include <hip/hip_runtime.h>
#include <rccl/rccl.h>
#include <cstdio>
#include <cstdint>

#define HIP_CHECK(expr) do { hipError_t _e = (expr); if (_e != hipSuccess) { \
    fprintf(stderr, "HIP error %s at %s:%d\n", hipGetErrorString(_e), __FILE__, __LINE__); \
    return; } } while (0)

extern "C" __device__ unsigned writelane(unsigned value, unsigned lane, unsigned old) __asm("llvm.amdgcn.writelane.i32");

namespace {

typedef unsigned u32;
typedef unsigned long long u64;
typedef unsigned short u16;
typedef u32 u32x4 __attribute__((ext_vector_type(4)));
typedef u32 u32x2 __attribute__((ext_vector_type(2)));
typedef u32 u32x8 __attribute__((ext_vector_type(8)));
typedef u32 u32x32 __attribute__((ext_vector_type(32)));
typedef u16 u16x2 __attribute__((ext_vector_type(2)));
typedef float f32x4 __attribute__((ext_vector_type(4)));
typedef float f32x2 __attribute__((ext_vector_type(2)));
typedef __bf16 bf16x2 __attribute__((ext_vector_type(2)));
typedef __bf16 bf16x8 __attribute__((ext_vector_type(8)));
typedef __amdgpu_buffer_rsrc_t rsrc_t;
#define LAS __attribute__((address_space(3)))
#define GAS __attribute__((address_space(1)))
#define DEV __device__ __forceinline__

#define PSLEEP 3
#define OAUX 17
#define TOPO 1
#define DIAG 0
#define XROT 1
#define FORCE_RAW_Y 0
#define FORCE_RAW_IN 0

constexpr int NR = 8;
constexpr int NTOT = 16384, KSH = 512;
constexpr int NWG = 256, NCOMM = 32, NPULL = 28, NGEMM = NWG - NCOMM, NTHREADS = 768;
constexpr int NCHIN = 384;
constexpr int NMSG = 256;
constexpr u32 NQY = NMSG / 4;
constexpr int NWORK = 11;

constexpr u32 WS_CNT = 0, WS_TMO = 128, WS_CLM = 1024;
constexpr u32 WS_SEAT = 2048;
constexpr u32 WS_FIN = 4096;
constexpr u32 WS_FY = 32768;
constexpr u32 WS_XR = 49152;
constexpr u32 WS_WR = 57344;
constexpr u32 WS_DBG = 65536;
constexpr u32 CHB = 16384;
constexpr u32 WS_PK = 1u << 20;
constexpr u32 PK_PAR = 1280u * CHB;
constexpr u32 WS_XA = WS_PK + 2 * PK_PAR;
constexpr u32 WS_WB = WS_XA + (32u << 20);
constexpr u32 YSLOT = 8192;
constexpr u32 WS_YP = WS_WB + (16u << 20);
constexpr u32 WS_END = WS_YP + 8u * NMSG * YSLOT;
constexpr u32 PKD_BYTES = 12288 + 128;

constexpr u32 STG = 28672, NSTG = 5;
constexpr u32 LSLOT = 6144;
constexpr u32 L_CTL = 143360;
static_assert(STG * NSTG <= L_CTL && NWORK * 2 * LSLOT <= L_CTL, "ring");
constexpr u32 C_CALL = 0, C_ABORT = 1, C_TICKET = 2, C_TICKET2 = 3, C_FULL = 8, C_FREE = 16,
              C_TK = 2, C_IS = 3, C_LAND = 4, C_YGO = 5, C_YT = 6, C_SEAT = 7, C_Q = 64, C_WORDS = 128,
              C_LUT = C_WORDS;
constexpr u32 LDS_BYTES = L_CTL + (C_LUT + 128) * 4;
#define ST2(c, i, v) do {} while (0)
constexpr u32 L_LUT = L_CTL + C_LUT * 4, L_YST = 3 * STG, YST = 4608;
static_assert(L_LUT % 512 == 0 && L_YST + 8 * YST <= 5 * STG && 4 * (32 + 64 * 17) <= YST && 1536 + 4 * (32 + 64 * 17) <= LSLOT, "y staging");
static_assert(LDS_BYTES <= 160 * 1024, "lds");

constexpr u64 DEADLINE = 40000000ull;
constexpr u64 DEADLINE_LDS = 100000000ull;

constexpr u32 YM_RAW = 1u << 16, YM_BAD = 1u << 17;

struct Order { u16 v[7][NCHIN]; };
#define D1V 6
#define D2V 2
#define D3V 0
constexpr Order make_order() {
    Order o{};
    for (int d = 1; d <= 7; ++d) {
        int n = 0;
        for (int pass = 0; pass < 4; ++pass)
            for (int s = 0; s < 8; ++s) {
                if (pass == (s >= 8 - D1V ? 3 : 0) && s) for (int q = 0; q < 4; ++q) o.v[d - 1][n++] = (u16)(s << 8 | q);
                for (int c = 0; c < 16; ++c) if (pass == ((c & 2) && s >= 8 - D3V ? 1 : 0)) o.v[d - 1][n++] = (u16)(0x8000 | s << 8 | c);
                if (pass == (s >= 8 - D1V ? 3 : 0) && !s) for (int q = 0; q < 4; ++q) o.v[d - 1][n++] = (u16)(s << 8 | q);
                for (int i = 0; i < 28; ++i) {
                    const int q = (i / 14) * 2 + (i & 1), own = 1 + ((i % 14) / 2 + d) % 7;
                    if (pass == (q >= 2 && own != 4 && s >= 8 - D2V ? 2 : 0)) o.v[d - 1][n++] = (u16)(s << 8 | own << 2 | q);
                }
            }
    }
    return o;
}
__constant__ const Order ORDER = make_order();

struct Args { const u16* x; const u16* w; u16* out; char* ws[NR]; int rank; int pad; };

DEV u64 rt() { return __builtin_amdgcn_s_memrealtime(); }
DEV u32 rfl(u32 v) { return __builtin_amdgcn_readfirstlane(v); }
DEV u32 perm(u32 hi, u32 lo, u32 sel) { return __builtin_amdgcn_perm(hi, lo, sel); }
DEV u32 bfi(u32 m, u32 a, u32 b) { return (m & a) | (~m & b); }
#define CBAR() asm volatile("" ::: "memory")
#define WAIT_VM0() asm volatile("s_waitcnt vmcnt(0)" ::: "memory")
#define WAIT_LGKM0() do { CBAR(); __builtin_amdgcn_s_waitcnt(0xc07f); CBAR(); } while (0)

#define STAMPW(c, i) do {} while (0)

struct Ctx {
    const u16* x; const u16* w; u16* out; char* me; char* peer_lane;
    rsrc_t rs, ro;
    LAS char* lds; volatile LAS u32* ctl;
    int rank, wg, wave, lane;
};

DEV u32 ld_sys(const void* p) { return __hip_atomic_load((const GAS u32*)p, __ATOMIC_RELAXED, __HIP_MEMORY_SCOPE_SYSTEM); }
DEV u64 ld_sys64(const void* p) { return __hip_atomic_load((const GAS u64*)p, __ATOMIC_RELAXED, __HIP_MEMORY_SCOPE_SYSTEM); }
DEV void st_sys(void* p, u32 v) { __hip_atomic_store((GAS u32*)p, v, __ATOMIC_RELAXED, __HIP_MEMORY_SCOPE_SYSTEM); }
DEV void st_sys64(void* p, u64 v) { __hip_atomic_store((GAS u64*)p, v, __ATOMIC_RELAXED, __HIP_MEMORY_SCOPE_SYSTEM); }
DEV void st16(const Ctx& c, u32x4 v, u32 voff, u32 soff) { __builtin_amdgcn_raw_buffer_store_b128(v, c.rs, voff + soff, 0, 17); }
DEV void st_out(const Ctx& c, u32 off, u32x2 v) { __builtin_amdgcn_raw_buffer_store_b64(v, c.ro, off, 0, OAUX); }
DEV u32 out_off(const Ctx& c, u32 row0, u32 col0) { return (row0 + (c.lane >> 4) * 4) * (NTOT * 2) + (col0 + (c.lane & 15) * 4) * 2; }
DEV bool tmo_set(const Ctx& c, u32 call) { return rfl(__hip_atomic_load((const GAS u32*)(c.me + WS_TMO), __ATOMIC_RELAXED, __HIP_MEMORY_SCOPE_AGENT)) == call; }
DEV void tmo_raise(const Ctx& c, u32 call) { if (c.lane == 0) __hip_atomic_store((GAS u32*)(c.me + WS_TMO), call, __ATOMIC_RELAXED, __HIP_MEMORY_SCOPE_AGENT); }

DEV bool expired(u64& t0, u64 limit) { const u64 t = rt(); if (!t0) t0 = t; return t - t0 > limit; }

DEV u32 lds_wait(const Ctx& c, u32 word, u32 want, bool look_abort, int nap) {
    u64 t0 = 0;
    for (u32 n = 1;; ++n) {
        const u32 v = rfl(c.ctl[word]);
        if (v >= want) return v;
        if (nap) __builtin_amdgcn_s_sleep(1);
        if ((n & 63) == 0) {
            if (look_abort && rfl(c.ctl[C_ABORT])) return 0;
            if (expired(t0, DEADLINE_LDS)) { c.ctl[C_ABORT] = 1; return 0; }
        }
    }
}
DEV u32 get_call(const Ctx& c) { return lds_wait(c, C_CALL, 1, false, 1); }

DEV void glds4(rsrc_t rs, u32 voff, u32 soff, u32 ldsaddr) {
    u32 keep;
    asm volatile("s_mov_b32 %0, m0\n\ts_mov_b32 m0, %4\n\ts_nop 2\n\t"
                 "buffer_load_dwordx4 %1, %2, %3 offen sc0 sc1 lds\n\t"
                 "buffer_load_dwordx4 %1, %2, %3 offen offset:1024 sc0 sc1 lds\n\t"
                 "buffer_load_dwordx4 %1, %2, %3 offen offset:2048 sc0 sc1 lds\n\t"
                 "buffer_load_dwordx4 %1, %2, %3 offen offset:3072 sc0 sc1 lds\n\t"
                 "s_mov_b32 m0, %0" : "=&s"(keep) : "v"(voff), "s"(rs), "s"(soff), "s"(ldsaddr) : "memory");
}
DEV void glds2(rsrc_t rs, u32 voff, u32 soff, u32 ldsaddr) {
    u32 keep;
    asm volatile("s_mov_b32 %0, m0\n\ts_mov_b32 m0, %4\n\ts_nop 2\n\t"
                 "buffer_load_dwordx4 %1, %2, %3 offen sc0 sc1 lds\n\t"
                 "buffer_load_dwordx4 %1, %2, %3 offen offset:1024 sc0 sc1 lds\n\t"
                 "s_mov_b32 m0, %0" : "=&s"(keep) : "v"(voff), "s"(rs), "s"(soff), "s"(ldsaddr) : "memory");
}
DEV u32x4 ld16(rsrc_t rs, u32 voff, u32 soff) { return __builtin_amdgcn_raw_buffer_load_b128(rs, voff, soff, 17); }
DEV u32 lds_inc(const Ctx& c, u32 word) {
    u32 v = 0;
    if (c.lane == 0) v = __hip_atomic_fetch_add((LAS u32*)&c.ctl[word], 1u, __ATOMIC_RELAXED, __HIP_MEMORY_SCOPE_WORKGROUP);
    return rfl(v);
}

DEV u32 wave_max(u32 v) {
    v = max(v, (u32)__builtin_amdgcn_update_dpp(0, (int)v, 0x111, 0xf, 0xf, true));
    v = max(v, (u32)__builtin_amdgcn_update_dpp(0, (int)v, 0x112, 0xf, 0xf, true));
    v = max(v, (u32)__builtin_amdgcn_update_dpp(0, (int)v, 0x114, 0xf, 0xf, true));
    v = max(v, (u32)__builtin_amdgcn_update_dpp(0, (int)v, 0x118, 0xf, 0xf, true));
    v = max(v, (u32)__builtin_amdgcn_update_dpp(0, (int)v, 0x142, 0xa, 0xf, false));
    v = max(v, (u32)__builtin_amdgcn_update_dpp(0, (int)v, 0x143, 0xc, 0xf, false));
    return __builtin_amdgcn_readlane(v, 63);
}

DEV u32 wave_scan(u32 v) {
    v += (u32)__builtin_amdgcn_update_dpp(0, (int)v, 0x111, 0xf, 0xf, true);
    v += (u32)__builtin_amdgcn_update_dpp(0, (int)v, 0x112, 0xf, 0xf, true);
    v += (u32)__builtin_amdgcn_update_dpp(0, (int)v, 0x114, 0xf, 0xf, true);
    v += (u32)__builtin_amdgcn_update_dpp(0, (int)v, 0x118, 0xf, 0xf, true);
    v += (u32)__builtin_amdgcn_update_dpp(0, (int)v, 0x142, 0xa, 0xf, false);
    v += (u32)__builtin_amdgcn_update_dpp(0, (int)v, 0x143, 0xc, 0xf, false);
    return v;
}
DEV u32 lane_rank(u64 m) { return __builtin_amdgcn_mbcnt_hi((u32)(m >> 32), __builtin_amdgcn_mbcnt_lo((u32)m, 0)); }

DEV void store_raw(const Ctx& c, const u32x4 (&R)[16], u32 off0, u32 off1) {
#pragma unroll
    for (int f = 0; f < 16; ++f) st16(c, R[f], c.lane * 16, (f < 8 ? off0 : off1) + (f & 7) * 1024);
}
DEV u32x4 join8(u32 ha, u32 la, u32 hb, u32 lb) {
    return (u32x4){perm(ha, la, 0x05010400u), perm(ha, la, 0x07030602u), perm(hb, lb, 0x05010400u), perm(hb, lb, 0x07030602u)};
}
DEV u32 pack_store(const Ctx& c, const u32x4 (&R)[16], u32 dst) {
    u32 H[32], L[32];
    u16x2 m0 = {0, 0}, m1 = {0, 0};
#pragma unroll
    for (int f = 0; f < 16; ++f) {
        L[2 * f] = perm(R[f][1], R[f][0], 0x06040200u); L[2 * f + 1] = perm(R[f][3], R[f][2], 0x06040200u);
        H[2 * f] = perm(R[f][1], R[f][0], 0x07050301u); H[2 * f + 1] = perm(R[f][3], R[f][2], 0x07050301u);
    }
#pragma unroll
    for (int i = 0; i < 8; ++i) st16(c, (u32x4){L[4 * i], L[4 * i + 1], L[4 * i + 2], L[4 * i + 3]}, c.lane * 16, dst + i * 1024);
#pragma unroll
    for (int i = 0; i < 32; ++i) {
        const u32 t = H[i] & 0x7f7f7f7fu;
        m0 = __builtin_elementwise_max(m0, __builtin_bit_cast(u16x2, t));
        m1 = __builtin_elementwise_max(m1, __builtin_bit_cast(u16x2, t << 8));
    }
    m0 = __builtin_elementwise_max(m0, m1);
    const u32 top = wave_max(max((u32)m0[0], (u32)m0[1])) >> 8;
    const u32 C = (0x80u + top) * 0x01010101u;
    u32 nexc = 0, hv = 0;
#pragma unroll
    for (int i = 0; i < 4; ++i) {
        u32x4 code;
        u32 chk = 0;
#pragma unroll
        for (int j = 0; j < 4; ++j) {
            const u32 d0 = C - H[8 * i + 2 * j], d1 = C - H[8 * i + 2 * j + 1];
            chk |= d0 | d1;
            code[j] = bfi(0x87878787u, d0, __builtin_amdgcn_alignbit(d1, d1, 4));
        }
        st16(c, code, c.lane * 16, dst + 8192 + i * 1024);
        for (u64 m = __ballot(chk & 0x78787878u); m && nexc <= 31; m &= m - 1) {
            const u32 ln = __builtin_ctzll(m);
            u32 th = top * 0x01010101u, tl = 0;
#pragma unroll
            for (int k = 0; k < 8; ++k) {
                th = writelane(__builtin_amdgcn_readlane(H[8 * i + k], ln), k, th);
                tl = writelane(__builtin_amdgcn_readlane(L[8 * i + k], ln), k, tl);
            }
            for (u32 mk = (u32)__ballot((C - th) & 0x78787878u); mk; mk &= mk - 1) {
                const u32 k = __builtin_ctz(mk), h = __builtin_amdgcn_readlane(th, k), l = __builtin_amdgcn_readlane(tl, k);
                for (u32 d = (C - h) & 0x78787878u; d;) {
                    const u32 b8 = __builtin_ctz(d) & 24, pos = (4 * i + (k >> 1)) << 9 | ln << 3 | (k & 1) << 2 | b8 >> 3;
                    d &= ~(0xffu << b8);
                    ++nexc;
                    hv = writelane(pos << 16 | ((h >> b8) & 0xff) << 8 | ((l >> b8) & 0xff), nexc & 63, hv);
                }
            }
        }
    }
    if (FORCE_RAW_IN || nexc > 31) {
#pragma unroll
        for (int f = 0; f < 16; ++f) st16(c, join8(H[2 * f], L[2 * f], H[2 * f + 1], L[2 * f + 1]), c.lane * 16, dst + f * 1024);
        return 1;
    }
    hv = writelane(top | nexc << 8, 0, hv);
    if (c.lane < 32) __builtin_amdgcn_raw_buffer_store_b32(hv, c.rs, c.lane * 4 + dst + 12288, 0, 17);
    return 0;
}

DEV u32 xa_off(int ks, int h, int rowfrag) { return WS_XA + (u32)((ks * 2 + h) * 256 + rowfrag) * 1024u; }
DEV u32 wb_off(int ks, int h, int colfrag) { return WS_WB + (u32)((ks * 2 + h) * 128 + colfrag) * 1024u; }

DEV void pack_job(const Ctx& c, int job) {
    const int s = job / 160, q = job % 160, ks = c.rank * 8 + s;
    const int l15 = c.lane & 15, lq = c.lane >> 4;
    const bool isx = q < 32;
    const int dst = (c.rank + 1 + (q - 32)) & 7, cc = (q - 32) >> 3;
    u32x4 R[16];
    if (isx) {
        const GAS char* src = (const GAS char*)c.x + (size_t)(q * 128 + l15) * (KSH * 2) + (s * 64 + lq * 8) * 2;
#pragma unroll
        for (int f = 0; f < 16; ++f) R[f] = *(const GAS u32x4*)(src + (f & 7) * 16 * (KSH * 2) + (f >> 3) * 64);
    } else {
        const GAS char* src = (const GAS char*)c.w + (size_t)(s * 64 + lq * 8) * (NTOT * 2) + (dst * 2048 + cc * 128 + l15 * 4) * 2;
        u32x2 r[4][8];
#pragma unroll
        for (int hg = 0; hg < 4; ++hg)
#pragma unroll
            for (int j = 0; j < 8; ++j) r[hg][j] = *(const GAS u32x2*)(src + (size_t)((hg >> 1) * 32 + j) * (NTOT * 2) + (hg & 1) * 128);
#pragma unroll
        for (int hg = 0; hg < 4; ++hg)
#pragma unroll
            for (int ii = 0; ii < 4; ++ii)
#pragma unroll
                for (int t = 0; t < 4; ++t)
                    R[hg * 4 + ii][t] = perm(r[hg][2 * t + 1][ii >> 1], r[hg][2 * t][ii >> 1], (ii & 1) ? 0x07060302u : 0x05040100u);
    }
    const bool mine = isx || dst == c.rank, theirs = isx || dst != c.rank;
    if (mine) store_raw(c, R, isx ? xa_off(ks, 0, q * 8) : wb_off(ks, 0, cc * 8), isx ? xa_off(ks, 1, q * 8) : wb_off(ks, 1, cc * 8));
    const u32 call = get_call(c), ch = isx ? s * 32 + q : 256 + s * 16 + cc;
    u32 raw = 0;
    if (theirs) raw = pack_store(c, R, WS_PK + (call & 1) * PK_PAR + (isx ? ch : ch + dst * 128) * CHB);
    if (job < 160) STAMPW(c, 5);
    WAIT_VM0();
    if (c.lane < NR && (isx || c.lane == dst)) {
        if (c.lane == c.rank) st_sys(c.me + (isx ? WS_XR + q * 256 : WS_WR + cc * 256) + ks * 4, call);
        else st_sys(c.peer_lane + WS_FIN + (((call & 1) * NR + c.rank) * NCHIN + ch) * 4, 2 * call + raw);
    }
}

struct Tile { int cc, owner, h, rf, rfo; };
constexpr u32 TK2 = 130;

template <bool BIG>
DEV void loader(const Ctx& c, const Tile& t, int L, u32 xr, u32 call) {
    const int which = c.lane & 3, myks = L * 16 + ((c.lane >> 2) ^ xr), rc = BIG && which == 2 ? t.rfo >> 3 : (t.rf >> 3) + (which & 1);
    const char* fp = c.me + (which == 3 ? WS_WR + t.cc * 256 : WS_XR + rc * 256) + myks * 4;
    const u32 ldsbase = (u32)(size_t)c.lds, voff = c.lane * 16;
    u64 todo = 0x1111111111111111ull, avail = 0, own = 0, late = 0;
    u32 pend = 0, pslot = 0;
    bool ok = true, second = false;
    while (ok) {
        if (!todo) {
            if (!BIG || second || !late) break;
            second = true; todo = late;
        }
        if (!avail) {
            if (pend) { WAIT_VM0(); c.ctl[C_FULL + pslot] = pend; pend = 0; }
            u64 t0 = 0;
            for (u32 n = 1;; ++n) {
                const u64 m = __ballot(ld_sys(fp) >= call);
                own = m >> 2;
                avail = (second ? own : m & m >> 1 & m >> 3) & todo;
                if (avail) break;
                if ((n & 15) == 0) {
                    if (rfl(c.ctl[C_ABORT]) || tmo_set(c, call)) { ok = false; break; }
                    if (expired(t0, DEADLINE)) { tmo_raise(c, call); ok = false; break; }
                }
                __builtin_amdgcn_s_sleep(4);
            }
            if (!ok) break;
            if (avail == todo && tmo_set(c, call)) { ok = false; break; }
        }
        const int bit = __builtin_ctzll(avail), ks = L * 16 + ((bit >> 2) ^ xr);
        avail &= avail - 1; todo &= ~(1ull << bit);
        const u32 full = BIG && !second && ((own >> bit) & 1);
        if (BIG && !second && !full) late |= 1ull << bit;
        u32 tk = 0;
        if (c.lane == 0) tk = __hip_atomic_fetch_add((LAS u32*)&c.ctl[second ? C_TICKET2 : C_TICKET], second ? 1u : 2u, __ATOMIC_RELAXED, __HIP_MEMORY_SCOPE_WORKGROUP);
        tk = rfl(tk) + (second ? TK2 : 0);
#pragma unroll
        for (int h = 0; h < 2; ++h) {
            const u32 tt = tk + h, slot = tt % NSTG, need = 8 * (tt / NSTG);
            if (rfl(c.ctl[C_FREE + slot]) < need) {
                if (pend) { WAIT_VM0(); c.ctl[C_FULL + pslot] = pend; pend = 0; }
                if (!lds_wait(c, C_FREE + slot, need, true, 1)) { ok = false; break; }
            }
            const u32 l = ldsbase + slot * STG;
            if (BIG && second) {
                glds4(c.rs, voff, xa_off(ks, 0, t.rfo), l); glds4(c.rs, voff, xa_off(ks, 1, t.rfo), l + 4096);
#pragma unroll
                for (int g = 0; g < 4; ++g) glds4(c.rs, voff, wb_off(ks, g >> 1, t.cc * 8 + (g & 1) * 4), l + 8192 + g * 4096);
            } else {
#pragma unroll
                for (int g = 0; g < 4; ++g) glds4(c.rs, voff, xa_off(ks, h, t.rf + g * 4), l + g * 4096);
                glds4(c.rs, voff, wb_off(ks, h, t.cc * 8), l + 20480); glds4(c.rs, voff, wb_off(ks, h, t.cc * 8 + 4), l + 24576);
            }
            if (full) {
                glds4(c.rs, voff, xa_off(ks, h, t.rfo), l + 16384);
                asm volatile("s_waitcnt vmcnt(28)" ::: "memory");
            } else asm volatile("s_waitcnt vmcnt(24)" ::: "memory");
            if (pend) c.ctl[C_FULL + pslot] = pend;
            pend = 2 * tt + 2 + full; pslot = slot;
            if (BIG && second) break;
        }
    }
    WAIT_VM0();
    if (!ok) c.ctl[C_ABORT] = 1;
    else if (pend) c.ctl[C_FULL + pslot] = pend;
}

DEV u32 pk_relu(float a, float b) {
    const f32x2 v = __builtin_elementwise_maximum((f32x2){a, b}, (f32x2){0.f, 0.f});
    return __builtin_bit_cast(u32, __builtin_convertvector(v, bf16x2));
}

DEV u32 nz8(u32 cb) { return (cb + 0x07070707u) & 0x08080808u; }
template <int NF>
DEV u32 encode_y(const Ctx& c, const u32 (&P)[NF * 8], u32 dst) {
    constexpr int G = NF * 4;
    constexpr u32 CODEB = NF == 5 ? 2048 : 1536;
    u16x2 mx = {0, 0}, mn[NF];
#pragma unroll
    for (int f = 0; f < NF; ++f) {
        mn[f] = (u16x2){0xffff, 0xffff};
#pragma unroll
        for (int j = 0; j < 8; ++j) {
            const u16x2 v = __builtin_bit_cast(u16x2, P[f * 8 + j]);
            mx = __builtin_elementwise_max(mx, v);
            mn[f] = __builtin_elementwise_min(mn[f], (u16x2)(v - (u16x2){1, 1}));
        }
    }
    const u32 base = max(wave_max(max((u32)mx[0], (u32)mx[1])) >> 8, 7u) - 7, lim = ((base + 1) << 8) - 1;
    const u32 ldsbase = (u32)(size_t)c.lds, stg = L_YST + c.wave * YST;
    u32 nexc = 0, hv = 0;
#pragma unroll
    for (int f = 0; f < NF; ++f)
        for (u64 m = __ballot(min((u32)mn[f][0], (u32)mn[f][1]) < lim); m && nexc <= 31; m &= m - 1) {
            const u32 ln = __builtin_ctzll(m);
            u32 t = 0;
#pragma unroll
            for (int k = 0; k < 8; ++k) t = writelane(__builtin_amdgcn_readlane(P[f * 8 + k], ln), k, t);
            for (u32 mk = (u32)__ballot((t & 0xffff) - 1 < lim) | (u32)__ballot((t >> 16) - 1 < lim) << 8; mk; mk &= mk - 1) {
                const u32 k = __builtin_ctz(mk), v = (__builtin_amdgcn_readlane(t, k & 7) >> (k & 8) * 2) & 0xffff;
                ++nexc;
                hv = writelane((f * 16 + (k & 7) * 2 + (k >> 3)) << 22 | ln << 16 | v, nexc & 63, hv);
            }
        }
    if (FORCE_RAW_Y || nexc > 31) {
#pragma unroll
        for (int i = 0; i < NF * 2; ++i) st16(c, (u32x4){P[4 * i], P[4 * i + 1], P[4 * i + 2], P[4 * i + 3]}, c.lane * 16, dst + i * 1024);
        return YM_RAW | NF * 128;
    }
    hv = writelane(base | nexc << 8, 0, hv);
    if ((u32)c.lane <= nexc) *(LAS u32*)(c.lds + stg + c.lane * 4) = hv;
    u32 cb[G], lb[G], cnt = 0;
    const u16x2 B = {(u16)(base << 8), (u16)(base << 8)};
#pragma unroll
    for (int g = 0; g < G; ++g) {
        const u32 t0 = __builtin_bit_cast(u32, __builtin_elementwise_sub_sat(__builtin_bit_cast(u16x2, P[2 * g]), B));
        const u32 t1 = __builtin_bit_cast(u32, __builtin_elementwise_sub_sat(__builtin_bit_cast(u16x2, P[2 * g + 1]), B));
        cb[g] = perm(t1, t0, 0x07050301u); lb[g] = perm(t1, t0, 0x06040200u);
        cnt += __builtin_popcount(nz8(cb[g]));
    }
    u32 D[8];
#pragma unroll
    for (int h = 0; h < 2; ++h) {
        const u32* q = cb + 8 * h;
        D[3 * h] = q[0] | q[1] << 3 | (q[6] & 0x03030303u) << 6;
        D[3 * h + 1] = q[2] | q[3] << 3 | (q[7] & 0x03030303u) << 6;
        D[3 * h + 2] = q[4] | q[5] << 3 | (q[6] & 0x04040404u) << 4 | (q[7] & 0x04040404u) << 5;
    }
    st16(c, (u32x4){D[0], D[1], D[2], D[3]}, c.lane * 16, dst);
    if (NF == 5) st16(c, (u32x4){D[4], D[5], cb[G - 4] | cb[G - 3] << 3, cb[G - 2] | cb[G - 1] << 3}, c.lane * 16, dst + 1024);
    else __builtin_amdgcn_raw_buffer_store_b64((u32x2){D[4], D[5]}, c.rs, c.lane * 8 + dst + 1024, 0, 17);
    const u32 dw = (cnt >> 2) + 1, incl = wave_scan(dw), total = 1 + nexc + __builtin_amdgcn_readlane(incl, 63);
    u32 pos = ldsbase + stg + 4 * (1 + nexc + incl - dw), adr[G];
    u32x2 sel[G];
#pragma unroll
    for (int g = 0; g < G; ++g) {
        const u32 nz = nz8(cb[g]);
        adr[g] = pos & ~3u;
        sel[g] = *(const LAS u32x2*)(size_t)__builtin_amdgcn_udot4(nz, 0x20100804u, ((pos & 3) << 3) | (ldsbase + L_LUT), false);
        pos += __builtin_popcount(nz);
    }
    __builtin_amdgcn_sched_barrier(0);
    u32 cur = 0;
#pragma unroll
    for (int g = 0; g < G; ++g) {
        const u32 o = perm(lb[g], cur, sel[g][0]);
        *(LAS u32*)(size_t)adr[g] = o;
        cur = perm(lb[g], o, sel[g][1]);
    }
    *(LAS u32*)(size_t)(pos & ~3u) = cur;
    CBAR();
    u32x4 v[5];
#pragma unroll
    for (int i = 0; i < 5; ++i) if (i < 3 || total > i * 256u) v[i] = *(const LAS u32x4*)(c.lds + stg + c.lane * 16 + min(i * 1024u, YST - 1024));
#pragma unroll
    for (int i = 0; i < 5; ++i) if (i < 3 || total > i * 256u) st16(c, v[i], c.lane * 16, dst + CODEB + min(i * 1024u, YST - 1024));
    return (CODEB + total * 4 + 15) >> 4;
}

DEV f32x4 mfma(u32x4 a, u32x4 b, f32x4 acc) { return __builtin_amdgcn_mfma_f32_16x16x32_bf16(__builtin_bit_cast(bf16x8, a), __builtin_bit_cast(bf16x8, b), acc, 0, 0, 0); }
DEV void slot_free(const Ctx& c, u32 slot) { if (c.lane == 0) __hip_atomic_fetch_add((LAS u32*)&c.ctl[C_FREE + slot], 1u, __ATOMIC_RELAXED, __HIP_MEMORY_SCOPE_WORKGROUP); }

template <bool BIG>
DEV void consumer(const Ctx& c, const Tile& t, int wr, int wc) {
    f32x4 acc[4][4], oacc[4];
#pragma unroll
    for (int i = 0; i < 4; ++i) {
        oacc[i] = (f32x4){0.f, 0.f, 0.f, 0.f};
#pragma unroll
        for (int f = 0; f < 4; ++f) acc[f][i] = (f32x4){0.f, 0.f, 0.f, 0.f};
    }
    const LAS char* pa = c.lds + c.lane * 16 + wr * 4096;
    const LAS char* po = c.lds + c.lane * 16 + wr * 1024;
    const LAS char* pb = c.lds + c.lane * 16 + wc * 4096;
    u32 slot = 0, seen = 0, nfull = 0;
    bool ok = true;
#pragma unroll 1
    for (u32 tk = 0; tk < 128; ++tk) {
        u32 sv = rfl(seen);
        if (ok && sv < 2 * tk + 2) { sv = lds_wait(c, C_FULL + slot, 2 * tk + 2, true, 0); ok = sv != 0; }
        CBAR();
        if (tk == 0) STAMPW(c, 1);
        if (tk == 64) STAMPW(c, 4);
        if (tk == 96) STAMPW(c, 5);
        if (tk == 112) STAMPW(c, 6);
        if (tk == 120) STAMPW(c, 7);
        u32x4 a[4], b[4], ao;
#pragma unroll
        for (int i = 0; i < 4; ++i) b[i] = *(const LAS u32x4*)(pb + slot * STG + 20480 + i * 1024);
#pragma unroll
        for (int f = 0; f < 4; ++f) a[f] = *(const LAS u32x4*)(pa + slot * STG + f * 1024);
        if (BIG) ao = *(const LAS u32x4*)(po + slot * STG + 16384);
        WAIT_LGKM0();
        slot_free(c, slot);
        slot = slot == NSTG - 1 ? 0 : slot + 1;
        seen = c.ctl[C_FULL + slot];
        __builtin_amdgcn_sched_barrier(0);
#pragma unroll
        for (int f = 0; f < 4; ++f)
#pragma unroll
            for (int i = 0; i < 4; ++i) acc[f][i] = mfma(a[f], b[i], acc[f][i]);
        if (BIG && (sv & 1)) {
            ++nfull;
#pragma unroll
            for (int i = 0; i < 4; ++i) oacc[i] = mfma(ao, b[i], oacc[i]);
        }
    }
    STAMPW(c, 2);
    ok = ok && !rfl(c.ctl[C_ABORT]);
    {
        const u32 call = get_call(c), msg = (t.cc * 2 + t.h) * 8 + c.wave;
        u32 P[32];
#pragma unroll
        for (int g = 0; g < 16; ++g) {
            P[2 * g] = pk_relu(acc[g >> 2][0][g & 3], acc[g >> 2][1][g & 3]);
            P[2 * g + 1] = pk_relu(acc[g >> 2][2][g & 3], acc[g >> 2][3][g & 3]);
        }
        ok = ok && lds_wait(c, C_FREE + 3, 200, true, 0) && lds_wait(c, C_FREE + 4, 200, true, 0);
        CBAR();
        STAMPW(c, 8);
        u32 meta = YM_BAD;
        if (ok) meta = encode_y<4>(c, P, WS_YP + (t.owner * NMSG + msg) * YSLOT);
        STAMPW(c, 9);
        WAIT_VM0();
        if (c.lane == t.owner) st_sys64(c.peer_lane + WS_FY + (c.rank * NMSG + msg) * 8, (u64)call << 32 | meta);
    }
    STAMPW(c, 3);
    if (BIG) {
        slot_free(c, 3); slot_free(c, 4);
        slot = 0;
#pragma unroll 1
        for (u32 tk = TK2; ok && tk < TK2 + 64 - (nfull >> 1); ++tk) {
            ok = lds_wait(c, C_FULL + slot, 2 * tk + 2, true, 0) != 0;
            CBAR();
            u32x4 a[2], b[8];
#pragma unroll
            for (int i = 0; i < 8; ++i) b[i] = *(const LAS u32x4*)(pb + slot * STG + 8192 + (i >> 2) * 8192 + (i & 3) * 1024);
#pragma unroll
            for (int f = 0; f < 2; ++f) a[f] = *(const LAS u32x4*)(po + slot * STG + f * 4096);
            WAIT_LGKM0();
            slot_free(c, slot);
            slot = slot == NSTG - 1 ? 0 : slot + 1;
#pragma unroll
            for (int i = 0; i < 8; ++i) oacc[i & 3] = mfma(a[i >> 2], b[i], oacc[i & 3]);
        }
        ok = ok && !rfl(c.ctl[C_ABORT]);
        const u32 o = out_off(c, ((t.rfo + wr) & 31) * 16, c.rank * 2048 + t.cc * 128 + wc * 64);
#pragma unroll
        for (int r = 0; r < 4; ++r)
            st_out(c, o + r * (NTOT * 2), ok ? (u32x2){pk_relu(oacc[0][r], oacc[1][r]), pk_relu(oacc[2][r], oacc[3][r])} : (u32x2){0x7fc07fc0u, 0x7fc07fc0u});
        STAMPW(c, 10);
    }
}

DEV void gemm_wg(const Ctx& c) {
    const int g = c.wg - NCOMM, xcd = g & 7, idx = g >> 3, grp = xcd >> 2, k = idx % 7;
    const bool big = k < 4;
    Tile t;
    t.cc = (xcd & 3) * 4 + idx / 7;
    t.h = grp * 4 + k >= 7 || k >= 4;
    t.owner = (c.rank + (grp ? (k < 3 ? 5 + k : 1 + k) : (k < 4 ? 1 + k : k - 3))) & 7;
    t.rf = t.owner * 32 + t.h * 16; t.rfo = c.rank * 32 + (grp * 4 + (k & 3)) * 4;
    if (c.wave >= 8) {
        const int L = c.wave - 8;
        if (L * NGEMM + g >= 160) for (int i = 0; i < PSLEEP + L; ++i) __builtin_amdgcn_s_sleep(24);
#pragma unroll 1
        for (int job = L * NGEMM + g; job < 1280; job += 4 * NGEMM) { pack_job(c, job); if (job < 4 * NGEMM) STAMPW(c, 3); }
        STAMPW(c, 1);
        const u32 call = get_call(c), xr = XROT ? (t.cc & 1) | (t.cc & 2) << 2 | (k & 3) << 1 : 0;
        if (big) loader<true>(c, t, L, xr, call); else loader<false>(c, t, L, xr, call);
        STAMPW(c, 2);
    } else {
        if (big) consumer<true>(c, t, c.wave >> 1, c.wave & 1); else consumer<false>(c, t, c.wave >> 1, c.wave & 1);
    }
}

#define WSUM 8
#define WCLV 5

DEV u32 take_seat(const Ctx& c, u32 call) {
    u32 xcc;
    asm volatile("s_getreg_b32 %0, hwreg(HW_REG_XCC_ID, 0, 4)" : "=s"(xcc));
    const u32 grp = (0xb2u >> c.rank) & 1 ? 0xb2u : 0x4du, j = (c.wg >> 3) * 4 + (xcc & 3);
    u32 near = TOPO ? (xcc & 4 ? ~grp & 0xff : grp & ~(1u << c.rank)) : 0, seat = NPULL;
    const u32 n = __builtin_popcount(near);
    if (!TOPO) seat = min((u32)c.wg, (u32)NPULL);
    else if (j < 4 * n) {
        for (u32 i = j % n; i; --i) near &= near - 1;
        seat = ((__builtin_ctz(near) - c.rank) & 7) - 1 + 7 * (j / n);
    }
    for (bool first = true;; first = false) {
        u64 old = ~0ull;
        if (c.lane == 0 ? seat < NPULL : first && c.lane < NR)
            old = __hip_atomic_fetch_max((GAS u64*)(c.me + (c.lane ? WS_CLM + c.lane * 128 : WS_SEAT + seat * 8)), (u64)call << 32, __ATOMIC_RELAXED, __HIP_MEMORY_SCOPE_AGENT);
        if (rfl((u32)(old >> 32)) < call) return seat;
        if (first) for (int i = 0; i < 4; ++i) __builtin_amdgcn_s_sleep(32);
        const u64 v = __hip_atomic_load((const GAS u64*)(c.me + WS_SEAT + min(c.lane, NPULL - 1) * 8), __ATOMIC_RELAXED, __HIP_MEMORY_SCOPE_AGENT);
        const u64 open = __ballot((u32)(v >> 32) < call);
        if (!open) return NPULL;
        seat = __builtin_ctzll(open);
    }
}

DEV void watcher(const Ctx& c, int p, int sub, u32 call) {
    static_assert(NQY == 64, "one message per lane");
    const u32 id = sub + 4 * c.lane;
    const char* fp = c.me + WS_FY + (p * NMSG + id) * 8;
    bool bad = !lds_wait(c, C_YGO, 1, false, 1);
    STAMPW(c, 1);
    u64 pending = ~0ull, t0 = 0;
    u32 qn = 0;
    for (u32 it = 1; pending; ++it) {
        const u64 v = bad ? 0 : ld_sys64(fp);
        const bool mine = ((pending >> c.lane) & 1) && (bad || (u32)(v >> 32) >= call);
        const u64 m = __ballot(mine);
        if (m) {
            const bool poison = bad || tmo_set(c, call);
            if (mine) c.ctl[C_Q + qn + lane_rank(m)] = 0x80000000u | id << 20 | (poison ? YM_BAD : (u32)v & (YM_BAD | YM_RAW | 0x3ff));
            qn += __popcll(m); pending &= ~m; t0 = 0;
        } else {
            if ((it & 15) == 0) {
                if (tmo_set(c, call)) bad = true;
                else if (expired(t0, DEADLINE)) { tmo_raise(c, call); bad = true; }
            }
            __builtin_amdgcn_s_sleep(2);
        }
    }
    STAMPW(c, 2);
}

DEV void pull_inputs(const Ctx& c, rsrc_t pr, int d, int sub, int p, u32 call) {
    GAS u64* ctr = (GAS u64*)(c.me + WS_CLM + d * 128);
    const u32 voff = c.lane * 16, win = WSUM / 4 + (sub < WSUM % 4), WCL = win + WCLV;
    bool bad = false;
#pragma unroll 1
    for (;;) {
        const u32 t = lds_inc(c, C_TK);
        if (t >= WCL) lds_wait(c, C_LAND, t - WCL + 1, false, 1);
        u32 n = 0;
        if (c.lane == 0) n = (u32)__hip_atomic_fetch_add(ctr, 1ull, __ATOMIC_RELAXED, __HIP_MEMORY_SCOPE_AGENT);
        n = rfl(n);
        if (n + 64 >= NCHIN) c.ctl[C_YGO] = 1;
        if (n >= NCHIN) break;
        const u32 code = rfl(ORDER.v[d - 1][n]), s = (code >> 8) & 7, ix = code & 31;
        const bool isw = code & 0x8000;
        const u32 rc = isw ? ix : ((((ix >> 2) + c.rank) & 7) << 2 | (ix & 3)), id = isw ? 256 + s * 16 + rc : s * 32 + rc;
        const char* fp = c.me + WS_FIN + (((call & 1) * NR + p) * NCHIN + id) * 4;
        u32 v = 0;
        u64 t0 = 0;
        for (u32 it = 1; !bad; ++it) {
            v = rfl(ld_sys(fp));
            if (v >= 2 * call) break;
            if ((it & 15) == 0) {
                if (tmo_set(c, call)) bad = true;
                else if (expired(t0, DEADLINE)) { tmo_raise(c, call); bad = true; }
            }
            __builtin_amdgcn_s_sleep(2);
        }
        if (bad) { lds_inc(c, C_LAND); continue; }
        const u32 u = lds_inc(c, C_IS);
        if (u >= win) lds_wait(c, C_LAND, u - win + 1, false, 0);
        ST2(c, 1024 + u, 13);
        const int ks = p * 8 + s;
        const u32 src = WS_PK + (call & 1) * PK_PAR + (isw ? id + c.rank * 128 : id) * CHB;
        const u32 d0 = isw ? wb_off(ks, 0, rc * 8) : xa_off(ks, 0, rc * 8), d1 = isw ? wb_off(ks, 1, rc * 8) : xa_off(ks, 1, rc * 8);
        u32x4 R[16];
        if (v & 1) {
#pragma unroll
            for (int f = 0; f < 16; ++f) R[f] = ld16(pr, voff + (f & 3) * 1024, src + (f >> 2) * 4096);
            WAIT_VM0();
            lds_inc(c, C_LAND);
            store_raw(c, R, d0, d1);
        } else {
            u32x4 L[8], code[4];
            const u32 hv = __builtin_amdgcn_raw_buffer_load_b32(pr, (c.lane & 31) * 4, src + 12288, 17);
#pragma unroll
            for (int i = 0; i < 4; ++i) code[i] = ld16(pr, voff + i * 1024, src + 8192);
#pragma unroll
            for (int i = 0; i < 8; ++i) L[i] = ld16(pr, voff + (i & 3) * 1024, src + (i >> 2) * 4096);
            WAIT_VM0();
            lds_inc(c, C_LAND);
            ST2(c, 1536 + u, 0); ST2(c, u, 0);
            const u32 h0 = rfl(hv), C = (0x80u + (h0 & 0x7f)) * 0x01010101u, nexc = min((h0 >> 8) & 0xff, 31u);
#pragma unroll
            for (int f = 0; f < 16; ++f) {
                const u32 n = code[f >> 2][f & 3];
                const u32 ha = C - (n & 0x87878787u), hb = C - (__builtin_amdgcn_alignbit(n, n, 28) & 0x87878787u);
                const u32 la = L[f >> 1][(f & 1) * 2], lb = L[f >> 1][(f & 1) * 2 + 1];
                R[f] = join8(ha, la, hb, lb);
            }
            store_raw(c, R, d0, d1);
#pragma unroll 1
            for (u32 i = 0; i < nexc; ++i) {
                const u32 x = __builtin_amdgcn_readlane(hv, i + 1), pos = x >> 16, f = (pos >> 9) & 15, ln = (pos >> 3) & 63;
                if ((u32)c.lane == ln) __builtin_amdgcn_raw_buffer_store_b16((u16)x, c.rs, ln * 16 + (f & 7) * 1024 + (pos & 7) * 2, f < 8 ? d0 : d1, 17);
            }
        }
        STAMPW(c, 3);
        WAIT_VM0();
        if (c.lane == 0) st_sys(c.me + (isw ? WS_WR : WS_XR) + rc * 256 + ks * 4, call);
        ST2(c, 512 + u, 0);
    }
    lds_inc(c, C_LAND);
}

template <int NF>
DEV void decode_y(const Ctx& c, u32 e, u32 lo, rsrc_t pr, u32 src, u32 o, u32 tk) {
    constexpr int G = NF * 4;
    constexpr u32 CODEB = NF == 5 ? 2048 : 1536;
    const u32 ldsbase = (u32)(size_t)c.lds;
    const LAS char* l = c.lds + lo;
    u32x2 P[G];
    u32 hv = 0, nexc = 0;
    if (e & YM_BAD) {
#pragma unroll
        for (int g = 0; g < G; ++g) P[g] = (u32x2){0x7fc07fc0u, 0x7fc07fc0u};
    } else if (e & YM_RAW) {
#pragma unroll
        for (int i = 0; i < G / 2; ++i) {
            const u32x4 v = ld16(pr, c.lane * 16 + (i & 3) * 1024, src + (i >> 2) * 4096);
            P[2 * i] = (u32x2){v[0], v[1]}; P[2 * i + 1] = (u32x2){v[2], v[3]};
        }
        __builtin_amdgcn_s_waitcnt(0x0f70);
    } else {
        const u32x4 da = *(const LAS u32x4*)(l + c.lane * 16);
        u32x4 db;
        if (NF == 5) db = *(const LAS u32x4*)(l + 1024 + c.lane * 16);
        else { const u32x2 t = *(const LAS u32x2*)(l + 1024 + c.lane * 8); db = (u32x4){t[0], t[1], 0, 0}; }
        hv = *(const LAS u32*)(l + CODEB + (c.lane & 31) * 4);
        const u32 D[8] = {da[0], da[1], da[2], da[3], db[0], db[1], db[2], db[3]};
        u32 cb[G], cnt = 0;
#pragma unroll
        for (int h = 0; h < 2; ++h) {
            u32* q = cb + 8 * h;
#pragma unroll
            for (int k = 0; k < 3; ++k) { q[2 * k] = D[3 * h + k] & 0x07070707u; q[2 * k + 1] = (D[3 * h + k] >> 3) & 0x07070707u; }
            q[6] = bfi(0x03030303u, D[3 * h] >> 6, D[3 * h + 2] >> 4) & 0x07070707u;
            q[7] = bfi(0x03030303u, D[3 * h + 1] >> 6, D[3 * h + 2] >> 5) & 0x07070707u;
        }
        if (NF == 5) { cb[G - 4] = D[6] & 0x07070707u; cb[G - 3] = (D[6] >> 3) & 0x07070707u; cb[G - 2] = D[7] & 0x07070707u; cb[G - 1] = (D[7] >> 3) & 0x07070707u; }
#pragma unroll
        for (int g = 0; g < G; ++g) cnt += __builtin_popcount(nz8(cb[g]));
        const u32 h0 = rfl(hv), B = (h0 & 0xff) * 0x01010101u, dw = (cnt >> 2) + 1;
        nexc = min((h0 >> 8) & 0xff, 31u);
        u32 pos = ldsbase + lo + CODEB + 4 * (1 + nexc + wave_scan(dw) - dw), ps[G];
        u32x2 sel[G];
#pragma unroll
        for (int g = 0; g < G; ++g) {
            const u32 nz = nz8(cb[g]);
            const LAS u32* q = (const LAS u32*)(size_t)(pos & ~3u);
            sel[g] = *(const LAS u32x2*)(size_t)__builtin_amdgcn_udot4(nz, 0x08040201u, ldsbase + L_LUT, false);
            P[g] = (u32x2){q[0], q[1]};
            ps[g] = pos;
            pos += __builtin_popcount(nz);
        }
        __builtin_amdgcn_sched_barrier(0);
#pragma unroll
        for (int g = 0; g < G; ++g) {
            const u32 by = __builtin_amdgcn_alignbyte(P[g][1], P[g][0], ps[g]), lb = perm(by, by, sel[g][0]), hb = (cb[g] + B) & sel[g][1];
            P[g] = (u32x2){perm(hb, lb, 0x05010400u), perm(hb, lb, 0x07030602u)};
        }
    }
#pragma unroll
    for (int g = 0; g < G; ++g) st_out(c, o + ((g >> 2) * 16 + (g & 3)) * (NTOT * 2), P[g]);
    ST2(c, 512 + 384 + tk, nexc);
#pragma unroll 1
    for (u32 i = 0; i < nexc; ++i) {
        const u32 x = __builtin_amdgcn_readlane(hv, i + 1), j = min(x >> 22, (u32)G * 4 - 1);
        if ((u32)c.lane == ((x >> 16) & 63)) __builtin_amdgcn_raw_buffer_store_b16((u16)x, c.ro, o + ((j >> 4) * 16 + ((j >> 2) & 3)) * (NTOT * 2) + (j & 3) * 2, 0, OAUX);
    }
}
DEV void fetch_y(const Ctx& c, char* peer, u32 e, u32 lo) {
    if (e & (YM_BAD | YM_RAW)) return;
    const u32 src = WS_YP + (c.rank * NMSG + ((e >> 20) & 0xff)) * YSLOT, bytes = min((e & 0x3ff) * 16, LSLOT);
    const rsrc_t rs = __builtin_amdgcn_make_buffer_rsrc(peer, 0, src + bytes, 0x00020000);
    glds4(rs, c.lane * 16, src, (u32)(size_t)c.lds + lo);
    if (bytes > 4096) glds2(rs, c.lane * 16, src + 4096, (u32)(size_t)c.lds + lo + 4096);
}

DEV void worker(const Ctx& c, char* peer, int d, int sub, int p, int w, u32 call) {
    const rsrc_t pr = __builtin_amdgcn_make_buffer_rsrc(peer, 0, WS_END, 0x00020000);
    pull_inputs(c, pr, d, sub, p, call);
    STAMPW(c, 1);
    u32 lo = w * 2 * LSLOT, lo2 = lo + LSLOT, e = 0;
#pragma unroll 1
    for (u32 tk = lds_inc(c, C_YT); tk < NQY;) {
        if (e) asm volatile("s_waitcnt vmcnt(16)" ::: "memory");
        else {
            e = lds_wait(c, C_Q + tk, 1, false, 1);
            if (!e) { tmo_raise(c, call); break; }
            ST2(c, 1024 + 448 + tk, 4);
            fetch_y(c, peer, e, lo);
            WAIT_VM0();
        }
        ST2(c, 1536 + 448 + tk, 0); ST2(c, 448 + tk, 0);
        if (tk == 0) STAMPW(c, 4);
        STAMPW(c, 5);
        const u32 nt = lds_inc(c, C_YT), ne = nt < NQY ? rfl(c.ctl[C_Q + nt]) : 0;
        if (ne) { ST2(c, 1024 + 448 + nt, 5); fetch_y(c, peer, ne, lo2); }
        const u32 id = (e >> 20) & 0xff, wc = id & 1, rg = (id >> 1) & 7, cc = id >> 4;
        decode_y<4>(c, DIAG & 1 ? e | YM_BAD : e, lo, pr, WS_YP + (c.rank * NMSG + id) * YSLOT, out_off(c, rg * 64, p * 2048 + cc * 128 + wc * 64), tk);
        ST2(c, 512 + 448 + tk, 0);
        tk = nt; e = ne;
        const u32 sw = lo; lo = lo2; lo2 = sw;
    }
    STAMPW(c, 2);
}

__global__ void __launch_bounds__(NTHREADS, 3) fused_kernel(Args a) {
    extern __shared__ __attribute__((aligned(128))) char smem[];
    Ctx c;
    c.x = a.x; c.w = a.w; c.out = a.out; c.rank = a.rank & 7; c.me = a.ws[c.rank];
    c.wg = blockIdx.x; c.wave = rfl(threadIdx.x >> 6); c.lane = threadIdx.x & 63;
    c.peer_lane = a.ws[c.lane & 7];
    c.rs = __builtin_amdgcn_make_buffer_rsrc(c.me, 0, WS_END, 0x00020000);
    c.ro = __builtin_amdgcn_make_buffer_rsrc(c.out, 0, DIAG & 2 ? 0 : 512 * NTOT * 2, 0x00020000);
    c.lds = (LAS char*)smem; c.ctl = (volatile LAS u32*)(c.lds + L_CTL);
    STAMPW(c, 0);
    u32 ticket = 0;
    if (threadIdx.x == 0) ticket = __hip_atomic_fetch_add((GAS u32*)(c.me + WS_CNT), 1u, __ATOMIC_RELAXED, __HIP_MEMORY_SCOPE_AGENT);
    for (u32 i = threadIdx.x; i < C_WORDS; i += NTHREADS) c.ctl[i] = 0;
    if (c.wave == 11) {
        u32 s0, s1;
        if (c.wg >= NCOMM) {
            const u32 mask = c.lane >> 2;
            u32 n = c.lane & 3;
            u64 sel = 0x0c0c0c0c03020100ull;
            for (int b = 0; b < 4; ++b)
                if ((mask >> b) & 1) { sel = (sel & ~(0xffull << (8 * n))) | (u64)(4 + b) << (8 * n); ++n; }
            s0 = (u32)sel; s1 = n >= 4 ? (u32)(sel >> 32) : 0x03020100u;
        } else {
            u32 n = 0;
            s0 = 0x0c0c0c0cu; s1 = 0;
            for (int b = 0; b < 4; ++b)
                if ((c.lane >> b) & 1) { s0 = (s0 & ~(0xffu << (8 * b))) | n << (8 * b); s1 |= 0xffu << (8 * b); ++n; }
        }
        c.ctl[C_LUT + 2 * c.lane] = s0; c.ctl[C_LUT + 2 * c.lane + 1] = s1;
    }
    WAIT_LGKM0();
    __builtin_amdgcn_s_barrier();
    CBAR();
    if (threadIdx.x == 0) c.ctl[C_CALL] = ticket / NWG + 1;
    if (c.wg >= NCOMM) gemm_wg(c);
    else {
        const u32 call = get_call(c);
        if (c.wave == 0) c.ctl[C_SEAT] = take_seat(c, call) + 1;
        const u32 seat = lds_wait(c, C_SEAT, 1, false, 1) - 1;
        if (seat >= NPULL) return;
        const int d = 1 + seat % 7, sub = seat / 7, p = (c.rank + d) & 7;
        if (c.wave == 0) watcher(c, p, sub, call);
        else worker(c, a.ws[p], d, sub, p, c.wave - 1, call);
    }
}

}

extern "C" void kernel_launch(int rank, int nranks, ncclComm_t comm, hipStream_t stream,
                              void* const* d_in, const int* in_sizes, int n_in,
                              void* d_out, int out_size, const int* in_dtypes, int out_dtype,
                              void* const* d_out_all, void* const* d_ws_all, size_t ws_bytes) {
    if (nranks != NR || (unsigned)rank >= (unsigned)NR || n_in != 2 || ws_bytes < WS_END || in_sizes[0] != 4096 * KSH ||
        in_sizes[1] != KSH * NTOT || out_size != 512 * NTOT) {
        fprintf(stderr, "unexpected problem shape\n");
        return;
    }
    HIP_CHECK(hipFuncSetAttribute((const void*)fused_kernel, hipFuncAttributeMaxDynamicSharedMemorySize, LDS_BYTES));
    Args a;
    a.x = (const u16*)d_in[0]; a.w = (const u16*)d_in[1]; a.out = (u16*)d_out;
    for (int p = 0; p < NR; ++p) a.ws[p] = (char*)d_ws_all[p];
    a.rank = rank; a.pad = 0;
    hipLaunchKernelGGL(fused_kernel, dim3(NWG), dim3(NTHREADS), LDS_BYTES, stream, a);
    HIP_CHECK(hipGetLastError());
}
```

```cpp
#include <hip/hip_runtime.h>
#include <rccl/rccl.h>
#include <cstdio>
#include <cstdint>

#define HIP_CHECK(expr) do { hipError_t _e = (expr); if (_e != hipSuccess) { \
    fprintf(stderr, "HIP error %s at %s:%d\n", hipGetErrorString(_e), __FILE__, __LINE__); \
    return; } } while (0)

extern "C" __device__ unsigned writelane(unsigned value, unsigned lane, unsigned old) __asm("llvm.amdgcn.writelane.i32");

namespace {

typedef unsigned u32;
typedef unsigned long long u64;
typedef unsigned short u16;
typedef u32 u32x4 __attribute__((ext_vector_type(4)));
typedef u32 u32x2 __attribute__((ext_vector_type(2)));
typedef u32 u32x8 __attribute__((ext_vector_type(8)));
typedef u32 u32x32 __attribute__((ext_vector_type(32)));
typedef u16 u16x2 __attribute__((ext_vector_type(2)));
typedef float f32x4 __attribute__((ext_vector_type(4)));
typedef float f32x2 __attribute__((ext_vector_type(2)));
typedef __bf16 bf16x2 __attribute__((ext_vector_type(2)));
typedef __bf16 bf16x8 __attribute__((ext_vector_type(8)));
typedef __amdgpu_buffer_rsrc_t rsrc_t;
#define LAS __attribute__((address_space(3)))
#define GAS __attribute__((address_space(1)))
#define DEV __device__ __forceinline__

#define PSLEEP 3
#define LSLEEP 4
#define SPREAD 0
#define RAW0 16
#define PSTEP 1
#define P2 0
#define YPRE 0
#define YWIN 7
#define OAUX 17
#define TOPO 1
#define DIAG 0
#define XROT 1
#define FORCE_RAW_Y 0
#define FORCE_RAW_IN 0

constexpr int NR = 8;
constexpr int NTOT = 16384, KSH = 512;
constexpr int NWG = 256, NCOMM = 32, NPULL = 28, NGEMM = NWG - NCOMM, NTHREADS = 768;
constexpr int NCHIN = 384;
constexpr int NMSG = 256;
constexpr u32 NQY = NMSG / 4;
constexpr int NWORK = 11;

constexpr u32 WS_TMO = 128, WS_CLM = 1024;
constexpr u32 WS_SEAT = 2048;
constexpr u32 WS_FIN = 4096;
constexpr u32 WS_FY = WS_FIN + 2 * NR * NCHIN * 8;
constexpr u32 WS_XR = WS_FY + NR * NMSG * 8;
constexpr u32 WS_WR = WS_XR + 8192;
constexpr u32 WS_CNT = WS_WR + 4096;
constexpr u32 WS_DBG = 131072;
static_assert(WS_CNT + NWG * 128 <= WS_DBG, "ws");
constexpr u32 CHB = 16384;
constexpr u32 WS_PK = 1u << 20;
constexpr u32 PK_PAR = 1280u * CHB;
constexpr u32 WS_XA = WS_PK + 2 * PK_PAR;
constexpr u32 WS_WB = WS_XA + (32u << 20);
constexpr u32 YSLOT = 8192;
constexpr u32 WS_YP = WS_WB + (16u << 20);
constexpr u32 WS_END = WS_YP + 8u * NMSG * YSLOT;
constexpr u32 PK_TAIL = 8192 + 2048;

constexpr u32 STG = 28672, NSTG = 5;
constexpr u32 LSLOT = 6144;
constexpr u32 L_CTL = 143360;
static_assert(STG * NSTG <= L_CTL && NWORK * 2 * LSLOT <= L_CTL, "ring");
constexpr u32 C_CALL = 0, C_ABORT = 1, C_TICKET = 2, C_TICKET2 = 3, C_FULL = 8, C_FREE = 16,
              C_TK = 2, C_IS = 3, C_LAND = 4, C_YGO = 5, C_YT = 6, C_SEAT = 7, C_YL = 8, C_Q = 64, C_WORDS = 128,
              C_LUT = C_WORDS;
constexpr u32 C_LUT2 = C_LUT + 128, L_PST = L_CTL + (C_LUT2 + 32) * 4, PST = 4352;
constexpr u32 LDS_BYTES = L_PST + 4 * PST;
#define ST2(c, i, v) do {} while (0)
#define ST3(c, k, tt, v) do {} while (0)
constexpr u32 L_LUT = L_CTL + C_LUT * 4, L_LUT2 = L_CTL + C_LUT2 * 4, L_YST = 3 * STG, YST = 4608;
static_assert(L_LUT % 512 == 0 && L_YST + 8 * YST <= 5 * STG && 4 * (32 + 64 * 17) <= YST && 1536 + 4 * (32 + 64 * 17) <= LSLOT, "y staging");
static_assert(LDS_BYTES <= 160 * 1024, "lds");

constexpr u64 DEADLINE = 40000000ull;
constexpr u64 DEADLINE_LDS = 100000000ull;

constexpr u32 YM_RAW = 1u << 16, YM_BAD = 1u << 17;

struct Order { u16 v[7][NCHIN]; };
#define D1V 6
#define D2V 2
#define D3V 0
constexpr Order make_order() {
    Order o{};
    for (int d = 1; d <= 7; ++d) {
        int n = 0;
        for (int pass = 0; pass < 4; ++pass)
            for (int s = 0; s < 8; ++s) {
                if (pass == (s >= 8 - D1V ? 3 : 0) && s) for (int q = 0; q < 4; ++q) o.v[d - 1][n++] = (u16)(s << 8 | q);
                for (int c = 0; c < 16; ++c) if (pass == ((c & 2) && s >= 8 - D3V ? 1 : 0)) o.v[d - 1][n++] = (u16)(0x8000 | s << 8 | c);
                if (pass == (s >= 8 - D1V ? 3 : 0) && !s) for (int q = 0; q < 4; ++q) o.v[d - 1][n++] = (u16)(s << 8 | q);
                for (int i = 0; i < 28; ++i) {
                    const int q = (i / 14) * 2 + (i & 1), own = 1 + ((i % 14) / 2 + d) % 7;
                    if (pass == (q >= 2 && own != 4 && s >= 8 - D2V ? 2 : 0)) o.v[d - 1][n++] = (u16)(s << 8 | own << 2 | q);
                }
            }
    }
    return o;
}
__constant__ const Order ORDER = make_order();

struct Args { const u16* x; const u16* w; u16* out; char* me; int rank; int pad; char* ws[NR]; };

DEV u64 rt() { return __builtin_amdgcn_s_memrealtime(); }
DEV u32 rfl(u32 v) { return __builtin_amdgcn_readfirstlane(v); }
DEV u32 perm(u32 hi, u32 lo, u32 sel) { return __builtin_amdgcn_perm(hi, lo, sel); }
DEV u32 bfi(u32 m, u32 a, u32 b) { return (m & a) | (~m & b); }
#define CBAR() asm volatile("" ::: "memory")
#define WAIT_VM0() asm volatile("s_waitcnt vmcnt(0)" ::: "memory")
#define WAIT_LGKM0() do { CBAR(); __builtin_amdgcn_s_waitcnt(0xc07f); CBAR(); } while (0)

#define STAMPW(c, i) do {} while (0)
#define STAMPC(c, i) do {} while (0)

struct Ctx {
    const u16* x; const u16* w; u16* out; char* me; char* peer_lane;
    rsrc_t rs, ro;
    LAS char* lds; volatile LAS u32* ctl;
    int rank, wg, wave, lane;
};

DEV u32 ld_sys(const void* p) { return __hip_atomic_load((const GAS u32*)p, __ATOMIC_RELAXED, __HIP_MEMORY_SCOPE_SYSTEM); }
DEV u64 ld_sys64(const void* p) { return __hip_atomic_load((const GAS u64*)p, __ATOMIC_RELAXED, __HIP_MEMORY_SCOPE_SYSTEM); }
DEV void st_sys(void* p, u32 v) { __hip_atomic_store((GAS u32*)p, v, __ATOMIC_RELAXED, __HIP_MEMORY_SCOPE_SYSTEM); }
DEV void st_sys64(void* p, u64 v) { __hip_atomic_store((GAS u64*)p, v, __ATOMIC_RELAXED, __HIP_MEMORY_SCOPE_SYSTEM); }
DEV void st16(const Ctx& c, u32x4 v, u32 voff, u32 soff) { __builtin_amdgcn_raw_buffer_store_b128(v, c.rs, voff + soff, 0, 17); }
DEV void st_out(const Ctx& c, u32 off, u32x2 a, u32x2 b) {
    u32x4 v;
#pragma unroll
    for (int i = 0; i < 2; ++i) {
        v[i] = __builtin_amdgcn_update_dpp(a[i], b[i], 0x128, 0xf, 0xc, false);
        v[2 + i] = __builtin_amdgcn_update_dpp(b[i], a[i], 0x128, 0xf, 0x3, false);
    }
    __builtin_amdgcn_raw_buffer_store_b128(v, c.ro, off, 0, OAUX);
}
DEV u32 out_off(const Ctx& c, u32 row0, u32 col0) { return (row0 + (c.lane >> 4) * 4 + (c.lane >> 3 & 1)) * (NTOT * 2) + (col0 + (c.lane & 7) * 8) * 2; }
DEV bool tmo_set(const Ctx& c, u32 call) { return rfl(__hip_atomic_load((const GAS u32*)(c.me + WS_TMO), __ATOMIC_RELAXED, __HIP_MEMORY_SCOPE_AGENT)) == call; }
DEV void tmo_raise(const Ctx& c, u32 call) { if (c.lane == 0) __hip_atomic_store((GAS u32*)(c.me + WS_TMO), call, __ATOMIC_RELAXED, __HIP_MEMORY_SCOPE_AGENT); }

DEV bool expired(u64& t0, u64 limit) { const u64 t = rt(); if (!t0) t0 = t; return t - t0 > limit; }

DEV u32 lds_wait(const Ctx& c, u32 word, u32 want, bool look_abort, int nap) {
    u64 t0 = 0;
    for (u32 n = 1;; ++n) {
        const u32 v = rfl(c.ctl[word]);
        if (v >= want) return v;
        if (nap) __builtin_amdgcn_s_sleep(1);
        if ((n & 63) == 0) {
            if (look_abort && rfl(c.ctl[C_ABORT])) return 0;
            if (expired(t0, DEADLINE_LDS)) { c.ctl[C_ABORT] = 1; return 0; }
        }
    }
}
DEV u32 get_call(const Ctx& c) { return rfl(c.ctl[C_CALL]); }

DEV void glds4(rsrc_t rs, u32 voff, u32 soff, u32 ldsaddr) {
    u32 keep;
    if ((DIAG & 4) && soff >= WS_XA && soff < WS_YP) return;
    if ((DIAG & 8) && soff >= WS_XA && soff < WS_YP) soff = WS_XA + (soff & 0x7ffff);
    asm volatile("s_mov_b32 %0, m0\n\ts_mov_b32 m0, %4\n\ts_nop 2\n\t"
                 "buffer_load_dwordx4 %1, %2, %3 offen sc0 sc1 lds\n\t"
                 "buffer_load_dwordx4 %1, %2, %3 offen offset:1024 sc0 sc1 lds\n\t"
                 "buffer_load_dwordx4 %1, %2, %3 offen offset:2048 sc0 sc1 lds\n\t"
                 "buffer_load_dwordx4 %1, %2, %3 offen offset:3072 sc0 sc1 lds\n\t"
                 "s_mov_b32 m0, %0" : "=&s"(keep) : "v"(voff), "s"(rs), "s"(soff), "s"(ldsaddr) : "memory");
}
DEV void glds2(rsrc_t rs, u32 voff, u32 soff, u32 ldsaddr) {
    u32 keep;
    asm volatile("s_mov_b32 %0, m0\n\ts_mov_b32 m0, %4\n\ts_nop 2\n\t"
                 "buffer_load_dwordx4 %1, %2, %3 offen sc0 sc1 lds\n\t"
                 "buffer_load_dwordx4 %1, %2, %3 offen offset:1024 sc0 sc1 lds\n\t"
                 "s_mov_b32 m0, %0" : "=&s"(keep) : "v"(voff), "s"(rs), "s"(soff), "s"(ldsaddr) : "memory");
}
DEV u32x4 ld16(rsrc_t rs, u32 voff, u32 soff) { return __builtin_amdgcn_raw_buffer_load_b128(rs, voff, soff, 17); }
DEV u32 lds_inc(const Ctx& c, u32 word) {
    u32 v = 0;
    if (c.lane == 0) v = __hip_atomic_fetch_add((LAS u32*)&c.ctl[word], 1u, __ATOMIC_RELAXED, __HIP_MEMORY_SCOPE_WORKGROUP);
    return rfl(v);
}

DEV u32 wave_max(u32 v) {
    v = max(v, (u32)__builtin_amdgcn_update_dpp(0, (int)v, 0x111, 0xf, 0xf, true));
    v = max(v, (u32)__builtin_amdgcn_update_dpp(0, (int)v, 0x112, 0xf, 0xf, true));
    v = max(v, (u32)__builtin_amdgcn_update_dpp(0, (int)v, 0x114, 0xf, 0xf, true));
    v = max(v, (u32)__builtin_amdgcn_update_dpp(0, (int)v, 0x118, 0xf, 0xf, true));
    v = max(v, (u32)__builtin_amdgcn_update_dpp(0, (int)v, 0x142, 0xa, 0xf, false));
    v = max(v, (u32)__builtin_amdgcn_update_dpp(0, (int)v, 0x143, 0xc, 0xf, false));
    return __builtin_amdgcn_readlane(v, 63);
}

DEV u32 wave_scan(u32 v) {
    v += (u32)__builtin_amdgcn_update_dpp(0, (int)v, 0x111, 0xf, 0xf, true);
    v += (u32)__builtin_amdgcn_update_dpp(0, (int)v, 0x112, 0xf, 0xf, true);
    v += (u32)__builtin_amdgcn_update_dpp(0, (int)v, 0x114, 0xf, 0xf, true);
    v += (u32)__builtin_amdgcn_update_dpp(0, (int)v, 0x118, 0xf, 0xf, true);
    v += (u32)__builtin_amdgcn_update_dpp(0, (int)v, 0x142, 0xa, 0xf, false);
    v += (u32)__builtin_amdgcn_update_dpp(0, (int)v, 0x143, 0xc, 0xf, false);
    return v;
}
DEV u32 lane_rank(u64 m) { return __builtin_amdgcn_mbcnt_hi((u32)(m >> 32), __builtin_amdgcn_mbcnt_lo((u32)m, 0)); }

DEV void store_raw(const Ctx& c, const u32x4 (&R)[16], u32 off0, u32 off1) {
#pragma unroll
    for (int f = 0; f < 16; ++f) st16(c, R[f], c.lane * 16, (f < 8 ? off0 : off1) + (f & 7) * 1024);
}
DEV u32 sad(u32 a, u32 acc) { return __builtin_amdgcn_sad_u8(a, 0, acc); }
DEV u32 pk_sub(u32 a, u32 b) { return __builtin_bit_cast(u32, (u16x2)(__builtin_bit_cast(u16x2, a) - __builtin_bit_cast(u16x2, b))); }
DEV u32 pack_store(const Ctx& c, u32x4 (&X)[16], u32 dst) {
    const u32 ldsbase = (u32)(size_t)c.lds, stg = L_PST + (c.wave & 3) * PST;
#pragma unroll
    for (int i = 0; i < 5; ++i) *(LAS u32x4*)(c.lds + stg + c.lane * 16 + min(i * 1024u, PST - 1024)) = (u32x4){0, 0, 0, 0};
    u32 sum = 0;
#pragma unroll
    for (int f = 0; f < 16; ++f)
#pragma unroll
        for (int i = 0; i < 4; ++i) X[f][i] = __builtin_amdgcn_alignbit(X[f][i], X[f][i], 31);
#pragma unroll
    for (int f = 0; f < 16; ++f) asm volatile("" : "+v"(X[f]));
#pragma unroll
    for (int f = 0; f < 16; f += 2) sum = sad(perm(X[f][1], X[f][0], 0x07050301u), sum);
#pragma unroll
    for (int i = 0; i < 8; ++i)
        st16(c, (u32x4){perm(X[2 * i][1], X[2 * i][0], 0x06040200u), perm(X[2 * i][3], X[2 * i][2], 0x06040200u),
                        perm(X[2 * i + 1][1], X[2 * i + 1][0], 0x06040200u), perm(X[2 * i + 1][3], X[2 * i + 1][2], 0x06040200u)}, c.lane * 16, dst + i * 1024);
    const u32 top = min(max(((__builtin_amdgcn_readlane(wave_scan(sum), 63) + 1946) >> 11) + 2, 12u), 255u), K = (top << 8 | 0xff) * 0x10001u;
    u32 nexc = 0, hv = 0, D[8], W[16], rel[16], bits = 0;
#pragma unroll
    for (int j = 0; j < 8; ++j) {
        u32 t[4], cf[4], es[4];
        u32x2 sel[4];
#pragma unroll
        for (int g = 0; g < 4; ++g) {
            const u32x4 x = X[2 * j + (g >> 1)];
            t[g] = perm(pk_sub(K, x[(g & 1) * 2 + 1]), pk_sub(K, x[(g & 1) * 2]), 0x07050301u);
            cf[g] = perm(0, 0x07060500u, t[g]);
            es[g] = ~cf[g] & 0x04040404u;
            sel[g] = *(const LAS u32x2*)(size_t)__builtin_amdgcn_udot4(es[g], 0x10080402u, ldsbase + L_LUT2, false);
        }
        D[j] = bfi(0x03030303u, cf[0], bfi(0x0c0c0c0cu, cf[1] << 2, bfi(0x30303030u, cf[2] << 4, cf[3] << 6)));
#pragma unroll
        for (int h = 0; h < 2; ++h) {
            const int f = 2 * j + h;
            for (u64 m = __ballot((cf[2 * h] | cf[2 * h + 1]) & 0x80808080u); m && nexc <= 31; m &= m - 1) {
                const u32 ln = __builtin_ctzll(m);
                const u32 r0 = __builtin_amdgcn_readlane(X[f][0], ln), r1 = __builtin_amdgcn_readlane(X[f][1], ln);
                const u32 r2 = __builtin_amdgcn_readlane(X[f][2], ln), r3 = __builtin_amdgcn_readlane(X[f][3], ln);
                for (u32 fl = (__builtin_amdgcn_readlane(cf[2 * h], ln) & 0x80808080u) >> 1 | (__builtin_amdgcn_readlane(cf[2 * h + 1], ln) & 0x80808080u); fl; fl &= fl - 1) {
                    const u32 b = __builtin_ctz(fl), x = b & 1 ? (b & 16 ? r3 : r2) : (b & 16 ? r1 : r0), r = x >> 1 | x << 31;
                    ++nexc;
                    hv = writelane((f << 9 | ln << 3 | (b & 1) << 2 | b >> 3) << 16 | ((r >> (b & 8) * 2) & 0xffff), nexc & 63, hv);
                }
            }
            const u32 sa = sad(es[2 * h], 0);
            const u32 wa = perm(0, t[2 * h], sel[2 * h][0]) | perm(0, t[2 * h], sel[2 * h][1]) << 4;
            const u32 wb = perm(0, t[2 * h + 1], sel[2 * h + 1][0]) | perm(0, t[2 * h + 1], sel[2 * h + 1][1]) << 4;
            W[f] = wa | wb << sa;
            rel[f] = bits;
            bits = sad(es[2 * h + 1], bits + sa);
            asm volatile("" : "+v"(W[f]), "+v"(bits));
        }
        asm volatile("" : "+v"(D[j]));
    }
    if (FORCE_RAW_IN || nexc > 31) {
#pragma unroll
        for (int f = 0; f < 16; ++f)
#pragma unroll
            for (int i = 0; i < 4; ++i) X[f][i] = __builtin_amdgcn_alignbit(X[f][i], X[f][i], 1);
        store_raw(c, X, dst, dst + 8192);
        return YM_RAW | 1024;
    }
    st16(c, (u32x4){D[0], D[1], D[2], D[3]}, c.lane * 16, dst + 8192);
    st16(c, (u32x4){D[4], D[5], D[6], D[7]}, c.lane * 16, dst + 9216);
    hv = writelane(top | nexc << 8, 0, hv);
    if ((u32)c.lane <= nexc) *(LAS u32*)(c.lds + stg + c.lane * 4) = hv;
    const u32 incl = wave_scan(bits), start = 32 * (1 + nexc) + incl - bits, bytes = 4 * (1 + nexc) + ((__builtin_amdgcn_readlane(incl, 63) + 7) >> 3);
#pragma unroll
    for (int f = 0; f < 16; ++f) {
        const u32 pb = start + rel[f];
        const u64 v = (u64)W[f] << (pb & 31);
        LAS u32* q = (LAS u32*)(size_t)(ldsbase + stg + (pb >> 5) * 4);
        __hip_atomic_fetch_or(q, (u32)v, __ATOMIC_RELAXED, __HIP_MEMORY_SCOPE_WAVEFRONT);
        __hip_atomic_fetch_or(q + 1, (u32)(v >> 32), __ATOMIC_RELAXED, __HIP_MEMORY_SCOPE_WAVEFRONT);
    }
    CBAR();
    u32x4 v[5];
#pragma unroll
    for (int i = 0; i < 5; ++i) if (i < 2 || bytes > i * 1024u) v[i] = *(const LAS u32x4*)(c.lds + stg + c.lane * 16 + min(i * 1024u, PST - 1024));
#pragma unroll
    for (int i = 0; i < 5; ++i) if (i < 2 || bytes > i * 1024u) st16(c, v[i], c.lane * 16, dst + PK_TAIL + min(i * 1024u, PST - 1024));
    return (PK_TAIL + bytes + 15) >> 4;
}

DEV u32 xa_off(int ks, int h, int rowfrag) { return WS_XA + (u32)((ks * 2 + h) * 256 + rowfrag) * 1024u; }
DEV u32 wb_off(int ks, int h, int colfrag) { return WS_WB + (u32)((ks * 2 + h) * 128 + colfrag) * 1024u; }

DEV void wg_barrier() { WAIT_LGKM0(); __builtin_amdgcn_s_barrier(); CBAR(); }

DEV void pack_job(const Ctx& c, int job) {
    const int s = job / 160, q = job % 160, ks = c.rank * 8 + s;
    const int l15 = c.lane & 15, lq = c.lane >> 4;
    const bool isx = q < 32;
    const int dst = (c.rank + 1 + (q - 32)) & 7, cc = (q - 32) >> 3;
    u32x4 R[16];
    if (job < 160) STAMPW(c, 13);
    if (isx) {
        const GAS char* src = (const GAS char*)c.x + (size_t)(q * 128 + l15) * (KSH * 2) + (s * 64 + lq * 8) * 2;
#pragma unroll
        for (int f = 0; f < 16; ++f) R[f] = *(const GAS u32x4*)(src + (f & 7) * 16 * (KSH * 2) + (f >> 3) * 64);
    } else {
        const GAS char* src = (const GAS char*)c.w + (size_t)(s * 64 + lq * 8) * (NTOT * 2) + (dst * 2048 + cc * 128 + (l15 & 7) * 8 + (l15 >> 3) * 4) * 2;
        u32x2 r[4][8];
#pragma unroll
        for (int hg = 0; hg < 4; ++hg)
#pragma unroll
            for (int j = 0; j < 8; ++j) r[hg][j] = *(const GAS u32x2*)(src + (size_t)((hg >> 1) * 32 + j) * (NTOT * 2) + (hg & 1) * 128);
#pragma unroll
        for (int hg = 0; hg < 4; ++hg)
#pragma unroll
            for (int ii = 0; ii < 4; ++ii)
#pragma unroll
                for (int t = 0; t < 4; ++t)
                    R[hg * 4 + ii][t] = perm(r[hg][2 * t + 1][ii >> 1], r[hg][2 * t][ii >> 1], (ii & 1) ? 0x07060302u : 0x05040100u);
    }
    if (job < 160) wg_barrier();
    __builtin_amdgcn_s_waitcnt(0x0f70);
    if (job < 160) STAMPW(c, 4);
    const u32 call = get_call(c), ch = isx ? s * 32 + q : 256 + s * 16 + cc;
    const u32 slot = WS_PK + (call & 1) * PK_PAR + (isx ? ch : ch + dst * 128) * CHB;
    const bool mine = isx || dst == c.rank, raw = !mine && ch < 256 + RAW0;
    if (mine || raw) store_raw(c, R, raw ? slot : isx ? xa_off(ks, 0, q * 8) : wb_off(ks, 0, cc * 8), raw ? slot + 8192 : isx ? xa_off(ks, 1, q * 8) : wb_off(ks, 1, cc * 8));
    u32 meta = YM_RAW | 1024;
    if (isx || !(mine || raw)) meta = pack_store(c, R, slot);
    if (job < 160) STAMPW(c, 5);
    WAIT_VM0();
    if (c.lane < NR && (isx || c.lane == dst)) {
        if (c.lane == c.rank) st_sys(c.me + (isx ? WS_XR + q * 256 : WS_WR + cc * 256) + ks * 4, call);
        else st_sys64(c.peer_lane + WS_FIN + (((call & 1) * NR + c.rank) * NCHIN + ch) * 8, (u64)call << 32 | meta);
    }
}

struct Tile { int cc, owner, h, rf, rfo; };
constexpr u32 TK2 = 130;

template <bool BIG>
DEV void loader(const Ctx& c, const Tile& t, int L, u32 xr, u32 call) {
    const int which = c.lane & 3, myks = L * 16 + ((c.lane >> 2) ^ xr), rc = BIG && which == 2 ? t.rfo >> 3 : (t.rf >> 3) + (which & 1);
    const char* fp = c.me + (which == 3 ? WS_WR + t.cc * 256 : WS_XR + rc * 256) + myks * 4;
    const u32 ldsbase = (u32)(size_t)c.lds, voff = c.lane * 16;
    u64 todo = 0x1111111111111111ull, avail = 0, own = 0, late = 0;
    u32 pend = 0, pslot = 0;
    bool ok = true, second = false;
    while (ok) {
        if (!todo) {
            if (!BIG || second || !late) break;
            second = true; todo = late;
        }
        if (!avail) {
            if (pend) { WAIT_VM0(); c.ctl[C_FULL + pslot] = pend; ST3(c, 2, pend / 2 - 1, 3); pend = 0; }
            u64 t0 = 0;
            for (u32 n = 1;; ++n) {
                const u64 m = __ballot(ld_sys(fp) >= call);
                own = m >> 2;
                avail = (second ? own : m & m >> 1 & m >> 3) & todo;
                if (avail) break;
                if ((n & 15) == 0) {
                    if (rfl(c.ctl[C_ABORT]) || tmo_set(c, call)) { ok = false; break; }
                    if (expired(t0, DEADLINE)) { tmo_raise(c, call); ok = false; break; }
                }
                __builtin_amdgcn_s_sleep(LSLEEP);
            }
            if (!ok) break;
            if (avail == todo && tmo_set(c, call)) { ok = false; break; }
        }
        const int bit = __builtin_ctzll(avail), ks = L * 16 + ((bit >> 2) ^ xr);
        avail &= avail - 1; todo &= ~(1ull << bit);
        const u32 full = BIG && !second && ((own >> bit) & 1);
        if (BIG && !second && !full) late |= 1ull << bit;
        u32 tk = 0;
        if (c.lane == 0) tk = __hip_atomic_fetch_add((LAS u32*)&c.ctl[second ? C_TICKET2 : C_TICKET], second ? 1u : 2u, __ATOMIC_RELAXED, __HIP_MEMORY_SCOPE_WORKGROUP);
        tk = rfl(tk) + (second ? TK2 : 0);
#pragma unroll
        for (int h = 0; h < 2; ++h) {
            const u32 tt = tk + h, slot = tt % NSTG, need = 8 * (tt / NSTG);
            ST3(c, 5, tt, L);
            if (rfl(c.ctl[C_FREE + slot]) < need) {
                if (pend) { WAIT_VM0(); c.ctl[C_FULL + pslot] = pend; ST3(c, 2, pend / 2 - 1, 1); pend = 0; }
                if (!lds_wait(c, C_FREE + slot, need, true, 1)) { ok = false; break; }
            }
            ST3(c, 0, tt, L);
            const u32 l = ldsbase + slot * STG;
            if (BIG && second) {
                glds4(c.rs, voff, xa_off(ks, 0, t.rfo), l); glds4(c.rs, voff, xa_off(ks, 1, t.rfo), l + 4096);
#pragma unroll
                for (int g = 0; g < 4; ++g) glds4(c.rs, voff, wb_off(ks, g >> 1, t.cc * 8 + (g & 1) * 4), l + 8192 + g * 4096);
            } else {
#pragma unroll
                for (int g = 0; g < 4; ++g) glds4(c.rs, voff, xa_off(ks, h, t.rf + g * 4), l + g * 4096);
                glds4(c.rs, voff, wb_off(ks, h, t.cc * 8), l + 20480); glds4(c.rs, voff, wb_off(ks, h, t.cc * 8 + 4), l + 24576);
            }
            if (full) {
                glds4(c.rs, voff, xa_off(ks, h, t.rfo), l + 16384);
                ST3(c, 1, tt, 1);
                asm volatile("s_waitcnt vmcnt(28)" ::: "memory");
            } else { ST3(c, 1, tt, 0); asm volatile("s_waitcnt vmcnt(24)" ::: "memory"); }
            if (pend) { c.ctl[C_FULL + pslot] = pend; ST3(c, 2, pend / 2 - 1, 2); }
            pend = 2 * tt + 2 + full; pslot = slot;
            if (BIG && second) break;
        }
    }
    WAIT_VM0();
    if (!ok) c.ctl[C_ABORT] = 1;
    else if (pend) { c.ctl[C_FULL + pslot] = pend; ST3(c, 2, pend / 2 - 1, 4); }
}

DEV u32 pk_relu(float a, float b) {
    const f32x2 v = __builtin_elementwise_maximum((f32x2){a, b}, (f32x2){0.f, 0.f});
    return __builtin_bit_cast(u32, __builtin_convertvector(v, bf16x2));
}

DEV u32 nz8(u32 cb) { return (cb + 0x07070707u) & 0x08080808u; }
template <int NF>
DEV u32 encode_y(const Ctx& c, const u32 (&P)[NF * 8], u32 dst) {
    constexpr int G = NF * 4;
    constexpr u32 CODEB = NF == 5 ? 2048 : 1536;
    u16x2 mx = {0, 0}, mn[NF];
#pragma unroll
    for (int f = 0; f < NF; ++f) {
        mn[f] = (u16x2){0xffff, 0xffff};
#pragma unroll
        for (int j = 0; j < 8; ++j) {
            const u16x2 v = __builtin_bit_cast(u16x2, P[f * 8 + j]);
            mx = __builtin_elementwise_max(mx, v);
            mn[f] = __builtin_elementwise_min(mn[f], (u16x2)(v - (u16x2){1, 1}));
        }
    }
    const u32 base = max(wave_max(max((u32)mx[0], (u32)mx[1])) >> 8, 7u) - 7, lim = ((base + 1) << 8) - 1;
    const u32 ldsbase = (u32)(size_t)c.lds, stg = L_YST + c.wave * YST;
    u32 nexc = 0, hv = 0;
#pragma unroll
    for (int f = 0; f < NF; ++f)
        for (u64 m = __ballot(min((u32)mn[f][0], (u32)mn[f][1]) < lim); m && nexc <= 31; m &= m - 1) {
            const u32 ln = __builtin_ctzll(m);
            u32 t = 0;
#pragma unroll
            for (int k = 0; k < 8; ++k) t = writelane(__builtin_amdgcn_readlane(P[f * 8 + k], ln), k, t);
            for (u32 mk = (u32)__ballot((t & 0xffff) - 1 < lim) | (u32)__ballot((t >> 16) - 1 < lim) << 8; mk; mk &= mk - 1) {
                const u32 k = __builtin_ctz(mk), v = (__builtin_amdgcn_readlane(t, k & 7) >> (k & 8) * 2) & 0xffff;
                ++nexc;
                hv = writelane((f * 16 + (k & 7) * 2 + (k >> 3)) << 22 | ln << 16 | v, nexc & 63, hv);
            }
        }
    if (FORCE_RAW_Y || nexc > 31) {
#pragma unroll
        for (int i = 0; i < NF * 2; ++i) st16(c, (u32x4){P[4 * i], P[4 * i + 1], P[4 * i + 2], P[4 * i + 3]}, c.lane * 16, dst + i * 1024);
        return YM_RAW | NF * 128;
    }
    hv = writelane(base | nexc << 8, 0, hv);
    if ((u32)c.lane <= nexc) *(LAS u32*)(c.lds + stg + c.lane * 4) = hv;
    u32 cb[G], lb[G], cnt = 0;
    const u16x2 B = {(u16)(base << 8), (u16)(base << 8)};
#pragma unroll
    for (int g = 0; g < G; ++g) {
        const u32 t0 = __builtin_bit_cast(u32, __builtin_elementwise_sub_sat(__builtin_bit_cast(u16x2, P[2 * g]), B));
        const u32 t1 = __builtin_bit_cast(u32, __builtin_elementwise_sub_sat(__builtin_bit_cast(u16x2, P[2 * g + 1]), B));
        cb[g] = perm(t1, t0, 0x07050301u); lb[g] = perm(t1, t0, 0x06040200u);
        cnt += __builtin_popcount(nz8(cb[g]));
    }
    u32 D[8];
#pragma unroll
    for (int h = 0; h < 2; ++h) {
        const u32* q = cb + 8 * h;
        D[3 * h] = q[0] | q[1] << 3 | (q[6] & 0x03030303u) << 6;
        D[3 * h + 1] = q[2] | q[3] << 3 | (q[7] & 0x03030303u) << 6;
        D[3 * h + 2] = q[4] | q[5] << 3 | (q[6] & 0x04040404u) << 4 | (q[7] & 0x04040404u) << 5;
    }
    st16(c, (u32x4){D[0], D[1], D[2], D[3]}, c.lane * 16, dst);
    if (NF == 5) st16(c, (u32x4){D[4], D[5], cb[G - 4] | cb[G - 3] << 3, cb[G - 2] | cb[G - 1] << 3}, c.lane * 16, dst + 1024);
    else __builtin_amdgcn_raw_buffer_store_b64((u32x2){D[4], D[5]}, c.rs, c.lane * 8 + dst + 1024, 0, 17);
    const u32 dw = (cnt >> 2) + 1, incl = wave_scan(dw), total = 1 + nexc + __builtin_amdgcn_readlane(incl, 63);
    u32 pos = ldsbase + stg + 4 * (1 + nexc + incl - dw), adr[G];
    u32x2 sel[G];
#pragma unroll
    for (int g = 0; g < G; ++g) {
        const u32 nz = nz8(cb[g]);
        adr[g] = pos & ~3u;
        sel[g] = *(const LAS u32x2*)(size_t)__builtin_amdgcn_udot4(nz, 0x20100804u, ((pos & 3) << 3) | (ldsbase + L_LUT), false);
        pos += __builtin_popcount(nz);
    }
    __builtin_amdgcn_sched_barrier(0);
    u32 cur = 0;
#pragma unroll
    for (int g = 0; g < G; ++g) {
        const u32 o = perm(lb[g], cur, sel[g][0]);
        *(LAS u32*)(size_t)adr[g] = o;
        cur = perm(lb[g], o, sel[g][1]);
    }
    *(LAS u32*)(size_t)(pos & ~3u) = cur;
    CBAR();
    u32x4 v[5];
#pragma unroll
    for (int i = 0; i < 5; ++i) if (i < 3 || total > i * 256u) v[i] = *(const LAS u32x4*)(c.lds + stg + c.lane * 16 + min(i * 1024u, YST - 1024));
#pragma unroll
    for (int i = 0; i < 5; ++i) if (i < 3 || total > i * 256u) st16(c, v[i], c.lane * 16, dst + CODEB + min(i * 1024u, YST - 1024));
    return (CODEB + total * 4 + 15) >> 4;
}

DEV f32x4 mfma(u32x4 a, u32x4 b, f32x4 acc) { return __builtin_amdgcn_mfma_f32_16x16x32_bf16(__builtin_bit_cast(bf16x8, a), __builtin_bit_cast(bf16x8, b), acc, 0, 0, 0); }
DEV void lds_add1(const Ctx& c, u32 word) {
    asm volatile("s_mov_b64 exec, 1\n\tds_add_u32 %0, %1\n\ts_mov_b64 exec, -1" :: "v"((u32)(size_t)&c.ctl[word]), "v"(1u) : "memory");
}
DEV void slot_free(const Ctx& c, u32 slot) { lds_add1(c, C_FREE + slot); }

template <bool BIG>
DEV void consumer(const Ctx& c, const Tile& t, int wr, int wc) {
    f32x4 acc[4][4], oacc[4];
#pragma unroll
    for (int i = 0; i < 4; ++i) {
        oacc[i] = (f32x4){0.f, 0.f, 0.f, 0.f};
#pragma unroll
        for (int f = 0; f < 4; ++f) acc[f][i] = (f32x4){0.f, 0.f, 0.f, 0.f};
    }
    const LAS char* pa = c.lds + c.lane * 16 + wr * 4096;
    const LAS char* po = c.lds + c.lane * 16 + wr * 1024;
    const LAS char* pb = c.lds + c.lane * 16 + wc * 4096;
    u32 slot = 0, seen = 0, nfull = 0;
    bool ok = true;
    struct Ops { u32x4 a[4], b[4], ao; u32 full; } X, Y;
    auto fetch = [&](Ops& o, u32 tn) {
        u32 sv = rfl(seen);
        if (ok && tn < 128 && sv < 2 * tn + 2) { sv = lds_wait(c, C_FULL + slot, 2 * tn + 2, true, 0); ok = sv != 0; }
        CBAR();
        if (c.wave == 0) ST3(c, 3, tn, 0);

        if (tn == 0) STAMPW(c, 1);
        if (tn == 64) STAMPW(c, 4);
        if (tn == 96) STAMPW(c, 5);
        if (tn == 112) { STAMPW(c, 6); STAMPC(c, 13); }
        if (tn == 0) STAMPC(c, 12);
        if (tn == 120) STAMPW(c, 7);
        o.full = sv & 1;
#pragma unroll
        for (int i = 0; i < 4; ++i) o.b[i] = *(const LAS u32x4*)(pb + slot * STG + 20480 + i * 1024);
#pragma unroll
        for (int f = 0; f < 4; ++f) o.a[f] = *(const LAS u32x4*)(pa + slot * STG + f * 1024);
        if (BIG) o.ao = *(const LAS u32x4*)(po + slot * STG + 16384);
        __builtin_amdgcn_sched_barrier(0);
    };
    auto release = [&](u32 tn) {
        __builtin_amdgcn_sched_barrier(0);
        WAIT_LGKM0();
        if (tn < 128) slot_free(c, slot);

        slot = slot == NSTG - 1 ? 0 : slot + 1;
        seen = c.ctl[C_FULL + slot];
        __builtin_amdgcn_sched_barrier(0);
    };
    auto step = [&](const Ops& o, Ops& nx, u32 tn) {
        fetch(nx, tn);
#pragma unroll
        for (int f = 0; f < 3; ++f)
#pragma unroll
            for (int i = 0; i < 4; ++i) acc[f][i] = mfma(o.a[f], o.b[i], acc[f][i]);
        release(tn);
#pragma unroll
        for (int i = 0; i < 4; ++i) acc[3][i] = mfma(o.a[3], o.b[i], acc[3][i]);
        if (BIG && o.full) {
            ++nfull;
#pragma unroll
            for (int i = 0; i < 4; ++i) oacc[i] = mfma(o.ao, o.b[i], oacc[i]);
        }
    };
    fetch(X, 0);
    release(0);
#pragma unroll 1
    for (u32 tk = 0; tk < 128; tk += 2) { step(X, Y, tk + 1); step(Y, X, tk + 2); }
    STAMPW(c, 2); STAMPC(c, 14);
    ok = ok && !rfl(c.ctl[C_ABORT]);
    {
        const u32 call = get_call(c), msg = (t.cc * 2 + t.h) * 8 + c.wave;
        u32 P[32];
#pragma unroll
        for (int g = 0; g < 16; ++g) {
            P[2 * g] = pk_relu(acc[g >> 2][0][g & 3], acc[g >> 2][1][g & 3]);
            P[2 * g + 1] = pk_relu(acc[g >> 2][2][g & 3], acc[g >> 2][3][g & 3]);
        }
        ok = ok && lds_wait(c, C_FREE + 3, 200, true, 0) && lds_wait(c, C_FREE + 4, 200, true, 0);
        CBAR();
        STAMPW(c, 8);
        u32 meta = YM_BAD;
        if (ok) meta = encode_y<4>(c, P, WS_YP + (t.owner * NMSG + msg) * YSLOT);
        STAMPW(c, 9);
        WAIT_VM0();
        if (c.lane == t.owner) st_sys64(c.peer_lane + WS_FY + (c.rank * NMSG + msg) * 8, (u64)call << 32 | meta);
    }
    STAMPW(c, 3);
    if (BIG) {
        slot_free(c, 3); slot_free(c, 4);
        slot = 0; seen = 0;
        const u32 end = TK2 + 64 - (nfull >> 1);
        struct Own { u32x4 a[2], b[8]; } U, V;
        auto fetch2 = [&](Own& o, u32 tn) {
            if (ok && tn < end && rfl(seen) < 2 * tn + 2) ok = lds_wait(c, C_FULL + slot, 2 * tn + 2, true, 0) != 0;
            CBAR();
            if (c.wave == 0) ST3(c, 3, tn, 0);

#pragma unroll
            for (int i = 0; i < 8; ++i) o.b[i] = *(const LAS u32x4*)(pb + slot * STG + 8192 + (i >> 2) * 8192 + (i & 3) * 1024);
#pragma unroll
            for (int f = 0; f < 2; ++f) o.a[f] = *(const LAS u32x4*)(po + slot * STG + f * 4096);
            __builtin_amdgcn_sched_barrier(0);
        };
        auto step2 = [&](const Own& o, Own& nx, u32 tn) {
            fetch2(nx, tn);
            if (tn <= end)
#pragma unroll
                for (int i = 0; i < 8; ++i) oacc[i & 3] = mfma(o.a[i >> 2], o.b[i], oacc[i & 3]);
            release(tn < end ? 0 : 128);
        };
        fetch2(U, TK2);
        release(TK2 < end ? 0 : 128);
#pragma unroll 1
        for (u32 tk = TK2; tk < end; tk += 2) { step2(U, V, tk + 1); step2(V, U, tk + 2); }
        ok = ok && !rfl(c.ctl[C_ABORT]);
        const u32 o = out_off(c, ((t.rfo + wr) & 31) * 16, c.rank * 2048 + t.cc * 128 + wc * 64);
#pragma unroll
        for (int r = 0; r < 4; r += 2)
            st_out(c, o + r * (NTOT * 2), ok ? (u32x2){pk_relu(oacc[0][r], oacc[1][r]), pk_relu(oacc[2][r], oacc[3][r])} : (u32x2){0x7fc07fc0u, 0x7fc07fc0u},
                   ok ? (u32x2){pk_relu(oacc[0][r + 1], oacc[1][r + 1]), pk_relu(oacc[2][r + 1], oacc[3][r + 1])} : (u32x2){0x7fc07fc0u, 0x7fc07fc0u});
        STAMPW(c, 10);
    }
}

DEV void gemm_wg(const Ctx& c) {
    const int g = c.wg - NCOMM, xcd = g & 7, idx = g >> 3, grp = xcd >> 2, k = idx % 7;
    const bool big = k < 4;
    Tile t;
    t.cc = (xcd & 3) * 4 + idx / 7;
    t.h = grp * 4 + k >= 7 || k >= 4;
    t.owner = (c.rank + (grp ? (k < 3 ? 5 + k : 1 + k) : (k < 4 ? 1 + k : k - 3))) & 7;
    t.rf = t.owner * 32 + t.h * 16; t.rfo = c.rank * 32 + (grp * 4 + (k & 3)) * 4;
    if (c.wave >= 8) {
        const int L = c.wave - 8;
        if (L * NGEMM + g >= 160) {
            wg_barrier();
            for (int i = 0; i < PSLEEP + PSTEP * L; ++i) __builtin_amdgcn_s_sleep(24);
        }
#pragma unroll 1
        for (int job = L * NGEMM + g; job < 1280; job += 4 * NGEMM) {
            if (job >= 4 * NGEMM) for (int i = 0; i < P2; ++i) __builtin_amdgcn_s_sleep(24);
            pack_job(c, job);
            if (job < 4 * NGEMM) STAMPW(c, 3);
        }
        STAMPW(c, 1);
        const u32 call = get_call(c), xr = XROT ? (t.cc & 1) | (t.cc & 2) << 2 | (k & 3) << 1 : 0;
        if (big) loader<true>(c, t, L, xr, call); else loader<false>(c, t, L, xr, call);
        STAMPW(c, 2);
    } else {
        wg_barrier();
        if (big) consumer<true>(c, t, c.wave >> 1, c.wave & 1); else consumer<false>(c, t, c.wave >> 1, c.wave & 1);
    }
}

#define WSUM 8
#define EARLY 0
#define WCLV 4

constexpr u32 NFIX = 4;

DEV u32 take_seat(const Ctx& c, u32 call) {
    u32 xcc;
    asm volatile("s_getreg_b32 %0, hwreg(HW_REG_XCC_ID, 0, 4)" : "=s"(xcc));
    const u32 grp = (0xb2u >> c.rank) & 1 ? 0xb2u : 0x4du, j = (c.wg >> 3) * 4 + (xcc & 3);
    u32 near = TOPO ? (xcc & 4 ? ~grp & 0xff : grp & ~(1u << c.rank)) : 0, seat = NPULL;
    const u32 n = __builtin_popcount(near);
    if (!TOPO) seat = min((u32)c.wg, (u32)NPULL);
    else if (j < 4 * n) {
        for (u32 i = (j + SPREAD * (n == 4 ? j / n : 0)) % n; i; --i) near &= near - 1;
        seat = ((__builtin_ctz(near) - c.rank) & 7) - 1 + 7 * (j / n);
    }
    for (bool first = true;; first = false) {
        u64 old = ~0ull;
        if (c.lane == 0 ? seat < NPULL : first && c.lane < NR)
            old = __hip_atomic_fetch_max((GAS u64*)(c.me + (c.lane ? WS_CLM + c.lane * 128 : WS_SEAT + seat * 8)), (u64)call << 32 | 4 * NFIX, __ATOMIC_RELAXED, __HIP_MEMORY_SCOPE_AGENT);
        if (rfl((u32)(old >> 32)) < call) return seat;
        if (first) for (int i = 0; i < 4; ++i) __builtin_amdgcn_s_sleep(32);
        const u64 v = __hip_atomic_load((const GAS u64*)(c.me + WS_SEAT + min(c.lane, NPULL - 1) * 8), __ATOMIC_RELAXED, __HIP_MEMORY_SCOPE_AGENT);
        const u64 open = __ballot((u32)(v >> 32) < call);
        if (!open) return NPULL;
        seat = __builtin_ctzll(open);
    }
}

DEV void watcher(const Ctx& c, int p, int sub, u32 call) {
    static_assert(NQY == 64, "one message per lane");
    const u32 id = sub + 4 * c.lane;
    const char* fp = c.me + WS_FY + (p * NMSG + id) * 8;
    bool bad = !lds_wait(c, C_YGO, 1, false, 1);
    STAMPW(c, 1);
    u64 pending = ~0ull, t0 = 0;
    u32 qn = 0;
    for (u32 it = 1; pending; ++it) {
        const u64 v = bad ? 0 : ld_sys64(fp);
        const bool mine = ((pending >> c.lane) & 1) && (bad || (u32)(v >> 32) >= call);
        const u64 m = __ballot(mine);
        if (m) {
            const bool poison = bad || tmo_set(c, call);
            if (mine) c.ctl[C_Q + qn + lane_rank(m)] = 0x80000000u | id << 20 | (poison ? YM_BAD : (u32)v & (YM_BAD | YM_RAW | 0x3ff));
            qn += __popcll(m); pending &= ~m; t0 = 0;
        } else {
            if ((it & 15) == 0) {
                if (tmo_set(c, call)) bad = true;
                else if (expired(t0, DEADLINE)) { tmo_raise(c, call); bad = true; }
            }
            __builtin_amdgcn_s_sleep(2);
        }
    }
    STAMPW(c, 2);
}

DEV void pull_inputs(const Ctx& c, char* peer, rsrc_t pr, u32 lo, int d, int sub, int p, u32 call) {
    GAS u64* ctr = (GAS u64*)(c.me + WS_CLM + d * 128);
    const u32 voff = c.lane * 16, win = WSUM / 4 + (sub < WSUM % 4), WCL = win + WCLV;
    bool bad = false;
#pragma unroll 1
    for (;;) {
        const u32 t = lds_inc(c, C_TK);
        if (t >= WCL) lds_wait(c, C_LAND, t - WCL + 1, false, 1);
        u32 n = t * 4 + sub;
        if (t >= NFIX) {
            if (c.lane == 0) n = (u32)__hip_atomic_fetch_add(ctr, 1ull, __ATOMIC_RELAXED, __HIP_MEMORY_SCOPE_AGENT);
            n = rfl(n);
        }
        if (n + 64 >= NCHIN) c.ctl[C_YGO] = 1;
        if (n >= NCHIN) break;
        const u32 code = rfl(ORDER.v[d - 1][n]), s = (code >> 8) & 7, ix = code & 31;
        const bool isw = code & 0x8000;
        const u32 rc = isw ? ix : ((((ix >> 2) + c.rank) & 7) << 2 | (ix & 3)), id = isw ? 256 + s * 16 + rc : s * 32 + rc;
        const char* fp = c.me + WS_FIN + (((call & 1) * NR + p) * NCHIN + id) * 8;
        u32 meta = 0;
        u64 t0 = 0;
        for (u32 it = 1; !bad; ++it) {
            const u64 v = ld_sys64(fp);
            meta = rfl((u32)v);
            if (rfl((u32)(v >> 32)) >= call) break;
            if ((it & 15) == 0) {
                if (tmo_set(c, call)) bad = true;
                else if (expired(t0, DEADLINE)) { tmo_raise(c, call); bad = true; }
            }
            __builtin_amdgcn_s_sleep(2);
        }
        if (bad) { lds_inc(c, C_LAND); continue; }
        const u32 u = lds_inc(c, C_IS);
        if (u >= win) lds_wait(c, C_LAND, u - win + 1, false, 0);
        ST2(c, 1024 + u, 13);
        const int ks = p * 8 + s;
        const u32 src = WS_PK + (call & 1) * PK_PAR + (isw ? id + c.rank * 128 : id) * CHB;
        const u32 d0 = isw ? wb_off(ks, 0, rc * 8) : xa_off(ks, 0, rc * 8), d1 = isw ? wb_off(ks, 1, rc * 8) : xa_off(ks, 1, rc * 8);
        u32x4 R[16];
        if (meta & YM_RAW) {
#pragma unroll
            for (int f = 0; f < 16; ++f) R[f] = ld16(pr, voff + (f & 3) * 1024, src + (f >> 2) * 4096);
            WAIT_VM0();
            lds_inc(c, C_LAND);
            ST2(c, 1536 + u, 0); ST2(c, u, 0);
            store_raw(c, R, d0, d1);
        } else {
            const u32 bytes = min((meta & 0x7ff) * 16, PK_TAIL + LSLOT), tl = (u32)(size_t)c.lds + lo;
            const rsrc_t tr = __builtin_amdgcn_make_buffer_rsrc(peer, 0, rfl(src + bytes), 0x00020000);
            if (bytes > PK_TAIL + 2048) glds4(tr, voff, src + PK_TAIL + 2048, tl + 2048);
            glds2(tr, voff, src + PK_TAIL, tl);
            u32x4 L[8], D[2];
#pragma unroll
            for (int i = 0; i < 2; ++i) D[i] = ld16(pr, voff + i * 1024, src + 8192);
#pragma unroll
            for (int i = 0; i < 8; ++i) L[i] = ld16(pr, voff + (i & 3) * 1024, src + (i >> 2) * 4096);
            asm volatile("s_waitcnt vmcnt(%0)" :: "n"(EARLY) : "memory");
            if (c.lane == 0) __hip_atomic_fetch_add((LAS u32*)&c.ctl[C_LAND], 1u, __ATOMIC_RELAXED, __HIP_MEMORY_SCOPE_WORKGROUP);
            WAIT_VM0();
            ST2(c, 1536 + u, 0); ST2(c, u, 0);
            const u32 hv = *(const LAS u32*)(size_t)(tl + (c.lane & 31) * 4);
            u32 cnt = 0;
#pragma unroll
            for (int j = 0; j < 8; ++j) cnt += __builtin_popcount(~(D[j >> 2][j & 3] | D[j >> 2][j & 3] >> 1) & 0x55555555u);
            const u32 h0 = rfl(hv), T = (h0 & 0xff) * 0x01010101u, nexc = min((h0 >> 8) & 0xff, 31u);
            u32 pb = 32 * (1 + nexc) + 4 * (wave_scan(cnt) - cnt);
#pragma unroll
            for (int hf = 0; hf < 16; hf += 8) {
                u32 cd[16], sel[16], sa[8], ps[8];
                u32x2 q[8];
#pragma unroll
                for (int f = 0; f < 8; ++f) {
                    const u32 w = D[hf >> 3][f >> 1] >> (f & 1) * 4;
                    cd[2 * f] = w & 0x03030303u; cd[2 * f + 1] = (w >> 2) & 0x03030303u;
                    const u32 ea = (0x04040404u - cd[2 * f]) & 0x04040404u, eb = (0x04040404u - cd[2 * f + 1]) & 0x04040404u;
                    sel[2 * f] = *(const LAS u32*)(size_t)__builtin_amdgcn_udot4(ea, 0x08040201u, (u32)(size_t)c.lds + L_LUT2, false);
                    sel[2 * f + 1] = *(const LAS u32*)(size_t)__builtin_amdgcn_udot4(eb, 0x08040201u, (u32)(size_t)c.lds + L_LUT2, false);
                    const LAS u32* z = (const LAS u32*)(size_t)(tl + (pb >> 5) * 4);
                    q[f] = (u32x2){z[0], z[1]};
                    sa[f] = sad(ea, 0); ps[f] = pb;
                    pb = sad(eb, pb + sa[f]);
                }
                __builtin_amdgcn_sched_barrier(0);
#pragma unroll
                for (int f = 0; f < 8; ++f) {
                    const u32 va = __builtin_amdgcn_alignbit(q[f][1], q[f][0], ps[f]), vb = va >> sa[f];
                    const u32 ha = T - ((perm(va >> 4, va, sel[2 * f]) & 0x0f0f0f0fu) | cd[2 * f]), hb = T - ((perm(vb >> 4, vb, sel[2 * f + 1]) & 0x0f0f0f0fu) | cd[2 * f + 1]);
                    const u32 la = L[(hf + f) >> 1][(f & 1) * 2], lb = L[(hf + f) >> 1][(f & 1) * 2 + 1];
                    const u32x4 x = {perm(ha, la, 0x05010400u), perm(ha, la, 0x07030602u), perm(hb, lb, 0x05010400u), perm(hb, lb, 0x07030602u)};
                    u32x4 r;
#pragma unroll
                    for (int i = 0; i < 4; ++i) r[i] = __builtin_amdgcn_alignbit(x[i], x[i], 1);
                    st16(c, r, voff, (hf ? d1 : d0) + f * 1024);
                }
            }
            if ((u32)c.lane - 1 < nexc)
                __builtin_amdgcn_raw_buffer_store_b16((u16)hv, c.rs, (hv >> 28 & 1 ? d1 : d0) + (hv >> 25 & 7) * 1024 + (hv >> 16 & 511) * 2, 0, 17);
        }
        STAMPW(c, 3);
        WAIT_VM0();
        if (c.lane == 0) st_sys(c.me + (isw ? WS_WR : WS_XR) + rc * 256 + ks * 4, call);
        ST2(c, 512 + u, 0);
    }
    lds_inc(c, C_LAND);
}

template <int NF>
DEV void decode_y(const Ctx& c, u32 e, u32 lo, rsrc_t pr, u32 src, u32 row0, u32 col0, u32 tk) {
    constexpr int G = NF * 4;
    constexpr u32 CODEB = NF == 5 ? 2048 : 1536;
    const u32 ldsbase = (u32)(size_t)c.lds;
    const LAS char* l = c.lds + lo;
    u32x2 P[G];
    u32 hv = 0, nexc = 0;
    if (e & YM_BAD) {
#pragma unroll
        for (int g = 0; g < G; ++g) P[g] = (u32x2){0x7fc07fc0u, 0x7fc07fc0u};
    } else if (e & YM_RAW) {
#pragma unroll
        for (int i = 0; i < G / 2; ++i) {
            const u32x4 v = ld16(pr, c.lane * 16 + (i & 3) * 1024, src + (i >> 2) * 4096);
            P[2 * i] = (u32x2){v[0], v[1]}; P[2 * i + 1] = (u32x2){v[2], v[3]};
        }
        __builtin_amdgcn_s_waitcnt(0x0f70);
    } else {
        const u32x4 da = *(const LAS u32x4*)(l + c.lane * 16);
        u32x4 db;
        if (NF == 5) db = *(const LAS u32x4*)(l + 1024 + c.lane * 16);
        else { const u32x2 t = *(const LAS u32x2*)(l + 1024 + c.lane * 8); db = (u32x4){t[0], t[1], 0, 0}; }
        hv = *(const LAS u32*)(l + CODEB + (c.lane & 31) * 4);
        const u32 D[8] = {da[0], da[1], da[2], da[3], db[0], db[1], db[2], db[3]};
        u32 cb[G], cnt = 0;
#pragma unroll
        for (int h = 0; h < 2; ++h) {
            u32* q = cb + 8 * h;
#pragma unroll
            for (int k = 0; k < 3; ++k) { q[2 * k] = D[3 * h + k] & 0x07070707u; q[2 * k + 1] = (D[3 * h + k] >> 3) & 0x07070707u; }
            q[6] = bfi(0x03030303u, D[3 * h] >> 6, D[3 * h + 2] >> 4) & 0x07070707u;
            q[7] = bfi(0x03030303u, D[3 * h + 1] >> 6, D[3 * h + 2] >> 5) & 0x07070707u;
        }
        if (NF == 5) { cb[G - 4] = D[6] & 0x07070707u; cb[G - 3] = (D[6] >> 3) & 0x07070707u; cb[G - 2] = D[7] & 0x07070707u; cb[G - 1] = (D[7] >> 3) & 0x07070707u; }
#pragma unroll
        for (int g = 0; g < G; ++g) cnt += __builtin_popcount(nz8(cb[g]));
        const u32 h0 = rfl(hv), B = (h0 & 0xff) * 0x01010101u, dw = (cnt >> 2) + 1;
        nexc = min((h0 >> 8) & 0xff, 31u);
        u32 pos = ldsbase + lo + CODEB + 4 * (1 + nexc + wave_scan(dw) - dw), ps[G];
        u32x2 sel[G];
#pragma unroll
        for (int g = 0; g < G; ++g) {
            const u32 nz = nz8(cb[g]);
            const LAS u32* q = (const LAS u32*)(size_t)(pos & ~3u);
            sel[g] = *(const LAS u32x2*)(size_t)__builtin_amdgcn_udot4(nz, 0x08040201u, ldsbase + L_LUT, false);
            P[g] = (u32x2){q[0], q[1]};
            ps[g] = pos;
            pos += __builtin_popcount(nz);
        }
        __builtin_amdgcn_sched_barrier(0);
#pragma unroll
        for (int g = 0; g < G; ++g) {
            const u32 by = __builtin_amdgcn_alignbyte(P[g][1], P[g][0], ps[g]), lb = perm(by, by, sel[g][0]), hb = (cb[g] + B) & sel[g][1];
            P[g] = (u32x2){perm(hb, lb, 0x05010400u), perm(hb, lb, 0x07030602u)};
        }
    }
#pragma unroll
    for (int g = 0; g < G; g += 2) st_out(c, out_off(c, row0, col0) + ((g >> 2) * 16 + (g & 3)) * (NTOT * 2), P[g], P[g + 1]);
    ST2(c, 512 + 384 + tk, nexc);
    const u32 j = (hv >> 22) & 63, ln = hv >> 16, xo = (row0 + (j & 48) + (ln & 48) / 4 + (j >> 2 & 3)) * (NTOT * 2) + (col0 + (ln & 7) * 8 + (ln & 8) / 2 + (j & 3)) * 2;
    __builtin_amdgcn_raw_buffer_store_b16((u16)hv, c.ro, (u32)c.lane - 1 < nexc ? xo : ~1u, 0, OAUX);
}
DEV void fetch_y(const Ctx& c, char* peer, u32 e, u32 lo) {
    if (e & (YM_BAD | YM_RAW)) return;
    const u32 src = WS_YP + (c.rank * NMSG + ((e >> 20) & 0xff)) * YSLOT, bytes = min((e & 0x3ff) * 16, LSLOT);
    const rsrc_t rs = __builtin_amdgcn_make_buffer_rsrc(peer, 0, src + bytes, 0x00020000);
    glds4(rs, c.lane * 16, src, (u32)(size_t)c.lds + lo);
    if (bytes > 4096) glds2(rs, c.lane * 16, src + 4096, (u32)(size_t)c.lds + lo + 4096);
}

DEV void worker(const Ctx& c, char* peer, int d, int sub, int p, int w, u32 call) {
    const rsrc_t pr = __builtin_amdgcn_make_buffer_rsrc(peer, 0, WS_END, 0x00020000);
    pull_inputs(c, peer, pr, w * 2 * LSLOT, d, sub, p, call);
    STAMPW(c, 1);
    u32 lo = w * 2 * LSLOT, lo2 = lo + LSLOT, e = 0;
#pragma unroll 1
    for (u32 tk = lds_inc(c, C_YT); tk < NQY;) {
        if (e) asm volatile("s_waitcnt vmcnt(9)" ::: "memory");
        else {
            e = lds_wait(c, C_Q + tk, 1, false, 1);
            if (!e) { tmo_raise(c, call); break; }
            if (YWIN && tk >= YWIN) lds_wait(c, C_YL, tk - YWIN + 1, false, 0);
            ST2(c, 1024 + 448 + tk, 4);
            fetch_y(c, peer, e, lo);
            WAIT_VM0();
        }
        if (YWIN) lds_add1(c, C_YL);
        ST2(c, 1536 + 448 + tk, 0); ST2(c, 448 + tk, 0);
        if (tk == 0) STAMPW(c, 4);
        STAMPW(c, 5);
        const u32 nt = lds_inc(c, C_YT), ne = YPRE && nt < NQY && (!YWIN || rfl(c.ctl[C_YL]) + YWIN > nt) ? rfl(c.ctl[C_Q + nt]) : 0;
        if (ne) { ST2(c, 1024 + 448 + nt, 5); fetch_y(c, peer, ne, lo2); }
        const u32 id = (e >> 20) & 0xff, wc = id & 1, rg = (id >> 1) & 7, cc = id >> 4;
        decode_y<4>(c, DIAG & 1 ? e | YM_BAD : e, lo, pr, WS_YP + (c.rank * NMSG + id) * YSLOT, rg * 64, p * 2048 + cc * 128 + wc * 64, tk);
        ST2(c, 512 + 448 + tk, 0);
        tk = nt; e = ne;
        const u32 sw = lo; lo = lo2; lo2 = sw;
    }
    STAMPW(c, 2);
}

__global__ void __launch_bounds__(NTHREADS, 3) fused_kernel(Args a) {
    extern __shared__ __attribute__((aligned(128))) char smem[];
    Ctx c;
    c.x = a.x; c.w = a.w; c.out = a.out; c.rank = a.rank & 7; c.me = a.me;
    c.wg = blockIdx.x; c.wave = rfl(threadIdx.x >> 6); c.lane = threadIdx.x & 63;
    c.peer_lane = a.ws[c.lane & 7];
    c.rs = __builtin_amdgcn_make_buffer_rsrc(c.me, 0, WS_END, 0x00020000);
    c.ro = __builtin_amdgcn_make_buffer_rsrc(c.out, 0, DIAG & 2 ? 0 : 512 * NTOT * 2, 0x00020000);
    c.lds = (LAS char*)smem; c.ctl = (volatile LAS u32*)(c.lds + L_CTL);
    STAMPW(c, 0);
    u32 init = 0;
    static_assert(C_CALL == 0, "thread 0");
    if (threadIdx.x == 0) {
        GAS u32* cnt = (GAS u32*)(c.me + WS_CNT + c.wg * 128);
        init = __hip_atomic_load(cnt, __ATOMIC_RELAXED, __HIP_MEMORY_SCOPE_AGENT) + 1;
        __hip_atomic_store(cnt, init, __ATOMIC_RELAXED, __HIP_MEMORY_SCOPE_AGENT);
    }
    if (threadIdx.x < C_WORDS) c.ctl[threadIdx.x] = init;
    if (c.wave == 11) {
        u32 s0, s1;
        if (c.wg >= NCOMM) {
            const u32 mask = c.lane >> 2;
            u32 n = c.lane & 3;
            u64 sel = 0x0c0c0c0c03020100ull;
            for (int b = 0; b < 4; ++b)
                if ((mask >> b) & 1) { sel = (sel & ~(0xffull << (8 * n))) | (u64)(4 + b) << (8 * n); ++n; }
            s0 = (u32)sel; s1 = n >= 4 ? (u32)(sel >> 32) : 0x03020100u;
        } else {
            u32 n = 0;
            s0 = 0x0c0c0c0cu; s1 = 0;
            for (int b = 0; b < 4; ++b)
                if ((c.lane >> b) & 1) { s0 = (s0 & ~(0xffu << (8 * b))) | n << (8 * b); s1 |= 0xffu << (8 * b); ++n; }
        }
        c.ctl[C_LUT + 2 * c.lane] = s0; c.ctl[C_LUT + 2 * c.lane + 1] = s1;
    }
    if (c.wave == 10 && c.lane < 16) {
        u32 n = 0, s0 = 0x0c0c0c0cu, s1 = 0x0c0c0c0cu, sd = 0x0c0c0c0cu;
        for (int b = 0; b < 4; ++b)
            if ((c.lane >> b) & 1) {
                u32& t = n & 1 ? s1 : s0;
                t = (t & ~(0xffu << (n >> 1) * 8)) | b << (n >> 1) * 8;
                sd = (sd & ~(0xffu << 8 * b)) | ((n & 1) * 4 + (n >> 1)) << 8 * b;
                ++n;
            }
        if (c.wg >= NCOMM) { c.ctl[C_LUT2 + 2 * c.lane] = s0; c.ctl[C_LUT2 + 2 * c.lane + 1] = s1; }
        else c.ctl[C_LUT2 + c.lane] = sd;
    }
    if (c.wg >= NCOMM) gemm_wg(c);
    else {
        wg_barrier();
        const u32 call = get_call(c);
        if (c.wave == 0) c.ctl[C_SEAT] = take_seat(c, call) + 1;
        const u32 seat = lds_wait(c, C_SEAT, 1, false, 1) - 1;
        if (seat >= NPULL) return;
        const int d = 1 + seat % 7, sub = seat / 7, p = (c.rank + d) & 7;
        if (c.wave == 0) watcher(c, p, sub, call);
        else worker(c, a.ws[p], d, sub, p, c.wave - 1, call);
    }
}

}

extern "C" void kernel_launch(int rank, int nranks, ncclComm_t comm, hipStream_t stream,
                              void* const* d_in, const int* in_sizes, int n_in,
                              void* d_out, int out_size, const int* in_dtypes, int out_dtype,
                              void* const* d_out_all, void* const* d_ws_all, size_t ws_bytes) {
    if (nranks != NR || (unsigned)rank >= (unsigned)NR || n_in != 2 || ws_bytes < WS_END || in_sizes[0] != 4096 * KSH ||
        in_sizes[1] != KSH * NTOT || out_size != 512 * NTOT) {
        fprintf(stderr, "unexpected problem shape\n");
        return;
    }
    HIP_CHECK(hipFuncSetAttribute((const void*)fused_kernel, hipFuncAttributeMaxDynamicSharedMemorySize, LDS_BYTES));
    Args a;
    a.x = (const u16*)d_in[0]; a.w = (const u16*)d_in[1]; a.out = (u16*)d_out;
    for (int p = 0; p < NR; ++p) a.ws[p] = (char*)d_ws_all[p];
    a.me = a.ws[rank]; a.rank = rank; a.pad = 0;
    hipLaunchKernelGGL(fused_kernel, dim3(NWG), dim3(NTHREADS), LDS_BYTES, stream, a);
    HIP_CHECK(hipGetLastError());
}
```

```cpp
#include <hip/hip_runtime.h>
#include <rccl/rccl.h>
#include <cstdio>
#include <cstdint>

#define HIP_CHECK(expr) do { hipError_t _e = (expr); if (_e != hipSuccess) { \
    fprintf(stderr, "HIP error %s at %s:%d\n", hipGetErrorString(_e), __FILE__, __LINE__); \
    return; } } while (0)

extern "C" __device__ unsigned writelane(unsigned value, unsigned lane, unsigned old) __asm("llvm.amdgcn.writelane.i32");

namespace {

typedef unsigned u32;
typedef unsigned long long u64;
typedef unsigned short u16;
typedef u32 u32x4 __attribute__((ext_vector_type(4)));
typedef u32 u32x2 __attribute__((ext_vector_type(2)));
typedef u32 u32x8 __attribute__((ext_vector_type(8)));
typedef u32 u32x32 __attribute__((ext_vector_type(32)));
typedef u16 u16x2 __attribute__((ext_vector_type(2)));
typedef float f32x4 __attribute__((ext_vector_type(4)));
typedef float f32x2 __attribute__((ext_vector_type(2)));
typedef __bf16 bf16x2 __attribute__((ext_vector_type(2)));
typedef __bf16 bf16x8 __attribute__((ext_vector_type(8)));
typedef __amdgpu_buffer_rsrc_t rsrc_t;
#define LAS __attribute__((address_space(3)))
#define GAS __attribute__((address_space(1)))
#define DEV __device__ __forceinline__

#define PSLEEP 3
#define LSLEEP 4
#define SPREAD 0
#define PSTEP 1
#define PS1 4
#define PS2 5
#define PS3 6
#define P2 0
#define L4 1
#define PACE 27
#define PFREE 3
#define NPUSH 16
#define JROT 2
#define YPAIR 0
#define YPEND 12
#define YWIN 7
#define RELF 3
#define LNAP 1
#define LPRIO 0
#define MAXP 5
#define OWNALL 0
#define UPRIO 2
#define EPRIO 2
#define WNAP 0
#define CNAP 0
#define CPRIO 0
#define OAUX 17
#define TOPO 1
#define DIAG 0
#define XROT 1
#define FORCE_RAW_Y 0
#define FORCE_RAW_IN 0

constexpr int NR = 8;
constexpr int NTOT = 16384, KSH = 512;
constexpr int NWG = 256, NCOMM = 32, NPULL = 28, NGEMM = NWG - NCOMM, NTHREADS = 768;
constexpr int NCHIN = 384;
constexpr int NMSG = 256;
constexpr u32 NQY = NMSG / 4;
constexpr int NWORK = 11;

constexpr u32 WS_TMO = 128, WS_CLM = 1024;
constexpr u32 WS_SEAT = 2048;
constexpr u32 WS_FIN = 4096;
constexpr u32 WS_FY = WS_FIN + 2 * NR * NCHIN * 8;
constexpr u32 WS_XR = WS_FY + NR * NMSG * 8;
constexpr u32 WS_WR = WS_XR + 8192;
constexpr u32 WS_CNT = WS_WR + 4096;
constexpr u32 WS_DBG = 131072;
static_assert(WS_CNT + NWG * 128 <= WS_DBG, "ws");
constexpr u32 CHB = 16384;
constexpr u32 WS_PK = 1u << 20;
constexpr u32 PK_PAR = 1280u * CHB;
constexpr u32 WS_XA = WS_PK + 2 * PK_PAR;
constexpr u32 WS_WB = WS_XA + (32u << 20);
constexpr u32 YSLOT = 8192;
constexpr u32 WS_YP = WS_WB + (20u << 20);
constexpr u32 WS_END = WS_YP + 8u * NMSG * YSLOT;
constexpr u32 PK_TAIL = 8192 + 2048;

constexpr u32 STG = 28672, NSTG = 5;
constexpr u32 LSLOT = 6144;
constexpr u32 L_CTL = STG * NSTG;
static_assert(NWORK * 2 * LSLOT + 8192 <= L_CTL, "ring");
constexpr u32 C_CALL = 0, C_ABORT = 1, C_FULL = 8, C_FREE = 16, C_LE = 24, C_LIST = 64,
              C_TK = 2, C_IS = 3, C_LAND = 4, C_YGO = 5, C_YT = 6, C_QN = 7, C_YL = 8, C_SEAT = 9, C_Q = 64, C_WORDS = 256,
              C_LUT = C_WORDS;
constexpr u32 C_LUT2 = C_LUT + 128, L_PST = L_CTL + (C_LUT2 + 32) * 4, PST = 4352;
constexpr u32 LDS_BYTES = L_PST + 4 * PST;
#define ST2(c, i, v) do {} while (0)
constexpr u32 YS = 128 % NSTG, L_LUT = L_CTL + C_LUT * 4, L_LUT2 = L_CTL + C_LUT2 * 4, L_YST = YS * STG, YST = 4608;
static_assert(L_LUT % 512 == 0 && YS + 1 < NSTG && 8 * YST <= 2 * STG && 4 * (32 + 64 * 17) <= YST && 1536 + 4 * (32 + 64 * 17) <= LSLOT, "y staging");
static_assert(LDS_BYTES <= 160 * 1024, "lds");

constexpr u64 DEADLINE = 40000000ull;
constexpr u64 DEADLINE_LDS = 100000000ull;

constexpr u32 YM_RAW = 1u << 16, YM_BAD = 1u << 17;

constexpr int NPOS = NCHIN - NPUSH;
static_assert(NPUSH <= 24, "push");
struct Order { u16 v[7][NPOS]; };
#define D1V 6
#define D2V 2
#define D3V 0
constexpr Order make_order() {
    Order o{};
    for (int d = 1; d <= 7; ++d) {
        int n = 0;
        for (int pass = 0; pass < 4; ++pass)
            for (int s = 0; s < 8; ++s) {
                if (pass == (s >= 8 - D1V ? 3 : 0) && s) for (int q = 0; q < 4; ++q) o.v[d - 1][n++] = (u16)(s << 8 | q);
                for (int c = 0; c < 16; ++c) if (pass == ((c & 2) && s >= 8 - D3V ? 1 : 0) && s * 16 + c >= NPUSH) o.v[d - 1][n++] = (u16)(0x8000 | s << 8 | c);
                if (pass == (s >= 8 - D1V ? 3 : 0) && !s) for (int q = 0; q < 4; ++q) o.v[d - 1][n++] = (u16)(s << 8 | q);
                for (int i = 0; i < 28; ++i) {
                    const int q = (i / 14) * 2 + (i & 1), own = 1 + ((i % 14) / 2 + d) % 7;
                    if (pass == (q >= 2 && own != 4 && s >= 8 - D2V ? 2 : 0)) o.v[d - 1][n++] = (u16)(s << 8 | own << 2 | q);
                }
            }
    }
    return o;
}
__constant__ const Order ORDER = make_order();

struct Args { const u16* x; const u16* w; u16* out; char* me; int rank; int pad; char* ws[NR]; };

DEV u64 rt() { return __builtin_amdgcn_s_memrealtime(); }
DEV u32 rfl(u32 v) { return __builtin_amdgcn_readfirstlane(v); }
DEV u32 perm(u32 hi, u32 lo, u32 sel) { return __builtin_amdgcn_perm(hi, lo, sel); }
DEV u32 bfi(u32 m, u32 a, u32 b) { return (m & a) | (~m & b); }
#define CBAR() asm volatile("" ::: "memory")
#define WAIT_VM0() asm volatile("s_waitcnt vmcnt(0)" ::: "memory")
#define WAIT_LGKM0() do { CBAR(); __builtin_amdgcn_s_waitcnt(0xc07f); CBAR(); } while (0)

#define STAMPW(c, i) do {} while (0)
#define STAMPC(c, i) do {} while (0)

struct Ctx {
    const u16* x; const u16* w; u16* out; char* me; char* peer_lane;
    rsrc_t rs, ro;
    LAS char* lds; volatile LAS u32* ctl;
    int rank, wg, wave, lane;
};

DEV u32 ld_sys(const void* p) { return __hip_atomic_load((const GAS u32*)p, __ATOMIC_RELAXED, __HIP_MEMORY_SCOPE_SYSTEM); }
DEV u64 ld_sys64(const void* p) { return __hip_atomic_load((const GAS u64*)p, __ATOMIC_RELAXED, __HIP_MEMORY_SCOPE_SYSTEM); }
DEV void st_sys(void* p, u32 v) { __hip_atomic_store((GAS u32*)p, v, __ATOMIC_RELAXED, __HIP_MEMORY_SCOPE_SYSTEM); }
DEV void st_sys64(void* p, u64 v) { __hip_atomic_store((GAS u64*)p, v, __ATOMIC_RELAXED, __HIP_MEMORY_SCOPE_SYSTEM); }
DEV void st16(const Ctx& c, u32x4 v, u32 voff, u32 soff) { __builtin_amdgcn_raw_buffer_store_b128(v, c.rs, voff + soff, 0, 17); }
DEV void st_out(const Ctx& c, u32 off, u32x2 a, u32x2 b) {
    u32x4 v;
#pragma unroll
    for (int i = 0; i < 2; ++i) {
        v[i] = __builtin_amdgcn_update_dpp(a[i], b[i], 0x128, 0xf, 0xc, false);
        v[2 + i] = __builtin_amdgcn_update_dpp(b[i], a[i], 0x128, 0xf, 0x3, false);
    }
    __builtin_amdgcn_raw_buffer_store_b128(v, c.ro, off, 0, OAUX);
}
DEV u32 out_off(const Ctx& c, u32 row0, u32 col0) { return (row0 + (c.lane >> 4) * 4 + (c.lane >> 3 & 1)) * (NTOT * 2) + (col0 + (c.lane & 7) * 8) * 2; }
DEV bool tmo_set(const Ctx& c, u32 call) { return rfl(__hip_atomic_load((const GAS u32*)(c.me + WS_TMO), __ATOMIC_RELAXED, __HIP_MEMORY_SCOPE_AGENT)) == call; }
DEV void tmo_raise(const Ctx& c, u32 call) { if (c.lane == 0) __hip_atomic_store((GAS u32*)(c.me + WS_TMO), call, __ATOMIC_RELAXED, __HIP_MEMORY_SCOPE_AGENT); }

DEV bool expired(u64& t0, u64 limit) { const u64 t = rt(); if (!t0) t0 = t; return t - t0 > limit; }

DEV u32 lds_wait(const Ctx& c, u32 word, u32 want, bool look_abort, int nap, u32 mask = ~0u) {
    u64 t0 = 0;
    for (u32 n = 1;; ++n) {
        const u32 v = rfl(c.ctl[word]) & mask;
        if (v >= want) return v;
        if (nap) __builtin_amdgcn_s_sleep(1);
        if ((n & 63) == 0) {
            if (look_abort && rfl(c.ctl[C_ABORT])) return 0;
            if (expired(t0, DEADLINE_LDS)) { c.ctl[C_ABORT] = 1; return 0; }
        }
    }
}
DEV u32 get_call(const Ctx& c) { return rfl(c.ctl[C_CALL]); }

DEV void glds4(rsrc_t rs, u32 voff, u32 soff, u32 ldsaddr) {
    u32 keep;
    asm volatile("s_mov_b32 %0, m0\n\ts_mov_b32 m0, %4\n\ts_nop 2\n\t"
                 "buffer_load_dwordx4 %1, %2, %3 offen sc0 sc1 lds\n\t"
                 "buffer_load_dwordx4 %1, %2, %3 offen offset:1024 sc0 sc1 lds\n\t"
                 "buffer_load_dwordx4 %1, %2, %3 offen offset:2048 sc0 sc1 lds\n\t"
                 "buffer_load_dwordx4 %1, %2, %3 offen offset:3072 sc0 sc1 lds\n\t"
                 "s_mov_b32 m0, %0" : "=&s"(keep) : "v"(voff), "s"(rs), "s"(soff), "s"(ldsaddr) : "memory");
}
#define GPOL " sc0 sc1"
DEV void glds4g(rsrc_t rs, u32 voff, u32 soff, u32 ldsaddr) {
    u32 keep;
    if (DIAG & 4) return;
    if (DIAG & 8) soff = WS_XA + (soff & 0x7ffff);
    asm volatile("s_mov_b32 %0, m0\n\ts_mov_b32 m0, %4\n\ts_nop 2\n\t"
                 "buffer_load_dwordx4 %1, %2, %3 offen" GPOL " lds\n\t"
                 "buffer_load_dwordx4 %1, %2, %3 offen offset:1024" GPOL " lds\n\t"
                 "buffer_load_dwordx4 %1, %2, %3 offen offset:2048" GPOL " lds\n\t"
                 "buffer_load_dwordx4 %1, %2, %3 offen offset:3072" GPOL " lds\n\t"
                 "s_mov_b32 m0, %0" : "=&s"(keep) : "v"(voff), "s"(rs), "s"(soff), "s"(ldsaddr) : "memory");
}
DEV void glds2(rsrc_t rs, u32 voff, u32 soff, u32 ldsaddr) {
    u32 keep;
    asm volatile("s_mov_b32 %0, m0\n\ts_mov_b32 m0, %4\n\ts_nop 2\n\t"
                 "buffer_load_dwordx4 %1, %2, %3 offen sc0 sc1 lds\n\t"
                 "buffer_load_dwordx4 %1, %2, %3 offen offset:1024 sc0 sc1 lds\n\t"
                 "s_mov_b32 m0, %0" : "=&s"(keep) : "v"(voff), "s"(rs), "s"(soff), "s"(ldsaddr) : "memory");
}
DEV u32x4 ld16(rsrc_t rs, u32 voff, u32 soff) { return __builtin_amdgcn_raw_buffer_load_b128(rs, voff, soff, 17); }
DEV u32 lds_inc(const Ctx& c, u32 word) {
    u32 v = 0;
    if (c.lane == 0) v = __hip_atomic_fetch_add((LAS u32*)&c.ctl[word], 1u, __ATOMIC_RELAXED, __HIP_MEMORY_SCOPE_WORKGROUP);
    return rfl(v);
}

DEV u32 wave_max(u32 v) {
    v = max(v, (u32)__builtin_amdgcn_update_dpp(0, (int)v, 0x111, 0xf, 0xf, true));
    v = max(v, (u32)__builtin_amdgcn_update_dpp(0, (int)v, 0x112, 0xf, 0xf, true));
    v = max(v, (u32)__builtin_amdgcn_update_dpp(0, (int)v, 0x114, 0xf, 0xf, true));
    v = max(v, (u32)__builtin_amdgcn_update_dpp(0, (int)v, 0x118, 0xf, 0xf, true));
    v = max(v, (u32)__builtin_amdgcn_update_dpp(0, (int)v, 0x142, 0xa, 0xf, false));
    v = max(v, (u32)__builtin_amdgcn_update_dpp(0, (int)v, 0x143, 0xc, 0xf, false));
    return __builtin_amdgcn_readlane(v, 63);
}

DEV u32 wave_scan(u32 v) {
    v += (u32)__builtin_amdgcn_update_dpp(0, (int)v, 0x111, 0xf, 0xf, true);
    v += (u32)__builtin_amdgcn_update_dpp(0, (int)v, 0x112, 0xf, 0xf, true);
    v += (u32)__builtin_amdgcn_update_dpp(0, (int)v, 0x114, 0xf, 0xf, true);
    v += (u32)__builtin_amdgcn_update_dpp(0, (int)v, 0x118, 0xf, 0xf, true);
    v += (u32)__builtin_amdgcn_update_dpp(0, (int)v, 0x142, 0xa, 0xf, false);
    v += (u32)__builtin_amdgcn_update_dpp(0, (int)v, 0x143, 0xc, 0xf, false);
    return v;
}
DEV u32 lane_rank(u64 m) { return __builtin_amdgcn_mbcnt_hi((u32)(m >> 32), __builtin_amdgcn_mbcnt_lo((u32)m, 0)); }

DEV void store_raw(const Ctx& c, const u32x4 (&R)[16], u32 off0, u32 off1) {
#pragma unroll
    for (int f = 0; f < 16; ++f) st16(c, R[f], c.lane * 16, (f < 8 ? off0 : off1) + (f & 7) * 1024);
}
DEV u32 sad(u32 a, u32 acc) { return __builtin_amdgcn_sad_u8(a, 0, acc); }
DEV u32 pk_sub(u32 a, u32 b) { return __builtin_bit_cast(u32, (u16x2)(__builtin_bit_cast(u16x2, a) - __builtin_bit_cast(u16x2, b))); }
DEV u32 pack_store(const Ctx& c, u32x4 (&X)[16], u32 dst) {
    const u32 ldsbase = (u32)(size_t)c.lds, stg = L_PST + (c.wave & 3) * PST;
#pragma unroll
    for (int i = 0; i < 5; ++i) *(LAS u32x4*)(c.lds + stg + c.lane * 16 + min(i * 1024u, PST - 1024)) = (u32x4){0, 0, 0, 0};
    u32 sum = 0;
#pragma unroll
    for (int f = 0; f < 16; ++f)
#pragma unroll
        for (int i = 0; i < 4; ++i) X[f][i] = __builtin_amdgcn_alignbit(X[f][i], X[f][i], 31);
#pragma unroll
    for (int f = 0; f < 16; ++f) asm volatile("" : "+v"(X[f]));
#pragma unroll
    for (int f = 0; f < 16; f += 2) sum = sad(perm(X[f][1], X[f][0], 0x07050301u), sum);
#pragma unroll
    for (int i = 0; i < 8; ++i)
        st16(c, (u32x4){perm(X[2 * i][1], X[2 * i][0], 0x06040200u), perm(X[2 * i][3], X[2 * i][2], 0x06040200u),
                        perm(X[2 * i + 1][1], X[2 * i + 1][0], 0x06040200u), perm(X[2 * i + 1][3], X[2 * i + 1][2], 0x06040200u)}, c.lane * 16, dst + i * 1024);
    const u32 top = min(max(((__builtin_amdgcn_readlane(wave_scan(sum), 63) + 1946) >> 11) + 2, 12u), 255u), K = (top << 8 | 0xff) * 0x10001u;
    u32 nexc = 0, hv = 0, D[8], W[16], rel[16], bits = 0;
#pragma unroll
    for (int j = 0; j < 8; ++j) {
        u32 t[4], cf[4], es[4];
        u32x2 sel[4];
#pragma unroll
        for (int g = 0; g < 4; ++g) {
            const u32x4 x = X[2 * j + (g >> 1)];
            t[g] = perm(pk_sub(K, x[(g & 1) * 2 + 1]), pk_sub(K, x[(g & 1) * 2]), 0x07050301u);
            cf[g] = perm(0, 0x07060500u, t[g]);
            es[g] = ~cf[g] & 0x04040404u;
            sel[g] = *(const LAS u32x2*)(size_t)__builtin_amdgcn_udot4(es[g], 0x10080402u, ldsbase + L_LUT2, false);
        }
        D[j] = bfi(0x03030303u, cf[0], bfi(0x0c0c0c0cu, cf[1] << 2, bfi(0x30303030u, cf[2] << 4, cf[3] << 6)));
#pragma unroll
        for (int h = 0; h < 2; ++h) {
            const int f = 2 * j + h;
            for (u64 m = __ballot((cf[2 * h] | cf[2 * h + 1]) & 0x80808080u); m && nexc <= 31; m &= m - 1) {
                const u32 ln = __builtin_ctzll(m);
                const u32 r0 = __builtin_amdgcn_readlane(X[f][0], ln), r1 = __builtin_amdgcn_readlane(X[f][1], ln);
                const u32 r2 = __builtin_amdgcn_readlane(X[f][2], ln), r3 = __builtin_amdgcn_readlane(X[f][3], ln);
                for (u32 fl = (__builtin_amdgcn_readlane(cf[2 * h], ln) & 0x80808080u) >> 1 | (__builtin_amdgcn_readlane(cf[2 * h + 1], ln) & 0x80808080u); fl; fl &= fl - 1) {
                    const u32 b = __builtin_ctz(fl), x = b & 1 ? (b & 16 ? r3 : r2) : (b & 16 ? r1 : r0), r = x >> 1 | x << 31;
                    ++nexc;
                    hv = writelane((f << 9 | ln << 3 | (b & 1) << 2 | b >> 3) << 16 | ((r >> (b & 8) * 2) & 0xffff), nexc & 63, hv);
                }
            }
            const u32 sa = sad(es[2 * h], 0);
            const u32 wa = perm(0, t[2 * h], sel[2 * h][0]) | perm(0, t[2 * h], sel[2 * h][1]) << 4;
            const u32 wb = perm(0, t[2 * h + 1], sel[2 * h + 1][0]) | perm(0, t[2 * h + 1], sel[2 * h + 1][1]) << 4;
            W[f] = wa | wb << sa;
            rel[f] = bits;
            bits = sad(es[2 * h + 1], bits + sa);
            asm volatile("" : "+v"(W[f]), "+v"(bits));
        }
        asm volatile("" : "+v"(D[j]));
    }
    if (FORCE_RAW_IN || nexc > 31) {
#pragma unroll
        for (int f = 0; f < 16; ++f)
#pragma unroll
            for (int i = 0; i < 4; ++i) X[f][i] = __builtin_amdgcn_alignbit(X[f][i], X[f][i], 1);
        store_raw(c, X, dst, dst + 8192);
        return YM_RAW | 1024;
    }
    st16(c, (u32x4){D[0], D[1], D[2], D[3]}, c.lane * 16, dst + 8192);
    st16(c, (u32x4){D[4], D[5], D[6], D[7]}, c.lane * 16, dst + 9216);
    hv = writelane(top | nexc << 8, 0, hv);
    if ((u32)c.lane <= nexc) *(LAS u32*)(c.lds + stg + c.lane * 4) = hv;
    const u32 incl = wave_scan(bits), start = 32 * (1 + nexc) + incl - bits, bytes = 4 * (1 + nexc) + ((__builtin_amdgcn_readlane(incl, 63) + 7) >> 3);
#pragma unroll
    for (int f = 0; f < 16; ++f) {
        const u32 pb = start + rel[f];
        const u64 v = (u64)W[f] << (pb & 31);
        LAS u32* q = (LAS u32*)(size_t)(ldsbase + stg + (pb >> 5) * 4);
        __hip_atomic_fetch_or(q, (u32)v, __ATOMIC_RELAXED, __HIP_MEMORY_SCOPE_WAVEFRONT);
        __hip_atomic_fetch_or(q + 1, (u32)(v >> 32), __ATOMIC_RELAXED, __HIP_MEMORY_SCOPE_WAVEFRONT);
    }
    CBAR();
    u32x4 v[5];
#pragma unroll
    for (int i = 0; i < 5; ++i) if (i < 2 || bytes > i * 1024u) v[i] = *(const LAS u32x4*)(c.lds + stg + c.lane * 16 + min(i * 1024u, PST - 1024));
#pragma unroll
    for (int i = 0; i < 5; ++i) if (i < 2 || bytes > i * 1024u) st16(c, v[i], c.lane * 16, dst + PK_TAIL + min(i * 1024u, PST - 1024));
    return (PK_TAIL + bytes + 15) >> 4;
}

DEV u32 xa_off(int ks, int h, int rowfrag) { return WS_XA + (u32)((ks * 2 + h) * 256 + rowfrag) * 1024u; }
DEV u32 wb_off(int ks, int h, int colfrag) { return WS_WB + (u32)((ks * 2 + h) * 128 + colfrag) * 1024u; }

DEV int wb_ks(int ks, u32 call) { return (call & 1) && (ks & 7) < 2 ? 64 + (ks >> 3) * 2 + (ks & 7) : ks; }
DEV void wg_barrier() { WAIT_LGKM0(); __builtin_amdgcn_s_barrier(); CBAR(); }

DEV void pack_job(const Ctx& c, int job) {
    const int s = job / 160, r0 = job % 160, q = s != 1 || r0 >= 96 ? r0 : r0 < 64 ? r0 + 32 : r0 - 64, ks = c.rank * 8 + s;
    const int l15 = c.lane & 15, lq = c.lane >> 4;
    const bool isx = q < 32;
    const int dst = (c.rank + 1 + (q - 32)) & 7, cc = (q - 32) >> 3;
    u32x4 R[16];
    if (job < NGEMM) STAMPW(c, 13);
    if (isx) {
        const GAS char* src = (const GAS char*)c.x + (size_t)(q * 128 + l15) * (KSH * 2) + (s * 64 + lq * 8) * 2;
#pragma unroll
        for (int f = 0; f < 16; ++f) R[f] = *(const GAS u32x4*)(src + (f & 7) * 16 * (KSH * 2) + (f >> 3) * 64);
    } else {
        const GAS char* src = (const GAS char*)c.w + (size_t)(s * 64 + lq * 8) * (NTOT * 2) + (dst * 2048 + cc * 128 + (l15 & 7) * 8 + (l15 >> 3) * 4) * 2;
        u32x2 r[4][8];
#pragma unroll
        for (int hg = 0; hg < 4; ++hg)
#pragma unroll
            for (int j = 0; j < 8; ++j) r[hg][j] = *(const GAS u32x2*)(src + (size_t)((hg >> 1) * 32 + j) * (NTOT * 2) + (hg & 1) * 128);
#pragma unroll
        for (int hg = 0; hg < 4; ++hg)
#pragma unroll
            for (int ii = 0; ii < 4; ++ii)
#pragma unroll
                for (int t = 0; t < 4; ++t)
                    R[hg * 4 + ii][t] = perm(r[hg][2 * t + 1][ii >> 1], r[hg][2 * t][ii >> 1], (ii & 1) ? 0x07060302u : 0x05040100u);
    }
    if (job < NGEMM) wg_barrier();
    __builtin_amdgcn_s_waitcnt(0x0f70);
    if (job < NGEMM) STAMPW(c, 4);
    const u32 call = get_call(c), ch = isx ? s * 32 + q : 256 + s * 16 + cc;
    const u32 slot = WS_PK + (call & 1) * PK_PAR + (isx ? ch : ch + dst * 128) * CHB;
    const bool done = !isx && (dst == c.rank || ch < 256 + NPUSH);
    if (done && dst != c.rank && ch > 256 + PFREE)
        for (const u64 t0 = rt(); rt() - t0 < (ch - 256 - PFREE) * PACE;) __builtin_amdgcn_s_sleep(2);
    if (isx || done) {
        const u64 base = (u64)c.peer_lane;
        const u32 to = isx ? c.rank : dst, d0 = isx ? xa_off(ks, 0, q * 8) : wb_off(wb_ks(ks, call), 0, cc * 8);
        Ctx e = c;
        e.rs = __builtin_amdgcn_make_buffer_rsrc((char*)((u64)(u32)__builtin_amdgcn_readlane((u32)(base >> 32), to) << 32 | (u32)__builtin_amdgcn_readlane((u32)base, to)), 0, WS_END, 0x00020000);
        store_raw(e, R, d0, d0 + (isx ? 256 : 128) * 1024);
    }
    u32 meta = 0;
    if (!done) meta = pack_store(c, R, slot);
    if (job < NGEMM) STAMPW(c, 5);
    WAIT_VM0();
    if (c.lane < NR && (isx || c.lane == dst)) {
        if (done || c.lane == c.rank) st_sys(c.peer_lane + (isx ? WS_XR + q * 256 : WS_WR + cc * 256) + ks * 4, call);
        else st_sys64(c.peer_lane + WS_FIN + (((call & 1) * NR + c.rank) * NCHIN + ch) * 8, (u64)call << 32 | meta);
    }
}

struct Tile { int cc, owner, h, rf, rfo; };
constexpr u32 TK2 = 130;

constexpr u32 LE_FULL = 64, LE_OK = 128, LE_END = 256, NISS = 3;
DEV void lds_add1(const Ctx& c, u32 word) {
    asm volatile("s_mov_b64 exec, 1\n\tds_add_u32 %0, %1\n\ts_mov_b64 exec, -1" :: "v"((u32)(size_t)&c.ctl[word]), "v"(1u) : "memory");
}
template <bool BIG>
DEV void lister(const Ctx& c, const Tile& t, u32 xr, u32 call) {
    const u32 ks = c.lane ^ xr;
    const char* fx = c.me + WS_XR + (t.rf >> 3) * 256 + ks * 4;
    const char* fo = c.me + WS_XR + (t.rfo >> 3) * 256 + ks * 4;
    const char* fw = c.me + WS_WR + t.cc * 256 + ks * 4;
    u64 todo = ~0ull, late = 0, t0 = 0;
    u32 n = 0;
    bool second = false;
    for (u32 it = 1;; ++it) {
        const u32 o = BIG ? ld_sys(fo) : 0;
        u64 m = 0;
        if (!second) { const u32 a = ld_sys(fx), b = ld_sys(fx + 256), w = ld_sys(fw); m = __ballot(min(min(a, b), w) >= call); }
        const u64 own = __ballot(o >= call);
        m = (second ? own : m) & todo;
        if (m) {
            if (m == todo && tmo_set(c, call)) break;
            if ((m >> c.lane) & 1) c.ctl[C_LIST + n + lane_rank(m)] = LE_OK | (BIG && o >= call ? LE_FULL : 0) | ks;
            n += __popcll(m); todo &= ~m; late |= m & ~own; t0 = 0;
            if (!todo) {
                if (!BIG || second || !late) { c.ctl[C_LIST + n + (c.lane & 3)] = LE_END; return; }
                second = true; todo = late;
            }
        } else {
            if ((it & 15) == 0) {
                if (rfl(c.ctl[C_ABORT]) || tmo_set(c, call)) break;
                if (expired(t0, DEADLINE)) { tmo_raise(c, call); break; }
            }
            __builtin_amdgcn_s_sleep(LSLEEP);
        }
    }
    c.ctl[C_ABORT] = 1;
}
template <bool BIG>
DEV void loader(const Ctx& c, const Tile& t, u32 call) {
    const u32 ldsbase = (u32)(size_t)c.lds, voff = c.lane * 16;
    u32 pend = 0, pslot = 0;
#pragma unroll 1
    for (;;) {
        const u32 n = lds_inc(c, C_LE), h = n & 1;
        const bool second = n >= 128;
        const u32 e = second ? n - 64 : n >> 1, tt = second ? n + (TK2 - 128) : n, slot = tt % NSTG, need = 8 * (tt / NSTG);
        const u32 look = c.ctl[c.lane ? C_LIST + e : C_FREE + slot];
        u32 v = __builtin_amdgcn_readlane(look, 1);
        if (!v || rfl(look) < need) {
            if (pend) { WAIT_VM0(); c.ctl[C_FULL + pslot] = pend; pend = 0; }
            if (!v) v = lds_wait(c, C_LIST + e, 1, true, LNAP);
            if (need && (v & LE_OK) && !lds_wait(c, C_FREE + slot, need, true, LNAP)) break;
        }
        if (!(v & LE_OK)) break;
        const int ks = v & 63, kw = wb_ks(ks, call);
        const u32 full = BIG && !second && (v & LE_FULL), l = ldsbase + slot * STG;
        if (BIG && second) {
            glds4g(c.rs, voff, xa_off(ks, 0, t.rfo), l); glds4g(c.rs, voff, xa_off(ks, 1, t.rfo), l + 4096);
#pragma unroll
            for (int g = 0; g < 4; ++g) glds4g(c.rs, voff, wb_off(kw, g >> 1, t.cc * 8 + (g & 1) * 4), l + 8192 + g * 4096);
        } else {
#pragma unroll
            for (int g = 0; g < 4; ++g) glds4g(c.rs, voff, xa_off(ks, h, t.rf + g * 4), l + g * 4096);
            glds4g(c.rs, voff, wb_off(kw, h, t.cc * 8), l + 16384); glds4g(c.rs, voff, wb_off(kw, h, t.cc * 8 + 4), l + 20480);
        }
        if (full) {
            glds4g(c.rs, voff, xa_off(ks, h, t.rfo), l + 24576);
            asm volatile("s_waitcnt vmcnt(28)" ::: "memory");
        } else asm volatile("s_waitcnt vmcnt(24)" ::: "memory");
        if (pend) c.ctl[C_FULL + pslot] = pend;
        pend = 2 * tt + 2 + full; pslot = slot;
    }
    WAIT_VM0();
    if (pend) c.ctl[C_FULL + pslot] = pend;
}

DEV u32 pk_relu(float a, float b) {
    const f32x2 v = __builtin_elementwise_maximum((f32x2){a, b}, (f32x2){0.f, 0.f});
    return __builtin_bit_cast(u32, __builtin_convertvector(v, bf16x2));
}

DEV u32 nz8(u32 cb) { return (cb + 0x07070707u) & 0x08080808u; }
template <int NF>
DEV u32 encode_y(const Ctx& c, const u32 (&P)[NF * 8], u32 dst) {
    constexpr int G = NF * 4;
    constexpr u32 CODEB = NF == 5 ? 2048 : 1536;
    const u32 ldsbase = (u32)(size_t)c.lds, stg = L_YST + c.wave * YST;
#pragma unroll
    for (int i = 0; i < 5; ++i) *(LAS u32x4*)(c.lds + stg + c.lane * 16 + min(i * 1024u, YST - 1024)) = (u32x4){0, 0, 0, 0};
    u16x2 mx = {0, 0}, mn[NF];
#pragma unroll
    for (int f = 0; f < NF; ++f) {
        mn[f] = (u16x2){0xffff, 0xffff};
#pragma unroll
        for (int j = 0; j < 8; ++j) {
            const u16x2 v = __builtin_bit_cast(u16x2, P[f * 8 + j]);
            mx = __builtin_elementwise_max(mx, v);
            mn[f] = __builtin_elementwise_min(mn[f], (u16x2)(v - (u16x2){1, 1}));
        }
    }
    const u32 base = max(wave_max(max((u32)mx[0], (u32)mx[1])) >> 8, 7u) - 7, lim = ((base + 1) << 8) - 1;
    u32 nexc = 0, hv = 0;
#pragma unroll
    for (int f = 0; f < NF; ++f)
        for (u64 m = __ballot(min((u32)mn[f][0], (u32)mn[f][1]) < lim); m && nexc <= 31; m &= m - 1) {
            const u32 ln = __builtin_ctzll(m);
            u32 t = 0;
#pragma unroll
            for (int k = 0; k < 8; ++k) t = writelane(__builtin_amdgcn_readlane(P[f * 8 + k], ln), k, t);
            for (u32 mk = (u32)__ballot((t & 0xffff) - 1 < lim) | (u32)__ballot((t >> 16) - 1 < lim) << 8; mk; mk &= mk - 1) {
                const u32 k = __builtin_ctz(mk), v = (__builtin_amdgcn_readlane(t, k & 7) >> (k & 8) * 2) & 0xffff;
                ++nexc;
                hv = writelane((f * 16 + (k & 7) * 2 + (k >> 3)) << 22 | ln << 16 | v, nexc & 63, hv);
            }
        }
    if (FORCE_RAW_Y || nexc > 31) {
#pragma unroll
        for (int i = 0; i < NF * 2; ++i) st16(c, (u32x4){P[4 * i], P[4 * i + 1], P[4 * i + 2], P[4 * i + 3]}, c.lane * 16, dst + i * 1024);
        return YM_RAW | NF * 128;
    }
    hv = writelane(base | nexc << 8, 0, hv);
    if ((u32)c.lane <= nexc) *(LAS u32*)(c.lds + stg + c.lane * 4) = hv;
    u32 cb[G], lb[G], cnt = 0;
    const u16x2 B = {(u16)(base << 8), (u16)(base << 8)};
#pragma unroll
    for (int g = 0; g < G; ++g) {
        const u32 t0 = __builtin_bit_cast(u32, __builtin_elementwise_sub_sat(__builtin_bit_cast(u16x2, P[2 * g]), B));
        const u32 t1 = __builtin_bit_cast(u32, __builtin_elementwise_sub_sat(__builtin_bit_cast(u16x2, P[2 * g + 1]), B));
        cb[g] = perm(t1, t0, 0x07050301u); lb[g] = perm(t1, t0, 0x06040200u);
        cnt += __builtin_popcount(nz8(cb[g]));
    }
    u32 D[8];
#pragma unroll
    for (int h = 0; h < 2; ++h) {
        const u32* q = cb + 8 * h;
        D[3 * h] = q[0] | q[1] << 3 | (q[6] & 0x03030303u) << 6;
        D[3 * h + 1] = q[2] | q[3] << 3 | (q[7] & 0x03030303u) << 6;
        D[3 * h + 2] = q[4] | q[5] << 3 | (q[6] & 0x04040404u) << 4 | (q[7] & 0x04040404u) << 5;
    }
    st16(c, (u32x4){D[0], D[1], D[2], D[3]}, c.lane * 16, dst);
    if (NF == 5) st16(c, (u32x4){D[4], D[5], cb[G - 4] | cb[G - 3] << 3, cb[G - 2] | cb[G - 1] << 3}, c.lane * 16, dst + 1024);
    else __builtin_amdgcn_raw_buffer_store_b64((u32x2){D[4], D[5]}, c.rs, c.lane * 8 + dst + 1024, 0, 17);
    const u32 incl = wave_scan(cnt), total = 4 * (1 + nexc) + __builtin_amdgcn_readlane(incl, 63);
    u32 pos = ldsbase + stg + 4 * (1 + nexc) + incl - cnt, adr[G];
    u32x2 sel[G];
#pragma unroll
    for (int g = 0; g < G; ++g) {
        const u32 nz = nz8(cb[g]);
        adr[g] = pos & ~3u;
        sel[g] = *(const LAS u32x2*)(size_t)__builtin_amdgcn_udot4(nz, 0x20100804u, ((pos & 3) << 3) | (ldsbase + L_LUT), false);
        pos += __builtin_popcount(nz);
    }
    __builtin_amdgcn_sched_barrier(0);
    u32 cur = 0;
#pragma unroll
    for (int g = 0; g < G; ++g) {
        const u32 o = perm(lb[g], cur, sel[g][0]);
        __hip_atomic_fetch_or((LAS u32*)(size_t)adr[g], o, __ATOMIC_RELAXED, __HIP_MEMORY_SCOPE_WAVEFRONT);
        cur = perm(lb[g], o, sel[g][1]);
    }
    __hip_atomic_fetch_or((LAS u32*)(size_t)(pos & ~3u), cur, __ATOMIC_RELAXED, __HIP_MEMORY_SCOPE_WAVEFRONT);
    CBAR();
    u32x4 v[5];
#pragma unroll
    for (int i = 0; i < 5; ++i) if (i < 3 || total > i * 1024u) v[i] = *(const LAS u32x4*)(c.lds + stg + c.lane * 16 + min(i * 1024u, YST - 1024));
#pragma unroll
    for (int i = 0; i < 5; ++i) if (i < 3 || total > i * 1024u) st16(c, v[i], c.lane * 16, dst + CODEB + min(i * 1024u, YST - 1024));
    return (CODEB + total + 15) >> 4;
}

DEV f32x4 mfma(u32x4 a, u32x4 b, f32x4 acc) { return __builtin_amdgcn_mfma_f32_16x16x32_bf16(__builtin_bit_cast(bf16x8, a), __builtin_bit_cast(bf16x8, b), acc, 0, 0, 0); }
DEV void slot_free(const Ctx& c, u32 slot) { lds_add1(c, C_FREE + slot); }

template <bool BIG>
DEV void consumer(const Ctx& c, const Tile& t, int wr, int wc) {
    f32x4 acc[4][4], oacc[4];
#pragma unroll
    for (int i = 0; i < 4; ++i) {
        oacc[i] = (f32x4){0.f, 0.f, 0.f, 0.f};
#pragma unroll
        for (int f = 0; f < 4; ++f) acc[f][i] = (f32x4){0.f, 0.f, 0.f, 0.f};
    }
    const LAS char* pa = c.lds + c.lane * 16 + wr * 4096;
    const LAS char* po = c.lds + c.lane * 16 + wr * 1024;
    const LAS char* pb = c.lds + c.lane * 16 + wc * 4096;
    u32 slot = 0, seen = 0, nfull = 0;
    bool ok = true;
    struct Ops { u32x4 a[4], b[4], ao; u32 full; } X, Y;
    auto fetch = [&](Ops& o, u32 tn) __attribute__((always_inline)) {
        u32 sv = rfl(seen);
        if (ok && tn < 128 && sv < 2 * tn + 2) {
            if (CPRIO) __builtin_amdgcn_s_setprio(0);
            sv = lds_wait(c, C_FULL + slot, 2 * tn + 2, true, CNAP); ok = sv != 0;
            if (CPRIO) __builtin_amdgcn_s_setprio(CPRIO);
        }
        CBAR();
        if (tn == 64) STAMPW(c, 4);
        if (tn == 96) STAMPW(c, 5);
        if (tn == 112) { STAMPW(c, 6); STAMPC(c, 13); }
        if (tn == 120) STAMPW(c, 7);
        o.full = sv & 1;
        __builtin_amdgcn_sched_barrier(0);
#pragma unroll
        for (int i = 0; i < 4; ++i) o.b[i] = *(const LAS u32x4*)(pb + slot * STG + 16384 + i * 1024);
#pragma unroll
        for (int f = 0; f < 4; ++f) o.a[f] = *(const LAS u32x4*)(pa + slot * STG + f * 1024);
        if (BIG) o.ao = *(const LAS u32x4*)(po + slot * STG + 24576);
        seen = *(const LAS u32*)&c.ctl[C_FULL + (slot == NSTG - 1 ? 0 : slot + 1)];
    };
    auto release = [&](u32 tn) __attribute__((always_inline)) {
        __builtin_amdgcn_sched_barrier(0);
        WAIT_LGKM0();
        if (tn < 128) slot_free(c, slot);
        slot = slot == NSTG - 1 ? 0 : slot + 1;
    };
    auto step = [&](const Ops& o, Ops& nx, u32 tn) __attribute__((always_inline)) {
        fetch(nx, tn);
#pragma unroll
        for (int f = 0; f < 4; ++f)
#pragma unroll
            for (int i = 0; i < 4; ++i) acc[f][i] = mfma(o.a[f], o.b[i], acc[f][i]);
#pragma unroll
        for (int i = 0; i < (BIG ? 10 : 9); ++i) { __builtin_amdgcn_sched_group_barrier(0x8, 1, 0); __builtin_amdgcn_sched_group_barrier(0x100, 1, 0); }
        __builtin_amdgcn_sched_barrier(0);
        if (BIG && o.full) {
            ++nfull;
#pragma unroll
            for (int i = 0; i < 4; ++i) oacc[i] = mfma(o.ao, o.b[i], oacc[i]);
        }
        release(tn);
    };
    fetch(X, 0);
    STAMPW(c, 1); STAMPC(c, 12);
    release(0);
#pragma unroll 1
    for (u32 tk = 0; tk < 128; tk += 2) { step(X, Y, tk + 1); step(Y, X, tk + 2); }
    STAMPW(c, 2); STAMPC(c, 14);
    __builtin_amdgcn_s_setprio(EPRIO);
    ok = ok && !rfl(c.ctl[C_ABORT]);
    {
        const u32 call = get_call(c), msg = (t.cc * 2 + t.h) * 8 + c.wave;
        u32 P[32];
#pragma unroll
        for (int g = 0; g < 16; ++g) {
            P[2 * g] = pk_relu(acc[g >> 2][0][g & 3], acc[g >> 2][1][g & 3]);
            P[2 * g + 1] = pk_relu(acc[g >> 2][2][g & 3], acc[g >> 2][3][g & 3]);
        }
        ok = ok && lds_wait(c, C_FREE + YS, 8 * (128 / NSTG), true, 0) && lds_wait(c, C_FREE + YS + 1, 8 * (128 / NSTG), true, 0);
        CBAR();
        STAMPW(c, 8);
        u32 meta = YM_BAD;
        if (ok) meta = encode_y<4>(c, P, WS_YP + (t.owner * NMSG + msg) * YSLOT);
        STAMPW(c, 9);
        WAIT_VM0();
        if (c.lane == t.owner) st_sys64(c.peer_lane + WS_FY + (c.rank * NMSG + msg) * 8, (u64)call << 32 | meta);
    }
    STAMPW(c, 3);
    if (BIG) {
        slot_free(c, YS); slot_free(c, YS + 1);
        slot = TK2 % NSTG; seen = 0;
        const u32 end = TK2 + 64 - (nfull >> 1);
        struct Own { u32x4 a[2], b[8]; } U, V;
        auto fetch2 = [&](Own& o, u32 tn) __attribute__((always_inline)) {
            if (ok && tn < end && rfl(seen) < 2 * tn + 2) {
                if (CPRIO) __builtin_amdgcn_s_setprio(0);
                ok = lds_wait(c, C_FULL + slot, 2 * tn + 2, true, CNAP) != 0;
                if (CPRIO) __builtin_amdgcn_s_setprio(CPRIO);
            }
            CBAR();
            __builtin_amdgcn_sched_barrier(0);
#pragma unroll
            for (int i = 0; i < 8; ++i) o.b[i] = *(const LAS u32x4*)(pb + slot * STG + 8192 + (i >> 2) * 8192 + (i & 3) * 1024);
#pragma unroll
            for (int f = 0; f < 2; ++f) o.a[f] = *(const LAS u32x4*)(po + slot * STG + f * 4096);
            seen = *(const LAS u32*)&c.ctl[C_FULL + (slot == NSTG - 1 ? 0 : slot + 1)];
        };
        auto step2 = [&](const Own& o, Own& nx, u32 tn) __attribute__((always_inline)) {
            fetch2(nx, tn);
#pragma unroll
            for (int i = 0; i < 8; ++i) oacc[i & 3] = mfma(o.a[i >> 2], o.b[i], oacc[i & 3]);
#pragma unroll
            for (int i = 0; i < 5; ++i) { __builtin_amdgcn_sched_group_barrier(0x100, 2, 0); __builtin_amdgcn_sched_group_barrier(0x8, 1, 0); }
            release(tn < end ? 0 : 128);
        };
        fetch2(U, TK2);
        release(TK2 < end ? 0 : 128);
        u32 tk = TK2;
#pragma unroll 1
        for (; tk + 1 < end; tk += 2) { step2(U, V, tk + 1); step2(V, U, tk + 2); }
        if (tk < end)
#pragma unroll
            for (int i = 0; i < 8; ++i) oacc[i & 3] = mfma(U.a[i >> 2], U.b[i], oacc[i & 3]);
        ok = ok && !rfl(c.ctl[C_ABORT]);
        const u32 o = out_off(c, ((t.rfo + wr) & 31) * 16, c.rank * 2048 + t.cc * 128 + wc * 64);
#pragma unroll
        for (int r = 0; r < 4; r += 2)
            st_out(c, o + r * (NTOT * 2), ok ? (u32x2){pk_relu(oacc[0][r], oacc[1][r]), pk_relu(oacc[2][r], oacc[3][r])} : (u32x2){0x7fc07fc0u, 0x7fc07fc0u},
                   ok ? (u32x2){pk_relu(oacc[0][r + 1], oacc[1][r + 1]), pk_relu(oacc[2][r + 1], oacc[3][r + 1])} : (u32x2){0x7fc07fc0u, 0x7fc07fc0u});
        STAMPW(c, 10);
    }
}

DEV void gemm_wg(const Ctx& c) {
    const int g = c.wg - NCOMM, xcd = g & 7, idx = g >> 3, grp = xcd >> 2, k = idx % 7;
    const bool big = k < 4;
    Tile t;
    t.cc = (xcd & 3) * 4 + idx / 7;
    t.h = grp * 4 + k >= 7 || k >= 4;
    t.owner = (c.rank + (grp ? (k < 3 ? 5 + k : 1 + k) : (k < 4 ? 1 + k : k - 3))) & 7;
    t.rf = t.owner * 32 + t.h * 16; t.rfo = c.rank * 32 + (grp * 4 + (k & 3)) * 4;
    if (c.wave >= 8) {
        const int L = c.wave - 8, J = (L + JROT) & 3;
        if (J) {
            wg_barrier();
            for (int i = 0; i < (J == 1 ? PS1 : J == 2 ? PS2 : PS3); ++i) __builtin_amdgcn_s_sleep(24);
        }
#pragma unroll 1
        for (int job = J * NGEMM + g; job < 1280; job += J < 2 ? 1280 : 2 * NGEMM) {
            if (job >= 4 * NGEMM) for (int i = 0; i < P2; ++i) __builtin_amdgcn_s_sleep(24);
            pack_job(c, job);
            if (job < 4 * NGEMM) STAMPW(c, 3);
        }
        STAMPW(c, 1);
        __builtin_amdgcn_s_setprio(LPRIO);
        const u32 call = get_call(c), xr = XROT ? (t.cc & 1) | (t.cc & 2) << 2 | (k & 3) << 1 | (t.cc & 3) << 4 : 0;
        if (L == NISS) { if (big) lister<true>(c, t, xr, call); else lister<false>(c, t, xr, call); }
        if (L < NISS || L4) { if (big) loader<true>(c, t, call); else loader<false>(c, t, call); }
        STAMPW(c, 2);
    } else {
        wg_barrier();
        if (big) consumer<true>(c, t, c.wave >> 1, c.wave & 1); else consumer<false>(c, t, c.wave >> 1, c.wave & 1);
    }
}

#define WSUM 8
#define EARLY 0
#define WCLV 4

constexpr u32 NFIX = 4;

DEV u32 take_seat(const Ctx& c, u32 call) {
    u32 xcc;
    asm volatile("s_getreg_b32 %0, hwreg(HW_REG_XCC_ID, 0, 4)" : "=s"(xcc));
    const u32 grp = (0xb2u >> c.rank) & 1 ? 0xb2u : 0x4du, j = (c.wg >> 3) * 4 + (xcc & 3);
    u32 near = TOPO ? (xcc & 4 ? ~grp & 0xff : grp & ~(1u << c.rank)) : 0, seat = NPULL;
    const u32 n = __builtin_popcount(near);
    if (!TOPO) seat = min((u32)c.wg, (u32)NPULL);
    else if (j < 4 * n) {
        for (u32 i = (j + SPREAD * (n == 4 ? j / n : 0)) % n; i; --i) near &= near - 1;
        seat = ((__builtin_ctz(near) - c.rank) & 7) - 1 + 7 * (j / n);
    }
    for (bool first = true;; first = false) {
        u64 old = ~0ull;
        if (c.lane == 0 ? seat < NPULL : first && c.lane < NR)
            old = __hip_atomic_fetch_max((GAS u64*)(c.me + (c.lane ? WS_CLM + c.lane * 128 : WS_SEAT + seat * 8)), (u64)call << 32 | 4 * NFIX, __ATOMIC_RELAXED, __HIP_MEMORY_SCOPE_AGENT);
        if (rfl((u32)(old >> 32)) < call) return seat;
        if (first) for (int i = 0; i < 4; ++i) __builtin_amdgcn_s_sleep(32);
        const u64 v = __hip_atomic_load((const GAS u64*)(c.me + WS_SEAT + min(c.lane, NPULL - 1) * 8), __ATOMIC_RELAXED, __HIP_MEMORY_SCOPE_AGENT);
        const u64 open = __ballot((u32)(v >> 32) < call);
        if (!open) return NPULL;
        seat = __builtin_ctzll(open);
    }
}

DEV void watcher(const Ctx& c, int p, int sub, u32 call) {
    static_assert(NQY == 64, "one message per lane");
    const u32 id = sub + 4 * c.lane;
    const char* fp = c.me + WS_FY + (p * NMSG + id) * 8;
    bool bad = !lds_wait(c, C_YGO, 1, false, 1);
    STAMPW(c, 1);
    u64 pending = ~0ull, t0 = 0;
    u32 qn = 0;
    for (u32 it = 1; pending; ++it) {
        const u64 v = bad ? 0 : ld_sys64(fp);
        const bool mine = ((pending >> c.lane) & 1) && (bad || (u32)(v >> 32) >= call);
        const u64 m = __ballot(mine);
        if (m) {
            const bool poison = bad || tmo_set(c, call);
            if (mine) c.ctl[C_Q + qn + lane_rank(m)] = 0x80000000u | id << 20 | (poison ? YM_BAD : (u32)v & (YM_BAD | YM_RAW | 0x3ff));
            qn += __popcll(m); pending &= ~m; t0 = 0;
            c.ctl[C_QN] = qn;
        } else {
            if ((it & 15) == 0) {
                if (tmo_set(c, call)) bad = true;
                else if (expired(t0, DEADLINE)) { tmo_raise(c, call); bad = true; }
            }
            __builtin_amdgcn_s_sleep(2);
        }
    }
    STAMPW(c, 2);
}

DEV void pull_inputs(const Ctx& c, char* peer, rsrc_t pr, u32 lo, int d, int sub, int p, u32 call) {
    GAS u64* ctr = (GAS u64*)(c.me + WS_CLM + d * 128);
    const u32 voff = c.lane * 16, win = WSUM / 4 + (sub < WSUM % 4), WCL = win + WCLV;
    bool bad = false;
#pragma unroll 1
    for (;;) {
        const u32 t = lds_inc(c, C_TK);
        if (t >= WCL) lds_wait(c, C_LAND, t - WCL + 1, false, 1);
        u32 n = t * 4 + sub;
        if (t >= NFIX) {
            if (c.lane == 0) n = (u32)__hip_atomic_fetch_add(ctr, 1ull, __ATOMIC_RELAXED, __HIP_MEMORY_SCOPE_AGENT);
            n = rfl(n);
        }
        if (n + 64 >= NPOS) c.ctl[C_YGO] = 1;
        if (n >= NPOS) break;
        const u32 code = rfl(ORDER.v[d - 1][n]), s = (code >> 8) & 7, ix = code & 31;
        const bool isw = code & 0x8000;
        const u32 rc = isw ? ix : ((((ix >> 2) + c.rank) & 7) << 2 | (ix & 3)), id = isw ? 256 + s * 16 + rc : s * 32 + rc;
        const char* fp = c.me + WS_FIN + (((call & 1) * NR + p) * NCHIN + id) * 8;
        u32 meta = 0;
        u64 t0 = 0;
        for (u32 it = 1; !bad; ++it) {
            const u64 v = ld_sys64(fp);
            meta = rfl((u32)v);
            if (rfl((u32)(v >> 32)) >= call) break;
            if ((it & 15) == 0) {
                if (tmo_set(c, call)) bad = true;
                else if (expired(t0, DEADLINE)) { tmo_raise(c, call); bad = true; }
            }
            __builtin_amdgcn_s_sleep(2);
        }
        const u32 u = lds_inc(c, C_IS);
        if (bad) { lds_inc(c, C_LAND); continue; }
        if (u >= win) lds_wait(c, C_LAND, u - win + 1, false, WNAP, 0xffff);
        __builtin_amdgcn_s_setprio(UPRIO);
        ST2(c, 1024 + u, (meta & 0x7ff) >> 3);
        const int ks = p * 8 + s;
        const u32 src = WS_PK + (call & 1) * PK_PAR + (isw ? id + c.rank * 128 : id) * CHB;
        const u32 d0 = isw ? wb_off(wb_ks(ks, call), 0, rc * 8) : xa_off(ks, 0, rc * 8), d1 = d0 + (isw ? 128 : 256) * 1024;
        u32x4 R[16];
        if (meta & YM_RAW) {
#pragma unroll
            for (int f = 0; f < 16; ++f) R[f] = ld16(pr, voff + (f & 3) * 1024, src + (f >> 2) * 4096);
            WAIT_VM0();
            lds_inc(c, C_LAND);
            ST2(c, 1536 + u, 0); ST2(c, u, 0);
            store_raw(c, R, d0, d1);
        } else {
            const u32 bytes = min((meta & 0x7ff) * 16, PK_TAIL + LSLOT), tl = (u32)(size_t)c.lds + lo;
            const rsrc_t tr = __builtin_amdgcn_make_buffer_rsrc(peer, 0, rfl(src + bytes), 0x00020000);
            if (bytes > PK_TAIL + 2048) glds4(tr, voff, src + PK_TAIL + 2048, tl + 2048);
            glds2(tr, voff, src + PK_TAIL, tl);
            u32x4 L[8], D[2];
#pragma unroll
            for (int i = 0; i < 2; ++i) D[i] = ld16(pr, voff + i * 1024, src + 8192);
#pragma unroll
            for (int i = 0; i < 8; ++i) L[i] = ld16(pr, voff + (i & 3) * 1024, src + (i >> 2) * 4096);
            asm volatile("s_waitcnt vmcnt(%0)" :: "n"(EARLY) : "memory");
            if (c.lane == 0) __hip_atomic_fetch_add((LAS u32*)&c.ctl[C_LAND], 1u, __ATOMIC_RELAXED, __HIP_MEMORY_SCOPE_WORKGROUP);
            WAIT_VM0();
            ST2(c, 1536 + u, 0); ST2(c, u, 0);
            const u32 hv = *(const LAS u32*)(size_t)(tl + (c.lane & 31) * 4);
            u32 cnt = 0;
#pragma unroll
            for (int j = 0; j < 8; ++j) cnt += __builtin_popcount(~(D[j >> 2][j & 3] | D[j >> 2][j & 3] >> 1) & 0x55555555u);
            const u32 h0 = rfl(hv), T = (h0 & 0xff) * 0x01010101u, nexc = min((h0 >> 8) & 0xff, 31u);
            u32 pb = 32 * (1 + nexc) + 4 * (wave_scan(cnt) - cnt);
#pragma unroll
            for (int hf = 0; hf < 16; hf += 8) {
                u32 cd[16], sel[16], sa[8], ps[8];
                u32x2 q[8];
#pragma unroll
                for (int f = 0; f < 8; ++f) {
                    const u32 w = D[hf >> 3][f >> 1] >> (f & 1) * 4;
                    cd[2 * f] = w & 0x03030303u; cd[2 * f + 1] = (w >> 2) & 0x03030303u;
                    const u32 ea = (0x04040404u - cd[2 * f]) & 0x04040404u, eb = (0x04040404u - cd[2 * f + 1]) & 0x04040404u;
                    sel[2 * f] = *(const LAS u32*)(size_t)__builtin_amdgcn_udot4(ea, 0x08040201u, (u32)(size_t)c.lds + L_LUT2, false);
                    sel[2 * f + 1] = *(const LAS u32*)(size_t)__builtin_amdgcn_udot4(eb, 0x08040201u, (u32)(size_t)c.lds + L_LUT2, false);
                    const LAS u32* z = (const LAS u32*)(size_t)(tl + (pb >> 5) * 4);
                    q[f] = (u32x2){z[0], z[1]};
                    sa[f] = sad(ea, 0); ps[f] = pb;
                    pb = sad(eb, pb + sa[f]);
                }
                __builtin_amdgcn_sched_barrier(0);
#pragma unroll
                for (int f = 0; f < 8; ++f) {
                    const u32 va = __builtin_amdgcn_alignbit(q[f][1], q[f][0], ps[f]), vb = va >> sa[f];
                    const u32 ha = T - ((perm(va >> 4, va, sel[2 * f]) & 0x0f0f0f0fu) | cd[2 * f]), hb = T - ((perm(vb >> 4, vb, sel[2 * f + 1]) & 0x0f0f0f0fu) | cd[2 * f + 1]);
                    const u32 la = L[(hf + f) >> 1][(f & 1) * 2], lb = L[(hf + f) >> 1][(f & 1) * 2 + 1];
                    const u32x4 x = {perm(ha, la, 0x05010400u), perm(ha, la, 0x07030602u), perm(hb, lb, 0x05010400u), perm(hb, lb, 0x07030602u)};
                    u32x4 r;
#pragma unroll
                    for (int i = 0; i < 4; ++i) r[i] = __builtin_amdgcn_alignbit(x[i], x[i], 1);
                    st16(c, r, voff, (hf ? d1 : d0) + f * 1024);
                }
            }
            if ((u32)c.lane - 1 < nexc)
                __builtin_amdgcn_raw_buffer_store_b16((u16)hv, c.rs, (hv >> 28 & 1 ? d1 : d0) + (hv >> 25 & 7) * 1024 + (hv >> 16 & 511) * 2, 0, 17);
        }
        STAMPW(c, 3);
        ST2(c, 128 + u, 0);
        WAIT_VM0();
        if (c.lane == 0) st_sys(c.me + (isw ? WS_WR : WS_XR) + rc * 256 + ks * 4, call);
        __builtin_amdgcn_s_setprio(0);
        ST2(c, 512 + u, 0);
    }
    if (c.lane == 0) __hip_atomic_fetch_or((LAS u32*)&c.ctl[C_LAND], 1u << 16, __ATOMIC_RELAXED, __HIP_MEMORY_SCOPE_WORKGROUP);
}

template <int NF>
DEV void decode_y(const Ctx& c, u32 e, u32 lo, rsrc_t pr, u32 src, u32 row0, u32 col0, u32 tk) {
    constexpr int G = NF * 4;
    constexpr u32 CODEB = NF == 5 ? 2048 : 1536;
    const u32 ldsbase = (u32)(size_t)c.lds;
    const LAS char* l = c.lds + lo;
    u32x2 P[G];
    u32 hv = 0, nexc = 0;
    if (e & YM_BAD) {
#pragma unroll
        for (int g = 0; g < G; ++g) P[g] = (u32x2){0x7fc07fc0u, 0x7fc07fc0u};
    } else if (e & YM_RAW) {
#pragma unroll
        for (int i = 0; i < G / 2; ++i) {
            const u32x4 v = ld16(pr, c.lane * 16 + (i & 3) * 1024, src + (i >> 2) * 4096);
            P[2 * i] = (u32x2){v[0], v[1]}; P[2 * i + 1] = (u32x2){v[2], v[3]};
        }
        __builtin_amdgcn_s_waitcnt(0x0f70);
    } else {
        const u32x4 da = *(const LAS u32x4*)(l + c.lane * 16);
        u32x4 db;
        if (NF == 5) db = *(const LAS u32x4*)(l + 1024 + c.lane * 16);
        else { const u32x2 t = *(const LAS u32x2*)(l + 1024 + c.lane * 8); db = (u32x4){t[0], t[1], 0, 0}; }
        hv = *(const LAS u32*)(l + CODEB + (c.lane & 31) * 4);
        const u32 D[8] = {da[0], da[1], da[2], da[3], db[0], db[1], db[2], db[3]};
        u32 cb[G], cnt = 0;
#pragma unroll
        for (int h = 0; h < 2; ++h) {
            u32* q = cb + 8 * h;
#pragma unroll
            for (int k = 0; k < 3; ++k) { q[2 * k] = D[3 * h + k] & 0x07070707u; q[2 * k + 1] = (D[3 * h + k] >> 3) & 0x07070707u; }
            q[6] = bfi(0x03030303u, D[3 * h] >> 6, D[3 * h + 2] >> 4) & 0x07070707u;
            q[7] = bfi(0x03030303u, D[3 * h + 1] >> 6, D[3 * h + 2] >> 5) & 0x07070707u;
        }
        if (NF == 5) { cb[G - 4] = D[6] & 0x07070707u; cb[G - 3] = (D[6] >> 3) & 0x07070707u; cb[G - 2] = D[7] & 0x07070707u; cb[G - 1] = (D[7] >> 3) & 0x07070707u; }
#pragma unroll
        for (int g = 0; g < G; ++g) cnt += __builtin_popcount(nz8(cb[g]));
        const u32 h0 = rfl(hv), B = (h0 & 0xff) * 0x01010101u;
        nexc = min((h0 >> 8) & 0xff, 31u);
        u32 pos = ldsbase + lo + CODEB + 4 * (1 + nexc) + wave_scan(cnt) - cnt, ps[G];
        u32x2 sel[G];
#pragma unroll
        for (int g = 0; g < G; ++g) {
            const u32 nz = nz8(cb[g]);
            const LAS u32* q = (const LAS u32*)(size_t)(pos & ~3u);
            sel[g] = *(const LAS u32x2*)(size_t)__builtin_amdgcn_udot4(nz, 0x08040201u, ldsbase + L_LUT, false);
            P[g] = (u32x2){q[0], q[1]};
            ps[g] = pos;
            pos += __builtin_popcount(nz);
        }
        __builtin_amdgcn_sched_barrier(0);
#pragma unroll
        for (int g = 0; g < G; ++g) {
            const u32 by = __builtin_amdgcn_alignbyte(P[g][1], P[g][0], ps[g]), lb = perm(by, by, sel[g][0]), hb = (cb[g] + B) & sel[g][1];
            P[g] = (u32x2){perm(hb, lb, 0x05010400u), perm(hb, lb, 0x07030602u)};
        }
    }
#pragma unroll
    for (int g = 0; g < G; g += 2) st_out(c, out_off(c, row0, col0) + ((g >> 2) * 16 + (g & 3)) * (NTOT * 2), P[g], P[g + 1]);
    ST2(c, 512 + 384 + tk, nexc);
    const u32 j = (hv >> 22) & 63, ln = hv >> 16, xo = (row0 + (j & 48) + (ln & 48) / 4 + (j >> 2 & 3)) * (NTOT * 2) + (col0 + (ln & 7) * 8 + (ln & 8) / 2 + (j & 3)) * 2;
    __builtin_amdgcn_raw_buffer_store_b16((u16)hv, c.ro, (u32)c.lane - 1 < nexc ? xo : ~1u, 0, OAUX);
}
DEV void fetch_y(const Ctx& c, char* peer, u32 e, u32 lo) {
    if (e & (YM_BAD | YM_RAW)) return;
    const u32 src = WS_YP + (c.rank * NMSG + ((e >> 20) & 0xff)) * YSLOT, bytes = min((e & 0x3ff) * 16, LSLOT);
    const rsrc_t rs = __builtin_amdgcn_make_buffer_rsrc(peer, 0, src + bytes, 0x00020000);
    glds4(rs, c.lane * 16, src, (u32)(size_t)c.lds + lo);
    if (bytes > 4096) glds2(rs, c.lane * 16, src + 4096, (u32)(size_t)c.lds + lo + 4096);
}

DEV void worker(const Ctx& c, char* peer, int d, int sub, int p, int w, u32 call) {
    const rsrc_t pr = __builtin_amdgcn_make_buffer_rsrc(peer, 0, WS_END, 0x00020000);
    pull_inputs(c, peer, pr, w * 2 * LSLOT, d, sub, p, call);
    STAMPW(c, 1);
    const u32 lo = w * 2 * LSLOT;
#pragma unroll 1
    for (;;) {
        const u32x2 st = *(const volatile LAS u32x2*)&c.ctl[C_YT];
        const u32 two = YPAIR && rfl(st[1]) >= rfl(st[0]) + YPAIR && rfl(st[0]) + YPEND <= NQY;
        u32 tk = 0;
        if (c.lane == 0) tk = __hip_atomic_fetch_add((LAS u32*)&c.ctl[C_YT], 1 + two, __ATOMIC_RELAXED, __HIP_MEMORY_SCOPE_WORKGROUP);
        tk = rfl(tk);
        if (tk >= NQY) break;
        const u32 n = min(1 + two, NQY - tk);
        u32 e[2] = {lds_wait(c, C_Q + tk, 1, false, 1), lds_wait(c, C_Q + tk + n - 1, 1, false, 1)};
        if (!e[0] || !e[1]) { tmo_raise(c, call); break; }
        if (tk + n > YWIN) lds_wait(c, C_YL, tk + n - YWIN, false, 0);
        ST2(c, 1024 + 448 + tk, 4 + n);
        fetch_y(c, peer, e[0], lo);
        if (n > 1) fetch_y(c, peer, e[1], lo + LSLOT);
        WAIT_VM0();
        if (c.lane == 0) __hip_atomic_fetch_add((LAS u32*)&c.ctl[C_YL], n, __ATOMIC_RELAXED, __HIP_MEMORY_SCOPE_WORKGROUP);
        ST2(c, 1536 + 448 + tk, 0);
        if (tk == 0) STAMPW(c, 4);
        STAMPW(c, 5);
#pragma unroll 1
        for (u32 i = 0; i < n; ++i) {
            const u32 q = i ? e[1] : e[0], id = (q >> 20) & 0xff, wc = id & 1, rg = (id >> 1) & 7, cc = id >> 4;
            decode_y<4>(c, DIAG & 1 ? q | YM_BAD : q, lo + i * LSLOT, pr, WS_YP + (c.rank * NMSG + id) * YSLOT, rg * 64, p * 2048 + cc * 128 + wc * 64, tk);
        }
        ST2(c, 512 + 448 + tk, 0);
    }
    STAMPW(c, 2);
}

__global__ void __launch_bounds__(NTHREADS, 3) fused_kernel(Args a) {
    extern __shared__ __attribute__((aligned(128))) char smem[];
    Ctx c;
    c.x = a.x; c.w = a.w; c.out = a.out; c.rank = a.rank & 7; c.me = a.me;
    c.wg = blockIdx.x; c.wave = rfl(threadIdx.x >> 6); c.lane = threadIdx.x & 63;
    c.peer_lane = a.ws[c.lane & 7];
    c.rs = __builtin_amdgcn_make_buffer_rsrc(c.me, 0, WS_END, 0x00020000);
    c.ro = __builtin_amdgcn_make_buffer_rsrc(c.out, 0, DIAG & 2 ? 0 : 512 * NTOT * 2, 0x00020000);
    c.lds = (LAS char*)smem; c.ctl = (volatile LAS u32*)(c.lds + L_CTL);
    STAMPW(c, 0);
    u32 init = 0;
    static_assert(C_CALL == 0, "thread 0");
    if (threadIdx.x == 0) {
        GAS u32* cnt = (GAS u32*)(c.me + WS_CNT + c.wg * 128);
        init = __hip_atomic_load(cnt, __ATOMIC_RELAXED, __HIP_MEMORY_SCOPE_AGENT) + 1;
        __hip_atomic_store(cnt, init, __ATOMIC_RELAXED, __HIP_MEMORY_SCOPE_AGENT);
    }
    if (threadIdx.x < C_WORDS) c.ctl[threadIdx.x] = init;
    if (c.wave == 11) {
        u32 s0, s1;
        if (c.wg >= NCOMM) {
            const u32 mask = c.lane >> 2;
            u32 n = c.lane & 3;
            u64 sel = 0x0c0c0c0c03020100ull;
            for (int b = 0; b < 4; ++b)
                if ((mask >> b) & 1) { sel = (sel & ~(0xffull << (8 * n))) | (u64)(4 + b) << (8 * n); ++n; }
            s0 = (u32)sel; s1 = n >= 4 ? (u32)(sel >> 32) : 0x03020100u;
        } else {
            u32 n = 0;
            s0 = 0x0c0c0c0cu; s1 = 0;
            for (int b = 0; b < 4; ++b)
                if ((c.lane >> b) & 1) { s0 = (s0 & ~(0xffu << (8 * b))) | n << (8 * b); s1 |= 0xffu << (8 * b); ++n; }
        }
        c.ctl[C_LUT + 2 * c.lane] = s0; c.ctl[C_LUT + 2 * c.lane + 1] = s1;
    }
    if (c.wave == 10 && c.lane < 16) {
        u32 n = 0, s0 = 0x0c0c0c0cu, s1 = 0x0c0c0c0cu, sd = 0x0c0c0c0cu;
        for (int b = 0; b < 4; ++b)
            if ((c.lane >> b) & 1) {
                u32& t = n & 1 ? s1 : s0;
                t = (t & ~(0xffu << (n >> 1) * 8)) | b << (n >> 1) * 8;
                sd = (sd & ~(0xffu << 8 * b)) | ((n & 1) * 4 + (n >> 1)) << 8 * b;
                ++n;
            }
        if (c.wg >= NCOMM) { c.ctl[C_LUT2 + 2 * c.lane] = s0; c.ctl[C_LUT2 + 2 * c.lane + 1] = s1; }
        else c.ctl[C_LUT2 + c.lane] = sd;
    }
    if (c.wg >= NCOMM) gemm_wg(c);
    else {
        wg_barrier();
        const u32 call = get_call(c);
        if (c.wave == 0) c.ctl[C_SEAT] = take_seat(c, call) + 1;
        const u32 seat = lds_wait(c, C_SEAT, 1, false, 1) - 1;
        if (seat >= NPULL) return;
        const int d = 1 + seat % 7, sub = seat / 7, p = (c.rank + d) & 7;
        if (c.wave == 0) watcher(c, p, sub, call);
        else worker(c, a.ws[p], d, sub, p, c.wave - 1, call);
    }
}

}

extern "C" void kernel_launch(int rank, int nranks, ncclComm_t comm, hipStream_t stream,
                              void* const* d_in, const int* in_sizes, int n_in,
                              void* d_out, int out_size, const int* in_dtypes, int out_dtype,
                              void* const* d_out_all, void* const* d_ws_all, size_t ws_bytes) {
    if (nranks != NR || (unsigned)rank >= (unsigned)NR || n_in != 2 || ws_bytes < WS_END || in_sizes[0] != 4096 * KSH ||
        in_sizes[1] != KSH * NTOT || out_size != 512 * NTOT) {
        fprintf(stderr, "unexpected problem shape\n");
        return;
    }
    HIP_CHECK(hipFuncSetAttribute((const void*)fused_kernel, hipFuncAttributeMaxDynamicSharedMemorySize, LDS_BYTES));
    Args a;
    a.x = (const u16*)d_in[0]; a.w = (const u16*)d_in[1]; a.out = (u16*)d_out;
    for (int p = 0; p < NR; ++p) a.ws[p] = (char*)d_ws_all[p];
    a.me = a.ws[rank]; a.rank = rank; a.pad = 0;
    hipLaunchKernelGGL(fused_kernel, dim3(NWG), dim3(NTHREADS), LDS_BYTES, stream, a);
    HIP_CHECK(hipGetLastError());
}
```

```cpp
#include <hip/hip_runtime.h>
#include <rccl/rccl.h>
#include <cstdio>
#include <cstdint>

#define HIP_CHECK(expr) do { hipError_t _e = (expr); if (_e != hipSuccess) { \
    fprintf(stderr, "HIP error %s at %s:%d\n", hipGetErrorString(_e), __FILE__, __LINE__); \
    return; } } while (0)

extern "C" __device__ unsigned writelane(unsigned value, unsigned lane, unsigned old) __asm("llvm.amdgcn.writelane.i32");

namespace {

typedef unsigned u32;
typedef unsigned long long u64;
typedef unsigned short u16;
typedef u32 u32x4 __attribute__((ext_vector_type(4)));
typedef u32 u32x2 __attribute__((ext_vector_type(2)));
typedef u32 u32x8 __attribute__((ext_vector_type(8)));
typedef u32 u32x32 __attribute__((ext_vector_type(32)));
typedef u16 u16x2 __attribute__((ext_vector_type(2)));
typedef short i16x2 __attribute__((ext_vector_type(2)));
typedef float f32x4 __attribute__((ext_vector_type(4)));
typedef float f32x2 __attribute__((ext_vector_type(2)));
typedef __bf16 bf16x2 __attribute__((ext_vector_type(2)));
typedef __bf16 bf16x8 __attribute__((ext_vector_type(8)));
typedef __amdgpu_buffer_rsrc_t rsrc_t;
#define LAS __attribute__((address_space(3)))
#define GAS __attribute__((address_space(1)))
#define DEV __device__ __forceinline__

#define PSLEEP 3
#define LSLEEP 4
#define SPREAD 0
#define PSTEP 1
#define PS1 4
#define PS2 5
#define PS3 6
#define P2 0
#define L4 1
#define PACE 27
#define PFREE 3
#define NPUSH 16
#define JROT 2
#define YPEND 12
#define YWIN 4
#define RELF 3
#define LNAP 1
#define LPRIO 0
#define MAXP 5
#define OWNALL 0
#define UPRIO 2
#define EPS 0
#define EFREE 1
#define YPRE 0
#define CP2 0
#define EPRIO 2
#define WNAP 0
#define CNAP 0
#define CPRIO 0
#define OAUX 17
#define TOPO 1
#define DIAG 0
#define XROT 1
#define FORCE_RAW_Y 0
#define FORCE_RAW_IN 0

constexpr int NR = 8;
constexpr int NTOT = 16384, KSH = 512;
constexpr int NWG = 256, NCOMM = 32, NPULL = 28, NGEMM = NWG - NCOMM, NTHREADS = 768;
constexpr int NCHIN = 384;
constexpr int NMSG = 256;
constexpr u32 NQY = NMSG / 4;
constexpr int NWORK = 11;

constexpr u32 WS_TMO = 128, WS_CLM = 1024;
constexpr u32 WS_SEAT = 2048;
constexpr u32 WS_FIN = 4096;
constexpr u32 WS_FY = WS_FIN + 2 * NR * NCHIN * 8;
constexpr u32 WS_XR = WS_FY + NR * NMSG * 8;
constexpr u32 WS_WR = WS_XR + 8192;
constexpr u32 WS_CNT = WS_WR + 4096;
constexpr u32 WS_DBG = 131072;
static_assert(WS_CNT + NWG * 128 <= WS_DBG, "ws");
#define CHBV 16384
constexpr u32 CHB = CHBV;
constexpr u32 WS_PK = 1u << 20;
constexpr u32 PK_PAR = 1280u * CHB;
constexpr u32 WS_XA = WS_PK + 2 * PK_PAR;
constexpr u32 WS_WB = WS_XA + (32u << 20);
#define YSL 8192
constexpr u32 YSLOT = YSL;
constexpr u32 WS_YP = WS_WB + (20u << 20);
constexpr u32 WS_END = WS_YP + 8u * NMSG * YSLOT;
constexpr u32 PK_TAIL = 8192 + 2048;

constexpr u32 STG = 28672, NSTG = 5;
constexpr u32 LSLOT = 6144;
constexpr u32 L_CTL = STG * NSTG;
static_assert(NWORK * 2 * LSLOT + 8192 <= L_CTL, "ring");
constexpr u32 C_CALL = 0, C_ABORT = 1, C_FULL = 8, C_FREE = 16, C_LE = 24, C_LIST = 64,
              C_TK = 2, C_IS = 3, C_LAND = 4, C_YGO = 5, C_YT = 6, C_QN = 7, C_YL = 8, C_SEAT = 9, C_Q = 64, C_WORDS = 256,
              C_LUT = C_WORDS;
constexpr u32 C_LUT2 = C_LUT + 128, L_PST = L_CTL + (C_LUT2 + 32) * 4, PST = 4352;
constexpr u32 LDS_BYTES = L_PST + 4 * PST;
#define ST2(c, i, v) do {} while (0)
constexpr u32 YS = 128 % NSTG, L_LUT = L_CTL + C_LUT * 4, L_LUT2 = L_CTL + C_LUT2 * 4, L_YST = YS * STG, YST = 4608;
static_assert(L_LUT % 512 == 0 && YS + 1 < NSTG && 8 * YST <= 2 * STG && 4 * (32 + 64 * 17) <= YST && 1536 + 4 * (32 + 64 * 17) <= LSLOT, "y staging");
static_assert(LDS_BYTES <= 160 * 1024, "lds");

constexpr u64 DEADLINE = 40000000ull;
constexpr u64 DEADLINE_LDS = 100000000ull;

constexpr u32 YM_RAW = 1u << 16, YM_BAD = 1u << 17;

constexpr int NPOS = NCHIN - NPUSH;
static_assert(NPUSH <= 24, "push");
struct Order { u16 v[7][NPOS]; };
#define D1V 6
#define D2V 2
#define D3V 0
#define SQV 2
#define SQA 2
constexpr Order make_order() {
    Order o{};
    for (int d = 1; d <= 7; ++d) {
        int n = 0;
        for (int pass = 0; pass < 4; ++pass)
            for (int s = 0; s < 8; ++s) {
                if (pass == (s >= 8 - D1V ? 3 : 0) && s) for (int q = 0; q < 4; ++q) o.v[d - 1][n++] = (u16)(s << 8 | q);
                const bool sq = s >= 8 - SQV;
                for (int c = 0; c < 16 && !sq; ++c) if (pass == ((c & 2) && s >= 8 - D3V ? 1 : 0) && s * 16 + c >= NPUSH) o.v[d - 1][n++] = (u16)(0x8000 | s << 8 | c);
                if (pass == (s >= 8 - D1V ? 3 : 0) && !s) for (int q = 0; q < 4; ++q) o.v[d - 1][n++] = (u16)(s << 8 | q);
                for (int j = 0; j < 8 && sq && !pass; ++j)
                    for (int i = 0; i < 4; ++i)
                        o.v[d - 1][n++] = (u16)(i < 2 ? 0x8000 | s << 8 | (2 * (j + SQA * d) + i) % 16 : j < 7 ? s << 8 | (1 + (j + d) % 7) << 2 | (i & 1) : s << 8 | 4 << 2 | i);
                for (int i = 0; i < 28; ++i) {
                    const int q = (i / 14) * 2 + (i & 1), own = 1 + ((i % 14) / 2 + d) % 7;
                    if (sq && (q < 2 || own == 4)) continue;
                    if (pass == (q >= 2 && own != 4 && s >= 8 - D2V ? 2 : 0)) o.v[d - 1][n++] = (u16)(s << 8 | own << 2 | q);
                }
            }
    }
    return o;
}
__constant__ const Order ORDER = make_order();

struct Args { const u16* x; const u16* w; u16* out; char* me; int rank; int pad; char* ws[NR]; };

#define RARE(x) __builtin_expect(!!(x), 0)
DEV u64 rt() { return __builtin_amdgcn_s_memrealtime(); }
DEV u32 rfl(u32 v) { return __builtin_amdgcn_readfirstlane(v); }
DEV u32 perm(u32 hi, u32 lo, u32 sel) { return __builtin_amdgcn_perm(hi, lo, sel); }
DEV u32 bfi(u32 m, u32 a, u32 b) { return (m & a) | (~m & b); }
#define CBAR() asm volatile("" ::: "memory")
#define WAIT_VM0() asm volatile("s_waitcnt vmcnt(0)" ::: "memory")
#define WAIT_LGKM0() do { CBAR(); __builtin_amdgcn_s_waitcnt(0xc07f); CBAR(); } while (0)

#define STAMPW(c, i) do {} while (0)
#define STAMPC(c, i) do {} while (0)

struct Ctx {
    const u16* x; const u16* w; u16* out; char* me; char* peer_lane;
    rsrc_t rs, ro;
    LAS char* lds; volatile LAS u32* ctl;
    int rank, wg, wave, lane;
};

DEV u32 ld_sys(const void* p) { return __hip_atomic_load((const GAS u32*)p, __ATOMIC_RELAXED, __HIP_MEMORY_SCOPE_SYSTEM); }
DEV u64 ld_sys64(const void* p) { return __hip_atomic_load((const GAS u64*)p, __ATOMIC_RELAXED, __HIP_MEMORY_SCOPE_SYSTEM); }
DEV void st_sys(void* p, u32 v) { __hip_atomic_store((GAS u32*)p, v, __ATOMIC_RELAXED, __HIP_MEMORY_SCOPE_SYSTEM); }
DEV void st_sys64(void* p, u64 v) { __hip_atomic_store((GAS u64*)p, v, __ATOMIC_RELAXED, __HIP_MEMORY_SCOPE_SYSTEM); }
DEV void st16(const Ctx& c, u32x4 v, u32 voff, u32 soff) { __builtin_amdgcn_raw_buffer_store_b128(v, c.rs, voff + soff, 0, 17); }
DEV void st_out(const Ctx& c, u32 off, u32x2 a, u32x2 b) {
    u32x4 v;
#pragma unroll
    for (int i = 0; i < 2; ++i) {
        v[i] = __builtin_amdgcn_update_dpp(a[i], b[i], 0x128, 0xf, 0xc, false);
        v[2 + i] = __builtin_amdgcn_update_dpp(b[i], a[i], 0x128, 0xf, 0x3, false);
    }
    __builtin_amdgcn_raw_buffer_store_b128(v, c.ro, off, 0, OAUX);
}
DEV u32 out_off(const Ctx& c, u32 row0, u32 col0) { return (row0 + (c.lane >> 4) * 4 + (c.lane >> 3 & 1)) * (NTOT * 2) + (col0 + (c.lane & 7) * 8) * 2; }
DEV bool tmo_set(const Ctx& c, u32 call) { return rfl(__hip_atomic_load((const GAS u32*)(c.me + WS_TMO), __ATOMIC_RELAXED, __HIP_MEMORY_SCOPE_AGENT)) == call; }
DEV void tmo_raise(const Ctx& c, u32 call) { if (c.lane == 0) __hip_atomic_store((GAS u32*)(c.me + WS_TMO), call, __ATOMIC_RELAXED, __HIP_MEMORY_SCOPE_AGENT); }

DEV bool expired(u64& t0, u64 limit) { const u64 t = rt(); if (!t0) t0 = t; return t - t0 > limit; }

DEV u32 lds_wait(const Ctx& c, u32 word, u32 want, bool look_abort, int nap, u32 mask = ~0u) {
    u64 t0 = 0;
    for (u32 n = 1;; ++n) {
        const u32 v = rfl(c.ctl[word]) & mask;
        if (v >= want) return v;
        if (nap) __builtin_amdgcn_s_sleep(1);
        if ((n & 63) == 0) {
            if (look_abort && rfl(c.ctl[C_ABORT])) return 0;
            if (expired(t0, DEADLINE_LDS)) { c.ctl[C_ABORT] = 1; return 0; }
        }
    }
}
DEV u32 get_call(const Ctx& c) { return rfl(c.ctl[C_CALL]); }

DEV void glds4(rsrc_t rs, u32 voff, u32 soff, u32 ldsaddr) {
    u32 keep;
    asm volatile("s_mov_b32 %0, m0\n\ts_mov_b32 m0, %4\n\ts_nop 2\n\t"
                 "buffer_load_dwordx4 %1, %2, %3 offen sc0 sc1 lds\n\t"
                 "buffer_load_dwordx4 %1, %2, %3 offen offset:1024 sc0 sc1 lds\n\t"
                 "buffer_load_dwordx4 %1, %2, %3 offen offset:2048 sc0 sc1 lds\n\t"
                 "buffer_load_dwordx4 %1, %2, %3 offen offset:3072 sc0 sc1 lds\n\t"
                 "s_mov_b32 m0, %0" : "=&s"(keep) : "v"(voff), "s"(rs), "s"(soff), "s"(ldsaddr) : "memory");
}
#define GPOL " sc0 sc1"
DEV void glds4g(rsrc_t rs, u32 voff, u32 soff, u32 ldsaddr) {
    u32 keep;
    if (DIAG & 4) return;
    if (DIAG & 8) soff = WS_XA + (soff & 0x7ffff);
    asm volatile("s_mov_b32 %0, m0\n\ts_mov_b32 m0, %4\n\ts_nop 2\n\t"
                 "buffer_load_dwordx4 %1, %2, %3 offen" GPOL " lds\n\t"
                 "buffer_load_dwordx4 %1, %2, %3 offen offset:1024" GPOL " lds\n\t"
                 "buffer_load_dwordx4 %1, %2, %3 offen offset:2048" GPOL " lds\n\t"
                 "buffer_load_dwordx4 %1, %2, %3 offen offset:3072" GPOL " lds\n\t"
                 "s_mov_b32 m0, %0" : "=&s"(keep) : "v"(voff), "s"(rs), "s"(soff), "s"(ldsaddr) : "memory");
}
DEV void glds2(rsrc_t rs, u32 voff, u32 soff, u32 ldsaddr) {
    u32 keep;
    asm volatile("s_mov_b32 %0, m0\n\ts_mov_b32 m0, %4\n\ts_nop 2\n\t"
                 "buffer_load_dwordx4 %1, %2, %3 offen sc0 sc1 lds\n\t"
                 "buffer_load_dwordx4 %1, %2, %3 offen offset:1024 sc0 sc1 lds\n\t"
                 "s_mov_b32 m0, %0" : "=&s"(keep) : "v"(voff), "s"(rs), "s"(soff), "s"(ldsaddr) : "memory");
}
DEV u32x4 ld16(rsrc_t rs, u32 voff, u32 soff) { return __builtin_amdgcn_raw_buffer_load_b128(rs, voff, soff, 17); }
DEV u32 lds_inc(const Ctx& c, u32 word) {
    u32 v = 0;
    if (c.lane == 0) v = __hip_atomic_fetch_add((LAS u32*)&c.ctl[word], 1u, __ATOMIC_RELAXED, __HIP_MEMORY_SCOPE_WORKGROUP);
    return rfl(v);
}

DEV u32 wave_max(u32 v) {
    v = max(v, (u32)__builtin_amdgcn_update_dpp(0, (int)v, 0x111, 0xf, 0xf, true));
    v = max(v, (u32)__builtin_amdgcn_update_dpp(0, (int)v, 0x112, 0xf, 0xf, true));
    v = max(v, (u32)__builtin_amdgcn_update_dpp(0, (int)v, 0x114, 0xf, 0xf, true));
    v = max(v, (u32)__builtin_amdgcn_update_dpp(0, (int)v, 0x118, 0xf, 0xf, true));
    v = max(v, (u32)__builtin_amdgcn_update_dpp(0, (int)v, 0x142, 0xa, 0xf, false));
    v = max(v, (u32)__builtin_amdgcn_update_dpp(0, (int)v, 0x143, 0xc, 0xf, false));
    return __builtin_amdgcn_readlane(v, 63);
}

DEV u32 wave_scan(u32 v) {
    v += (u32)__builtin_amdgcn_update_dpp(0, (int)v, 0x111, 0xf, 0xf, true);
    v += (u32)__builtin_amdgcn_update_dpp(0, (int)v, 0x112, 0xf, 0xf, true);
    v += (u32)__builtin_amdgcn_update_dpp(0, (int)v, 0x114, 0xf, 0xf, true);
    v += (u32)__builtin_amdgcn_update_dpp(0, (int)v, 0x118, 0xf, 0xf, true);
    v += (u32)__builtin_amdgcn_update_dpp(0, (int)v, 0x142, 0xa, 0xf, false);
    v += (u32)__builtin_amdgcn_update_dpp(0, (int)v, 0x143, 0xc, 0xf, false);
    return v;
}
DEV u32 lane_rank(u64 m) { return __builtin_amdgcn_mbcnt_hi((u32)(m >> 32), __builtin_amdgcn_mbcnt_lo((u32)m, 0)); }

DEV void store_raw(const Ctx& c, const u32x4 (&R)[16], u32 off0, u32 off1) {
#pragma unroll
    for (int f = 0; f < 16; ++f) st16(c, R[f], c.lane * 16, (f < 8 ? off0 : off1) + (f & 7) * 1024);
}
DEV u32 sad(u32 a, u32 acc) { return __builtin_amdgcn_sad_u8(a, 0, acc); }
DEV u32 pk_sub(u32 a, u32 b) { return __builtin_bit_cast(u32, (u16x2)(__builtin_bit_cast(u16x2, a) - __builtin_bit_cast(u16x2, b))); }
DEV u32 pack_store(const Ctx& c, u32x4 (&X)[16], u32 dst) {
    const u32 ldsbase = (u32)(size_t)c.lds, stg = L_PST + (c.wave & 3) * PST;
#pragma unroll
    for (int i = 0; i < 5; ++i) *(LAS u32x4*)(c.lds + stg + c.lane * 16 + min(i * 1024u, PST - 1024)) = (u32x4){0, 0, 0, 0};
    u32 sum = 0;
#pragma unroll
    for (int f = 0; f < 16; ++f)
#pragma unroll
        for (int i = 0; i < 4; ++i) X[f][i] = __builtin_amdgcn_alignbit(X[f][i], X[f][i], 31);
#pragma unroll
    for (int f = 0; f < 16; ++f) asm volatile("" : "+v"(X[f]));
#pragma unroll
    for (int f = 0; f < 16; f += 2) sum = sad(perm(X[f][1], X[f][0], 0x07050301u), sum);
#pragma unroll
    for (int i = 0; i < 8; ++i)
        st16(c, (u32x4){perm(X[2 * i][1], X[2 * i][0], 0x06040200u), perm(X[2 * i][3], X[2 * i][2], 0x06040200u),
                        perm(X[2 * i + 1][1], X[2 * i + 1][0], 0x06040200u), perm(X[2 * i + 1][3], X[2 * i + 1][2], 0x06040200u)}, c.lane * 16, dst + i * 1024);
    const u32 top = min(max(((__builtin_amdgcn_readlane(wave_scan(sum), 63) + 1946) >> 11) + 2, 12u), 255u), K = (top << 8 | 0xff) * 0x10001u;
    u32 nexc = 0, hv = 0, D[8], W[16], rel[16], bits = 0;
#pragma unroll
    for (int j = 0; j < 8; ++j) {
        u32 t[4], cf[4], es[4];
        u32x2 sel[4];
#pragma unroll
        for (int g = 0; g < 4; ++g) {
            const u32x4 x = X[2 * j + (g >> 1)];
            t[g] = perm(pk_sub(K, x[(g & 1) * 2 + 1]), pk_sub(K, x[(g & 1) * 2]), 0x07050301u);
            cf[g] = perm(0, 0x07060500u, t[g]);
            es[g] = ~cf[g] & 0x04040404u;
            sel[g] = *(const LAS u32x2*)(size_t)__builtin_amdgcn_udot4(es[g], 0x10080402u, ldsbase + L_LUT2, false);
        }
        D[j] = bfi(0x03030303u, cf[0], bfi(0x0c0c0c0cu, cf[1] << 2, bfi(0x30303030u, cf[2] << 4, cf[3] << 6)));
#pragma unroll
        for (int h = 0; h < 2; ++h) {
            const int f = 2 * j + h;
            for (u64 m = __ballot((cf[2 * h] | cf[2 * h + 1]) & 0x80808080u); m && nexc <= 31; m &= m - 1) {
                const u32 ln = __builtin_ctzll(m);
                const u32 r0 = __builtin_amdgcn_readlane(X[f][0], ln), r1 = __builtin_amdgcn_readlane(X[f][1], ln);
                const u32 r2 = __builtin_amdgcn_readlane(X[f][2], ln), r3 = __builtin_amdgcn_readlane(X[f][3], ln);
                for (u32 fl = (__builtin_amdgcn_readlane(cf[2 * h], ln) & 0x80808080u) >> 1 | (__builtin_amdgcn_readlane(cf[2 * h + 1], ln) & 0x80808080u); fl; fl &= fl - 1) {
                    const u32 b = __builtin_ctz(fl), x = b & 1 ? (b & 16 ? r3 : r2) : (b & 16 ? r1 : r0), r = x >> 1 | x << 31;
                    ++nexc;
                    hv = writelane((f << 9 | ln << 3 | (b & 1) << 2 | b >> 3) << 16 | ((r >> (b & 8) * 2) & 0xffff), nexc & 63, hv);
                }
            }
            const u32 sa = sad(es[2 * h], 0);
            const u32 wa = perm(0, t[2 * h], sel[2 * h][0]) | perm(0, t[2 * h], sel[2 * h][1]) << 4;
            const u32 wb = perm(0, t[2 * h + 1], sel[2 * h + 1][0]) | perm(0, t[2 * h + 1], sel[2 * h + 1][1]) << 4;
            W[f] = wa | wb << sa;
            rel[f] = bits;
            bits = sad(es[2 * h + 1], bits + sa);
            asm volatile("" : "+v"(W[f]), "+v"(bits));
        }
        asm volatile("" : "+v"(D[j]));
    }
    if (FORCE_RAW_IN || nexc > 31) {
#pragma unroll
        for (int f = 0; f < 16; ++f)
#pragma unroll
            for (int i = 0; i < 4; ++i) X[f][i] = __builtin_amdgcn_alignbit(X[f][i], X[f][i], 1);
        store_raw(c, X, dst, dst + 8192);
        return YM_RAW | 1024;
    }
    st16(c, (u32x4){D[0], D[1], D[2], D[3]}, c.lane * 16, dst + 8192);
    st16(c, (u32x4){D[4], D[5], D[6], D[7]}, c.lane * 16, dst + 9216);
    hv = writelane(top | nexc << 8, 0, hv);
    if ((u32)c.lane <= nexc) *(LAS u32*)(c.lds + stg + c.lane * 4) = hv;
    const u32 incl = wave_scan(bits), start = 32 * (1 + nexc) + incl - bits, bytes = 4 * (1 + nexc) + ((__builtin_amdgcn_readlane(incl, 63) + 7) >> 3);
#pragma unroll
    for (int f = 0; f < 16; ++f) {
        const u32 pb = start + rel[f];
        const u64 v = (u64)W[f] << (pb & 31);
        LAS u32* q = (LAS u32*)(size_t)(ldsbase + stg + (pb >> 5) * 4);
        __hip_atomic_fetch_or(q, (u32)v, __ATOMIC_RELAXED, __HIP_MEMORY_SCOPE_WAVEFRONT);
        __hip_atomic_fetch_or(q + 1, (u32)(v >> 32), __ATOMIC_RELAXED, __HIP_MEMORY_SCOPE_WAVEFRONT);
    }
    CBAR();
    u32x4 v[5];
#pragma unroll
    for (int i = 0; i < 5; ++i) if (i < 2 || bytes > i * 1024u) v[i] = *(const LAS u32x4*)(c.lds + stg + c.lane * 16 + min(i * 1024u, PST - 1024));
#pragma unroll
    for (int i = 0; i < 5; ++i) if (i < 2 || bytes > i * 1024u) st16(c, v[i], c.lane * 16, dst + PK_TAIL + min(i * 1024u, PST - 1024));
    return (PK_TAIL + bytes + 15) >> 4;
}

DEV u32 xa_off(int ks, int h, int rowfrag) { return WS_XA + (u32)((ks * 2 + h) * 256 + rowfrag) * 1024u; }
DEV u32 wb_off(int ks, int h, int colfrag) { return WS_WB + (u32)((ks * 2 + h) * 128 + colfrag) * 1024u; }

DEV int wb_ks(int ks, u32 call) { return (call & 1) && (ks & 7) < 2 ? 64 + (ks >> 3) * 2 + (ks & 7) : ks; }
DEV void wg_barrier() { WAIT_LGKM0(); __builtin_amdgcn_s_barrier(); CBAR(); }

DEV void pack_job(const Ctx& c, int job) {
    const int s = job / 160, r0 = job % 160, q = s != 1 || r0 >= 96 ? r0 : r0 < 64 ? r0 + 32 : r0 - 64, ks = c.rank * 8 + s;
    const int l15 = c.lane & 15, lq = c.lane >> 4;
    const bool isx = q < 32;
    const int dst = (c.rank + 1 + (q - 32)) & 7, cc = (q - 32) >> 3;
    u32x4 R[16];
    if (job < NGEMM) STAMPW(c, 13);
    if (isx) {
        const GAS char* src = (const GAS char*)c.x + (size_t)(q * 128 + l15) * (KSH * 2) + (s * 64 + lq * 8) * 2;
#pragma unroll
        for (int f = 0; f < 16; ++f) R[f] = *(const GAS u32x4*)(src + (f & 7) * 16 * (KSH * 2) + (f >> 3) * 64);
    } else {
        const GAS char* src = (const GAS char*)c.w + (size_t)(s * 64 + lq * 8) * (NTOT * 2) + (dst * 2048 + cc * 128 + (l15 & 7) * 8 + (l15 >> 3) * 4) * 2;
        u32x2 r[4][8];
#pragma unroll
        for (int hg = 0; hg < 4; ++hg)
#pragma unroll
            for (int j = 0; j < 8; ++j) r[hg][j] = *(const GAS u32x2*)(src + (size_t)((hg >> 1) * 32 + j) * (NTOT * 2) + (hg & 1) * 128);
#pragma unroll
        for (int hg = 0; hg < 4; ++hg)
#pragma unroll
            for (int ii = 0; ii < 4; ++ii)
#pragma unroll
                for (int t = 0; t < 4; ++t)
                    R[hg * 4 + ii][t] = perm(r[hg][2 * t + 1][ii >> 1], r[hg][2 * t][ii >> 1], (ii & 1) ? 0x07060302u : 0x05040100u);
    }
    if (job < NGEMM) wg_barrier();
    __builtin_amdgcn_s_waitcnt(0x0f70);
    if (job < NGEMM) STAMPW(c, 4);
    const u32 call = get_call(c), ch = isx ? s * 32 + q : 256 + s * 16 + cc;
    const u32 slot = WS_PK + (call & 1) * PK_PAR + (isx ? ch : ch + dst * 128) * CHB;
    const bool done = !isx && (dst == c.rank || ch < 256 + NPUSH);
    if (done && dst != c.rank && ch > 256 + PFREE)
        for (const u64 t0 = rt(); rt() - t0 < (ch - 256 - PFREE) * PACE;) __builtin_amdgcn_s_sleep(2);
    if (isx || done) {
        const u64 base = (u64)c.peer_lane;
        const u32 to = isx ? c.rank : dst, d0 = isx ? xa_off(ks, 0, q * 8) : wb_off(wb_ks(ks, call), 0, cc * 8);
        Ctx e = c;
        e.rs = __builtin_amdgcn_make_buffer_rsrc((char*)((u64)(u32)__builtin_amdgcn_readlane((u32)(base >> 32), to) << 32 | (u32)__builtin_amdgcn_readlane((u32)base, to)), 0, WS_END, 0x00020000);
        store_raw(e, R, d0, d0 + (isx ? 256 : 128) * 1024);
    }
    u32 meta = 0;
    if (!done) meta = pack_store(c, R, slot);
    if (job < NGEMM) STAMPW(c, 5);
    WAIT_VM0();
    if (c.lane < NR && (isx || c.lane == dst)) {
        if (done || c.lane == c.rank) st_sys(c.peer_lane + (isx ? WS_XR + q * 256 : WS_WR + cc * 256) + ks * 4, call);
        else st_sys64(c.peer_lane + WS_FIN + (((call & 1) * NR + c.rank) * NCHIN + ch) * 8, (u64)call << 32 | meta);
    }
}

struct Tile { int cc, owner, h, rf, rfo; };
constexpr u32 TK2 = 130;

constexpr u32 LE_FULL = 64, LE_OK = 128, LE_END = 256, NISS = 3;
DEV void lds_add1(const Ctx& c, u32 word) {
    asm volatile("s_mov_b64 exec, 1\n\tds_add_u32 %0, %1\n\ts_mov_b64 exec, -1" :: "v"((u32)(size_t)&c.ctl[word]), "v"(1u) : "memory");
}
template <bool BIG>
DEV void lister(const Ctx& c, const Tile& t, u32 xr, u32 call) {
    const u32 ks = c.lane ^ xr;
    const char* fx = c.me + WS_XR + (t.rf >> 3) * 256 + ks * 4;
    const char* fo = c.me + WS_XR + (t.rfo >> 3) * 256 + ks * 4;
    const char* fw = c.me + WS_WR + t.cc * 256 + ks * 4;
    u64 todo = ~0ull, late = 0, t0 = 0;
    u32 n = 0;
    bool second = false;
    for (u32 it = 1;; ++it) {
        const u32 o = BIG ? ld_sys(fo) : 0;
        u64 m = 0;
        if (!second) { const u32 a = ld_sys(fx), b = ld_sys(fx + 256), w = ld_sys(fw); m = __ballot(min(min(a, b), w) >= call); }
        const u64 own = __ballot(o >= call);
        m = (second ? own : m) & todo;
        if (m) {
            if (m == todo && tmo_set(c, call)) break;
            if ((m >> c.lane) & 1) c.ctl[C_LIST + n + lane_rank(m)] = LE_OK | (BIG && o >= call ? LE_FULL : 0) | ks;
            n += __popcll(m); todo &= ~m; late |= m & ~own; t0 = 0;
            if (!todo) {
                if (!BIG || second || !late) { c.ctl[C_LIST + n + (c.lane & 3)] = LE_END; return; }
                second = true; todo = late;
            }
        } else {
            if ((it & 15) == 0) {
                if (rfl(c.ctl[C_ABORT]) || tmo_set(c, call)) break;
                if (expired(t0, DEADLINE)) { tmo_raise(c, call); break; }
            }
            __builtin_amdgcn_s_sleep(LSLEEP);
        }
    }
    c.ctl[C_ABORT] = 1;
}
#define OS(i, n) do {} while (0)
template <bool BIG>
DEV void loader(const Ctx& c, const Tile& t, u32 call) {
    const u32 ldsbase = (u32)(size_t)c.lds, voff = c.lane * 16;
    u32 pend = 0, pslot = 0;
#pragma unroll 1
    for (;;) {
        const u32 n = lds_inc(c, C_LE), h = n & 1;
        const bool second = n >= 128;
        const u32 e = second ? n - 64 : n >> 1, tt = second ? n + (TK2 - 128) : n, slot = tt % NSTG, need = 8 * (tt / NSTG);
        const u32 look = c.ctl[c.lane ? C_LIST + e : C_FREE + slot];
        u32 v = __builtin_amdgcn_readlane(look, 1);
        if (!v || rfl(look) < need) {
            if (pend) { WAIT_VM0(); c.ctl[C_FULL + pslot] = pend; pend = 0; OS(2, pn); }
            if (!v) v = lds_wait(c, C_LIST + e, 1, true, LNAP);
            if (need && (v & LE_OK) && !lds_wait(c, C_FREE + slot, need, true, LNAP)) break;
        }
        if (!(v & LE_OK)) break;
        const int ks = v & 63, kw = wb_ks(ks, call);
        const u32 full = BIG && !second && (v & LE_FULL), l = ldsbase + slot * STG;
        OS(0, n);
        if (BIG && second) {
            glds4g(c.rs, voff, xa_off(ks, 0, t.rfo), l); glds4g(c.rs, voff, xa_off(ks, 1, t.rfo), l + 4096);
#pragma unroll
            for (int g = 0; g < 4; ++g) glds4g(c.rs, voff, wb_off(kw, g >> 1, t.cc * 8 + (g & 1) * 4), l + 8192 + g * 4096);
        } else {
#pragma unroll
            for (int g = 0; g < 4; ++g) glds4g(c.rs, voff, xa_off(ks, h, t.rf + g * 4), l + g * 4096);
            glds4g(c.rs, voff, wb_off(kw, h, t.cc * 8), l + 16384); glds4g(c.rs, voff, wb_off(kw, h, t.cc * 8 + 4), l + 20480);
        }
        OS(1, n);
        if (full) {
            glds4g(c.rs, voff, xa_off(ks, h, t.rfo), l + 24576);
            asm volatile("s_waitcnt vmcnt(28)" ::: "memory");
        } else asm volatile("s_waitcnt vmcnt(24)" ::: "memory");
        if (pend) { c.ctl[C_FULL + pslot] = pend; OS(2, pn); }
        pend = 2 * tt + 2 + full; pslot = slot;
    }
    WAIT_VM0();
    if (pend) { c.ctl[C_FULL + pslot] = pend; OS(2, pn); }
}

DEV u32 pk_relu(float a, float b) {
    const f32x2 v = __builtin_elementwise_maximum((f32x2){a, b}, (f32x2){0.f, 0.f});
    return __builtin_bit_cast(u32, __builtin_convertvector(v, bf16x2));
}

DEV u32 bcnt(u32 a, u32 acc) { u32 r; asm("v_bcnt_u32_b32 %0, %1, %2" : "=v"(r) : "v"(a), "v"(acc)); return r; }
DEV u32 nz8(u32 cb) { return (cb + 0x07070707u) & 0x08080808u; }
DEV u32 pk_bf16(float a, float b) { return __builtin_bit_cast(u32, __builtin_convertvector((f32x2){a, b}, bf16x2)); }
DEV u32 pk_maxi(u32 a, u32 b) { return __builtin_bit_cast(u32, __builtin_elementwise_max(__builtin_bit_cast(i16x2, a), __builtin_bit_cast(i16x2, b))); }
#define ES(i) do {} while (0)
template <int NF>
DEV u32 encode_y(const Ctx& c, const u32 (&Q)[NF * 8], u32 dst) {
    constexpr int G = NF * 4;
    constexpr u32 CODEB = NF == 5 ? 2048 : 1536;
    const u32 ldsbase = (u32)(size_t)c.lds, stg = L_YST + c.wave * YST;
    ES(0);
    const u32 look = c.ctl[c.lane < 2 ? C_FREE + YS + c.lane : C_ABORT];
    u32 mx[NF];
    u16x2 mn[NF];
#pragma unroll
    for (int j = 0; j < 8; ++j)
#pragma unroll
        for (int f = 0; f < NF; ++f) {
            mx[f] = j ? pk_maxi(mx[f], Q[f * 8 + j]) : Q[f * 8];
            mn[f] = j ? __builtin_elementwise_min(mn[f], __builtin_bit_cast(u16x2, Q[f * 8 + j])) : __builtin_bit_cast(u16x2, Q[f * 8]);
        }
    u32 mnh[NF], mxa = 0;
#pragma unroll
    for (int f = 0; f < NF; ++f) {
        mnh[f] = min((u32)mn[f][0], (u32)mn[f][1]);
        asm volatile("" : "+v"(mnh[f]));
        mxa = pk_maxi(mxa, mx[f]);
    }
    const u32 base = max(wave_max(max(mxa & 0xffff, mxa >> 16)) >> 8, 7u) - 7, lim = ((base + 1) << 8) - 1;
    if (RARE(__builtin_amdgcn_readlane(look, 2))) return YM_BAD;
    if (RARE(min(rfl(look), (u32)__builtin_amdgcn_readlane(look, 1)) < 8 * (128 / NSTG)) &&
        !(lds_wait(c, C_FREE + YS, 8 * (128 / NSTG), true, 0) && lds_wait(c, C_FREE + YS + 1, 8 * (128 / NSTG), true, 0))) return YM_BAD;
    CBAR();
#pragma unroll
    for (int i = 0; i < 5; ++i) *(LAS u32x4*)(c.lds + stg + c.lane * 16 + min(i * 1024u, YST - 1024)) = (u32x4){0, 0, 0, 0};
    ES(1);
    u32 nexc = 0, hv = 0;
    u64 mb[NF];
#pragma unroll
    for (int f = 0; f < NF; ++f) mb[f] = __ballot(mnh[f] <= lim);
#pragma unroll
    for (int f = 0; f < NF; ++f) {
        u64 m = mb[f];
        if (!RARE(m)) continue;
        if (RARE(__builtin_popcountll(m) > 4)) {
            u16x2 e = {0xffff, 0xffff};
#pragma unroll
            for (int j = 0; j < 8; ++j) e = __builtin_elementwise_min(e, (u16x2)(__builtin_bit_cast(u16x2, Q[f * 8 + j]) - (u16x2){1, 1}));
            m = __ballot(min((u32)e[0], (u32)e[1]) < lim);
        }
        for (; m && nexc <= 31; m &= m - 1) {
            const u32 ln = __builtin_ctzll(m);
            u32 t = 0, r[8];
#pragma unroll
            for (int k = 0; k < 8; ++k) r[k] = __builtin_amdgcn_readlane(Q[f * 8 + k], ln);
            __builtin_amdgcn_sched_barrier(0);
#pragma unroll
            for (int k = 0; k < 8; ++k) t = writelane(r[k], k, t);
            for (u32 mk = (u32)__ballot((t & 0xffff) - 1 < lim) | (u32)__ballot((t >> 16) - 1 < lim) << 8; mk; mk &= mk - 1) {
                const u32 k = __builtin_ctz(mk), v = (__builtin_amdgcn_readlane(t, k & 7) >> (k & 8) * 2) & 0xffff;
                ++nexc;
                hv = writelane((f * 16 + (k & 7) * 2 + (k >> 3)) << 22 | ln << 16 | v, nexc & 63, hv);
            }
        }
    }
    if (FORCE_RAW_Y || RARE(nexc > 31)) {
#pragma unroll
        for (int i = 0; i < NF * 2; ++i) st16(c, (u32x4){pk_maxi(Q[4 * i], 0), pk_maxi(Q[4 * i + 1], 0), pk_maxi(Q[4 * i + 2], 0), pk_maxi(Q[4 * i + 3], 0)}, c.lane * 16, dst + i * 1024);
        return YM_RAW | NF * 128;
    }
    ES(2);
    hv = writelane(base | nexc << 8, 0, hv);
    if ((u32)c.lane <= nexc) *(LAS u32*)(c.lds + stg + c.lane * 4) = hv;
    u32 cb[G], lb[G], nz[G], cnt = 0;
#pragma unroll
    for (int g = 0; g < G; ++g) {
        const u32 t0 = pk_maxi(Q[2 * g], base * 0x01000100u), t1 = pk_maxi(Q[2 * g + 1], base * 0x01000100u);
        cb[g] = perm(t1, t0, 0x07050301u) - base * 0x01010101u; lb[g] = perm(t1, t0, 0x06040200u);
        nz[g] = nz8(cb[g]);
        cnt = bcnt(nz[g], cnt);
    }
    const u32 incl = wave_scan(cnt), total = 4 * (1 + nexc) + __builtin_amdgcn_readlane(incl, 63);
    u32 D[8];
#pragma unroll
    for (int h = 0; h < 2; ++h) {
        const u32* q = cb + 8 * h;
        D[3 * h] = (q[0] | q[1] << 3) | (q[6] & 0x03030303u) << 6;
        D[3 * h + 1] = (q[2] | q[3] << 3) | (q[7] & 0x03030303u) << 6;
        D[3 * h + 2] = (q[4] | q[5] << 3) | ((q[6] & 0x04040404u) | (q[7] & 0x04040404u) << 1) << 4;
    }
    if (NF == 5) { D[6] = cb[G - 4] | cb[G - 3] << 3; D[7] = cb[G - 2] | cb[G - 1] << 3; }
    ES(3);
    u32 pos = ldsbase + stg + 4 * (1 + nexc) + incl - cnt, adr[G], lut = ldsbase + L_LUT;
    asm volatile("" : "+v"(lut));
    u32x2 sel[G];
#pragma unroll
    for (int g = 0; g < G; ++g) {
        adr[g] = pos & ~3u;
        sel[g] = *(const LAS u32x2*)(size_t)__builtin_amdgcn_udot4((pos & 3) | nz[g], 0x20100408u, lut, false);
        pos = bcnt(nz[g], pos);
    }
    __builtin_amdgcn_sched_barrier(0);
    ES(4);
    u32 cur = 0;
#pragma unroll
    for (int g = 0; g < G; ++g) {
        const u32 o = perm(lb[g], cur, sel[g][0]);
        __hip_atomic_fetch_or((LAS u32*)(size_t)adr[g], o, __ATOMIC_RELAXED, __HIP_MEMORY_SCOPE_WAVEFRONT);
        cur = perm(lb[g], o, sel[g][1]);
    }
    __hip_atomic_fetch_or((LAS u32*)(size_t)(pos & ~3u), cur, __ATOMIC_RELAXED, __HIP_MEMORY_SCOPE_WAVEFRONT);
    CBAR();
    ES(5);
    u32x4 v[5];
#pragma unroll
    for (int i = 0; i < 5; ++i) if (i < 3 || RARE(total > i * 1024u)) v[i] = *(const LAS u32x4*)(c.lds + stg + c.lane * 16 + min(i * 1024u, YST - 1024));
    __builtin_amdgcn_sched_barrier(0);
    st16(c, (u32x4){D[0], D[1], D[2], D[3]}, c.lane * 16, dst);
    if (NF == 5) st16(c, (u32x4){D[4], D[5], D[6], D[7]}, c.lane * 16, dst + 1024);
    else __builtin_amdgcn_raw_buffer_store_b64((u32x2){D[4], D[5]}, c.rs, c.lane * 8 + dst + 1024, 0, 17);
#pragma unroll
    for (int i = 0; i < 5; ++i) if (i < 3 || RARE(total > i * 1024u)) st16(c, v[i], c.lane * 16, dst + CODEB + min(i * 1024u, YST - 1024));
    return (CODEB + total + 15) >> 4;
}

DEV f32x4 mfma(u32x4 a, u32x4 b, f32x4 acc) { return __builtin_amdgcn_mfma_f32_16x16x32_bf16(__builtin_bit_cast(bf16x8, a), __builtin_bit_cast(bf16x8, b), acc, 0, 0, 0); }
DEV void slot_free(const Ctx& c, u32 slot) { lds_add1(c, C_FREE + slot); }

template <bool BIG>
DEV void consumer(const Ctx& c, const Tile& t, int wr, int wc) {
    f32x4 acc[4][4], oacc[4];
#pragma unroll
    for (int i = 0; i < 4; ++i) {
        oacc[i] = (f32x4){0.f, 0.f, 0.f, 0.f};
#pragma unroll
        for (int f = 0; f < 4; ++f) acc[f][i] = (f32x4){0.f, 0.f, 0.f, 0.f};
    }
    const u32 call = get_call(c);
    const LAS char* pa = c.lds + c.lane * 16 + wr * 4096;
    const LAS char* po = c.lds + c.lane * 16 + wr * 1024;
    const LAS char* pb = c.lds + c.lane * 16 + wc * 4096;
    u32 slot = 0, seen = 0, nfull = 0;
    bool ok = true;
    struct Ops { u32x4 a[4], b[4], ao; u32 full; } X, Y;
    auto fetch = [&](Ops& o, u32 tn) __attribute__((always_inline)) {
        u32 sv = rfl(seen);
        if (RARE(sv < 2 * tn + 2) && ok && tn < 128) {
            if (CPRIO) __builtin_amdgcn_s_setprio(0);
            sv = lds_wait(c, C_FULL + slot, 2 * tn + 2, true, CNAP); ok = sv != 0;
            if (CPRIO) __builtin_amdgcn_s_setprio(CPRIO);
        }
        CBAR();
        if (tn == 64) STAMPW(c, 4);
        if (tn == 96) STAMPW(c, 5);
        if (tn == 112) { STAMPW(c, 6); STAMPC(c, 13); }
        if (tn == 120) STAMPW(c, 7);
        o.full = sv & 1;
        OS(0, tn);
        __builtin_amdgcn_sched_barrier(0);
#pragma unroll
        for (int i = 0; i < 4; ++i) o.b[i] = *(const LAS u32x4*)(pb + slot * STG + 16384 + i * 1024);
#pragma unroll
        for (int f = 0; f < 4; ++f) o.a[f] = *(const LAS u32x4*)(pa + slot * STG + f * 1024);
        if (BIG) o.ao = *(const LAS u32x4*)(po + slot * STG + 24576);
        seen = *(const LAS u32*)&c.ctl[C_FULL + (slot == NSTG - 1 ? 0 : slot + 1)];
    };
    auto release = [&](u32 tn) __attribute__((always_inline)) {
        __builtin_amdgcn_sched_barrier(0);
        WAIT_LGKM0();
        if (tn < 128) slot_free(c, slot);
        OS(1, cn);
        slot = slot == NSTG - 1 ? 0 : slot + 1;
    };
    auto step = [&](const Ops& o, Ops& nx, u32 tn) __attribute__((always_inline)) {
        fetch(nx, tn);
#pragma unroll
        for (int f = 0; f < 4; ++f)
#pragma unroll
            for (int i = 0; i < 4; ++i) acc[f][i] = mfma(o.a[f], o.b[i], acc[f][i]);
#pragma unroll
        for (int i = 0; i < (BIG ? 10 : 9); ++i) { __builtin_amdgcn_sched_group_barrier(0x8, 1, 0); __builtin_amdgcn_sched_group_barrier(0x100, 1, 0); }
        __builtin_amdgcn_sched_barrier(0);
        if (BIG && RARE(o.full)) {
            ++nfull;
#pragma unroll
            for (int i = 0; i < 4; ++i) oacc[i] = mfma(o.ao, o.b[i], oacc[i]);
        }
        release(tn);
    };
    if (CP2 == 1 && c.wave >= 4) __builtin_amdgcn_s_setprio(1);
    if (CP2 == 2 && c.wave < 4) __builtin_amdgcn_s_setprio(1);
    fetch(X, 0);
    STAMPW(c, 1); STAMPC(c, 12);
    release(0);
#pragma unroll 1
    for (u32 tk = 0; tk < 128; tk += 2) { step(X, Y, tk + 1); step(Y, X, tk + 2); }
    STAMPW(c, 2); STAMPC(c, 14);
    __builtin_amdgcn_s_setprio(EPRIO);
    {
        const u32 msg = (t.cc * 2 + t.h) * 8 + c.wave;
        u32 P[32];
#pragma unroll
        for (int g = 0; g < 16; ++g) {
            P[2 * g] = pk_bf16(acc[g >> 2][0][g & 3], acc[g >> 2][1][g & 3]);
            P[2 * g + 1] = pk_bf16(acc[g >> 2][2][g & 3], acc[g >> 2][3][g & 3]);
        }
        STAMPW(c, 8);
        u32 meta = YM_BAD;
        if (ok) meta = encode_y<4>(c, P, WS_YP + (t.owner * NMSG + msg) * YSLOT);
        ok = meta != YM_BAD;
        if (BIG && EFREE) { WAIT_LGKM0(); slot_free(c, YS); slot_free(c, YS + 1); }
        STAMPW(c, 9);
        WAIT_VM0();
        if (c.lane == t.owner) st_sys64(c.peer_lane + WS_FY + (c.rank * NMSG + msg) * 8, (u64)call << 32 | meta);
    }
    STAMPW(c, 3);
    if (BIG) {
        if (!EFREE) { slot_free(c, YS); slot_free(c, YS + 1); }
        slot = TK2 % NSTG; seen = 0;
        const u32 end = TK2 + 64 - (nfull >> 1);
        struct Own { u32x4 a[2], b[8]; } U, V;
        auto fetch2 = [&](Own& o, u32 tn) __attribute__((always_inline)) {
            if (RARE(rfl(seen) < 2 * tn + 2) && ok && tn < end) {
                if (CPRIO) __builtin_amdgcn_s_setprio(0);
                ok = lds_wait(c, C_FULL + slot, 2 * tn + 2, true, CNAP) != 0;
                if (CPRIO) __builtin_amdgcn_s_setprio(CPRIO);
            }
            CBAR();
            OS(0, tn - 2);
            __builtin_amdgcn_sched_barrier(0);
#pragma unroll
            for (int i = 0; i < 8; ++i) o.b[i] = *(const LAS u32x4*)(pb + slot * STG + 8192 + (i >> 2) * 8192 + (i & 3) * 1024);
#pragma unroll
            for (int f = 0; f < 2; ++f) o.a[f] = *(const LAS u32x4*)(po + slot * STG + f * 4096);
            seen = *(const LAS u32*)&c.ctl[C_FULL + (slot == NSTG - 1 ? 0 : slot + 1)];
        };
        auto step2 = [&](const Own& o, Own& nx, u32 tn) __attribute__((always_inline)) {
            fetch2(nx, tn);
#pragma unroll
            for (int i = 0; i < 8; ++i) oacc[i & 3] = mfma(o.a[i >> 2], o.b[i], oacc[i & 3]);
#pragma unroll
            for (int i = 0; i < 5; ++i) { __builtin_amdgcn_sched_group_barrier(0x100, 2, 0); __builtin_amdgcn_sched_group_barrier(0x8, 1, 0); }
            release(tn < end ? 0 : 128);
        };
        fetch2(U, TK2);
        release(TK2 < end ? 0 : 128);
        u32 tk = TK2;
#pragma unroll 1
        for (; tk + 1 < end; tk += 2) { step2(U, V, tk + 1); step2(V, U, tk + 2); }
        if (tk < end)
#pragma unroll
            for (int i = 0; i < 8; ++i) oacc[i & 3] = mfma(U.a[i >> 2], U.b[i], oacc[i & 3]);
        ok = ok && !rfl(c.ctl[C_ABORT]);
        const u32 o = out_off(c, ((t.rfo + wr) & 31) * 16, c.rank * 2048 + t.cc * 128 + wc * 64);
#pragma unroll
        for (int r = 0; r < 4; r += 2)
            st_out(c, o + r * (NTOT * 2), ok ? (u32x2){pk_relu(oacc[0][r], oacc[1][r]), pk_relu(oacc[2][r], oacc[3][r])} : (u32x2){0x7fc07fc0u, 0x7fc07fc0u},
                   ok ? (u32x2){pk_relu(oacc[0][r + 1], oacc[1][r + 1]), pk_relu(oacc[2][r + 1], oacc[3][r + 1])} : (u32x2){0x7fc07fc0u, 0x7fc07fc0u});
        STAMPW(c, 10);
    }
}

DEV void gemm_wg(const Ctx& c) {
    const int g = c.wg - NCOMM, xcd = g & 7, idx = g >> 3, grp = xcd >> 2, k = idx % 7;
    const bool big = k < 4;
    Tile t;
    t.cc = (xcd & 3) * 4 + idx / 7;
    t.h = grp * 4 + k >= 7 || k >= 4;
    t.owner = (c.rank + (grp ? (k < 3 ? 5 + k : 1 + k) : (k < 4 ? 1 + k : k - 3))) & 7;
    t.rf = t.owner * 32 + t.h * 16; t.rfo = c.rank * 32 + (grp * 4 + (k & 3)) * 4;
    if (c.wave >= 8) {
        const int L = c.wave - 8, J = (L + JROT) & 3;
        if (J) {
            wg_barrier();
            for (int i = 0; i < (J == 1 ? PS1 : J == 2 ? PS2 : PS3); ++i) __builtin_amdgcn_s_sleep(24);
        }
#pragma unroll 1
        for (int job = J * NGEMM + g; job < 1280; job += J < 2 ? 1280 : 2 * NGEMM) {
            if (job >= 4 * NGEMM) for (int i = 0; i < P2; ++i) __builtin_amdgcn_s_sleep(24);
            pack_job(c, job);
            if (job < 4 * NGEMM) STAMPW(c, 3);
        }
        STAMPW(c, 1);
        __builtin_amdgcn_s_setprio(LPRIO);
        const u32 call = get_call(c), xr = XROT ? (t.cc & 1) | (t.cc & 2) << 2 | (k & 3) << 1 | (t.cc & 3) << 4 : 0;
        if (L == NISS) { if (big) lister<true>(c, t, xr, call); else lister<false>(c, t, xr, call); }
        if (L < NISS || L4) { if (big) loader<true>(c, t, call); else loader<false>(c, t, call); }
        STAMPW(c, 2);
    } else {
        wg_barrier();
        if (big) consumer<true>(c, t, c.wave >> 1, c.wave & 1); else consumer<false>(c, t, c.wave >> 1, c.wave & 1);
    }
}

#define WSUM 8
#define EARLY 0
#define WCLV 4

constexpr u32 NFIX = 4;

DEV u32 take_seat(const Ctx& c, u32 call) {
    u32 xcc;
    asm volatile("s_getreg_b32 %0, hwreg(HW_REG_XCC_ID, 0, 4)" : "=s"(xcc));
    const u32 grp = (0xb2u >> c.rank) & 1 ? 0xb2u : 0x4du, j = (c.wg >> 3) * 4 + (xcc & 3);
    u32 near = TOPO ? (xcc & 4 ? ~grp & 0xff : grp & ~(1u << c.rank)) : 0, seat = NPULL;
    const u32 n = __builtin_popcount(near);
    if (!TOPO) seat = min((u32)c.wg, (u32)NPULL);
    else if (j < 4 * n) {
        for (u32 i = (j + SPREAD * (n == 4 ? j / n : 0)) % n; i; --i) near &= near - 1;
        seat = ((__builtin_ctz(near) - c.rank) & 7) - 1 + 7 * (j / n);
    }
    for (bool first = true;; first = false) {
        u64 old = ~0ull;
        if (c.lane == 0 ? seat < NPULL : first && c.lane < NR)
            old = __hip_atomic_fetch_max((GAS u64*)(c.me + (c.lane ? WS_CLM + c.lane * 128 : WS_SEAT + seat * 8)), (u64)call << 32 | 4 * NFIX, __ATOMIC_RELAXED, __HIP_MEMORY_SCOPE_AGENT);
        if (rfl((u32)(old >> 32)) < call) return seat;
        if (first) for (int i = 0; i < 4; ++i) __builtin_amdgcn_s_sleep(32);
        const u64 v = __hip_atomic_load((const GAS u64*)(c.me + WS_SEAT + min(c.lane, NPULL - 1) * 8), __ATOMIC_RELAXED, __HIP_MEMORY_SCOPE_AGENT);
        const u64 open = __ballot((u32)(v >> 32) < call);
        if (!open) return NPULL;
        seat = __builtin_ctzll(open);
    }
}

DEV void watcher(const Ctx& c, int p, int sub, u32 call) {
    static_assert(NQY == 64, "one message per lane");
    const u32 id = sub + 4 * c.lane;
    const char* fp = c.me + WS_FY + (p * NMSG + id) * 8;
    bool bad = !lds_wait(c, C_YGO, 1, false, 1);
    STAMPW(c, 1);
    u64 pending = ~0ull, t0 = 0;
    u32 qn = 0;
    for (u32 it = 1; pending; ++it) {
        const u64 v = bad ? 0 : ld_sys64(fp);
        const bool mine = ((pending >> c.lane) & 1) && (bad || (u32)(v >> 32) >= call);
        const u64 m = __ballot(mine);
        if (m) {
            const bool poison = bad || tmo_set(c, call);
            if (mine) c.ctl[C_Q + qn + lane_rank(m)] = 0x80000000u | id << 20 | (poison ? YM_BAD : (u32)v & (YM_BAD | YM_RAW | 0x3ff));
            qn += __popcll(m); pending &= ~m; t0 = 0;
            c.ctl[C_QN] = qn;
        } else {
            if ((it & 15) == 0) {
                if (tmo_set(c, call)) bad = true;
                else if (expired(t0, DEADLINE)) { tmo_raise(c, call); bad = true; }
            }
            __builtin_amdgcn_s_sleep(2);
        }
    }
    STAMPW(c, 2);
}

DEV void pull_inputs(const Ctx& c, char* peer, rsrc_t pr, u32 lo, int d, int sub, int p, u32 call) {
    GAS u64* ctr = (GAS u64*)(c.me + WS_CLM + d * 128);
    const u32 voff = c.lane * 16, win = WSUM / 4 + (sub < WSUM % 4), WCL = win + WCLV;
    bool bad = false;
#pragma unroll 1
    for (;;) {
        const u32 t = lds_inc(c, C_TK);
        if (t >= WCL) lds_wait(c, C_LAND, t - WCL + 1, false, 1);
        u32 n = t * 4 + sub;
        if (t >= NFIX) {
            if (c.lane == 0) n = (u32)__hip_atomic_fetch_add(ctr, 1ull, __ATOMIC_RELAXED, __HIP_MEMORY_SCOPE_AGENT);
            n = rfl(n);
        }
        if (n + 64 >= NPOS) c.ctl[C_YGO] = 1;
        if (n >= NPOS) break;
        const u32 code = rfl(ORDER.v[d - 1][n]), s = (code >> 8) & 7, ix = code & 31;
        const bool isw = code & 0x8000;
        const u32 rc = isw ? ix : ((((ix >> 2) + c.rank) & 7) << 2 | (ix & 3)), id = isw ? 256 + s * 16 + rc : s * 32 + rc;
        const char* fp = c.me + WS_FIN + (((call & 1) * NR + p) * NCHIN + id) * 8;
        u32 meta = 0;
        u64 t0 = 0;
        for (u32 it = 1; !bad; ++it) {
            const u64 v = ld_sys64(fp);
            meta = rfl((u32)v);
            if (rfl((u32)(v >> 32)) >= call) break;
            if ((it & 15) == 0) {
                if (tmo_set(c, call)) bad = true;
                else if (expired(t0, DEADLINE)) { tmo_raise(c, call); bad = true; }
            }
            __builtin_amdgcn_s_sleep(2);
        }
        const u32 u = lds_inc(c, C_IS);
        if (bad) { lds_inc(c, C_LAND); continue; }
        if (u >= win) lds_wait(c, C_LAND, u - win + 1, false, WNAP, 0xffff);
        __builtin_amdgcn_s_setprio(UPRIO);
        ST2(c, 1024 + u, (meta & 0x7ff) >> 3);
        const int ks = p * 8 + s;
        const u32 src = WS_PK + (call & 1) * PK_PAR + (isw ? id + c.rank * 128 : id) * CHB;
        const u32 d0 = isw ? wb_off(wb_ks(ks, call), 0, rc * 8) : xa_off(ks, 0, rc * 8), d1 = d0 + (isw ? 128 : 256) * 1024;
        u32x4 R[16];
        if (meta & YM_RAW) {
#pragma unroll
            for (int f = 0; f < 16; ++f) R[f] = ld16(pr, voff + (f & 3) * 1024, src + (f >> 2) * 4096);
            WAIT_VM0();
            lds_inc(c, C_LAND);
            ST2(c, 1536 + u, 0); ST2(c, u, 0);
            store_raw(c, R, d0, d1);
        } else {
            const u32 bytes = min((meta & 0x7ff) * 16, PK_TAIL + LSLOT), tl = (u32)(size_t)c.lds + lo;
            const rsrc_t tr = __builtin_amdgcn_make_buffer_rsrc(peer, 0, rfl(src + bytes), 0x00020000);
            if (bytes > PK_TAIL + 2048) glds4(tr, voff, src + PK_TAIL + 2048, tl + 2048);
            glds2(tr, voff, src + PK_TAIL, tl);
            u32x4 L[8], D[2];
#pragma unroll
            for (int i = 0; i < 2; ++i) D[i] = ld16(pr, voff + i * 1024, src + 8192);
#pragma unroll
            for (int i = 0; i < 8; ++i) L[i] = ld16(pr, voff + (i & 3) * 1024, src + (i >> 2) * 4096);
            asm volatile("s_waitcnt vmcnt(%0)" :: "n"(EARLY) : "memory");
            if (c.lane == 0) __hip_atomic_fetch_add((LAS u32*)&c.ctl[C_LAND], 1u, __ATOMIC_RELAXED, __HIP_MEMORY_SCOPE_WORKGROUP);
            WAIT_VM0();
            ST2(c, 1536 + u, 0); ST2(c, u, 0);
            const u32 hv = *(const LAS u32*)(size_t)(tl + (c.lane & 31) * 4);
            u32 cnt = 0;
#pragma unroll
            for (int j = 0; j < 8; ++j) cnt += __builtin_popcount(~(D[j >> 2][j & 3] | D[j >> 2][j & 3] >> 1) & 0x55555555u);
            const u32 h0 = rfl(hv), T = (h0 & 0xff) * 0x01010101u, nexc = min((h0 >> 8) & 0xff, 31u);
            u32 pb = 32 * (1 + nexc) + 4 * (wave_scan(cnt) - cnt);
#pragma unroll
            for (int hf = 0; hf < 16; hf += 8) {
                u32 cd[16], sel[16], sa[8], ps[8];
                u32x2 q[8];
#pragma unroll
                for (int f = 0; f < 8; ++f) {
                    const u32 w = D[hf >> 3][f >> 1] >> (f & 1) * 4;
                    cd[2 * f] = w & 0x03030303u; cd[2 * f + 1] = (w >> 2) & 0x03030303u;
                    const u32 ea = (0x04040404u - cd[2 * f]) & 0x04040404u, eb = (0x04040404u - cd[2 * f + 1]) & 0x04040404u;
                    sel[2 * f] = *(const LAS u32*)(size_t)__builtin_amdgcn_udot4(ea, 0x08040201u, (u32)(size_t)c.lds + L_LUT2, false);
                    sel[2 * f + 1] = *(const LAS u32*)(size_t)__builtin_amdgcn_udot4(eb, 0x08040201u, (u32)(size_t)c.lds + L_LUT2, false);
                    const LAS u32* z = (const LAS u32*)(size_t)(tl + (pb >> 5) * 4);
                    q[f] = (u32x2){z[0], z[1]};
                    sa[f] = sad(ea, 0); ps[f] = pb;
                    pb = sad(eb, pb + sa[f]);
                }
                __builtin_amdgcn_sched_barrier(0);
#pragma unroll
                for (int f = 0; f < 8; ++f) {
                    const u32 va = __builtin_amdgcn_alignbit(q[f][1], q[f][0], ps[f]), vb = va >> sa[f];
                    const u32 ha = T - ((perm(va >> 4, va, sel[2 * f]) & 0x0f0f0f0fu) | cd[2 * f]), hb = T - ((perm(vb >> 4, vb, sel[2 * f + 1]) & 0x0f0f0f0fu) | cd[2 * f + 1]);
                    const u32 la = L[(hf + f) >> 1][(f & 1) * 2], lb = L[(hf + f) >> 1][(f & 1) * 2 + 1];
                    const u32x4 x = {perm(ha, la, 0x05010400u), perm(ha, la, 0x07030602u), perm(hb, lb, 0x05010400u), perm(hb, lb, 0x07030602u)};
                    u32x4 r;
#pragma unroll
                    for (int i = 0; i < 4; ++i) r[i] = __builtin_amdgcn_alignbit(x[i], x[i], 1);
                    st16(c, r, voff, (hf ? d1 : d0) + f * 1024);
                }
            }
            if ((u32)c.lane - 1 < nexc)
                __builtin_amdgcn_raw_buffer_store_b16((u16)hv, c.rs, (hv >> 28 & 1 ? d1 : d0) + (hv >> 25 & 7) * 1024 + (hv >> 16 & 511) * 2, 0, 17);
        }
        STAMPW(c, 3);
        ST2(c, 128 + u, 0);
        WAIT_VM0();
        if (c.lane == 0) st_sys(c.me + (isw ? WS_WR : WS_XR) + rc * 256 + ks * 4, call);
        __builtin_amdgcn_s_setprio(0);
        ST2(c, 512 + u, 0);
    }
    if (c.lane == 0) __hip_atomic_fetch_or((LAS u32*)&c.ctl[C_LAND], 1u << 16, __ATOMIC_RELAXED, __HIP_MEMORY_SCOPE_WORKGROUP);
}

template <int NF>
DEV void decode_y(const Ctx& c, u32 e, u32 lo, rsrc_t pr, u32 src, u32 row0, u32 col0, u32 tk) {
    constexpr int G = NF * 4;
    constexpr u32 CODEB = NF == 5 ? 2048 : 1536;
    const u32 ldsbase = (u32)(size_t)c.lds;
    const LAS char* l = c.lds + lo;
    u32x2 P[G];
    u32 hv = 0, nexc = 0;
    if (e & YM_BAD) {
#pragma unroll
        for (int g = 0; g < G; ++g) P[g] = (u32x2){0x7fc07fc0u, 0x7fc07fc0u};
    } else if (e & YM_RAW) {
#pragma unroll
        for (int i = 0; i < G / 2; ++i) {
            const u32x4 v = ld16(pr, c.lane * 16 + (i & 3) * 1024, src + (i >> 2) * 4096);
            P[2 * i] = (u32x2){v[0], v[1]}; P[2 * i + 1] = (u32x2){v[2], v[3]};
        }
        __builtin_amdgcn_s_waitcnt(0x0f70);
    } else {
        const u32x4 da = *(const LAS u32x4*)(l + c.lane * 16);
        u32x4 db;
        if (NF == 5) db = *(const LAS u32x4*)(l + 1024 + c.lane * 16);
        else { const u32x2 t = *(const LAS u32x2*)(l + 1024 + c.lane * 8); db = (u32x4){t[0], t[1], 0, 0}; }
        hv = *(const LAS u32*)(l + CODEB + (c.lane & 31) * 4);
        const u32 D[8] = {da[0], da[1], da[2], da[3], db[0], db[1], db[2], db[3]};
        u32 cb[G], cnt = 0;
#pragma unroll
        for (int h = 0; h < 2; ++h) {
            u32* q = cb + 8 * h;
#pragma unroll
            for (int k = 0; k < 3; ++k) { q[2 * k] = D[3 * h + k] & 0x07070707u; q[2 * k + 1] = (D[3 * h + k] >> 3) & 0x07070707u; }
            q[6] = bfi(0x03030303u, D[3 * h] >> 6, D[3 * h + 2] >> 4) & 0x07070707u;
            q[7] = bfi(0x03030303u, D[3 * h + 1] >> 6, D[3 * h + 2] >> 5) & 0x07070707u;
        }
        if (NF == 5) { cb[G - 4] = D[6] & 0x07070707u; cb[G - 3] = (D[6] >> 3) & 0x07070707u; cb[G - 2] = D[7] & 0x07070707u; cb[G - 1] = (D[7] >> 3) & 0x07070707u; }
#pragma unroll
        for (int g = 0; g < G; ++g) cnt += __builtin_popcount(nz8(cb[g]));
        const u32 h0 = rfl(hv), B = (h0 & 0xff) * 0x01010101u;
        nexc = min((h0 >> 8) & 0xff, 31u);
        u32 pos = ldsbase + lo + CODEB + 4 * (1 + nexc) + wave_scan(cnt) - cnt, ps[G];
        u32x2 sel[G];
#pragma unroll
        for (int g = 0; g < G; ++g) {
            const u32 nz = nz8(cb[g]);
            const LAS u32* q = (const LAS u32*)(size_t)(pos & ~3u);
            sel[g] = *(const LAS u32x2*)(size_t)__builtin_amdgcn_udot4(nz, 0x08040201u, ldsbase + L_LUT, false);
            P[g] = (u32x2){q[0], q[1]};
            ps[g] = pos;
            pos += __builtin_popcount(nz);
        }
        __builtin_amdgcn_sched_barrier(0);
#pragma unroll
        for (int g = 0; g < G; ++g) {
            const u32 by = __builtin_amdgcn_alignbyte(P[g][1], P[g][0], ps[g]), lb = perm(by, by, sel[g][0]), hb = (cb[g] + B) & sel[g][1];
            P[g] = (u32x2){perm(hb, lb, 0x05010400u), perm(hb, lb, 0x07030602u)};
        }
    }
#pragma unroll
    for (int g = 0; g < G; g += 2) st_out(c, out_off(c, row0, col0) + ((g >> 2) * 16 + (g & 3)) * (NTOT * 2), P[g], P[g + 1]);
    ST2(c, 512 + 384 + tk, nexc);
    const u32 j = (hv >> 22) & 63, ln = hv >> 16, xo = (row0 + (j & 48) + (ln & 48) / 4 + (j >> 2 & 3)) * (NTOT * 2) + (col0 + (ln & 7) * 8 + (ln & 8) / 2 + (j & 3)) * 2;
    __builtin_amdgcn_raw_buffer_store_b16((u16)hv, c.ro, (u32)c.lane - 1 < nexc ? xo : ~1u, 0, OAUX);
}
DEV void fetch_y(const Ctx& c, char* peer, u32 e, u32 lo) {
    if (e & (YM_BAD | YM_RAW)) return;
    const u32 src = WS_YP + (c.rank * NMSG + ((e >> 20) & 0xff)) * YSLOT, bytes = min((e & 0x3ff) * 16, LSLOT);
    const rsrc_t rs = __builtin_amdgcn_make_buffer_rsrc(peer, 0, src + bytes, 0x00020000);
    glds4(rs, c.lane * 16, src, (u32)(size_t)c.lds + lo);
    if (bytes > 4096) glds2(rs, c.lane * 16, src + 4096, (u32)(size_t)c.lds + lo + 4096);
}

DEV void worker(const Ctx& c, char* peer, int d, int sub, int p, int w, u32 call) {
    const rsrc_t pr = __builtin_amdgcn_make_buffer_rsrc(peer, 0, WS_END, 0x00020000);
    pull_inputs(c, peer, pr, w * 2 * LSLOT, d, sub, p, call);
    STAMPW(c, 1);
    const u32 both = (w * 4 + 1) * LSLOT;
    static_assert(C_QN == C_YT + 1 && C_YL == C_YT + 2, "one read");
    u32 lo = w * 2 * LSLOT, e = 0, tk = 0;
#pragma unroll 1
    for (bool first = true;; first = false) {
        u32 tn = ~0u, en = 0;
        if (!first) {
            const u32 st = c.ctl[C_YT + (c.lane & 3)], yt = __builtin_amdgcn_readlane(st, 0);
            if (YPRE && yt < (u32)__builtin_amdgcn_readlane(st, 1) && yt + YPEND <= NQY && yt < (u32)__builtin_amdgcn_readlane(st, 2) + YWIN) {
                tn = lds_inc(c, C_YT);
                if (tn < NQY) en = rfl(c.ctl[C_Q + tn]);
                if (en) { ST2(c, 1024 + 448 + tn, 5); fetch_y(c, peer, en, both - lo); }
            }
            const u32 id = (e >> 20) & 0xff, wc = id & 1, rg = (id >> 1) & 7, cc = id >> 4;
            decode_y<4>(c, DIAG & 1 ? e | YM_BAD : e, lo, pr, WS_YP + (c.rank * NMSG + id) * YSLOT, rg * 64, p * 2048 + cc * 128 + wc * 64, tk);
            ST2(c, 512 + 448 + tk, 0);
        }
        if (en) asm volatile("s_waitcnt vmcnt(9)" ::: "memory");
        else {
            if (tn == ~0u) tn = lds_inc(c, C_YT);
            if (tn >= NQY) break;
            en = lds_wait(c, C_Q + tn, 1, false, 1);
            if (!en) { tmo_raise(c, call); break; }
            if (tn + 1 > YWIN) lds_wait(c, C_YL, tn + 1 - YWIN, false, 0);
            ST2(c, 1024 + 448 + tn, 5);
            fetch_y(c, peer, en, both - lo);
            WAIT_VM0();
        }
        lds_add1(c, C_YL);
        ST2(c, 1536 + 448 + tn, 0);
        if (tn == 0) STAMPW(c, 4);
        STAMPW(c, 5);
        e = en; tk = tn; lo = both - lo;
    }
    STAMPW(c, 2);
}

__global__ void __launch_bounds__(NTHREADS, 3) fused_kernel(Args a) {
    extern __shared__ __attribute__((aligned(128))) char smem[];
    Ctx c;
    c.x = a.x; c.w = a.w; c.out = a.out; c.rank = a.rank & 7; c.me = a.me;
    c.wg = blockIdx.x; c.wave = rfl(threadIdx.x >> 6); c.lane = threadIdx.x & 63;
    c.peer_lane = a.ws[c.lane & 7];
    c.rs = __builtin_amdgcn_make_buffer_rsrc(c.me, 0, WS_END, 0x00020000);
    c.ro = __builtin_amdgcn_make_buffer_rsrc(c.out, 0, DIAG & 2 ? 0 : 512 * NTOT * 2, 0x00020000);
    c.lds = (LAS char*)smem; c.ctl = (volatile LAS u32*)(c.lds + L_CTL);
    STAMPW(c, 0);
    u32 init = 0;
    static_assert(C_CALL == 0, "thread 0");
    if (threadIdx.x == 0) {
        GAS u32* cnt = (GAS u32*)(c.me + WS_CNT + c.wg * 128);
        init = __hip_atomic_load(cnt, __ATOMIC_RELAXED, __HIP_MEMORY_SCOPE_AGENT) + 1;
        __hip_atomic_store(cnt, init, __ATOMIC_RELAXED, __HIP_MEMORY_SCOPE_AGENT);
    }
    if (threadIdx.x < C_WORDS) c.ctl[threadIdx.x] = init;
    if (c.wave == 1) {
        u32 s0, s1;
        if (c.wg >= NCOMM) {
            const u32 mask = (c.lane >> 2 & 12) | (c.lane >> 3 & 1) | (c.lane >> 1 & 2);
            u32 n = c.lane & 3;
            u64 sel = 0x0c0c0c0c03020100ull;
            for (int b = 0; b < 4; ++b)
                if ((mask >> b) & 1) { sel = (sel & ~(0xffull << (8 * n))) | (u64)(4 + b) << (8 * n); ++n; }
            s0 = (u32)sel; s1 = n >= 4 ? (u32)(sel >> 32) : 0x03020100u;
        } else {
            u32 n = 0;
            s0 = 0x0c0c0c0cu; s1 = 0;
            for (int b = 0; b < 4; ++b)
                if ((c.lane >> b) & 1) { s0 = (s0 & ~(0xffu << (8 * b))) | n << (8 * b); s1 |= 0xffu << (8 * b); ++n; }
        }
        c.ctl[C_LUT + 2 * c.lane] = s0; c.ctl[C_LUT + 2 * c.lane + 1] = s1;
    }
    if (c.wave == 2 && c.lane < 16) {
        u32 n = 0, s0 = 0x0c0c0c0cu, s1 = 0x0c0c0c0cu, sd = 0x0c0c0c0cu;
        for (int b = 0; b < 4; ++b)
            if ((c.lane >> b) & 1) {
                u32& t = n & 1 ? s1 : s0;
                t = (t & ~(0xffu << (n >> 1) * 8)) | b << (n >> 1) * 8;
                sd = (sd & ~(0xffu << 8 * b)) | ((n & 1) * 4 + (n >> 1)) << 8 * b;
                ++n;
            }
        if (c.wg >= NCOMM) { c.ctl[C_LUT2 + 2 * c.lane] = s0; c.ctl[C_LUT2 + 2 * c.lane + 1] = s1; }
        else c.ctl[C_LUT2 + c.lane] = sd;
    }
    if (c.wg >= NCOMM) gemm_wg(c);
    else {
        wg_barrier();
        const u32 call = get_call(c);
        if (c.wave == 0) c.ctl[C_SEAT] = take_seat(c, call) + 1;
        const u32 seat = lds_wait(c, C_SEAT, 1, false, 1) - 1;
        if (seat >= NPULL) return;
        const int d = 1 + seat % 7, sub = seat / 7, p = (c.rank + d) & 7;
        if (c.wave == 0) watcher(c, p, sub, call);
        else worker(c, a.ws[p], d, sub, p, c.wave - 1, call);
    }
}

}

extern "C" void kernel_launch(int rank, int nranks, ncclComm_t comm, hipStream_t stream,
                              void* const* d_in, const int* in_sizes, int n_in,
                              void* d_out, int out_size, const int* in_dtypes, int out_dtype,
                              void* const* d_out_all, void* const* d_ws_all, size_t ws_bytes) {
    if (nranks != NR || (unsigned)rank >= (unsigned)NR || n_in != 2 || ws_bytes < WS_END || in_sizes[0] != 4096 * KSH ||
        in_sizes[1] != KSH * NTOT || out_size != 512 * NTOT) {
        fprintf(stderr, "unexpected problem shape\n");
        return;
    }
    HIP_CHECK(hipFuncSetAttribute((const void*)fused_kernel, hipFuncAttributeMaxDynamicSharedMemorySize, LDS_BYTES));
    Args a;
    a.x = (const u16*)d_in[0]; a.w = (const u16*)d_in[1]; a.out = (u16*)d_out;
    for (int p = 0; p < NR; ++p) a.ws[p] = (char*)d_ws_all[p];
    a.me = a.ws[rank]; a.rank = rank; a.pad = 0;
    hipLaunchKernelGGL(fused_kernel, dim3(NWG), dim3(NTHREADS), LDS_BYTES, stream, a);
    HIP_CHECK(hipGetLastError());
}
```

```cpp
#include <hip/hip_runtime.h>
#include <rccl/rccl.h>
#include <cstdio>
#include <cstdint>

#define HIP_CHECK(expr) do { hipError_t _e = (expr); if (_e != hipSuccess) { \
    fprintf(stderr, "HIP error %s at %s:%d\n", hipGetErrorString(_e), __FILE__, __LINE__); \
    return; } } while (0)

extern "C" __device__ unsigned writelane(unsigned value, unsigned lane, unsigned old) __asm("llvm.amdgcn.writelane.i32");

namespace {

typedef unsigned u32;
typedef unsigned long long u64;
typedef unsigned short u16;
typedef u32 u32x4 __attribute__((ext_vector_type(4)));
typedef u32 u32x2 __attribute__((ext_vector_type(2)));
typedef u32 u32x8 __attribute__((ext_vector_type(8)));
typedef u32 u32x32 __attribute__((ext_vector_type(32)));
typedef u16 u16x2 __attribute__((ext_vector_type(2)));
typedef short i16x2 __attribute__((ext_vector_type(2)));
typedef float f32x4 __attribute__((ext_vector_type(4)));
typedef float f32x2 __attribute__((ext_vector_type(2)));
typedef __bf16 bf16x2 __attribute__((ext_vector_type(2)));
typedef __bf16 bf16x8 __attribute__((ext_vector_type(8)));
typedef __amdgpu_buffer_rsrc_t rsrc_t;
#define LAS __attribute__((address_space(3)))
#define GAS __attribute__((address_space(1)))
#define DEV __device__ __forceinline__

#define PSLEEP 3
#define LSLEEP 4
#define SPREAD 0
#define PSTEP 1
#define PS1 3
#define PS2 4
#define PS3 5
#define P2 0
#define L4 1
#define PACE 27
#define PFREE 3
#define NPUSH 12
#define JROT 2
#define YPEND 12
#define YWIN 4
#define RELF 3
#define LNAP 1
#define LPRIO 0
#define MAXP 5
#define OWNALL 0
#define UPRIO 2
#define EPS 0
#define EFREE 1
#define YPRE 0
#define CP2 0
#define PKL 1
#define EPRIO 2
#define WNAP 0
#define CNAP 0
#define CPRIO 0
#define OAUX 17
#define TOPO 1
#define DIAG 0
#define XROT 1
#define FORCE_RAW_Y 0
#define FORCE_RAW_IN 0

constexpr int NR = 8;
constexpr int NTOT = 16384, KSH = 512;
constexpr int NWG = 256, NCOMM = 32, NPULL = 28, NGEMM = NWG - NCOMM, NTHREADS = 768;
constexpr int NCHIN = 384;
constexpr int NMSG = 256;
constexpr u32 NQY = NMSG / 4;
constexpr int NWORK = 11;

constexpr u32 WS_TMO = 128, WS_CLM = 1024;
constexpr u32 WS_SEAT = 2048;
constexpr u32 WS_FIN = 4096;
constexpr u32 WS_FY = WS_FIN + 2 * NR * NCHIN * 8;
constexpr u32 WS_XR = WS_FY + NR * NMSG * 8;
constexpr u32 WS_WR = WS_XR + 8192;
constexpr u32 WS_CNT = WS_WR + 4096;
constexpr u32 WS_DBG = 131072;
static_assert(WS_CNT + NWG * 128 <= WS_DBG, "ws");
#define CHBV 16384
constexpr u32 CHB = CHBV;
constexpr u32 WS_PK = 1u << 20;
constexpr u32 PK_PAR = 1280u * CHB;
constexpr u32 WS_XA = WS_PK + 2 * PK_PAR;
constexpr u32 WS_WB = WS_XA + (32u << 20);
#define YSL 8192
constexpr u32 YSLOT = YSL;
constexpr u32 WS_YP = WS_WB + (20u << 20);
constexpr u32 WS_END = WS_YP + 8u * NMSG * YSLOT;
constexpr u32 PK_TAIL = 8192 + 2048;

constexpr u32 STG = 28672, NSTG = 5;
constexpr u32 LSLOT = 6144;
constexpr u32 L_CTL = STG * NSTG;
static_assert(NWORK * 2 * LSLOT + 8192 <= L_CTL, "ring");
constexpr u32 C_CALL = 0, C_ABORT = 1, C_FULL = 8, C_FREE = 16, C_LE = 24, C_LIST = 64,
              C_TK = 2, C_IS = 3, C_LAND = 4, C_YGO = 5, C_YT = 6, C_QN = 7, C_YL = 8, C_SEAT = 9, C_Q = 64, C_WORDS = 256,
              C_LUT = C_WORDS;
constexpr u32 C_LUT2 = C_LUT + 128, L_PST = L_CTL + (C_LUT2 + 32) * 4, PST = 4352;
constexpr u32 LDS_BYTES = L_PST + 4 * PST;
#define ST2(c, i, v) do {} while (0)
constexpr u32 YS = 128 % NSTG, L_LUT = L_CTL + C_LUT * 4, L_LUT2 = L_CTL + C_LUT2 * 4, L_YST = YS * STG, YST = 4608;
static_assert(L_LUT % 512 == 0 && YS + 1 < NSTG && 8 * YST <= 2 * STG && 4 * (32 + 64 * 17) <= YST && 1536 + 4 * (32 + 64 * 17) <= LSLOT, "y staging");
static_assert(LDS_BYTES <= 160 * 1024, "lds");

constexpr u64 DEADLINE = 40000000ull;
constexpr u64 DEADLINE_LDS = 100000000ull;

constexpr u32 YM_RAW = 1u << 16, YM_BAD = 1u << 17;

constexpr int NPOS = NCHIN - NPUSH;
static_assert(NPUSH <= 24, "push");
struct Order { u16 v[7][NPOS]; };
#define D1V 6
#define D2V 2
#define D3V 0
#define SQV 2
#define SQA 2
constexpr Order make_order() {
    Order o{};
    for (int d = 1; d <= 7; ++d) {
        int n = 0;
        for (int pass = 0; pass < 4; ++pass)
            for (int s = 0; s < 8; ++s) {
                if (pass == (s >= 8 - D1V ? 3 : 0) && s) for (int q = 0; q < 4; ++q) o.v[d - 1][n++] = (u16)(s << 8 | q);
                const bool sq = s >= 8 - SQV;
                for (int c = 0; c < 16 && !sq; ++c) if (pass == ((c & 2) && s >= 8 - D3V ? 1 : 0) && s * 16 + c >= NPUSH) o.v[d - 1][n++] = (u16)(0x8000 | s << 8 | c);
                if (pass == (s >= 8 - D1V ? 3 : 0) && !s) for (int q = 0; q < 4; ++q) o.v[d - 1][n++] = (u16)(s << 8 | q);
                for (int j = 0; j < 8 && sq && !pass; ++j)
                    for (int i = 0; i < 4; ++i)
                        o.v[d - 1][n++] = (u16)(i < 2 ? 0x8000 | s << 8 | (2 * (j + SQA * d) + i) % 16 : j < 7 ? s << 8 | (1 + (j + d) % 7) << 2 | (i & 1) : s << 8 | 4 << 2 | i);
                for (int i = 0; i < 28; ++i) {
                    const int q = (i / 14) * 2 + (i & 1), own = 1 + ((i % 14) / 2 + d) % 7;
                    if (sq && (q < 2 || own == 4)) continue;
                    if (pass == (q >= 2 && own != 4 && s >= 8 - D2V ? 2 : 0)) o.v[d - 1][n++] = (u16)(s << 8 | own << 2 | q);
                }
            }
    }
    return o;
}
__constant__ const Order ORDER = make_order();

struct Args { const u16* x; const u16* w; u16* out; char* me; int rank; int pad; char* ws[NR]; };

#define RARE(x) __builtin_expect(!!(x), 0)
DEV u64 rt() { return __builtin_amdgcn_s_memrealtime(); }
DEV u32 rfl(u32 v) { return __builtin_amdgcn_readfirstlane(v); }
DEV u32 perm(u32 hi, u32 lo, u32 sel) { return __builtin_amdgcn_perm(hi, lo, sel); }
DEV u32 bfi(u32 m, u32 a, u32 b) { return (m & a) | (~m & b); }
#define CBAR() asm volatile("" ::: "memory")
#define WAIT_VM0() asm volatile("s_waitcnt vmcnt(0)" ::: "memory")
#define WAIT_LGKM0() do { CBAR(); __builtin_amdgcn_s_waitcnt(0xc07f); CBAR(); } while (0)

#define STAMPW(c, i) do {} while (0)
#define STAMPC(c, i) do {} while (0)

struct Ctx {
    const u16* x; const u16* w; u16* out; char* me; char* peer_lane;
    rsrc_t rs, ro;
    LAS char* lds; volatile LAS u32* ctl;
    int rank, wg, wave, lane;
};

DEV u32 ld_sys(const void* p) { return __hip_atomic_load((const GAS u32*)p, __ATOMIC_RELAXED, __HIP_MEMORY_SCOPE_SYSTEM); }
DEV u64 ld_sys64(const void* p) { return __hip_atomic_load((const GAS u64*)p, __ATOMIC_RELAXED, __HIP_MEMORY_SCOPE_SYSTEM); }
DEV void st_sys(void* p, u32 v) { __hip_atomic_store((GAS u32*)p, v, __ATOMIC_RELAXED, __HIP_MEMORY_SCOPE_SYSTEM); }
DEV void st_sys64(void* p, u64 v) { __hip_atomic_store((GAS u64*)p, v, __ATOMIC_RELAXED, __HIP_MEMORY_SCOPE_SYSTEM); }
DEV void st16(const Ctx& c, u32x4 v, u32 voff, u32 soff) { __builtin_amdgcn_raw_buffer_store_b128(v, c.rs, voff + soff, 0, 17); }
DEV void st_out(const Ctx& c, u32 off, u32x2 a, u32x2 b) {
    u32x4 v;
#pragma unroll
    for (int i = 0; i < 2; ++i) {
        v[i] = __builtin_amdgcn_update_dpp(a[i], b[i], 0x128, 0xf, 0xc, false);
        v[2 + i] = __builtin_amdgcn_update_dpp(b[i], a[i], 0x128, 0xf, 0x3, false);
    }
    __builtin_amdgcn_raw_buffer_store_b128(v, c.ro, off, 0, OAUX);
}
DEV u32 out_off(const Ctx& c, u32 row0, u32 col0) { return (row0 + (c.lane >> 4) * 4 + (c.lane >> 3 & 1)) * (NTOT * 2) + (col0 + (c.lane & 7) * 8) * 2; }
DEV bool tmo_set(const Ctx& c, u32 call) { return rfl(__hip_atomic_load((const GAS u32*)(c.me + WS_TMO), __ATOMIC_RELAXED, __HIP_MEMORY_SCOPE_AGENT)) == call; }
DEV void tmo_raise(const Ctx& c, u32 call) { if (c.lane == 0) __hip_atomic_store((GAS u32*)(c.me + WS_TMO), call, __ATOMIC_RELAXED, __HIP_MEMORY_SCOPE_AGENT); }

DEV bool expired(u64& t0, u64 limit) { const u64 t = rt(); if (!t0) t0 = t; return t - t0 > limit; }

DEV u32 lds_wait(const Ctx& c, u32 word, u32 want, bool look_abort, int nap, u32 mask = ~0u) {
    u64 t0 = 0;
    for (u32 n = 1;; ++n) {
        const u32 v = rfl(c.ctl[word]) & mask;
        if (v >= want) return v;
        if (nap) __builtin_amdgcn_s_sleep(1);
        if ((n & 63) == 0) {
            if (look_abort && rfl(c.ctl[C_ABORT])) return 0;
            if (expired(t0, DEADLINE_LDS)) { c.ctl[C_ABORT] = 1; return 0; }
        }
    }
}
DEV u32 get_call(const Ctx& c) { return rfl(c.ctl[C_CALL]); }

DEV void glds4(rsrc_t rs, u32 voff, u32 soff, u32 ldsaddr) {
    u32 keep;
    asm volatile("s_mov_b32 %0, m0\n\ts_mov_b32 m0, %4\n\ts_nop 2\n\t"
                 "buffer_load_dwordx4 %1, %2, %3 offen sc0 sc1 lds\n\t"
                 "buffer_load_dwordx4 %1, %2, %3 offen offset:1024 sc0 sc1 lds\n\t"
                 "buffer_load_dwordx4 %1, %2, %3 offen offset:2048 sc0 sc1 lds\n\t"
                 "buffer_load_dwordx4 %1, %2, %3 offen offset:3072 sc0 sc1 lds\n\t"
                 "s_mov_b32 m0, %0" : "=&s"(keep) : "v"(voff), "s"(rs), "s"(soff), "s"(ldsaddr) : "memory");
}
#define GPOL " sc0 sc1"
DEV void glds4g(rsrc_t rs, u32 voff, u32 soff, u32 ldsaddr) {
    u32 keep;
    if (DIAG & 4) return;
    if (DIAG & 8) soff = WS_XA + (soff & 0x7ffff);
    asm volatile("s_mov_b32 %0, m0\n\ts_mov_b32 m0, %4\n\ts_nop 2\n\t"
                 "buffer_load_dwordx4 %1, %2, %3 offen" GPOL " lds\n\t"
                 "buffer_load_dwordx4 %1, %2, %3 offen offset:1024" GPOL " lds\n\t"
                 "buffer_load_dwordx4 %1, %2, %3 offen offset:2048" GPOL " lds\n\t"
                 "buffer_load_dwordx4 %1, %2, %3 offen offset:3072" GPOL " lds\n\t"
                 "s_mov_b32 m0, %0" : "=&s"(keep) : "v"(voff), "s"(rs), "s"(soff), "s"(ldsaddr) : "memory");
}
DEV void glds2(rsrc_t rs, u32 voff, u32 soff, u32 ldsaddr) {
    u32 keep;
    asm volatile("s_mov_b32 %0, m0\n\ts_mov_b32 m0, %4\n\ts_nop 2\n\t"
                 "buffer_load_dwordx4 %1, %2, %3 offen sc0 sc1 lds\n\t"
                 "buffer_load_dwordx4 %1, %2, %3 offen offset:1024 sc0 sc1 lds\n\t"
                 "s_mov_b32 m0, %0" : "=&s"(keep) : "v"(voff), "s"(rs), "s"(soff), "s"(ldsaddr) : "memory");
}
DEV u32x4 ld16(rsrc_t rs, u32 voff, u32 soff) { return __builtin_amdgcn_raw_buffer_load_b128(rs, voff, soff, 17); }
DEV u32 lds_inc(const Ctx& c, u32 word) {
    u32 v = 0;
    if (c.lane == 0) v = __hip_atomic_fetch_add((LAS u32*)&c.ctl[word], 1u, __ATOMIC_RELAXED, __HIP_MEMORY_SCOPE_WORKGROUP);
    return rfl(v);
}

DEV u32 wave_max(u32 v) {
    v = max(v, (u32)__builtin_amdgcn_update_dpp(0, (int)v, 0x111, 0xf, 0xf, true));
    v = max(v, (u32)__builtin_amdgcn_update_dpp(0, (int)v, 0x112, 0xf, 0xf, true));
    v = max(v, (u32)__builtin_amdgcn_update_dpp(0, (int)v, 0x114, 0xf, 0xf, true));
    v = max(v, (u32)__builtin_amdgcn_update_dpp(0, (int)v, 0x118, 0xf, 0xf, true));
    v = max(v, (u32)__builtin_amdgcn_update_dpp(0, (int)v, 0x142, 0xa, 0xf, false));
    v = max(v, (u32)__builtin_amdgcn_update_dpp(0, (int)v, 0x143, 0xc, 0xf, false));
    return __builtin_amdgcn_readlane(v, 63);
}

DEV u32 wave_scan(u32 v) {
    v += (u32)__builtin_amdgcn_update_dpp(0, (int)v, 0x111, 0xf, 0xf, true);
    v += (u32)__builtin_amdgcn_update_dpp(0, (int)v, 0x112, 0xf, 0xf, true);
    v += (u32)__builtin_amdgcn_update_dpp(0, (int)v, 0x114, 0xf, 0xf, true);
    v += (u32)__builtin_amdgcn_update_dpp(0, (int)v, 0x118, 0xf, 0xf, true);
    v += (u32)__builtin_amdgcn_update_dpp(0, (int)v, 0x142, 0xa, 0xf, false);
    v += (u32)__builtin_amdgcn_update_dpp(0, (int)v, 0x143, 0xc, 0xf, false);
    return v;
}
DEV u32 lane_rank(u64 m) { return __builtin_amdgcn_mbcnt_hi((u32)(m >> 32), __builtin_amdgcn_mbcnt_lo((u32)m, 0)); }

DEV void store_raw(const Ctx& c, const u32x4 (&R)[16], u32 off0, u32 off1) {
#pragma unroll
    for (int f = 0; f < 16; ++f) st16(c, R[f], c.lane * 16, (f < 8 ? off0 : off1) + (f & 7) * 1024);
}
DEV u32 sad(u32 a, u32 acc) { return __builtin_amdgcn_sad_u8(a, 0, acc); }
DEV u32 pk_sub(u32 a, u32 b) { return __builtin_bit_cast(u32, (u16x2)(__builtin_bit_cast(u16x2, a) - __builtin_bit_cast(u16x2, b))); }
DEV u32 pack_store(const Ctx& c, u32x4 (&X)[16], u32 dst) {
    const u32 ldsbase = (u32)(size_t)c.lds, stg = L_PST + (c.wave & 3) * PST;
#pragma unroll
    for (int i = 0; i < 5; ++i) *(LAS u32x4*)(c.lds + stg + c.lane * 16 + min(i * 1024u, PST - 1024)) = (u32x4){0, 0, 0, 0};
    u32 sum = 0;
#pragma unroll
    for (int f = 0; f < 16; ++f)
#pragma unroll
        for (int i = 0; i < 4; ++i) X[f][i] = __builtin_amdgcn_alignbit(X[f][i], X[f][i], 31);
#pragma unroll
    for (int f = 0; f < 16; ++f) asm volatile("" : "+v"(X[f]));
#pragma unroll
    for (int f = 0; f < 16; f += 2) sum = sad(perm(X[f][1], X[f][0], 0x07050301u), sum);
    const u32 top = min(max(((__builtin_amdgcn_readlane(wave_scan(sum), 63) + 1946) >> 11) + 2, 12u), 255u), K = (top << 8 | 0xff) * 0x10001u;
    u32 nexc = 0, hv = 0, D[8], W[16], rel[16], bits = 0;
#pragma unroll
    for (int j = 0; j < 8; ++j) {
        u32 t[4], cf[4], es[4];
        u32x2 sel[4];
#pragma unroll
        for (int g = 0; g < 4; ++g) {
            const u32x4 x = X[2 * j + (g >> 1)];
            t[g] = perm(pk_sub(K, x[(g & 1) * 2 + 1]), pk_sub(K, x[(g & 1) * 2]), 0x07050301u);
            cf[g] = perm(0, 0x07060500u, t[g]);
            es[g] = ~cf[g] & 0x04040404u;
            sel[g] = *(const LAS u32x2*)(size_t)__builtin_amdgcn_udot4(es[g], 0x10080402u, ldsbase + L_LUT2, false);
        }
        D[j] = bfi(0x03030303u, cf[0], bfi(0x0c0c0c0cu, cf[1] << 2, bfi(0x30303030u, cf[2] << 4, cf[3] << 6)));
#pragma unroll
        for (int h = 0; h < 2; ++h) {
            const int f = 2 * j + h;
            u64 m = __ballot((cf[2 * h] | cf[2 * h + 1]) & 0x80808080u);
            if (RARE(m)) for (; m && nexc <= 31; m &= m - 1) {
                const u32 ln = __builtin_ctzll(m);
                const u32 r0 = __builtin_amdgcn_readlane(X[f][0], ln), r1 = __builtin_amdgcn_readlane(X[f][1], ln);
                const u32 r2 = __builtin_amdgcn_readlane(X[f][2], ln), r3 = __builtin_amdgcn_readlane(X[f][3], ln);
                for (u32 fl = (__builtin_amdgcn_readlane(cf[2 * h], ln) & 0x80808080u) >> 1 | (__builtin_amdgcn_readlane(cf[2 * h + 1], ln) & 0x80808080u); fl; fl &= fl - 1) {
                    const u32 b = __builtin_ctz(fl), x = b & 1 ? (b & 16 ? r3 : r2) : (b & 16 ? r1 : r0), r = x >> 1 | x << 31;
                    ++nexc;
                    hv = writelane((f << 9 | ln << 3 | (b & 1) << 2 | b >> 3) << 16 | ((r >> (b & 8) * 2) & 0xffff), nexc & 63, hv);
                }
            }
            const u32 sa = sad(es[2 * h], 0);
            const u32 wa = perm(0, t[2 * h], sel[2 * h][0]) | perm(0, t[2 * h], sel[2 * h][1]) << 4;
            const u32 wb = perm(0, t[2 * h + 1], sel[2 * h + 1][0]) | perm(0, t[2 * h + 1], sel[2 * h + 1][1]) << 4;
            W[f] = wa | wb << sa;
            rel[f] = bits;
            bits = sad(es[2 * h + 1], bits + sa);
            asm volatile("" : "+v"(W[f]), "+v"(bits));
        }
        asm volatile("" : "+v"(D[j]));
    }
    if (FORCE_RAW_IN || RARE(nexc > 31)) {
#pragma unroll
        for (int f = 0; f < 16; ++f)
#pragma unroll
            for (int i = 0; i < 4; ++i) X[f][i] = __builtin_amdgcn_alignbit(X[f][i], X[f][i], 1);
        store_raw(c, X, dst, dst + 8192);
        return YM_RAW | 1024;
    }
#pragma unroll
    for (int i = 0; i < 8; ++i)
        st16(c, (u32x4){perm(X[2 * i][1], X[2 * i][0], 0x06040200u), perm(X[2 * i][3], X[2 * i][2], 0x06040200u),
                        perm(X[2 * i + 1][1], X[2 * i + 1][0], 0x06040200u), perm(X[2 * i + 1][3], X[2 * i + 1][2], 0x06040200u)}, c.lane * 16, dst + i * 1024);
    st16(c, (u32x4){D[0], D[1], D[2], D[3]}, c.lane * 16, dst + 8192);
    st16(c, (u32x4){D[4], D[5], D[6], D[7]}, c.lane * 16, dst + 9216);
    hv = writelane(top | nexc << 8, 0, hv);
    if ((u32)c.lane <= nexc) *(LAS u32*)(c.lds + stg + c.lane * 4) = hv;
    const u32 incl = wave_scan(bits), start = 32 * (1 + nexc) + incl - bits, bytes = 4 * (1 + nexc) + ((__builtin_amdgcn_readlane(incl, 63) + 7) >> 3);
#pragma unroll
    for (int f = 0; f < 16; ++f) {
        const u32 pb = start + rel[f];
        const u64 v = (u64)W[f] << (pb & 31);
        LAS u32* q = (LAS u32*)(size_t)(ldsbase + stg + (pb >> 5) * 4);
        __hip_atomic_fetch_or(q, (u32)v, __ATOMIC_RELAXED, __HIP_MEMORY_SCOPE_WAVEFRONT);
        __hip_atomic_fetch_or(q + 1, (u32)(v >> 32), __ATOMIC_RELAXED, __HIP_MEMORY_SCOPE_WAVEFRONT);
    }
    CBAR();
    u32x4 v[5];
#pragma unroll
    for (int i = 0; i < 5; ++i) if (i < 2 || RARE(bytes > i * 1024u)) v[i] = *(const LAS u32x4*)(c.lds + stg + c.lane * 16 + min(i * 1024u, PST - 1024));
#pragma unroll
    for (int i = 0; i < 5; ++i) if (i < 2 || RARE(bytes > i * 1024u)) st16(c, v[i], c.lane * 16, dst + PK_TAIL + min(i * 1024u, PST - 1024));
    return (PK_TAIL + bytes + 15) >> 4;
}

DEV u32 xa_off(int ks, int h, int rowfrag) { return WS_XA + (u32)((ks * 2 + h) * 256 + rowfrag) * 1024u; }
DEV u32 wb_off(int ks, int h, int colfrag) { return WS_WB + (u32)((ks * 2 + h) * 128 + colfrag) * 1024u; }

DEV int wb_ks(int ks, u32 call) { return (call & 1) && (ks & 7) < 2 ? 64 + (ks >> 3) * 2 + (ks & 7) : ks; }
DEV void wg_barrier() { WAIT_LGKM0(); __builtin_amdgcn_s_barrier(); CBAR(); }

DEV void pack_job(const Ctx& c, int job) {
    const int s = job / 160, r0 = job % 160, q = s != 1 || r0 >= 96 ? r0 : r0 < 64 ? r0 + 32 : r0 - 64, ks = c.rank * 8 + s;
    const int l15 = c.lane & 15, lq = c.lane >> 4;
    const bool isx = q < 32;
    const int dst = (c.rank + 1 + (q - 32)) & 7, cc = (q - 32) >> 3;
    u32x4 R[16];
    if (job < NGEMM) STAMPW(c, 13);
    if (isx) {
        const GAS char* src = (const GAS char*)c.x + (size_t)(q * 128 + l15) * (KSH * 2) + (s * 64 + lq * 8) * 2;
#pragma unroll
        for (int f = 0; f < 16; ++f) R[f] = *(const GAS u32x4*)(src + (f & 7) * 16 * (KSH * 2) + (f >> 3) * 64);
    } else {
        const GAS char* src = (const GAS char*)c.w + (size_t)(s * 64 + lq * 8) * (NTOT * 2) + (dst * 2048 + cc * 128 + (l15 & 7) * 8 + (l15 >> 3) * 4) * 2;
        u32x2 r[4][8];
#pragma unroll
        for (int hg = 0; hg < 4; ++hg)
#pragma unroll
            for (int j = 0; j < 8; ++j) r[hg][j] = *(const GAS u32x2*)(src + (size_t)((hg >> 1) * 32 + j) * (NTOT * 2) + (hg & 1) * 128);
#pragma unroll
        for (int hg = 0; hg < 4; ++hg)
#pragma unroll
            for (int ii = 0; ii < 4; ++ii)
#pragma unroll
                for (int t = 0; t < 4; ++t)
                    R[hg * 4 + ii][t] = perm(r[hg][2 * t + 1][ii >> 1], r[hg][2 * t][ii >> 1], (ii & 1) ? 0x07060302u : 0x05040100u);
    }
    if (job < NGEMM) wg_barrier();
    __builtin_amdgcn_s_waitcnt(0x0f70);
    if (job < NGEMM) STAMPW(c, 4);
    const u32 call = get_call(c), ch = isx ? s * 32 + q : 256 + s * 16 + cc;
    const u32 slot = WS_PK + (call & 1) * PK_PAR + (isx ? ch : ch + dst * 128) * CHB;
    const bool done = !isx && (dst == c.rank || ch < 256 + NPUSH);
    if (done && dst != c.rank && ch > 256 + PFREE)
        for (const u64 t0 = rt(); rt() - t0 < (ch - 256 - PFREE) * PACE;) __builtin_amdgcn_s_sleep(2);
    if (!done) {
        const u32 meta = pack_store(c, R, slot);
        if (job < NGEMM) STAMPW(c, 5);
        WAIT_VM0();
        if (c.lane < NR && c.lane != c.rank && (isx || c.lane == dst)) st_sys64(c.peer_lane + WS_FIN + (((call & 1) * NR + c.rank) * NCHIN + ch) * 8, (u64)call << 32 | meta);
        if (!isx) return;
        if (!(meta & YM_RAW))
#pragma unroll
            for (int f = 0; f < 16; ++f)
#pragma unroll
                for (int i = 0; i < 4; ++i) R[f][i] = __builtin_amdgcn_alignbit(R[f][i], R[f][i], 1);
    }
    const u64 base = (u64)c.peer_lane;
    const u32 to = isx ? c.rank : dst, d0 = isx ? xa_off(ks, 0, q * 8) : wb_off(wb_ks(ks, call), 0, cc * 8);
    Ctx e = c;
    e.rs = __builtin_amdgcn_make_buffer_rsrc((char*)((u64)(u32)__builtin_amdgcn_readlane((u32)(base >> 32), to) << 32 | (u32)__builtin_amdgcn_readlane((u32)base, to)), 0, WS_END, 0x00020000);
    store_raw(e, R, d0, d0 + (isx ? 256 : 128) * 1024);
    if (done && job < NGEMM) STAMPW(c, 5);
    WAIT_VM0();
    if ((u32)c.lane == to) st_sys(c.peer_lane + (isx ? WS_XR + q * 256 : WS_WR + cc * 256) + ks * 4, call);
}

struct Tile { int cc, owner, h, rf, rfo; };
constexpr u32 TK2 = 130;

constexpr u32 LE_FULL = 64, LE_OK = 128, LE_END = 256, NISS = 3;
DEV void lds_add1(const Ctx& c, u32 word) {
    asm volatile("s_mov_b64 exec, 1\n\tds_add_u32 %0, %1\n\ts_mov_b64 exec, -1" :: "v"((u32)(size_t)&c.ctl[word]), "v"(1u) : "memory");
}
template <bool BIG>
DEV void lister(const Ctx& c, const Tile& t, u32 xr, u32 call) {
    const u32 ks = c.lane ^ xr;
    const char* fx = c.me + WS_XR + (t.rf >> 3) * 256 + ks * 4;
    const char* fo = c.me + WS_XR + (t.rfo >> 3) * 256 + ks * 4;
    const char* fw = c.me + WS_WR + t.cc * 256 + ks * 4;
    u64 todo = ~0ull, late = 0, t0 = 0;
    u32 n = 0;
    bool second = false;
    for (u32 it = 1;; ++it) {
        const u32 o = BIG ? ld_sys(fo) : 0;
        u64 m = 0;
        if (!second) { const u32 a = ld_sys(fx), b = ld_sys(fx + 256), w = ld_sys(fw); m = __ballot(min(min(a, b), w) >= call); }
        const u64 own = __ballot(o >= call);
        m = (second ? own : m) & todo;
        if (m) {
            if (m == todo && tmo_set(c, call)) break;
            if ((m >> c.lane) & 1) c.ctl[C_LIST + n + lane_rank(m)] = LE_OK | (BIG && o >= call ? LE_FULL : 0) | ks;
            n += __popcll(m); todo &= ~m; late |= m & ~own; t0 = 0;
            if (!todo) {
                if (!BIG || second || !late) { c.ctl[C_LIST + n + (c.lane & 3)] = LE_END; return; }
                second = true; todo = late;
            }
        } else {
            if ((it & 15) == 0) {
                if (rfl(c.ctl[C_ABORT]) || tmo_set(c, call)) break;
                if (expired(t0, DEADLINE)) { tmo_raise(c, call); break; }
            }
            __builtin_amdgcn_s_sleep(LSLEEP);
        }
    }
    c.ctl[C_ABORT] = 1;
}
#define OS(i, n) do {} while (0)
template <bool BIG>
DEV void loader(const Ctx& c, const Tile& t, u32 call) {
    const u32 ldsbase = (u32)(size_t)c.lds, voff = c.lane * 16;
    u32 pend = 0, pslot = 0;
#pragma unroll 1
    for (;;) {
        const u32 n = lds_inc(c, C_LE), h = n & 1;
        const bool second = n >= 128;
        const u32 e = second ? n - 64 : n >> 1, tt = second ? n + (TK2 - 128) : n, slot = tt % NSTG, need = 8 * (tt / NSTG);
        const u32 look = c.ctl[c.lane ? C_LIST + e : C_FREE + slot];
        u32 v = __builtin_amdgcn_readlane(look, 1);
        if (!v || rfl(look) < need) {
            if (pend) { WAIT_VM0(); c.ctl[C_FULL + pslot] = pend; pend = 0; OS(2, pn); }
            if (!v) v = lds_wait(c, C_LIST + e, 1, true, LNAP);
            if (need && (v & LE_OK) && !lds_wait(c, C_FREE + slot, need, true, LNAP)) break;
        }
        if (!(v & LE_OK)) break;
        const int ks = v & 63, kw = wb_ks(ks, call);
        const u32 full = BIG && !second && (v & LE_FULL), l = ldsbase + slot * STG;
        OS(0, n);
        if (BIG && second) {
            glds4g(c.rs, voff, xa_off(ks, 0, t.rfo), l); glds4g(c.rs, voff, xa_off(ks, 1, t.rfo), l + 4096);
#pragma unroll
            for (int g = 0; g < 4; ++g) glds4g(c.rs, voff, wb_off(kw, g >> 1, t.cc * 8 + (g & 1) * 4), l + 8192 + g * 4096);
        } else {
#pragma unroll
            for (int g = 0; g < 4; ++g) glds4g(c.rs, voff, xa_off(ks, h, t.rf + g * 4), l + g * 4096);
            glds4g(c.rs, voff, wb_off(kw, h, t.cc * 8), l + 16384); glds4g(c.rs, voff, wb_off(kw, h, t.cc * 8 + 4), l + 20480);
        }
        OS(1, n);
        if (full) {
            glds4g(c.rs, voff, xa_off(ks, h, t.rfo), l + 24576);
            asm volatile("s_waitcnt vmcnt(28)" ::: "memory");
        } else asm volatile("s_waitcnt vmcnt(24)" ::: "memory");
        if (pend) { c.ctl[C_FULL + pslot] = pend; OS(2, pn); }
        pend = 2 * tt + 2 + full; pslot = slot;
    }
    WAIT_VM0();
    if (pend) { c.ctl[C_FULL + pslot] = pend; OS(2, pn); }
}

DEV u32 pk_relu(float a, float b) {
    const f32x2 v = __builtin_elementwise_maximum((f32x2){a, b}, (f32x2){0.f, 0.f});
    return __builtin_bit_cast(u32, __builtin_convertvector(v, bf16x2));
}

DEV u32 bcnt(u32 a, u32 acc) { u32 r; asm("v_bcnt_u32_b32 %0, %1, %2" : "=v"(r) : "v"(a), "v"(acc)); return r; }
DEV u32 nz8(u32 cb) { return (cb + 0x07070707u) & 0x08080808u; }
DEV u32 pk_bf16(float a, float b) { return __builtin_bit_cast(u32, __builtin_convertvector((f32x2){a, b}, bf16x2)); }
DEV u32 pk_maxi(u32 a, u32 b) { return __builtin_bit_cast(u32, __builtin_elementwise_max(__builtin_bit_cast(i16x2, a), __builtin_bit_cast(i16x2, b))); }
#define ES(i) do {} while (0)
template <int NF>
DEV u32 encode_y(const Ctx& c, const u32 (&Q)[NF * 8], u32 dst) {
    constexpr int G = NF * 4;
    constexpr u32 CODEB = NF == 5 ? 2048 : 1536;
    const u32 ldsbase = (u32)(size_t)c.lds, stg = L_YST + c.wave * YST;
    ES(0);
    const u32 look = c.ctl[c.lane < 2 ? C_FREE + YS + c.lane : C_ABORT];
    u32 mx[NF];
    u16x2 mn[NF];
#pragma unroll
    for (int j = 0; j < 8; ++j)
#pragma unroll
        for (int f = 0; f < NF; ++f) {
            mx[f] = j ? pk_maxi(mx[f], Q[f * 8 + j]) : Q[f * 8];
            mn[f] = j ? __builtin_elementwise_min(mn[f], __builtin_bit_cast(u16x2, Q[f * 8 + j])) : __builtin_bit_cast(u16x2, Q[f * 8]);
        }
    u32 mnh[NF], mxa = 0;
#pragma unroll
    for (int f = 0; f < NF; ++f) {
        mnh[f] = min((u32)mn[f][0], (u32)mn[f][1]);
        asm volatile("" : "+v"(mnh[f]));
        mxa = pk_maxi(mxa, mx[f]);
    }
    const u32 base = max(wave_max(max(mxa & 0xffff, mxa >> 16)) >> 8, 7u) - 7, lim = ((base + 1) << 8) - 1;
    if (RARE(__builtin_amdgcn_readlane(look, 2))) return YM_BAD;
    if (RARE(min(rfl(look), (u32)__builtin_amdgcn_readlane(look, 1)) < 8 * (128 / NSTG)) &&
        !(lds_wait(c, C_FREE + YS, 8 * (128 / NSTG), true, 0) && lds_wait(c, C_FREE + YS + 1, 8 * (128 / NSTG), true, 0))) return YM_BAD;
    CBAR();
#pragma unroll
    for (int i = 0; i < 5; ++i) *(LAS u32x4*)(c.lds + stg + c.lane * 16 + min(i * 1024u, YST - 1024)) = (u32x4){0, 0, 0, 0};
    ES(1);
    u32 nexc = 0, hv = 0;
    u64 mb[NF];
#pragma unroll
    for (int f = 0; f < NF; ++f) mb[f] = __ballot(mnh[f] <= lim);
#pragma unroll
    for (int f = 0; f < NF; ++f) {
        u64 m = mb[f];
        if (!RARE(m)) continue;
        if (RARE(__builtin_popcountll(m) > 4)) {
            u16x2 e = {0xffff, 0xffff};
#pragma unroll
            for (int j = 0; j < 8; ++j) e = __builtin_elementwise_min(e, (u16x2)(__builtin_bit_cast(u16x2, Q[f * 8 + j]) - (u16x2){1, 1}));
            m = __ballot(min((u32)e[0], (u32)e[1]) < lim);
        }
        for (; m && nexc <= 31; m &= m - 1) {
            const u32 ln = __builtin_ctzll(m);
            u32 t = 0, r[8];
#pragma unroll
            for (int k = 0; k < 8; ++k) r[k] = __builtin_amdgcn_readlane(Q[f * 8 + k], ln);
            __builtin_amdgcn_sched_barrier(0);
#pragma unroll
            for (int k = 0; k < 8; ++k) t = writelane(r[k], k, t);
            for (u32 mk = (u32)__ballot((t & 0xffff) - 1 < lim) | (u32)__ballot((t >> 16) - 1 < lim) << 8; mk; mk &= mk - 1) {
                const u32 k = __builtin_ctz(mk), v = (__builtin_amdgcn_readlane(t, k & 7) >> (k & 8) * 2) & 0xffff;
                ++nexc;
                hv = writelane((f * 16 + (k & 7) * 2 + (k >> 3)) << 22 | ln << 16 | v, nexc & 63, hv);
            }
        }
    }
    if (FORCE_RAW_Y || RARE(nexc > 31)) {
#pragma unroll
        for (int i = 0; i < NF * 2; ++i) st16(c, (u32x4){pk_maxi(Q[4 * i], 0), pk_maxi(Q[4 * i + 1], 0), pk_maxi(Q[4 * i + 2], 0), pk_maxi(Q[4 * i + 3], 0)}, c.lane * 16, dst + i * 1024);
        return YM_RAW | NF * 128;
    }
    ES(2);
    hv = writelane(base | nexc << 8, 0, hv);
    if ((u32)c.lane <= nexc) *(LAS u32*)(c.lds + stg + c.lane * 4) = hv;
    u32 cb[G], lb[G], nz[G], cnt = 0;
#pragma unroll
    for (int g = 0; g < G; ++g) {
        const u32 t0 = pk_maxi(Q[2 * g], base * 0x01000100u), t1 = pk_maxi(Q[2 * g + 1], base * 0x01000100u);
        cb[g] = perm(t1, t0, 0x07050301u) - base * 0x01010101u; lb[g] = perm(t1, t0, 0x06040200u);
        nz[g] = nz8(cb[g]);
        cnt = bcnt(nz[g], cnt);
    }
    const u32 incl = wave_scan(cnt), total = 4 * (1 + nexc) + __builtin_amdgcn_readlane(incl, 63);
    u32 D[8];
#pragma unroll
    for (int h = 0; h < 2; ++h) {
        const u32* q = cb + 8 * h;
        D[3 * h] = (q[0] | q[1] << 3) | (q[6] & 0x03030303u) << 6;
        D[3 * h + 1] = (q[2] | q[3] << 3) | (q[7] & 0x03030303u) << 6;
        D[3 * h + 2] = (q[4] | q[5] << 3) | ((q[6] & 0x04040404u) | (q[7] & 0x04040404u) << 1) << 4;
    }
    if (NF == 5) { D[6] = cb[G - 4] | cb[G - 3] << 3; D[7] = cb[G - 2] | cb[G - 1] << 3; }
    ES(3);
    u32 pos = ldsbase + stg + 4 * (1 + nexc) + incl - cnt, adr[G], lut = ldsbase + L_LUT;
    asm volatile("" : "+v"(lut));
    u32x2 sel[G];
#pragma unroll
    for (int g = 0; g < G; ++g) {
        adr[g] = pos & ~3u;
        sel[g] = *(const LAS u32x2*)(size_t)__builtin_amdgcn_udot4((pos & 3) | nz[g], 0x20100408u, lut, false);
        pos = bcnt(nz[g], pos);
    }
    __builtin_amdgcn_sched_barrier(0);
    ES(4);
    u32 cur = 0;
#pragma unroll
    for (int g = 0; g < G; ++g) {
        const u32 o = perm(lb[g], cur, sel[g][0]);
        __hip_atomic_fetch_or((LAS u32*)(size_t)adr[g], o, __ATOMIC_RELAXED, __HIP_MEMORY_SCOPE_WAVEFRONT);
        cur = perm(lb[g], o, sel[g][1]);
    }
    __hip_atomic_fetch_or((LAS u32*)(size_t)(pos & ~3u), cur, __ATOMIC_RELAXED, __HIP_MEMORY_SCOPE_WAVEFRONT);
    CBAR();
    ES(5);
    u32x4 v[5];
#pragma unroll
    for (int i = 0; i < 5; ++i) if (i < 3 || RARE(total > i * 1024u)) v[i] = *(const LAS u32x4*)(c.lds + stg + c.lane * 16 + min(i * 1024u, YST - 1024));
    __builtin_amdgcn_sched_barrier(0);
    st16(c, (u32x4){D[0], D[1], D[2], D[3]}, c.lane * 16, dst);
    if (NF == 5) st16(c, (u32x4){D[4], D[5], D[6], D[7]}, c.lane * 16, dst + 1024);
    else __builtin_amdgcn_raw_buffer_store_b64((u32x2){D[4], D[5]}, c.rs, c.lane * 8 + dst + 1024, 0, 17);
#pragma unroll
    for (int i = 0; i < 5; ++i) if (i < 3 || RARE(total > i * 1024u)) st16(c, v[i], c.lane * 16, dst + CODEB + min(i * 1024u, YST - 1024));
    return (CODEB + total + 15) >> 4;
}

DEV f32x4 mfma(u32x4 a, u32x4 b, f32x4 acc) { return __builtin_amdgcn_mfma_f32_16x16x32_bf16(__builtin_bit_cast(bf16x8, a), __builtin_bit_cast(bf16x8, b), acc, 0, 0, 0); }
DEV void slot_free(const Ctx& c, u32 slot) { lds_add1(c, C_FREE + slot); }

template <bool BIG>
DEV void consumer(const Ctx& c, const Tile& t, int wr, int wc) {
    f32x4 acc[4][4], oacc[4];
#pragma unroll
    for (int i = 0; i < 4; ++i) {
        oacc[i] = (f32x4){0.f, 0.f, 0.f, 0.f};
#pragma unroll
        for (int f = 0; f < 4; ++f) acc[f][i] = (f32x4){0.f, 0.f, 0.f, 0.f};
    }
    const u32 call = get_call(c);
    const LAS char* pa = c.lds + c.lane * 16 + wr * 4096;
    const LAS char* po = c.lds + c.lane * 16 + wr * 1024;
    const LAS char* pb = c.lds + c.lane * 16 + wc * 4096;
    u32 slot = 0, seen = 0, nfull = 0;
    bool ok = true;
    struct Ops { u32x4 a[4], b[4], ao; u32 full; } X, Y;
    auto fetch = [&](Ops& o, u32 tn) __attribute__((always_inline)) {
        u32 sv = rfl(seen);
        if (RARE(sv < 2 * tn + 2) && ok && tn < 128) {
            if (CPRIO) __builtin_amdgcn_s_setprio(0);
            sv = lds_wait(c, C_FULL + slot, 2 * tn + 2, true, CNAP); ok = sv != 0;
            if (CPRIO) __builtin_amdgcn_s_setprio(CPRIO);
        }
        CBAR();
        if (tn == 64) STAMPW(c, 4);
        if (tn == 96) STAMPW(c, 5);
        if (tn == 112) { STAMPW(c, 6); STAMPC(c, 13); }
        if (tn == 120) STAMPW(c, 7);
        o.full = sv & 1;
        OS(0, tn);
        __builtin_amdgcn_sched_barrier(0);
#pragma unroll
        for (int i = 0; i < 4; ++i) o.b[i] = *(const LAS u32x4*)(pb + slot * STG + 16384 + i * 1024);
#pragma unroll
        for (int f = 0; f < 4; ++f) o.a[f] = *(const LAS u32x4*)(pa + slot * STG + f * 1024);
        if (BIG) o.ao = *(const LAS u32x4*)(po + slot * STG + 24576);
        seen = *(const LAS u32*)&c.ctl[C_FULL + (slot == NSTG - 1 ? 0 : slot + 1)];
    };
    auto release = [&](u32 tn) __attribute__((always_inline)) {
        __builtin_amdgcn_sched_barrier(0);
        WAIT_LGKM0();
        if (tn < 128) slot_free(c, slot);
        OS(1, cn);
        slot = slot == NSTG - 1 ? 0 : slot + 1;
    };
    auto step = [&](const Ops& o, Ops& nx, u32 tn) __attribute__((always_inline)) {
        fetch(nx, tn);
#pragma unroll
        for (int f = 0; f < 4; ++f)
#pragma unroll
            for (int i = 0; i < 4; ++i) acc[f][i] = mfma(o.a[f], o.b[i], acc[f][i]);
#pragma unroll
        for (int i = 0; i < (BIG ? 10 : 9); ++i) { __builtin_amdgcn_sched_group_barrier(0x8, 1, 0); __builtin_amdgcn_sched_group_barrier(0x100, 1, 0); }
        __builtin_amdgcn_sched_barrier(0);
        if (BIG && RARE(o.full)) {
            ++nfull;
#pragma unroll
            for (int i = 0; i < 4; ++i) oacc[i] = mfma(o.ao, o.b[i], oacc[i]);
        }
        release(tn);
    };
    if (CP2 == 1 && c.wave >= 4) __builtin_amdgcn_s_setprio(1);
    if (CP2 == 2 && c.wave < 4) __builtin_amdgcn_s_setprio(1);
    fetch(X, 0);
    STAMPW(c, 1); STAMPC(c, 12);
    release(0);
#pragma unroll 1
    for (u32 tk = 0; tk < 128; tk += 2) { step(X, Y, tk + 1); step(Y, X, tk + 2); }
    STAMPW(c, 2); STAMPC(c, 14);
    __builtin_amdgcn_s_setprio(EPRIO);
    {
        const u32 msg = (t.cc * 2 + t.h) * 8 + c.wave;
        u32 P[32];
#pragma unroll
        for (int g = 0; g < 16; ++g) {
            P[2 * g] = pk_bf16(acc[g >> 2][0][g & 3], acc[g >> 2][1][g & 3]);
            P[2 * g + 1] = pk_bf16(acc[g >> 2][2][g & 3], acc[g >> 2][3][g & 3]);
        }
        STAMPW(c, 8);
        u32 meta = YM_BAD;
        if (ok) meta = encode_y<4>(c, P, WS_YP + (t.owner * NMSG + msg) * YSLOT);
        ok = meta != YM_BAD;
        if (BIG && EFREE) { WAIT_LGKM0(); slot_free(c, YS); slot_free(c, YS + 1); }
        STAMPW(c, 9);
        WAIT_VM0();
        if (c.lane == t.owner) st_sys64(c.peer_lane + WS_FY + (c.rank * NMSG + msg) * 8, (u64)call << 32 | meta);
    }
    STAMPW(c, 3);
    if (BIG) {
        if (!EFREE) { slot_free(c, YS); slot_free(c, YS + 1); }
        slot = TK2 % NSTG; seen = 0;
        const u32 end = TK2 + 64 - (nfull >> 1);
        struct Own { u32x4 a[2], b[8]; } U, V;
        auto fetch2 = [&](Own& o, u32 tn) __attribute__((always_inline)) {
            if (RARE(rfl(seen) < 2 * tn + 2) && ok && tn < end) {
                if (CPRIO) __builtin_amdgcn_s_setprio(0);
                ok = lds_wait(c, C_FULL + slot, 2 * tn + 2, true, CNAP) != 0;
                if (CPRIO) __builtin_amdgcn_s_setprio(CPRIO);
            }
            CBAR();
            OS(0, tn - 2);
            __builtin_amdgcn_sched_barrier(0);
#pragma unroll
            for (int i = 0; i < 8; ++i) o.b[i] = *(const LAS u32x4*)(pb + slot * STG + 8192 + (i >> 2) * 8192 + (i & 3) * 1024);
#pragma unroll
            for (int f = 0; f < 2; ++f) o.a[f] = *(const LAS u32x4*)(po + slot * STG + f * 4096);
            seen = *(const LAS u32*)&c.ctl[C_FULL + (slot == NSTG - 1 ? 0 : slot + 1)];
        };
        auto step2 = [&](const Own& o, Own& nx, u32 tn) __attribute__((always_inline)) {
            fetch2(nx, tn);
#pragma unroll
            for (int i = 0; i < 8; ++i) oacc[i & 3] = mfma(o.a[i >> 2], o.b[i], oacc[i & 3]);
#pragma unroll
            for (int i = 0; i < 5; ++i) { __builtin_amdgcn_sched_group_barrier(0x100, 2, 0); __builtin_amdgcn_sched_group_barrier(0x8, 1, 0); }
            release(tn < end ? 0 : 128);
        };
        fetch2(U, TK2);
        release(TK2 < end ? 0 : 128);
        u32 tk = TK2;
#pragma unroll 1
        for (; tk + 1 < end; tk += 2) { step2(U, V, tk + 1); step2(V, U, tk + 2); }
        if (tk < end)
#pragma unroll
            for (int i = 0; i < 8; ++i) oacc[i & 3] = mfma(U.a[i >> 2], U.b[i], oacc[i & 3]);
        ok = ok && !rfl(c.ctl[C_ABORT]);
        const u32 o = out_off(c, ((t.rfo + wr) & 31) * 16, c.rank * 2048 + t.cc * 128 + wc * 64);
#pragma unroll
        for (int r = 0; r < 4; r += 2)
            st_out(c, o + r * (NTOT * 2), ok ? (u32x2){pk_relu(oacc[0][r], oacc[1][r]), pk_relu(oacc[2][r], oacc[3][r])} : (u32x2){0x7fc07fc0u, 0x7fc07fc0u},
                   ok ? (u32x2){pk_relu(oacc[0][r + 1], oacc[1][r + 1]), pk_relu(oacc[2][r + 1], oacc[3][r + 1])} : (u32x2){0x7fc07fc0u, 0x7fc07fc0u});
        STAMPW(c, 10);
    }
}

DEV void gemm_wg(const Ctx& c) {
    const int g = c.wg - NCOMM, xcd = g & 7, idx = g >> 3, grp = xcd >> 2, k = idx % 7;
    const bool big = k < 4;
    Tile t;
    t.cc = (xcd & 3) * 4 + idx / 7;
    t.h = grp * 4 + k >= 7 || k >= 4;
    t.owner = (c.rank + (grp ? (k < 3 ? 5 + k : 1 + k) : (k < 4 ? 1 + k : k - 3))) & 7;
    t.rf = t.owner * 32 + t.h * 16; t.rfo = c.rank * 32 + (grp * 4 + (k & 3)) * 4;
    if (c.wave >= 8) {
        const int L = c.wave - 8, J = (L + JROT) & 3;
        if (J) {
            wg_barrier();
            for (int i = 0; i < (J == 1 ? PS1 : J == 2 ? PS2 : PS3); ++i) __builtin_amdgcn_s_sleep(24);
        }
#pragma unroll 1
        for (int job = J * NGEMM + g; job < 1280; job += J < 2 ? 1280 : 2 * NGEMM) {
            if (job >= 4 * NGEMM) for (int i = 0; i < P2; ++i) __builtin_amdgcn_s_sleep(24);
            pack_job(c, job);
            if (job < 4 * NGEMM) STAMPW(c, 3);
        }
        STAMPW(c, 1);
        __builtin_amdgcn_s_setprio(LPRIO);
        const u32 call = get_call(c), xr = XROT ? (t.cc & 1) | (t.cc & 2) << 2 | (k & 3) << 1 | (t.cc & 3) << 4 : 0;
        if (L == NISS) { if (big) lister<true>(c, t, xr, call); else lister<false>(c, t, xr, call); }
        if (L < NISS || L4) { if (big) loader<true>(c, t, call); else loader<false>(c, t, call); }
        STAMPW(c, 2);
    } else {
        wg_barrier();
        if (big) consumer<true>(c, t, c.wave >> 1, c.wave & 1); else consumer<false>(c, t, c.wave >> 1, c.wave & 1);
    }
}

#define WSUM 8
#define EARLY 0
#define WCLV 4

constexpr u32 NFIX = 4;

DEV u32 take_seat(const Ctx& c, u32 call) {
    u32 xcc;
    asm volatile("s_getreg_b32 %0, hwreg(HW_REG_XCC_ID, 0, 4)" : "=s"(xcc));
    const u32 grp = (0xb2u >> c.rank) & 1 ? 0xb2u : 0x4du, j = (c.wg >> 3) * 4 + (xcc & 3);
    u32 near = TOPO ? (xcc & 4 ? ~grp & 0xff : grp & ~(1u << c.rank)) : 0, seat = NPULL;
    const u32 n = __builtin_popcount(near);
    if (!TOPO) seat = min((u32)c.wg, (u32)NPULL);
    else if (j < 4 * n) {
        for (u32 i = (j + SPREAD * (n == 4 ? j / n : 0)) % n; i; --i) near &= near - 1;
        seat = ((__builtin_ctz(near) - c.rank) & 7) - 1 + 7 * (j / n);
    }
    for (bool first = true;; first = false) {
        u64 old = ~0ull;
        if (c.lane == 0 ? seat < NPULL : first && c.lane < NR)
            old = __hip_atomic_fetch_max((GAS u64*)(c.me + (c.lane ? WS_CLM + c.lane * 128 : WS_SEAT + seat * 8)), (u64)call << 32 | 4 * NFIX, __ATOMIC_RELAXED, __HIP_MEMORY_SCOPE_AGENT);
        if (rfl((u32)(old >> 32)) < call) return seat;
        if (first) for (int i = 0; i < 4; ++i) __builtin_amdgcn_s_sleep(32);
        const u64 v = __hip_atomic_load((const GAS u64*)(c.me + WS_SEAT + min(c.lane, NPULL - 1) * 8), __ATOMIC_RELAXED, __HIP_MEMORY_SCOPE_AGENT);
        const u64 open = __ballot((u32)(v >> 32) < call);
        if (!open) return NPULL;
        seat = __builtin_ctzll(open);
    }
}

DEV void watcher(const Ctx& c, int p, int sub, u32 call) {
    static_assert(NQY == 64, "one message per lane");
    const u32 id = sub + 4 * c.lane;
    const char* fp = c.me + WS_FY + (p * NMSG + id) * 8;
    bool bad = !lds_wait(c, C_YGO, 1, false, 1);
    STAMPW(c, 1);
    u64 pending = ~0ull, t0 = 0;
    u32 qn = 0;
    for (u32 it = 1; pending; ++it) {
        const u64 v = bad ? 0 : ld_sys64(fp);
        const bool mine = ((pending >> c.lane) & 1) && (bad || (u32)(v >> 32) >= call);
        const u64 m = __ballot(mine);
        if (m) {
            const bool poison = bad || tmo_set(c, call);
            if (mine) c.ctl[C_Q + qn + lane_rank(m)] = 0x80000000u | id << 20 | (poison ? YM_BAD : (u32)v & (YM_BAD | YM_RAW | 0x3ff));
            qn += __popcll(m); pending &= ~m; t0 = 0;
            c.ctl[C_QN] = qn;
        } else {
            if ((it & 15) == 0) {
                if (tmo_set(c, call)) bad = true;
                else if (expired(t0, DEADLINE)) { tmo_raise(c, call); bad = true; }
            }
            __builtin_amdgcn_s_sleep(2);
        }
    }
    STAMPW(c, 2);
}

DEV void pull_inputs(const Ctx& c, char* peer, rsrc_t pr, u32 lo, int d, int sub, int p, u32 call) {
    GAS u64* ctr = (GAS u64*)(c.me + WS_CLM + d * 128);
    const u32 voff = c.lane * 16, win = WSUM / 4 + (sub < WSUM % 4), WCL = win + WCLV;
    bool bad = false;
#pragma unroll 1
    for (;;) {
        const u32 t = lds_inc(c, C_TK);
        if (t >= WCL) lds_wait(c, C_LAND, t - WCL + 1, false, 1);
        u32 n = t * 4 + sub;
        if (t >= NFIX) {
            if (c.lane == 0) n = (u32)__hip_atomic_fetch_add(ctr, 1ull, __ATOMIC_RELAXED, __HIP_MEMORY_SCOPE_AGENT);
            n = rfl(n);
        }
        if (n + 64 >= NPOS) c.ctl[C_YGO] = 1;
        if (n >= NPOS) break;
        const u32 code = rfl(ORDER.v[d - 1][n]), s = (code >> 8) & 7, ix = code & 31;
        const bool isw = code & 0x8000;
        const u32 rc = isw ? ix : ((((ix >> 2) + c.rank) & 7) << 2 | (ix & 3)), id = isw ? 256 + s * 16 + rc : s * 32 + rc;
        const char* fp = c.me + WS_FIN + (((call & 1) * NR + p) * NCHIN + id) * 8;
        u32 meta = 0;
        u64 t0 = 0;
        for (u32 it = 1; !bad; ++it) {
            const u64 v = ld_sys64(fp);
            meta = rfl((u32)v);
            if (rfl((u32)(v >> 32)) >= call) break;
            if ((it & 15) == 0) {
                if (tmo_set(c, call)) bad = true;
                else if (expired(t0, DEADLINE)) { tmo_raise(c, call); bad = true; }
            }
            __builtin_amdgcn_s_sleep(2);
        }
        const u32 u = lds_inc(c, C_IS);
        if (bad) { lds_inc(c, C_LAND); continue; }
        if (u >= win) lds_wait(c, C_LAND, u - win + 1, false, WNAP, 0xffff);
        __builtin_amdgcn_s_setprio(UPRIO);
        ST2(c, 1024 + u, (meta & 0x7ff) >> 3);
        const int ks = p * 8 + s;
        const u32 src = WS_PK + (call & 1) * PK_PAR + (isw ? id + c.rank * 128 : id) * CHB;
        const u32 d0 = isw ? wb_off(wb_ks(ks, call), 0, rc * 8) : xa_off(ks, 0, rc * 8), d1 = d0 + (isw ? 128 : 256) * 1024;
        u32x4 R[16];
        if (meta & YM_RAW) {
#pragma unroll
            for (int f = 0; f < 16; ++f) R[f] = ld16(pr, voff + (f & 3) * 1024, src + (f >> 2) * 4096);
            WAIT_VM0();
            lds_inc(c, C_LAND);
            ST2(c, 1536 + u, 0); ST2(c, u, 0);
            store_raw(c, R, d0, d1);
        } else {
            const u32 bytes = min((meta & 0x7ff) * 16, PK_TAIL + LSLOT), tl = (u32)(size_t)c.lds + lo;
            const rsrc_t tr = __builtin_amdgcn_make_buffer_rsrc(peer, 0, rfl(src + bytes), 0x00020000);
            if (bytes > PK_TAIL + 2048) glds4(tr, voff, src + PK_TAIL + 2048, tl + 2048);
            glds2(tr, voff, src + PK_TAIL, tl);
            u32x4 L[8], D[2];
#pragma unroll
            for (int i = 0; i < 2; ++i) D[i] = ld16(pr, voff + i * 1024, src + 8192);
#pragma unroll
            for (int i = 0; i < 8; ++i) L[i] = ld16(pr, voff + (i & 3) * 1024, src + (i >> 2) * 4096);
            asm volatile("s_waitcnt vmcnt(%0)" :: "n"(EARLY) : "memory");
            if (c.lane == 0) __hip_atomic_fetch_add((LAS u32*)&c.ctl[C_LAND], 1u, __ATOMIC_RELAXED, __HIP_MEMORY_SCOPE_WORKGROUP);
            WAIT_VM0();
            ST2(c, 1536 + u, 0); ST2(c, u, 0);
            const u32 hv = *(const LAS u32*)(size_t)(tl + (c.lane & 31) * 4);
            u32 cnt = 0;
#pragma unroll
            for (int j = 0; j < 8; ++j) cnt += __builtin_popcount(~(D[j >> 2][j & 3] | D[j >> 2][j & 3] >> 1) & 0x55555555u);
            const u32 h0 = rfl(hv), T = (h0 & 0xff) * 0x01010101u, nexc = min((h0 >> 8) & 0xff, 31u);
            u32 pb = 32 * (1 + nexc) + 4 * (wave_scan(cnt) - cnt);
#pragma unroll
            for (int hf = 0; hf < 16; hf += 8) {
                u32 cd[16], sel[16], sa[8], ps[8];
                u32x2 q[8];
#pragma unroll
                for (int f = 0; f < 8; ++f) {
                    const u32 w = D[hf >> 3][f >> 1] >> (f & 1) * 4;
                    cd[2 * f] = w & 0x03030303u; cd[2 * f + 1] = (w >> 2) & 0x03030303u;
                    const u32 ea = (0x04040404u - cd[2 * f]) & 0x04040404u, eb = (0x04040404u - cd[2 * f + 1]) & 0x04040404u;
                    sel[2 * f] = *(const LAS u32*)(size_t)__builtin_amdgcn_udot4(ea, 0x08040201u, (u32)(size_t)c.lds + L_LUT2, false);
                    sel[2 * f + 1] = *(const LAS u32*)(size_t)__builtin_amdgcn_udot4(eb, 0x08040201u, (u32)(size_t)c.lds + L_LUT2, false);
                    const LAS u32* z = (const LAS u32*)(size_t)(tl + (pb >> 5) * 4);
                    q[f] = (u32x2){z[0], z[1]};
                    sa[f] = sad(ea, 0); ps[f] = pb;
                    pb = sad(eb, pb + sa[f]);
                }
                __builtin_amdgcn_sched_barrier(0);
#pragma unroll
                for (int f = 0; f < 8; ++f) {
                    const u32 va = __builtin_amdgcn_alignbit(q[f][1], q[f][0], ps[f]), vb = va >> sa[f];
                    const u32 ha = T - ((perm(va >> 4, va, sel[2 * f]) & 0x0f0f0f0fu) | cd[2 * f]), hb = T - ((perm(vb >> 4, vb, sel[2 * f + 1]) & 0x0f0f0f0fu) | cd[2 * f + 1]);
                    const u32 la = L[(hf + f) >> 1][(f & 1) * 2], lb = L[(hf + f) >> 1][(f & 1) * 2 + 1];
                    const u32x4 x = {perm(ha, la, 0x05010400u), perm(ha, la, 0x07030602u), perm(hb, lb, 0x05010400u), perm(hb, lb, 0x07030602u)};
                    u32x4 r;
#pragma unroll
                    for (int i = 0; i < 4; ++i) r[i] = __builtin_amdgcn_alignbit(x[i], x[i], 1);
                    st16(c, r, voff, (hf ? d1 : d0) + f * 1024);
                }
            }
            if ((u32)c.lane - 1 < nexc)
                __builtin_amdgcn_raw_buffer_store_b16((u16)hv, c.rs, (hv >> 28 & 1 ? d1 : d0) + (hv >> 25 & 7) * 1024 + (hv >> 16 & 511) * 2, 0, 17);
        }
        STAMPW(c, 3);
        ST2(c, 128 + u, 0);
        WAIT_VM0();
        if (c.lane == 0) st_sys(c.me + (isw ? WS_WR : WS_XR) + rc * 256 + ks * 4, call);
        __builtin_amdgcn_s_setprio(0);
        ST2(c, 512 + u, 0);
    }
    if (c.lane == 0) __hip_atomic_fetch_or((LAS u32*)&c.ctl[C_LAND], 1u << 16, __ATOMIC_RELAXED, __HIP_MEMORY_SCOPE_WORKGROUP);
}

template <int NF>
DEV void decode_y(const Ctx& c, u32 e, u32 lo, rsrc_t pr, u32 src, u32 row0, u32 col0, u32 tk) {
    constexpr int G = NF * 4;
    constexpr u32 CODEB = NF == 5 ? 2048 : 1536;
    const u32 ldsbase = (u32)(size_t)c.lds;
    const LAS char* l = c.lds + lo;
    u32x2 P[G];
    u32 hv = 0, nexc = 0;
    if (e & YM_BAD) {
#pragma unroll
        for (int g = 0; g < G; ++g) P[g] = (u32x2){0x7fc07fc0u, 0x7fc07fc0u};
    } else if (e & YM_RAW) {
#pragma unroll
        for (int i = 0; i < G / 2; ++i) {
            const u32x4 v = ld16(pr, c.lane * 16 + (i & 3) * 1024, src + (i >> 2) * 4096);
            P[2 * i] = (u32x2){v[0], v[1]}; P[2 * i + 1] = (u32x2){v[2], v[3]};
        }
        __builtin_amdgcn_s_waitcnt(0x0f70);
    } else {
        const u32x4 da = *(const LAS u32x4*)(l + c.lane * 16);
        u32x4 db;
        if (NF == 5) db = *(const LAS u32x4*)(l + 1024 + c.lane * 16);
        else { const u32x2 t = *(const LAS u32x2*)(l + 1024 + c.lane * 8); db = (u32x4){t[0], t[1], 0, 0}; }
        hv = *(const LAS u32*)(l + CODEB + (c.lane & 31) * 4);
        const u32 D[8] = {da[0], da[1], da[2], da[3], db[0], db[1], db[2], db[3]};
        u32 cb[G], cnt = 0;
#pragma unroll
        for (int h = 0; h < 2; ++h) {
            u32* q = cb + 8 * h;
#pragma unroll
            for (int k = 0; k < 3; ++k) { q[2 * k] = D[3 * h + k] & 0x07070707u; q[2 * k + 1] = (D[3 * h + k] >> 3) & 0x07070707u; }
            q[6] = bfi(0x03030303u, D[3 * h] >> 6, D[3 * h + 2] >> 4) & 0x07070707u;
            q[7] = bfi(0x03030303u, D[3 * h + 1] >> 6, D[3 * h + 2] >> 5) & 0x07070707u;
        }
        if (NF == 5) { cb[G - 4] = D[6] & 0x07070707u; cb[G - 3] = (D[6] >> 3) & 0x07070707u; cb[G - 2] = D[7] & 0x07070707u; cb[G - 1] = (D[7] >> 3) & 0x07070707u; }
#pragma unroll
        for (int g = 0; g < G; ++g) cnt += __builtin_popcount(nz8(cb[g]));
        const u32 h0 = rfl(hv), B = (h0 & 0xff) * 0x01010101u;
        nexc = min((h0 >> 8) & 0xff, 31u);
        u32 pos = ldsbase + lo + CODEB + 4 * (1 + nexc) + wave_scan(cnt) - cnt, ps[G];
        u32x2 sel[G];
#pragma unroll
        for (int g = 0; g < G; ++g) {
            const u32 nz = nz8(cb[g]);
            const LAS u32* q = (const LAS u32*)(size_t)(pos & ~3u);
            sel[g] = *(const LAS u32x2*)(size_t)__builtin_amdgcn_udot4(nz, 0x08040201u, ldsbase + L_LUT, false);
            P[g] = (u32x2){q[0], q[1]};
            ps[g] = pos;
            pos += __builtin_popcount(nz);
        }
        __builtin_amdgcn_sched_barrier(0);
#pragma unroll
        for (int g = 0; g < G; ++g) {
            const u32 by = __builtin_amdgcn_alignbyte(P[g][1], P[g][0], ps[g]), lb = perm(by, by, sel[g][0]), hb = (cb[g] + B) & sel[g][1];
            P[g] = (u32x2){perm(hb, lb, 0x05010400u), perm(hb, lb, 0x07030602u)};
        }
    }
#pragma unroll
    for (int g = 0; g < G; g += 2) st_out(c, out_off(c, row0, col0) + ((g >> 2) * 16 + (g & 3)) * (NTOT * 2), P[g], P[g + 1]);
    ST2(c, 512 + 384 + tk, nexc);
    const u32 j = (hv >> 22) & 63, ln = hv >> 16, xo = (row0 + (j & 48) + (ln & 48) / 4 + (j >> 2 & 3)) * (NTOT * 2) + (col0 + (ln & 7) * 8 + (ln & 8) / 2 + (j & 3)) * 2;
    __builtin_amdgcn_raw_buffer_store_b16((u16)hv, c.ro, (u32)c.lane - 1 < nexc ? xo : ~1u, 0, OAUX);
}
DEV void fetch_y(const Ctx& c, char* peer, u32 e, u32 lo) {
    if (e & (YM_BAD | YM_RAW)) return;
    const u32 src = WS_YP + (c.rank * NMSG + ((e >> 20) & 0xff)) * YSLOT, bytes = min((e & 0x3ff) * 16, LSLOT);
    const rsrc_t rs = __builtin_amdgcn_make_buffer_rsrc(peer, 0, src + bytes, 0x00020000);
    glds4(rs, c.lane * 16, src, (u32)(size_t)c.lds + lo);
    if (bytes > 4096) glds2(rs, c.lane * 16, src + 4096, (u32)(size_t)c.lds + lo + 4096);
}

DEV void worker(const Ctx& c, char* peer, int d, int sub, int p, int w, u32 call) {
    const rsrc_t pr = __builtin_amdgcn_make_buffer_rsrc(peer, 0, WS_END, 0x00020000);
    pull_inputs(c, peer, pr, w * 2 * LSLOT, d, sub, p, call);
    STAMPW(c, 1);
    const u32 both = (w * 4 + 1) * LSLOT;
    static_assert(C_QN == C_YT + 1 && C_YL == C_YT + 2, "one read");
    u32 lo = w * 2 * LSLOT, e = 0, tk = 0;
#pragma unroll 1
    for (bool first = true;; first = false) {
        u32 tn = ~0u, en = 0;
        if (!first) {
            const u32 st = c.ctl[C_YT + (c.lane & 3)], yt = __builtin_amdgcn_readlane(st, 0);
            if (YPRE && yt < (u32)__builtin_amdgcn_readlane(st, 1) && yt + YPEND <= NQY && yt < (u32)__builtin_amdgcn_readlane(st, 2) + YWIN) {
                tn = lds_inc(c, C_YT);
                if (tn < NQY) en = rfl(c.ctl[C_Q + tn]);
                if (en) { ST2(c, 1024 + 448 + tn, 5); fetch_y(c, peer, en, both - lo); }
            }
            const u32 id = (e >> 20) & 0xff, wc = id & 1, rg = (id >> 1) & 7, cc = id >> 4;
            decode_y<4>(c, DIAG & 1 ? e | YM_BAD : e, lo, pr, WS_YP + (c.rank * NMSG + id) * YSLOT, rg * 64, p * 2048 + cc * 128 + wc * 64, tk);
            ST2(c, 512 + 448 + tk, 0);
        }
        if (en) asm volatile("s_waitcnt vmcnt(9)" ::: "memory");
        else {
            if (tn == ~0u) tn = lds_inc(c, C_YT);
            if (tn >= NQY) break;
            en = lds_wait(c, C_Q + tn, 1, false, 1);
            if (!en) { tmo_raise(c, call); break; }
            if (tn + 1 > YWIN) lds_wait(c, C_YL, tn + 1 - YWIN, false, 0);
            ST2(c, 1024 + 448 + tn, 5);
            fetch_y(c, peer, en, both - lo);
            WAIT_VM0();
        }
        lds_add1(c, C_YL);
        ST2(c, 1536 + 448 + tn, 0);
        if (tn == 0) STAMPW(c, 4);
        STAMPW(c, 5);
        e = en; tk = tn; lo = both - lo;
    }
    STAMPW(c, 2);
}

__global__ void __launch_bounds__(NTHREADS, 3) fused_kernel(Args a) {
    extern __shared__ __attribute__((aligned(128))) char smem[];
    Ctx c;
    c.x = a.x; c.w = a.w; c.out = a.out; c.rank = a.rank & 7; c.me = a.me;
    c.wg = blockIdx.x; c.wave = rfl(threadIdx.x >> 6); c.lane = threadIdx.x & 63;
    c.peer_lane = a.ws[c.lane & 7];
    c.rs = __builtin_amdgcn_make_buffer_rsrc(c.me, 0, WS_END, 0x00020000);
    c.ro = __builtin_amdgcn_make_buffer_rsrc(c.out, 0, DIAG & 2 ? 0 : 512 * NTOT * 2, 0x00020000);
    c.lds = (LAS char*)smem; c.ctl = (volatile LAS u32*)(c.lds + L_CTL);
    STAMPW(c, 0);
    u32 init = 0;
    static_assert(C_CALL == 0, "thread 0");
    if (threadIdx.x == 0) {
        GAS u32* cnt = (GAS u32*)(c.me + WS_CNT + c.wg * 128);
        init = __hip_atomic_load(cnt, __ATOMIC_RELAXED, __HIP_MEMORY_SCOPE_AGENT) + 1;
        __hip_atomic_store(cnt, init, __ATOMIC_RELAXED, __HIP_MEMORY_SCOPE_AGENT);
    }
    if (threadIdx.x < C_WORDS) c.ctl[threadIdx.x] = init;
    if (c.wave == 1) {
        u32 s0, s1;
        if (c.wg >= NCOMM) {
            const u32 mask = (c.lane >> 2 & 12) | (c.lane >> 3 & 1) | (c.lane >> 1 & 2);
            u32 n = c.lane & 3;
            u64 sel = 0x0c0c0c0c03020100ull;
            for (int b = 0; b < 4; ++b)
                if ((mask >> b) & 1) { sel = (sel & ~(0xffull << (8 * n))) | (u64)(4 + b) << (8 * n); ++n; }
            s0 = (u32)sel; s1 = n >= 4 ? (u32)(sel >> 32) : 0x03020100u;
        } else {
            u32 n = 0;
            s0 = 0x0c0c0c0cu; s1 = 0;
            for (int b = 0; b < 4; ++b)
                if ((c.lane >> b) & 1) { s0 = (s0 & ~(0xffu << (8 * b))) | n << (8 * b); s1 |= 0xffu << (8 * b); ++n; }
        }
        c.ctl[C_LUT + 2 * c.lane] = s0; c.ctl[C_LUT + 2 * c.lane + 1] = s1;
    }
    if (c.wave == 2 && c.lane < 16) {
        u32 n = 0, s0 = 0x0c0c0c0cu, s1 = 0x0c0c0c0cu, sd = 0x0c0c0c0cu;
        for (int b = 0; b < 4; ++b)
            if ((c.lane >> b) & 1) {
                u32& t = n & 1 ? s1 : s0;
                t = (t & ~(0xffu << (n >> 1) * 8)) | b << (n >> 1) * 8;
                sd = (sd & ~(0xffu << 8 * b)) | ((n & 1) * 4 + (n >> 1)) << 8 * b;
                ++n;
            }
        if (c.wg >= NCOMM) { c.ctl[C_LUT2 + 2 * c.lane] = s0; c.ctl[C_LUT2 + 2 * c.lane + 1] = s1; }
        else c.ctl[C_LUT2 + c.lane] = sd;
    }
    if (c.wg >= NCOMM) gemm_wg(c);
    else {
        wg_barrier();
        const u32 call = get_call(c);
        if (c.wave == 0) c.ctl[C_SEAT] = take_seat(c, call) + 1;
        const u32 seat = lds_wait(c, C_SEAT, 1, false, 1) - 1;
        if (seat >= NPULL) return;
        const int d = 1 + seat % 7, sub = seat / 7, p = (c.rank + d) & 7;
        if (c.wave == 0) watcher(c, p, sub, call);
        else worker(c, a.ws[p], d, sub, p, c.wave - 1, call);
    }
}

}

extern "C" void kernel_launch(int rank, int nranks, ncclComm_t comm, hipStream_t stream,
                              void* const* d_in, const int* in_sizes, int n_in,
                              void* d_out, int out_size, const int* in_dtypes, int out_dtype,
                              void* const* d_out_all, void* const* d_ws_all, size_t ws_bytes) {
    if (nranks != NR || (unsigned)rank >= (unsigned)NR || n_in != 2 || ws_bytes < WS_END || in_sizes[0] != 4096 * KSH ||
        in_sizes[1] != KSH * NTOT || out_size != 512 * NTOT) {
        fprintf(stderr, "unexpected problem shape\n");
        return;
    }
    HIP_CHECK(hipFuncSetAttribute((const void*)fused_kernel, hipFuncAttributeMaxDynamicSharedMemorySize, LDS_BYTES));
    Args a;
    a.x = (const u16*)d_in[0]; a.w = (const u16*)d_in[1]; a.out = (u16*)d_out;
    for (int p = 0; p < NR; ++p) a.ws[p] = (char*)d_ws_all[p];
    a.me = a.ws[rank]; a.rank = rank; a.pad = 0;
    hipLaunchKernelGGL(fused_kernel, dim3(NWG), dim3(NTHREADS), LDS_BYTES, stream, a);
    HIP_CHECK(hipGetLastError());
}
```
